```python
import numpy as np

kernel_name = "gpu_ag_gemm_m8192_k8192_n8192_f32_relu"


ABSMAX_RTOL = 0.001

COMPUTE_DTYPE = "float32"
EPILOGUE = "relu"
IN_QUANT = None
OUT_QUANT = None
IN_DTYPES = {"x": "float32", "w_mat": "float32"}
WORLD_SIZE = 8

def setup_inputs():

    x = np.random.randn(8192, 8192).astype(np.float32)
    w_mat = (np.random.randn(8192, 8192) / np.sqrt(8192)).astype(np.float32)
    return {"x": x, "w_mat": w_mat}

def _epilogue(y):
    return np.maximum(y, 0.0).astype(np.float32)

def reference(x, w_mat):
    y = (x @ w_mat).astype(np.float32)
    return _epilogue(y)
```

```cpp
#include <hip/hip_runtime.h>
#include <rccl/rccl.h>
#include <cstdio>

#define HIP_CHECK(expr) do { hipError_t _e = (expr); if (_e != hipSuccess) { \
    fprintf(stderr, "HIP error %s at %s:%d\n", hipGetErrorString(_e), __FILE__, __LINE__); \
    return; } } while (0)

#define LAS __attribute__((address_space(3)))
typedef _Float16 f16x8 __attribute__((ext_vector_type(8)));
typedef float f32x4 __attribute__((ext_vector_type(4)));
typedef int i32x4 __attribute__((ext_vector_type(4)));
typedef unsigned u32x4 __attribute__((ext_vector_type(4)));

namespace {

constexpr int NR = 8;
constexpr int KDIM = 8192, NLOC = 1024;
constexpr int KSTEPS = KDIM / 64;
constexpr unsigned KSTEP_B = 32768, HALF_B = 16384;
constexpr unsigned SLOT_B = KSTEPS * KSTEP_B;
constexpr int NSLOT = 25;
constexpr int SLOT_OWN_AWAY = 16, SLOT_W = 17, SLOT_WR = 21;
constexpr unsigned PAR_B = NSLOT * SLOT_B;
constexpr size_t CTRL_B = 1u << 20;
constexpr size_t WS_NEED = CTRL_B + 2 * (size_t)PAR_B;
constexpr unsigned OFF_TICKET = 0;
constexpr unsigned OFF_TMO = 1024;
constexpr unsigned OFF_PROG = 4096;
constexpr unsigned OFF_DONE = 16384;
constexpr unsigned OFF_EXP = 65536;
constexpr unsigned long long DEADLINE = 50000000ull;
constexpr int NWG = 256, NPUSH = 7, NGEMM = 16;
constexpr int LDS_B = 131072 + 256;
constexpr int AUX_SYS = 17;
constexpr unsigned RSRC_W3 = 0x00020000u;

struct Params {
  const float* x;
  const float* w;
  float* out[NR];
  unsigned char* ws[NR];
  int rank;
};

__device__ __forceinline__ unsigned ld_sys(const unsigned* p) { return __hip_atomic_load(p, __ATOMIC_RELAXED, __HIP_MEMORY_SCOPE_SYSTEM); }
__device__ __forceinline__ void st_sys(unsigned* p, unsigned v) { __hip_atomic_store(p, v, __ATOMIC_RELAXED, __HIP_MEMORY_SCOPE_SYSTEM); }
#define DRAIN() asm volatile("s_waitcnt vmcnt(0)" ::: "memory")

__device__ __noinline__ int wait_prog(const unsigned* pa, const unsigned* pb, unsigned base, int need, unsigned* tmo, unsigned seq) {
  const int lane = threadIdx.x & 63;
  const unsigned long long t0 = __builtin_amdgcn_s_memrealtime();
  for (;;) {
    const int da = (int)(ld_sys(pa + lane) - base), db = (int)(ld_sys(pb + lane) - base);
    const int d = da < db ? da : db;
    if (__all(d >= need)) {
      int m = 0;
#pragma unroll
      for (int bit = 128; bit >= 1; bit >>= 1) if (__all(d >= m + bit)) m += bit;
      return m > KSTEPS ? KSTEPS : m;
    }
    if (__builtin_amdgcn_readfirstlane(__hip_atomic_load(tmo, __ATOMIC_RELAXED, __HIP_MEMORY_SCOPE_AGENT)) == seq ||
        __builtin_amdgcn_s_memrealtime() - t0 > DEADLINE) {
      if (lane == 0) __hip_atomic_store(tmo, seq, __ATOMIC_RELAXED, __HIP_MEMORY_SCOPE_AGENT);
      return KSTEPS;
    }
    __builtin_amdgcn_s_sleep(2);
  }
}

__device__ __forceinline__ unsigned wave_max(unsigned m) {
#pragma unroll
  for (int o = 32; o >= 1; o >>= 1) { const unsigned t = (unsigned)__shfl_xor((int)m, o); m = t > m ? t : m; }
  return m;
}
__device__ __forceinline__ unsigned absbits(float f) { return __float_as_uint(f) & 0x7fffffffu; }
__device__ __forceinline__ int scale_eb(unsigned amax) { const int eb = (int)(amax >> 23); return eb < 14 ? 14 : eb; }

__device__ __forceinline__ void prep_w(const Params& P, unsigned char* ws, unsigned seq, int ux, int ui, LAS unsigned* red) {
  const int tid = threadIdx.x, wid = __builtin_amdgcn_readfirstlane(tid >> 6), lane = tid & 63;
  const int col0 = ux * 128 + ui * 4;
  const float* wp = P.w + (size_t)(wid * 1024 + lane * 8) * NLOC + col0;
  f32x4 v[2][8];
#pragma unroll
  for (int g = 0; g < 2; ++g)
#pragma unroll
    for (int j = 0; j < 8; ++j) v[g][j] = *(const f32x4*)(wp + (size_t)(g * 512 + j) * NLOC);
  unsigned m[4] = {0u, 0u, 0u, 0u};
#pragma unroll
  for (int g = 0; g < 2; ++g)
#pragma unroll
    for (int j = 0; j < 8; ++j)
#pragma unroll
      for (int c = 0; c < 4; ++c) { const unsigned a = absbits(v[g][j][c]); m[c] = a > m[c] ? a : m[c]; }
#pragma unroll
  for (int c = 0; c < 4; ++c) m[c] = wave_max(m[c]);
  __syncthreads();
  if (lane < 4) red[wid * 4 + lane] = lane == 0 ? m[0] : lane == 1 ? m[1] : lane == 2 ? m[2] : m[3];
  __syncthreads();
  int eb[4];
#pragma unroll
  for (int c = 0; c < 4; ++c) {
    unsigned a = 0;
#pragma unroll
    for (int q = 0; q < 8; ++q) { const unsigned t = red[q * 4 + c]; a = t > a ? t : a; }
    eb[c] = scale_eb(a);
  }
  const int slot = SLOT_W + (col0 >> 8);
  const unsigned par = seq & 1u;
  const __amdgpu_buffer_rsrc_t rd = __builtin_amdgcn_make_buffer_rsrc(ws + CTRL_B + (size_t)par * PAR_B + (size_t)slot * SLOT_B, 0, SLOT_B, RSRC_W3);
#pragma unroll
  for (int c = 0; c < 4; ++c) {
    const float s = __uint_as_float((unsigned)(268 - eb[c]) << 23);
    const int n = (col0 & 255) + c;
#pragma unroll
    for (int g = 0; g < 2; ++g) {
      f16x8 h;
#pragma unroll
      for (int j = 0; j < 8; ++j) h[j] = (_Float16)(v[g][j][c] * s);
      const unsigned off = (unsigned)(wid * 16 + g * 8 + (lane >> 3)) * KSTEP_B + (unsigned)n * 128u + (unsigned)(((lane & 7) ^ (n & 7)) * 16);
      __builtin_amdgcn_raw_buffer_store_b128(__builtin_bit_cast(u32x4, h), rd, off, 0, AUX_SYS);
    }
  }
  if (tid < 4) {
    const int e = tid == 0 ? eb[0] : tid == 1 ? eb[1] : tid == 2 ? eb[2] : eb[3];
    st_sys((unsigned*)(ws + OFF_EXP) + (par * NSLOT + slot) * 256 + (col0 & 255) + tid, (unsigned)(e - 141));
  }
  DRAIN();
  __syncthreads();
  if (tid == 0) st_sys((unsigned*)(ws + OFF_PROG) + slot * 64 + (ux & 1) * 32 + ui, seq * 256u + KSTEPS);
}

constexpr int KC = 4;
constexpr int UPT = KSTEPS / KC;
constexpr int PD = 4;
constexpr int PL = 2;

struct Unit { unsigned src, dst; int dslot, done; };

template <class Seq>
__device__ __forceinline__ void push_run(const Seq& seqf, int u0, int u1, __amdgpu_buffer_rsrc_t rs, __amdgpu_buffer_rsrc_t rd,
                                         unsigned voff, unsigned* pprog  , unsigned base) {
  const int lane = threadIdx.x & 63;
  u32x4 buf[PD][2];
#pragma unroll
  for (int d = 0; d < PD; ++d) {
    const Unit q = seqf(u0 + d < u1 ? u0 + d : u1 - 1);
    buf[d][0] = __builtin_amdgcn_raw_buffer_load_b128(rs, voff, q.src, AUX_SYS);
    buf[d][1] = __builtin_amdgcn_raw_buffer_load_b128(rs, voff + 1024, q.src, AUX_SYS);
  }
  for (int u = u0; u < u1; u += PD) {
#pragma unroll
    for (int d = 0; d < PD; ++d) {
      const int uu = u + d;
      const Unit q = seqf(uu);
      __builtin_amdgcn_raw_buffer_store_b128(buf[d][0], rd, voff, q.dst, AUX_SYS);
      __builtin_amdgcn_raw_buffer_store_b128(buf[d][1], rd, voff + 1024, q.dst, AUX_SYS);
      const Unit n = seqf(uu + PD < u1 ? uu + PD : u1 - 1);
      buf[d][0] = __builtin_amdgcn_raw_buffer_load_b128(rs, voff, n.src, AUX_SYS);
      buf[d][1] = __builtin_amdgcn_raw_buffer_load_b128(rs, voff + 1024, n.src, AUX_SYS);
      asm volatile("s_waitcnt vmcnt(%0)" :: "n"(5 * PL + 2) : "memory");
      const Unit f = seqf(uu - PL >= u0 ? uu - PL : u0);
      const unsigned val = uu - PL >= u0 ? base + (unsigned)f.done : base;
      if (lane == 0) st_sys(pprog + f.dslot * 64, val);
    }
  }
  DRAIN();
#pragma unroll
  for (int d = PL; d >= 1; --d) {
    const Unit f = seqf(u1 - d);
    if (lane == 0) st_sys(pprog + f.dslot * 64, base + (unsigned)f.done);
    DRAIN();
  }
}

__device__ __forceinline__ void push_role(const Params& P, unsigned char* ws, unsigned seq, int peer, int wl  ) {
  const int lane = threadIdx.x & 63;
  const int rank = P.rank, me = rank >> 1;
  const unsigned par = seq & 1u, base = seq * 256u;
  unsigned char* pw = P.ws[peer];
  const __amdgpu_buffer_rsrc_t rs = __builtin_amdgcn_make_buffer_rsrc(ws + CTRL_B + (size_t)par * PAR_B, 0, PAR_B, RSRC_W3);
  const __amdgpu_buffer_rsrc_t rd = __builtin_amdgcn_make_buffer_rsrc(pw + CTRL_B + (size_t)par * PAR_B, 0, PAR_B, RSRC_W3);
  const unsigned voff = (unsigned)wl * (KC * 512u) + (unsigned)lane * 16u;
  const unsigned* prog = (const unsigned*)(ws + OFF_PROG);
  unsigned* pprog = (unsigned*)(pw + OFF_PROG) + wl;
  unsigned* tmo = (unsigned*)(ws + OFF_TMO);
  const unsigned* ex = (const unsigned*)(ws + OFF_EXP) + par * NSLOT * 256;
  unsigned* pex = (unsigned*)(pw + OFF_EXP) + par * NSLOT * 256;
  if (((peer ^ rank) & 1) == 0) {
    const int s0 = me * 3;
    wait_prog(prog + s0 * 64, prog + (s0 + 1) * 64, base, KSTEPS, tmo, seq);
    wait_prog(prog + (s0 + 2) * 64, prog + (s0 + 2) * 64, base, KSTEPS, tmo, seq);
    if (wl < 12) st_sys(pex + s0 * 256 + wl * 64 + lane, ld_sys(ex + s0 * 256 + wl * 64 + lane));
    auto sq = [=](int u) {
      const int t = u < UPT ? 0 : 1 + ((u - UPT) & 1), c = u < UPT ? u : (u - UPT) >> 1;
      const unsigned o = (unsigned)(s0 + t) * SLOT_B + (unsigned)c * (KC * KSTEP_B);
      return Unit{o, o, s0 + t, (c + 1) * KC};
    };
    push_run(sq, 0, 3 * UPT, rs, rd, voff, pprog, base);
  } else {
    constexpr int LEAD = 4;
    const int sw = SLOT_W + (peer >> 1), dx = 12 + me, dw = SLOT_WR + me;
    wait_prog(prog + SLOT_OWN_AWAY * 64, prog + SLOT_OWN_AWAY * 64, base, KSTEPS, tmo, seq);
    if (wl < 4) st_sys(pex + dx * 256 + wl * 64 + lane, ld_sys(ex + SLOT_OWN_AWAY * 256 + wl * 64 + lane));
    auto sq = [=](int u) {
      bool isw; int c;
      if (u < LEAD) { isw = false; c = u; }
      else if (u < 2 * UPT - LEAD) { const int v = u - LEAD; isw = (v & 1) == 0; c = isw ? v >> 1 : LEAD + (v >> 1); }
      else { isw = true; c = u - UPT; }
      const unsigned k = (unsigned)c * (KC * KSTEP_B);
      return Unit{(unsigned)(isw ? sw : SLOT_OWN_AWAY) * SLOT_B + k, (unsigned)(isw ? dw : dx) * SLOT_B + k, isw ? dw : dx, (c + 1) * KC};
    };
    push_run(sq, 0, LEAD, rs, rd, voff, pprog, base);
    wait_prog(prog + sw * 64, prog + sw * 64, base, KSTEPS, tmo, seq);
    if (wl < 4) st_sys(pex + dw * 256 + wl * 64 + lane, ld_sys(ex + sw * 256 + wl * 64 + lane));
    push_run(sq, LEAD, 2 * UPT, rs, rd, voff, pprog, base);
  }
}

__device__ __forceinline__ void gemm_role(const Params& P, unsigned char* ws, unsigned seq, int xs, int wsl, LAS unsigned char* lds) {
  const int tid = threadIdx.x, wid = __builtin_amdgcn_readfirstlane(tid >> 6), lane = tid & 63;
  const int wr = wid >> 2, wc = wid & 3, fr = lane & 15, fq = lane >> 4;
  const unsigned par = seq & 1u, base = seq * 256u;
  const __amdgpu_buffer_rsrc_t ra = __builtin_amdgcn_make_buffer_rsrc(ws + CTRL_B + (size_t)par * PAR_B + (size_t)xs * SLOT_B, 0, SLOT_B, RSRC_W3);
  const __amdgpu_buffer_rsrc_t rb = __builtin_amdgcn_make_buffer_rsrc(ws + CTRL_B + (size_t)par * PAR_B + (size_t)wsl * SLOT_B, 0, SLOT_B, RSRC_W3);
  const unsigned* pa = (const unsigned*)(ws + OFF_PROG) + xs * 64;
  const unsigned* pb = (const unsigned*)(ws + OFF_PROG) + wsl * 64;
  unsigned* tmo = (unsigned*)(ws + OFF_TMO);
  const unsigned voff = (unsigned)wid * 2048u + (unsigned)lane * 16u;
  const unsigned ldsw = (unsigned)wid * 2048u;
  const int sw0 = (fq ^ (fr & 7)) * 16;
  const int aoff0 = (wr * 64 + fr) * 128 + sw0, aoff1 = aoff0 ^ 64;
  const int boff0 = 4 * HALF_B + (wc * 32 + fr) * 128 + sw0, boff1 = boff0 ^ 64;
#define SA(b, h) (((b) * 2 + (h)) * HALF_B)
#define SB(b, h) ((4 + (b) * 2 + (h)) * HALF_B)
#define STAGE(bufoff, rs, kt, h) do { const unsigned _so = (unsigned)(kt) * KSTEP_B + (h) * HALF_B; \
    __builtin_amdgcn_raw_ptr_buffer_load_lds(rs, (LAS void*)(lds + (bufoff) + ldsw), 16, voff, _so, 0, AUX_SYS); \
    __builtin_amdgcn_raw_ptr_buffer_load_lds(rs, (LAS void*)(lds + (bufoff) + ldsw), 16, voff, _so, 1024, AUX_SYS); } while (0)
#define LDA(dst, b, h) do { _Pragma("unroll") for (int m = 0; m < 4; ++m) { \
    dst[m][0] = *(const LAS f16x8*)(lds + SA(b, h) + aoff0 + m * 2048); dst[m][1] = *(const LAS f16x8*)(lds + SA(b, h) + aoff1 + m * 2048); } } while (0)
#define LDB(dst, b, h) do { _Pragma("unroll") for (int n = 0; n < 2; ++n) { \
    dst[n][0] = *(const LAS f16x8*)(lds + SA(b, h) + boff0 + n * 2048); dst[n][1] = *(const LAS f16x8*)(lds + SA(b, h) + boff1 + n * 2048); } } while (0)
#define MMA(ai, bj, At, Bt) do { __builtin_amdgcn_s_setprio(1); \
    _Pragma("unroll") for (int m = 0; m < 4; ++m) _Pragma("unroll") for (int n = 0; n < 2; ++n) _Pragma("unroll") for (int k = 0; k < 2; ++k) \
      acc[ai][bj][m][n] = __builtin_amdgcn_mfma_f32_16x16x32_f16(Bt[n][k], At[m][k], acc[ai][bj][m][n], 0, 0, 0); \
    __builtin_amdgcn_s_setprio(0); } while (0)
#define WAIT_V(n) asm volatile("s_waitcnt vmcnt(" #n ")" ::: "memory")
#define WAIT_L(n) asm volatile("s_waitcnt lgkmcnt(" #n ")" ::: "memory")
#define BAR __builtin_amdgcn_s_barrier()
#define SCHED __builtin_amdgcn_sched_barrier(0)
  f32x4 acc[2][2][4][2];
#pragma unroll
  for (int a = 0; a < 2; ++a)
#pragma unroll
    for (int b = 0; b < 2; ++b)
#pragma unroll
      for (int m = 0; m < 4; ++m)
#pragma unroll
        for (int n = 0; n < 2; ++n) acc[a][b][m][n] = (f32x4){0.f, 0.f, 0.f, 0.f};
  f16x8 At[4][2], B0[2][2], B1[2][2];
  int avail = wait_prog(pa, pb, base, 2, tmo, seq);
  STAGE(SB(0, 0), rb, 0, 0); STAGE(SA(0, 0), ra, 0, 0); STAGE(SB(0, 1), rb, 0, 1); STAGE(SA(0, 1), ra, 0, 1);
  if (wr == 1) BAR;
  WAIT_V(4); BAR;
  STAGE(SB(1, 0), rb, 1, 0); STAGE(SA(1, 0), ra, 1, 0); STAGE(SB(1, 1), rb, 1, 1);
  WAIT_V(6); BAR;
  for (int t = 0; t < KSTEPS; t += 2) {
    const int t2 = (t + 2) & (KSTEPS - 1), t3 = (t + 3) & (KSTEPS - 1);
    if (t + 4 > avail && t + 4 <= KSTEPS) avail = wait_prog(pa, pb, base, t + 4, tmo, seq);
    LDB(B0, 0, 0); SCHED; LDA(At, 0, 0); STAGE(SA(1, 1), ra, t + 1, 1);
    WAIT_L(8); BAR; WAIT_L(0); MMA(0, 0, At, B0); BAR; SCHED;
    LDB(B1, 0, 1); STAGE(SB(0, 0), rb, t2, 0);
    BAR; WAIT_L(0); MMA(0, 1, At, B1); BAR;
    LDA(At, 0, 1); STAGE(SA(0, 0), ra, t2, 0);
    BAR; WAIT_L(0); MMA(1, 0, At, B0); BAR; SCHED;
    STAGE(SB(0, 1), rb, t2, 1);
    WAIT_V(6); BAR; MMA(1, 1, At, B1); BAR;
    LDB(B0, 1, 0); SCHED; LDA(At, 1, 0); STAGE(SA(0, 1), ra, t2, 1);
    WAIT_L(8); BAR; WAIT_L(0); MMA(0, 0, At, B0); BAR; SCHED;
    LDB(B1, 1, 1); STAGE(SB(1, 0), rb, t3, 0);
    BAR; WAIT_L(0); MMA(0, 1, At, B1); BAR;
    LDA(At, 1, 1); STAGE(SA(1, 0), ra, t3, 0);
    BAR; WAIT_L(0); MMA(1, 0, At, B0); BAR; SCHED;
    STAGE(SB(1, 1), rb, t3, 1);
    WAIT_V(6); BAR; MMA(1, 1, At, B1); BAR;
  }
  WAIT_V(0);
  if (wr == 0) BAR;
  const int rank = P.rank, g = rank & 1, me = rank >> 1;
  const bool remote = wsl >= SLOT_WR;
  const int owner = remote ? 2 * (wsl - SLOT_WR) + (1 - g) : rank;
  const int ctile = remote ? me : wsl - SLOT_W;
  const int grow0 = xs < 12 ? (2 * (xs / 3) + g) * 1024 + (xs % 3) * 256 : (2 * (xs - 12) + (1 - g)) * 1024 + 768;
  const __amdgpu_buffer_rsrc_t ro = __builtin_amdgcn_make_buffer_rsrc(P.out[owner] + (size_t)grow0 * NLOC + ctile * 256, 0, 256 * NLOC * 4, RSRC_W3);
  const unsigned* exa = (const unsigned*)(ws + OFF_EXP) + (par * NSLOT + xs) * 256;
  const unsigned* exb = (const unsigned*)(ws + OFF_EXP) + (par * NSLOT + wsl) * 256;
  int tz = threadIdx.x; asm volatile("" : "+v"(tz));
  const int l2 = tz & 63, fr2 = l2 & 15, fq2 = l2 >> 4;
  int er[2][4], ec[2][2][4];
#pragma unroll
  for (int ai = 0; ai < 2; ++ai)
#pragma unroll
    for (int m = 0; m < 4; ++m) er[ai][m] = (int)ld_sys(exa + ai * 128 + wr * 64 + m * 16 + fr2);
#pragma unroll
  for (int bj = 0; bj < 2; ++bj)
#pragma unroll
    for (int n = 0; n < 2; ++n)
#pragma unroll
      for (int j = 0; j < 4; ++j) ec[bj][n][j] = (int)ld_sys(exb + bj * 128 + wc * 32 + n * 16 + fq2 * 4 + j);
  const bool bad = __hip_atomic_load(tmo, __ATOMIC_RELAXED, __HIP_MEMORY_SCOPE_AGENT) == seq;
#pragma unroll
  for (int ai = 0; ai < 2; ++ai)
#pragma unroll
    for (int m = 0; m < 4; ++m) {
      const unsigned ro_row = (unsigned)(ai * 128 + wr * 64 + m * 16 + fr2) * (NLOC * 4u) + (unsigned)(wc * 32 + fq2 * 4) * 4u;
#pragma unroll
      for (int bj = 0; bj < 2; ++bj)
#pragma unroll
        for (int n = 0; n < 2; ++n) {
          f32x4 o;
#pragma unroll
          for (int j = 0; j < 4; ++j) {
            const float v = __builtin_amdgcn_ldexpf(acc[ai][bj][m][n][j], er[ai][m] + ec[bj][n][j]);
            o[j] = bad ? __builtin_nanf("") : (v < 0.f ? 0.f : v);
          }
          if (remote) __builtin_amdgcn_raw_buffer_store_b128(__builtin_bit_cast(u32x4, o), ro, ro_row + (bj * 128 + n * 16) * 4, 0, AUX_SYS);
          else __builtin_amdgcn_raw_buffer_store_b128(__builtin_bit_cast(u32x4, o), ro, ro_row + (bj * 128 + n * 16) * 4, 0, 0);
        }
    }
  if (remote) {
    DRAIN();
    __syncthreads();
    if (tz == 0) st_sys((unsigned*)(P.ws[owner] + OFF_DONE) + me * 16 + xs, seq);
  }
#undef SA
#undef SB
#undef STAGE
#undef LDA
#undef LDB
#undef MMA
}

__global__ __launch_bounds__(512, 2) void ag_gemm_kernel(Params P) {
  extern __shared__ __attribute__((aligned(16))) unsigned char shm[];
  LAS unsigned char* lds = (LAS unsigned char*)shm;
  LAS unsigned* red = (LAS unsigned*)(lds + 131072);
  const int tid = threadIdx.x, wid = __builtin_amdgcn_readfirstlane(tid >> 6), lane = tid & 63;
  const int b = blockIdx.x, xcd = b & 7, bi = b >> 3;
  const int rank = P.rank, me = rank >> 1;
  unsigned char* ws = P.ws[rank];

  unsigned seq;
  {
    const int row = 4 * b + (wid >> 1), half = wid & 1;
    const float* xr = P.x + (size_t)row * KDIM + half * 4096 + lane * 8;
    f32x4 v[8][2];
#pragma unroll
    for (int g = 0; g < 8; ++g) { v[g][0] = *(const f32x4*)(xr + g * 512); v[g][1] = *(const f32x4*)(xr + g * 512 + 4); }
    if (tid == 0) {
      const unsigned ticket = __hip_atomic_fetch_add((unsigned*)(ws + OFF_TICKET) + xcd * 32, 1u, __ATOMIC_RELAXED, __HIP_MEMORY_SCOPE_AGENT);
      red[32] = ticket / (NWG / 8) + 1u;
      __builtin_amdgcn_fence(__ATOMIC_ACQUIRE, "");
      DRAIN();
    }
    unsigned m = 0;
#pragma unroll
    for (int g = 0; g < 8; ++g)
#pragma unroll
      for (int h = 0; h < 2; ++h)
#pragma unroll
        for (int j = 0; j < 4; ++j) { const unsigned a = absbits(v[g][h][j]); m = a > m ? a : m; }
    m = wave_max(m);
    if (lane == 0) red[wid] = m;
    __syncthreads();
    seq = __builtin_amdgcn_readfirstlane(red[32]);
    const unsigned m2 = red[wid ^ 1];
    const int eb = scale_eb(m2 > m ? m2 : m);
    const float s = __uint_as_float((unsigned)(268 - eb) << 23);
    const int t = row >> 8, rin = row & 255;
    const int slot = t < 3 ? me * 3 + t : SLOT_OWN_AWAY;
    const unsigned par = seq & 1u;
    const __amdgpu_buffer_rsrc_t rd = __builtin_amdgcn_make_buffer_rsrc(ws + CTRL_B + (size_t)par * PAR_B + (size_t)slot * SLOT_B, 0, SLOT_B, RSRC_W3);
    const unsigned off = (unsigned)(half * 64 + (lane >> 3)) * KSTEP_B + (unsigned)rin * 128u + (unsigned)(((lane & 7) ^ (rin & 7)) * 16);
#pragma unroll
    for (int g = 0; g < 8; ++g) {
      f16x8 h;
#pragma unroll
      for (int j = 0; j < 4; ++j) { h[j] = (_Float16)(v[g][0][j] * s); h[4 + j] = (_Float16)(v[g][1][j] * s); }
      __builtin_amdgcn_raw_buffer_store_b128(__builtin_bit_cast(u32x4, h), rd, off, g * 8 * KSTEP_B, AUX_SYS);
    }
    if (lane == 0 && half == 0) st_sys((unsigned*)(ws + OFF_EXP) + (par * NSLOT + slot) * 256 + rin, (unsigned)(eb - 141));
    DRAIN();
    __syncthreads();
    if (tid == 0) st_sys((unsigned*)(ws + OFF_PROG) + slot * 64 + (b & 63), seq * 256u + KSTEPS);
  }

  if (bi < NPUSH) {
    const int peer = (rank + 1 + bi) & (NR - 1);
    push_role(P, ws, seq, peer, xcd * 8 + wid);
    return;
  }
  prep_w(P, ws, seq, xcd, bi, red);
  if (bi < NPUSH + NGEMM) {
    const int j = bi - NPUSH;
    gemm_role(P, ws, seq, (xcd >> 1) * 4 + (j & 3), SLOT_W + (xcd & 1) * 4 + (j >> 2), lds);
    return;
  }
  if (bi < NPUSH + NGEMM + NPUSH) prep_w(P, ws, seq, xcd, bi - (NPUSH + NGEMM), red);
  if (b == NWG - 1 && wid == 0) {
    const unsigned* dn = (const unsigned*)(ws + OFF_DONE);
    unsigned* tmo = (unsigned*)(ws + OFF_TMO);
    const unsigned long long t0 = __builtin_amdgcn_s_memrealtime();
    for (;;) {
      if (__all((int)(ld_sys(dn + lane) - seq) >= 0)) break;
      if (__builtin_amdgcn_readfirstlane(__hip_atomic_load(tmo, __ATOMIC_RELAXED, __HIP_MEMORY_SCOPE_AGENT)) == seq ||
          __builtin_amdgcn_s_memrealtime() - t0 > DEADLINE) {
        if (lane == 0) {
          __hip_atomic_store(tmo, seq, __ATOMIC_RELAXED, __HIP_MEMORY_SCOPE_AGENT);
          P.out[rank][(size_t)rank * 1024 * NLOC] = __builtin_nanf("");
        }
        break;
      }
      __builtin_amdgcn_s_sleep(2);
    }
    __builtin_amdgcn_fence(__ATOMIC_ACQUIRE, "");
  }
}

}

extern "C" void kernel_launch(int rank, int nranks, ncclComm_t comm,
                              hipStream_t stream,
                              void* const* d_in, const int* in_sizes, int n_in,
                              void* d_out, int out_size,
                              const int* in_dtypes, int out_dtype,
                              void* const* d_out_all,
                              void* const* d_ws_all, size_t ws_bytes) {
  (void)comm; (void)in_dtypes; (void)out_dtype; (void)d_out;
  if (nranks != NR || (unsigned)rank >= (unsigned)NR || n_in < 2 || ws_bytes < WS_NEED ||
      in_sizes[0] != 1024 * KDIM || in_sizes[1] != KDIM * NLOC || out_size != 8192 * NLOC) {
    fprintf(stderr, "kernel_launch: unexpected problem shape\n");
    return;
  }
  HIP_CHECK(hipFuncSetAttribute((const void*)ag_gemm_kernel, hipFuncAttributeMaxDynamicSharedMemorySize, LDS_B));
  Params P;
  P.x = (const float*)d_in[0];
  P.w = (const float*)d_in[1];
  for (int p = 0; p < NR; ++p) { P.out[p] = (float*)d_out_all[p]; P.ws[p] = (unsigned char*)d_ws_all[p]; }
  P.rank = rank;
  hipLaunchKernelGGL(ag_gemm_kernel, dim3(NWG), dim3(512), LDS_B, stream, P);
  HIP_CHECK(hipGetLastError());
}
```

```cpp
#include <hip/hip_runtime.h>
#include <rccl/rccl.h>
#include <cstdio>

#define HIP_CHECK(expr) do { hipError_t _e = (expr); if (_e != hipSuccess) { \
    fprintf(stderr, "HIP error %s at %s:%d\n", hipGetErrorString(_e), __FILE__, __LINE__); \
    return; } } while (0)

#define LAS __attribute__((address_space(3)))
typedef _Float16 f16x8 __attribute__((ext_vector_type(8)));
typedef float f32x4 __attribute__((ext_vector_type(4)));
typedef unsigned u32x4 __attribute__((ext_vector_type(4)));

namespace {

constexpr int NR = 8;
constexpr int KDIM = 8192, NLOC = 1024;
constexpr int KSTEPS = KDIM / 64;
constexpr unsigned KSTEP_B = 32768, HALF_B = 16384;
constexpr unsigned SLOT_B = KSTEPS * KSTEP_B;
constexpr int NSLOT = 25;
constexpr int SLOT_AWAY = 12, SLOT_OWN_AWAY = 16, SLOT_W = 17, SLOT_WR = 21;
constexpr unsigned PAR_B = NSLOT * SLOT_B;
constexpr size_t CTRL_B = 1u << 20;
constexpr size_t WS_NEED = CTRL_B + 2 * (size_t)PAR_B;
constexpr unsigned OFF_TICKET = 0;
constexpr unsigned OFF_TMO = 1024;
constexpr unsigned OFF_EXP = 2048;
constexpr unsigned OFF_FLAG = 4096;
constexpr unsigned FLAG_B = (NSLOT + 1) * 1024;
constexpr unsigned OFF_DONE = 32768;
constexpr unsigned long long DEADLINE = 50000000ull;
constexpr int NWG = 256, NPUSH = 7, NFULL = 13, NPREP = 32 - NPUSH;
constexpr int LDS_MAIN = 131072;
constexpr int LDS_B = LDS_MAIN + 256;
constexpr int AUX_SYS = 17;
constexpr unsigned RSRC_W3 = 0x00020000u;

struct Params {
  const float* x;
  const float* w;
  float* out[NR];
  unsigned char* ws[NR];
  int rank;
};

__device__ __forceinline__ unsigned ld_sys(const unsigned* p) { return __hip_atomic_load(p, __ATOMIC_RELAXED, __HIP_MEMORY_SCOPE_SYSTEM); }
__device__ __forceinline__ void st_sys(unsigned* p, unsigned v) { __hip_atomic_store(p, v, __ATOMIC_RELAXED, __HIP_MEMORY_SCOPE_SYSTEM); }
#define DRAIN() asm volatile("s_waitcnt vmcnt(0)" ::: "memory")
#define WAIT_V(n) asm volatile("s_waitcnt vmcnt(%0)" :: "n"(n) : "memory")
#define WAIT_L(n) asm volatile("s_waitcnt lgkmcnt(%0)" :: "n"(n) : "memory")
#define BAR __builtin_amdgcn_s_barrier()
#define SCHED __builtin_amdgcn_sched_barrier(0)
#define TSV(i, v)
#define TS(i)

__device__ __forceinline__ bool timed_out(unsigned* tmo, unsigned seq, unsigned long long t0) {
  if (__builtin_amdgcn_readfirstlane(__hip_atomic_load(tmo, __ATOMIC_RELAXED, __HIP_MEMORY_SCOPE_AGENT)) == seq ||
      __builtin_amdgcn_s_memrealtime() - t0 > DEADLINE) {
    if ((threadIdx.x & 63) == 0) __hip_atomic_store(tmo, seq, __ATOMIC_RELAXED, __HIP_MEMORY_SCOPE_AGENT);
    return true;
  }
  return false;
}

__device__ __noinline__ int wait_flags(const unsigned char* wsv, unsigned fo, int sel, int k0v, int needv, unsigned seqv) {
  const unsigned long long wa = (unsigned long long)wsv;
  const unsigned whi = (unsigned)__builtin_amdgcn_readfirstlane((unsigned)(wa >> 32)), wlo = (unsigned)__builtin_amdgcn_readfirstlane((unsigned)wa);
  const unsigned char* ws = (const unsigned char*)(((unsigned long long)whi << 32) | (unsigned long long)wlo);
  const int k0 = __builtin_amdgcn_readfirstlane(k0v), need = __builtin_amdgcn_readfirstlane(needv);
  const unsigned seq = __builtin_amdgcn_readfirstlane(seqv);
  const __amdgpu_buffer_rsrc_t rf = __builtin_amdgcn_make_buffer_rsrc((void*)(ws + OFF_FLAG), 0, FLAG_B, RSRC_W3);
  const bool past = k0 + 2 * (int)(threadIdx.x & 31) >= KSTEPS;
  const unsigned long long t0 = __builtin_amdgcn_s_memrealtime();
  for (;;) {
    asm volatile("" ::: "memory");
    const u32x4 f = __builtin_amdgcn_raw_buffer_load_b128(rf, fo, (unsigned)k0 * 8u, AUX_SYS);
    const bool h0 = (int)(f[0] - seq) >= 0 && (int)(f[2] - seq) >= 0, h1 = (int)(f[1] - seq) >= 0 && (int)(f[3] - seq) >= 0;
    const unsigned long long m = __ballot(past || ((sel == 2 || h0) && (sel == 1 || h1)));
    const unsigned miss = ~((unsigned)m & (unsigned)(m >> 32));
    int av = k0 + 2 * (miss ? __builtin_ctz(miss) : 32);
    av = av > KSTEPS ? KSTEPS : av;
    if (av >= need) return av;
    if (timed_out((unsigned*)(ws + OFF_TMO), seq, t0)) return KSTEPS;
    __builtin_amdgcn_s_sleep(4);
  }
}

__device__ __forceinline__ unsigned wave_max(unsigned m) {
#pragma unroll
  for (int o = 32; o >= 1; o >>= 1) { const unsigned t = (unsigned)__shfl_xor((int)m, o); m = t > m ? t : m; }
  return m;
}
__device__ __forceinline__ unsigned absbits(float f) { return __float_as_uint(f) & 0x7fffffffu; }

__device__ __forceinline__ unsigned take_seq(unsigned char* ws, int xcd) {
  return __hip_atomic_fetch_add((unsigned*)(ws + OFF_TICKET) + xcd * 32, 1u, __ATOMIC_RELAXED, __HIP_MEMORY_SCOPE_AGENT) / (NWG / 8) + 1u;
}

__device__ __forceinline__ unsigned prep_role(const Params& P, unsigned char* ws, int tl8, int pi, LAS unsigned char* lds) {
  const int tid = threadIdx.x, wid = __builtin_amdgcn_readfirstlane(tid >> 6), lane = tid & 63;
  LAS unsigned* red = (LAS unsigned*)(lds + LDS_MAIN);
  const bool isw = tl8 >= 4;
  const int tl = tl8 & 3;
  const float* src = isw ? P.w + tl * 256 : P.x + (size_t)tl * 256 * KDIM;
  const __amdgpu_buffer_rsrc_t rs = __builtin_amdgcn_make_buffer_rsrc((void*)src, 0, isw ? (KDIM * NLOC - tl * 256) * 4 : 256 * KDIM * 4, RSRC_W3);
  const int r8 = lane >> 3, c8 = lane & 7;
  const unsigned lo = isw ? (unsigned)(wid * 8 * NLOC + lane * 4) * 4u : (unsigned)((wid * 8 + r8) * KDIM + (c8 ^ r8) * 8) * 4u;
  const unsigned ks_b = isw ? 64u * NLOC * 4u : 64u * 4u;
  const unsigned st_a = isw ? 2u * NLOC * 4u : 64u * KDIM * 4u, st_b = isw ? NLOC * 4u : 16u;
  const unsigned so = isw ? (unsigned)((tid * 16 + (tid & 15)) * NLOC + ((tid * 5) & 63) * 4) * 4u
                          : (unsigned)((tid >> 1) * KDIM + ((tid * 37 + 11) & 2047) * 4) * 4u;
  const u32x4 sv = __builtin_amdgcn_raw_buffer_load_b128(rs, so, 0, 0);
  u32x4 v[2][8];
#define PLOAD(j) do { const unsigned _k = (unsigned)(pi + (j) * NPREP) * ks_b, _lo = (j) < 5 || pi + (j) * NPREP < KSTEPS ? lo : 0x80000000u; \
    _Pragma("unroll") for (int i = 0; i < 8; ++i) \
      v[(j) & 1][i] = __builtin_amdgcn_raw_buffer_load_b128(rs, _lo, _k + (i >> 1) * st_a + (i & 1) * st_b, 0); } while (0)
  PLOAD(0);
  if (tid == 0) red[8] = take_seq(ws, tl8);
  PLOAD(1);
  unsigned m = sv[0] & 0x7fffffffu;
#pragma unroll
  for (int j = 1; j < 4; ++j) { const unsigned a = sv[j] & 0x7fffffffu; m = a > m ? a : m; }
  m = wave_max(m);
  if (lane == 0) red[wid] = m;
  WAIT_L(0);
  BAR;
  const unsigned seq = __builtin_amdgcn_readfirstlane(red[8]);
#pragma unroll
  for (int q = 0; q < 8; ++q) { const unsigned a = red[q]; m = a > m ? a : m; }
  m = __builtin_amdgcn_readfirstlane(m);
  int eb = (int)(m >> 23);
  eb = m == 0 ? 131 : eb < 5 ? 5 : eb;
  const float s = __uint_as_float((unsigned)(258 - eb) << 23);
  const int slot = isw ? SLOT_W + tl : tl < 3 ? (P.rank >> 1) * 3 + tl : SLOT_OWN_AWAY;
  const unsigned par = seq & 1u;
  const __amdgpu_buffer_rsrc_t rd = __builtin_amdgcn_make_buffer_rsrc(ws + CTRL_B + (size_t)par * PAR_B + (size_t)slot * SLOT_B, 0, SLOT_B, RSRC_W3);
  if (pi == 0 && tid == 0) st_sys((unsigned*)(ws + OFF_EXP) + par * 32 + slot, (unsigned)(eb - 131));
  const unsigned dst = (unsigned)tid * 16u;
  const int tw = lane * 512 + ((wid ^ c8) * 16);
  const int tr = tid * 16 ^ ((r8 ^ ((wid * 2 + (r8 >> 2)) & 7)) * 16);
#pragma unroll
  for (int j = 0; j < 6; ++j) {
    const int ks = pi + j * NPREP;
    if (j < 5 || ks < KSTEPS) {
      u32x4 h[4];
      if (isw) {
#pragma unroll
        for (int cc = 0; cc < 4; ++cc) {
          f16x8 t;
#pragma unroll
          for (int i = 0; i < 8; ++i) t[i] = (_Float16)(__uint_as_float(v[j & 1][i][cc]) * s);
          *(LAS f16x8*)(lds + (j & 1) * KSTEP_B + tw + cc * 128) = t;
        }
        WAIT_L(0);
        BAR;
#pragma unroll
        for (int it = 0; it < 4; ++it) h[it] = *(const LAS u32x4*)(lds + (j & 1) * KSTEP_B + tr + it * 8192);
      } else {
#pragma unroll
        for (int it = 0; it < 4; ++it) {
          f16x8 t;
#pragma unroll
          for (int i = 0; i < 4; ++i) {
            t[i] = (_Float16)(__uint_as_float(v[j & 1][2 * it][i]) * s);
            t[4 + i] = (_Float16)(__uint_as_float(v[j & 1][2 * it + 1][i]) * s);
          }
          h[it] = __builtin_bit_cast(u32x4, t);
        }
      }
#pragma unroll
      for (int it = 0; it < 4; ++it) __builtin_amdgcn_raw_buffer_store_b128(h[it], rd, dst, (unsigned)ks * KSTEP_B + it * 8192, AUX_SYS);
      if (j + 2 < 6) { PLOAD(j + 2); WAIT_V(8); } else DRAIN();
      BAR;
      if (tid < 2) st_sys((unsigned*)(ws + OFF_FLAG) + slot * 256 + ks * 2 + tid, seq);
    }
  }
#undef PLOAD
  return seq;
}

constexpr int PD = 4;
constexpr int PL = 2;

struct Piece { int src, dst, p; };

__device__ __forceinline__ void push_role(const Params& P, unsigned char* ws, unsigned seq, int peer, int wgi) {
  const int tid = threadIdx.x, wid = __builtin_amdgcn_readfirstlane(tid >> 6), lane = tid & 63;
  const int rank = P.rank, me = rank >> 1;
  const unsigned par = seq & 1u;
  unsigned char* pw = P.ws[peer];
  const __amdgpu_buffer_rsrc_t rs = __builtin_amdgcn_make_buffer_rsrc(ws + CTRL_B + (size_t)par * PAR_B, 0, PAR_B, RSRC_W3);
  const __amdgpu_buffer_rsrc_t rd = __builtin_amdgcn_make_buffer_rsrc(pw + CTRL_B + (size_t)par * PAR_B, 0, PAR_B, RSRC_W3);
  const unsigned voff = (unsigned)wid * 2048u + (unsigned)lane * 16u;
  const unsigned* flag = (const unsigned*)(ws + OFF_FLAG);
  unsigned* pflag = (unsigned*)(pw + OFF_FLAG);
  unsigned* tmo = (unsigned*)(ws + OFF_TMO);
  const bool same = ((peer ^ rank) & 1) == 0;
  const int sA = same ? me * 3 : SLOT_OWN_AWAY, dA = same ? me * 3 : SLOT_AWAY + me;
  const int sB = same ? me * 3 + 1 : SLOT_W + (peer >> 1), dB = same ? me * 3 + 1 : SLOT_WR + me;
  const int sC = me * 3 + 2;
  const int R = same ? 96 : 64;
  auto piece = [=](int r) {
    r = r < R ? r : R - 1;
    const int q = r * 8 + wgi;
    if (q >= 512) return Piece{sC, sC, q - 512};
    const bool b = (q & 2) != 0;
    return Piece{b ? sB : sA, b ? dB : dA, (q >> 2) * 2 + (q & 1)};
  };
  int ravail = 0;
  auto need = [&](int rn) {
    if (wid == 0 && ravail < rn) {
      const unsigned long long t0 = __builtin_amdgcn_s_memrealtime();
      for (;;) {
        const Piece q = piece(ravail + lane);
        const unsigned long long miss = ~__ballot((int)(ld_sys(flag + q.src * 256 + q.p) - seq) >= 0);
        const int n = ravail + (miss ? __builtin_ctzll(miss) : 64);
        if (n >= rn) { ravail = n; break; }
        if (timed_out(tmo, seq, t0)) { ravail = R + PD + 1; break; }
        __builtin_amdgcn_s_sleep(2);
      }
    }
  };
  auto publish = [&](int r) {
    if (tid == 0 && r >= 0) {
      const Piece q = piece(r);
      if (q.p == 0) st_sys((unsigned*)(pw + OFF_EXP) + par * 32 + q.dst, ld_sys((const unsigned*)(ws + OFF_EXP) + par * 32 + q.src));
      st_sys(pflag + q.dst * 256 + q.p, seq);
    }
  };
  need(PD + 1);
  BAR;
  TS(3);
  u32x4 buf[PD][2];
#pragma unroll
  for (int d = 0; d < PD; ++d) {
    const Piece q = piece(d);
    const unsigned o = (unsigned)q.src * SLOT_B + (unsigned)q.p * HALF_B;
    buf[d][0] = __builtin_amdgcn_raw_buffer_load_b128(rs, voff, o, AUX_SYS);
    buf[d][1] = __builtin_amdgcn_raw_buffer_load_b128(rs, voff + 1024, o, AUX_SYS);
  }
  for (int r0 = 0; r0 < R; r0 += PD) {
#pragma unroll
    for (int d = 0; d < PD; ++d) {
      const int r = r0 + d;
      const Piece q = piece(r), n = piece(r + PD);
      const unsigned od = (unsigned)q.dst * SLOT_B + (unsigned)q.p * HALF_B, os = (unsigned)n.src * SLOT_B + (unsigned)n.p * HALF_B;
      __builtin_amdgcn_raw_buffer_store_b128(buf[d][0], rd, voff, od, AUX_SYS);
      __builtin_amdgcn_raw_buffer_store_b128(buf[d][1], rd, voff + 1024, od, AUX_SYS);
      buf[d][0] = __builtin_amdgcn_raw_buffer_load_b128(rs, voff, os, AUX_SYS);
      buf[d][1] = __builtin_amdgcn_raw_buffer_load_b128(rs, voff + 1024, os, AUX_SYS);
      WAIT_V(4 * PL + 2);
      need(r + PD + 2);
      BAR;
      publish(r - PL);
    }
  }
  DRAIN();
  BAR;
#pragma unroll
  for (int d = PL; d >= 1; --d) publish(R - d);
}

struct OutTile { __amdgpu_buffer_rsrc_t ro; int e; bool bad, remote; };

__device__ __forceinline__ OutTile out_tile(const Params& P, unsigned char* ws, unsigned seq, int xs, int wsl) {
  const int rank = P.rank, g = rank & 1, me = rank >> 1;
  OutTile o;
  o.remote = wsl >= SLOT_WR;
  const int owner = o.remote ? 2 * (wsl - SLOT_WR) + (1 - g) : rank;
  const int ctile = o.remote ? me : wsl - SLOT_W;
  const int grow0 = xs < SLOT_AWAY ? (2 * (xs / 3) + g) * 1024 + (xs % 3) * 256 : (2 * (xs - SLOT_AWAY) + (1 - g)) * 1024 + 768;
  o.ro = __builtin_amdgcn_make_buffer_rsrc(P.out[owner] + (size_t)grow0 * NLOC + ctile * 256, 0, 255 * NLOC * 4 + 1024, RSRC_W3);
  const unsigned* ex = (const unsigned*)(ws + OFF_EXP) + (seq & 1u) * 32;
  o.e = (int)(ld_sys(ex + xs) + ld_sys(ex + wsl));
  o.bad = __hip_atomic_load((unsigned*)(ws + OFF_TMO), __ATOMIC_RELAXED, __HIP_MEMORY_SCOPE_AGENT) == seq;
  return o;
}

__device__ __forceinline__ void store_quarter(const OutTile& o, const f32x4 (&acc)[4][2], int ai, int bj, int wr, int wc, int lane) {
  const unsigned base = (unsigned)(ai * 128 + wr * 64 + (lane & 15)) * (NLOC * 4u) + (unsigned)(bj * 128 + wc * 32 + (lane >> 4) * 4) * 4u;
#pragma unroll
  for (int m = 0; m < 4; ++m)
#pragma unroll
    for (int n = 0; n < 2; ++n) {
      f32x4 v;
#pragma unroll
      for (int j = 0; j < 4; ++j) {
        const float t = __builtin_amdgcn_ldexpf(acc[m][n][j], o.e);
        v[j] = o.bad ? __builtin_nanf("") : (t < 0.f ? 0.f : t);
      }
      if (o.remote) __builtin_amdgcn_raw_buffer_store_b128(__builtin_bit_cast(u32x4, v), o.ro, base + n * 64, m * 16 * NLOC * 4, AUX_SYS);
      else __builtin_amdgcn_raw_buffer_store_b128(__builtin_bit_cast(u32x4, v), o.ro, base + n * 64, m * 16 * NLOC * 4, 0);
    }
}

#define DMA2(ldsoff, rs, so) do { \
    __builtin_amdgcn_raw_ptr_buffer_load_lds(rs, (LAS void*)(lds + (ldsoff) + ldsw), 16, voff, so, 0, AUX_SYS); \
    __builtin_amdgcn_raw_ptr_buffer_load_lds(rs, (LAS void*)(lds + (ldsoff) + ldsw), 16, voff, so, 1024, AUX_SYS); } while (0)
#define MMA16(ac, At, Bt) do { __builtin_amdgcn_s_setprio(1); \
    _Pragma("unroll") for (int m = 0; m < 4; ++m) _Pragma("unroll") for (int n = 0; n < 2; ++n) _Pragma("unroll") for (int k = 0; k < 2; ++k) \
      ac[m][n] = __builtin_amdgcn_mfma_f32_16x16x32_f16(Bt[n][k], At[m][k], ac[m][n], 0, 0, 0); \
    __builtin_amdgcn_s_setprio(0); } while (0)

__device__ __forceinline__ void gemm_full(const Params& P, unsigned char* ws, unsigned seq, int xs, int wsl, LAS unsigned char* lds) {
  const int tid = threadIdx.x, wid = __builtin_amdgcn_readfirstlane(tid >> 6), lane = tid & 63;
  const int wr = wid >> 2, wc = wid & 3, fr = lane & 15, fq = lane >> 4;
  const unsigned par = seq & 1u;
  const __amdgpu_buffer_rsrc_t ra = __builtin_amdgcn_make_buffer_rsrc(ws + CTRL_B + (size_t)par * PAR_B + (size_t)xs * SLOT_B, 0, SLOT_B, RSRC_W3);
  const __amdgpu_buffer_rsrc_t rb = __builtin_amdgcn_make_buffer_rsrc(ws + CTRL_B + (size_t)par * PAR_B + (size_t)wsl * SLOT_B, 0, SLOT_B, RSRC_W3);
  const unsigned fo = (unsigned)(lane < 32 ? xs : wsl) * 1024u + (unsigned)(lane & 31) * 16u;
  const unsigned voff = (unsigned)wid * 2048u + (unsigned)lane * 16u;
  const unsigned ldsw = (unsigned)wid * 2048u;
  const int sw0 = (fq ^ (fr & 7)) * 16;
  const int aoff0 = (wr * 64 + fr) * 128 + sw0, aoff1 = aoff0 ^ 64;
  const int boff0 = 4 * HALF_B + (wc * 32 + fr) * 128 + sw0, boff1 = boff0 ^ 64;
#define SA(b, h) (((b) * 2 + (h)) * HALF_B)
#define SB(b, h) ((4 + (b) * 2 + (h)) * HALF_B)
#define STAGE(bufoff, rs, kt, h) DMA2(bufoff, rs, (unsigned)(kt) * KSTEP_B + (h) * HALF_B)
#define LDA(dst, b, h) do { _Pragma("unroll") for (int m = 0; m < 4; ++m) { \
    dst[m][0] = *(const LAS f16x8*)(lds + SA(b, h) + aoff0 + m * 2048); dst[m][1] = *(const LAS f16x8*)(lds + SA(b, h) + aoff1 + m * 2048); } } while (0)
#define LDB(dst, b, h) do { _Pragma("unroll") for (int n = 0; n < 2; ++n) { \
    dst[n][0] = *(const LAS f16x8*)(lds + SA(b, h) + boff0 + n * 2048); dst[n][1] = *(const LAS f16x8*)(lds + SA(b, h) + boff1 + n * 2048); } } while (0)
#define MMA(ai, bj, At, Bt) MMA16(acc[ai][bj], At, Bt)
  f32x4 acc[2][2][4][2];
#pragma unroll
  for (int a = 0; a < 2; ++a)
#pragma unroll
    for (int b = 0; b < 2; ++b)
#pragma unroll
      for (int m = 0; m < 4; ++m)
#pragma unroll
        for (int n = 0; n < 2; ++n) acc[a][b][m][n] = (f32x4){0.f, 0.f, 0.f, 0.f};
  f16x8 At[4][2], B0[2][2], B1[2][2];
  int avail = 0;
  if (wid == 0) avail = wait_flags(ws, fo, 0, 0, 2, seq);
  BAR;
  STAGE(SB(0, 0), rb, 0, 0); STAGE(SA(0, 0), ra, 0, 0); STAGE(SB(0, 1), rb, 0, 1); STAGE(SA(0, 1), ra, 0, 1);
  if (wr == 1) BAR;
  WAIT_V(4); BAR;
  STAGE(SB(1, 0), rb, 1, 0); STAGE(SA(1, 0), ra, 1, 0); STAGE(SB(1, 1), rb, 1, 1);
  WAIT_V(6); BAR;
  for (int t = 0; t < KSTEPS; t += 2) {
    const int t2 = (t + 2) & (KSTEPS - 1), t3 = (t + 3) & (KSTEPS - 1);
    if (wid == 0 && t + 4 > avail && t + 4 <= KSTEPS) avail = wait_flags(ws, fo, 0, avail, t + 4, seq);
    LDB(B0, 0, 0); SCHED; LDA(At, 0, 0); STAGE(SA(1, 1), ra, t + 1, 1);
    WAIT_L(8); BAR; WAIT_L(0); MMA(0, 0, At, B0); BAR; SCHED;
    LDB(B1, 0, 1); STAGE(SB(0, 0), rb, t2, 0);
    BAR; WAIT_L(0); MMA(0, 1, At, B1); BAR;
    LDA(At, 0, 1); STAGE(SA(0, 0), ra, t2, 0);
    BAR; WAIT_L(0); MMA(1, 0, At, B0); BAR; SCHED;
    STAGE(SB(0, 1), rb, t2, 1);
    WAIT_V(6); BAR; MMA(1, 1, At, B1); BAR;
    LDB(B0, 1, 0); SCHED; LDA(At, 1, 0); STAGE(SA(0, 1), ra, t2, 1);
    WAIT_L(8); BAR; WAIT_L(0); MMA(0, 0, At, B0); BAR; SCHED;
    LDB(B1, 1, 1); STAGE(SB(1, 0), rb, t3, 0);
    BAR; WAIT_L(0); MMA(0, 1, At, B1); BAR;
    LDA(At, 1, 1); STAGE(SA(1, 0), ra, t3, 0);
    BAR; WAIT_L(0); MMA(1, 0, At, B0); BAR; SCHED;
    STAGE(SB(1, 1), rb, t3, 1);
    WAIT_V(6); BAR; MMA(1, 1, At, B1); BAR;
  }
  WAIT_V(0);
  if (wr == 0) BAR;
  const OutTile o = out_tile(P, ws, seq, xs, wsl);
  int tz = threadIdx.x; asm volatile("" : "+v"(tz));
#pragma unroll
  for (int ai = 0; ai < 2; ++ai)
#pragma unroll
    for (int bj = 0; bj < 2; ++bj) store_quarter(o, acc[ai][bj], ai, bj, wr, wc, tz & 63);
  if (o.remote) {
    DRAIN();
    BAR;
    if (tz < 4) st_sys((unsigned*)(P.ws[2 * (wsl - SLOT_WR) + (1 - (P.rank & 1))] + OFF_DONE) + ((P.rank >> 1) * 16 + xs) * 4 + tz, seq);
  }
#undef SA
#undef SB
#undef STAGE
#undef LDA
#undef LDB
#undef MMA
}

constexpr int NBQ = 4, QD = NBQ - 1;
__device__ __forceinline__ void gemm_quarter(const Params& P, unsigned char* ws, unsigned seq, int xs, int ah, int wsl, int bh, LAS unsigned char* lds) {
  const int tid = threadIdx.x, wid = __builtin_amdgcn_readfirstlane(tid >> 6), lane = tid & 63;
  const int wr = wid >> 2, wc = wid & 3, fr = lane & 15, fq = lane >> 4;
  const unsigned par = seq & 1u;
  const __amdgpu_buffer_rsrc_t ra = __builtin_amdgcn_make_buffer_rsrc(ws + CTRL_B + (size_t)par * PAR_B + (size_t)xs * SLOT_B + ah * HALF_B, 0, SLOT_B - ah * HALF_B, RSRC_W3);
  const __amdgpu_buffer_rsrc_t rb = __builtin_amdgcn_make_buffer_rsrc(ws + CTRL_B + (size_t)par * PAR_B + (size_t)wsl * SLOT_B + bh * HALF_B, 0, SLOT_B - bh * HALF_B, RSRC_W3);
  const unsigned fo = (unsigned)(lane < 32 ? xs : wsl) * 1024u + (unsigned)(lane & 31) * 16u;
  const int sel = 1 + (lane < 32 ? ah : bh);
  const unsigned voff = (unsigned)wid * 2048u + (unsigned)lane * 16u;
  const unsigned ldsw = (unsigned)wid * 2048u;
  const int sw0 = (fq ^ (fr & 7)) * 16;
  const int aoff0 = (wr * 64 + fr) * 128 + sw0, aoff1 = aoff0 ^ 64;
  const int boff0 = HALF_B + (wc * 32 + fr) * 128 + sw0, boff1 = boff0 ^ 64;
#define STAGEQ(b, kt) do { const unsigned _so = (unsigned)((kt) & (KSTEPS - 1)) * KSTEP_B; \
    DMA2((b) * KSTEP_B, ra, _so); DMA2((b) * KSTEP_B + HALF_B, rb, _so); } while (0)
  f32x4 acc[4][2];
#pragma unroll
  for (int m = 0; m < 4; ++m)
#pragma unroll
    for (int n = 0; n < 2; ++n) acc[m][n] = (f32x4){0.f, 0.f, 0.f, 0.f};
  f16x8 At[4][2], Bt[2][2];
  int avail = 0;
  if (wid == 0) avail = wait_flags(ws, fo, sel, 0, QD + 1, seq);
  BAR;
#pragma unroll
  for (int b = 0; b < QD; ++b) STAGEQ(b, b);
  WAIT_V(4 * (QD - 1)); BAR;
  if (wr == 1) BAR;
  for (int t0 = 0; t0 < KSTEPS; t0 += NBQ) {
#pragma unroll
    for (int b = 0; b < NBQ; ++b) {
      const int t = t0 + b;
#pragma unroll
      for (int n = 0; n < 2; ++n) {
        Bt[n][0] = *(const LAS f16x8*)(lds + b * KSTEP_B + boff0 + n * 2048); Bt[n][1] = *(const LAS f16x8*)(lds + b * KSTEP_B + boff1 + n * 2048);
      }
#pragma unroll
      for (int m = 0; m < 4; ++m) {
        At[m][0] = *(const LAS f16x8*)(lds + b * KSTEP_B + aoff0 + m * 2048); At[m][1] = *(const LAS f16x8*)(lds + b * KSTEP_B + aoff1 + m * 2048);
      }
      STAGEQ((b + QD) % NBQ, t + QD);
      WAIT_V(4 * (QD - 1)); WAIT_L(0); BAR;
      MMA16(acc, At, Bt);
      if (wid == 0 && t + QD + 2 > avail && avail < KSTEPS) avail = wait_flags(ws, fo, sel, avail, t + QD + 2 < KSTEPS ? t + QD + 2 : KSTEPS, seq);
      BAR; SCHED;
    }
  }
  WAIT_V(0);
  if (wr == 0) BAR;
  const OutTile o = out_tile(P, ws, seq, xs, wsl);
  int tz = threadIdx.x; asm volatile("" : "+v"(tz));
  store_quarter(o, acc, ah, bh, wr, wc, tz & 63);
  if (o.remote) {
    DRAIN();
    BAR;
    if (tz == 0) st_sys((unsigned*)(P.ws[2 * (wsl - SLOT_WR) + (1 - (P.rank & 1))] + OFF_DONE) + ((P.rank >> 1) * 16 + xs) * 4 + ah * 2 + bh, seq);
  }
#undef STAGEQ
}

__global__ __launch_bounds__(512, 2) void ag_gemm_kernel(Params P) {
  extern __shared__ __attribute__((aligned(16))) unsigned char shm[];
  LAS unsigned char* lds = (LAS unsigned char*)shm;
  const int tid = threadIdx.x;
  const int b = blockIdx.x, xcd = b & 7, bi = b >> 3;
  const int rank = P.rank, me = rank >> 1;
  unsigned char* ws = P.ws[rank];
  TS(0);
  if (bi < NPUSH) {
    LAS unsigned* red = (LAS unsigned*)(lds + LDS_MAIN);
    if (tid == 0) red[8] = take_seq(ws, xcd);
    WAIT_L(0);
    BAR;
    const unsigned seq = __builtin_amdgcn_readfirstlane(red[8]);
    push_role(P, ws, seq, (rank + 1 + bi) & (NR - 1), xcd);
    TS(9);
    if (b == 0 && tid < 64) {
      const __amdgpu_buffer_rsrc_t rdn = __builtin_amdgcn_make_buffer_rsrc(ws + OFF_DONE, 0, 1024, RSRC_W3);
      const unsigned long long t0 = __builtin_amdgcn_s_memrealtime();
      for (;;) {
        asm volatile("" ::: "memory");
        const u32x4 f = __builtin_amdgcn_raw_buffer_load_b128(rdn, tid * 16, 0, AUX_SYS);
        if (__all((int)(f[0] - seq) >= 0 && (int)(f[1] - seq) >= 0 && (int)(f[2] - seq) >= 0 && (int)(f[3] - seq) >= 0)) break;
        if (timed_out((unsigned*)(ws + OFF_TMO), seq, t0)) {
          if (tid == 0) P.out[rank][(size_t)rank * 1024 * NLOC] = __builtin_nanf("");
          break;
        }
        __builtin_amdgcn_s_sleep(2);
      }
      __builtin_amdgcn_fence(__ATOMIC_ACQUIRE, "");
      TS(12);
    }
    return;
  }
  const unsigned seq = prep_role(P, ws, xcd, bi - NPUSH, lds);
  TS(2);
  const int cp = xcd & 3, rh = xcd >> 2;
  if (bi < NPUSH + NFULL) {
    const int j = bi - NPUSH;
    const int ri = j < 12 ? rh * 7 + (j >> 1) : 6, c = j < 12 ? j & 1 : rh;
    const int xs = ri < 4 ? SLOT_AWAY + ri : ri < 6 ? ((me + 1) & 3) * 3 + ri - 4 : ri == 6 ? me * 3
                 : ri < 11 ? ((me + 2 + ((ri - 7) >> 1)) & 3) * 3 + ((ri - 7) & 1) : me * 3 + ri - 10;
    gemm_full(P, ws, seq, xs, (c ? SLOT_WR : SLOT_W) + cp, lds);
  } else {
    const int j = bi - NPUSH - NFULL;
    const int hr = rh * 3 + (j >> 2);
    gemm_quarter(P, ws, seq, ((me + 1 + (hr >> 1)) & 3) * 3 + 2, hr & 1, ((j & 2) ? SLOT_WR : SLOT_W) + cp, j & 1, lds);
  }
}

}

extern "C" void kernel_launch(int rank, int nranks, ncclComm_t comm,
                              hipStream_t stream,
                              void* const* d_in, const int* in_sizes, int n_in,
                              void* d_out, int out_size,
                              const int* in_dtypes, int out_dtype,
                              void* const* d_out_all,
                              void* const* d_ws_all, size_t ws_bytes) {
  (void)comm; (void)in_dtypes; (void)out_dtype; (void)d_out;
  if (nranks != NR || (unsigned)rank >= (unsigned)NR || n_in < 2 || ws_bytes < WS_NEED ||
      in_sizes[0] != 1024 * KDIM || in_sizes[1] != KDIM * NLOC || out_size != 8192 * NLOC) {
    fprintf(stderr, "kernel_launch: unexpected problem shape\n");
    return;
  }
  HIP_CHECK(hipFuncSetAttribute((const void*)ag_gemm_kernel, hipFuncAttributeMaxDynamicSharedMemorySize, LDS_B));
  Params P;
  P.x = (const float*)d_in[0];
  P.w = (const float*)d_in[1];
  for (int p = 0; p < NR; ++p) { P.out[p] = (float*)d_out_all[p]; P.ws[p] = (unsigned char*)d_ws_all[p]; }
  P.rank = rank;
  hipLaunchKernelGGL(ag_gemm_kernel, dim3(NWG), dim3(512), LDS_B, stream, P);
  HIP_CHECK(hipGetLastError());
}
```

```cpp
#include <hip/hip_runtime.h>
#include <rccl/rccl.h>
#include <cstdio>

#define HIP_CHECK(expr) do { hipError_t _e = (expr); if (_e != hipSuccess) { \
    fprintf(stderr, "HIP error %s at %s:%d\n", hipGetErrorString(_e), __FILE__, __LINE__); \
    return; } } while (0)

#define LAS __attribute__((address_space(3)))
typedef _Float16 f16x8 __attribute__((ext_vector_type(8)));
typedef float f32x4 __attribute__((ext_vector_type(4)));
typedef unsigned u32x4 __attribute__((ext_vector_type(4)));

namespace {

constexpr int NR = 8;
constexpr int KDIM = 8192, NLOC = 1024;
constexpr int KSTEPS = KDIM / 64;
constexpr unsigned KSTEP_B = 32768, HALF_B = 16384;
constexpr unsigned SLOT_B = KSTEPS * KSTEP_B;
constexpr int NSLOT = 25;
constexpr int SLOT_AWAY = 12, SLOT_OWN_AWAY = 16, SLOT_W = 17, SLOT_WR = 21;
constexpr unsigned PAR_B = NSLOT * SLOT_B;
constexpr size_t CTRL_B = 1u << 20;
constexpr unsigned UNIT_B = 65536;
constexpr unsigned OUTS_B = 4 * 64 * UNIT_B;
constexpr size_t OFF_OUTS = CTRL_B + 2 * (size_t)PAR_B;
constexpr size_t WS_NEED = OFF_OUTS + OUTS_B;
constexpr unsigned OFF_TICKET = 0;
constexpr unsigned OFF_TMO = 1024;
constexpr unsigned OFF_QUEUE = 2048;
constexpr unsigned OFF_UHEAD = 3072;
constexpr unsigned OFF_ULIST = 4096;
constexpr unsigned OFF_DONE = 8192;
constexpr unsigned OFF_FLAG = 16384;
constexpr unsigned FLAG_B = (NSLOT + 1) * 1024;
constexpr unsigned CTRL_USED = OFF_FLAG + 2 * FLAG_B;
constexpr unsigned long long DEADLINE = 50000000ull;
constexpr int NWG = 256, NPUSH = 7, NFULL = 10, NPREP = 32 - NPUSH - NFULL, NQUART = 12;
constexpr int NUNIT = 58;
constexpr int LDS_MAIN = 131072;
constexpr int LDS_B = LDS_MAIN + 256;
constexpr int AUX_SYS = 17;
constexpr unsigned RSRC_W3 = 0x00020000u;
constexpr unsigned OOB = 0x80000000u;

struct Params {
  const float* x;
  const float* w;
  float* out[NR];
  unsigned char* ws[NR];
  int rank;
};

__device__ __forceinline__ unsigned ld_sys(const unsigned* p) { return __hip_atomic_load(p, __ATOMIC_RELAXED, __HIP_MEMORY_SCOPE_SYSTEM); }
__device__ __forceinline__ void st_sys(unsigned* p, unsigned v) { __hip_atomic_store(p, v, __ATOMIC_RELAXED, __HIP_MEMORY_SCOPE_SYSTEM); }
#define DRAIN() asm volatile("s_waitcnt vmcnt(0)" ::: "memory")
#define WAIT_V(n) asm volatile("s_waitcnt vmcnt(%0)" :: "n"(n) : "memory")
#define WAIT_L(n) asm volatile("s_waitcnt lgkmcnt(%0)" :: "n"(n) : "memory")
#define BAR __builtin_amdgcn_s_barrier()
#define SCHED __builtin_amdgcn_sched_barrier(0)
#define TSV(i, v)
#define TS(i)

__device__ __forceinline__ bool timed_out(unsigned* tmo, unsigned seq, unsigned long long t0) {
  if (__builtin_amdgcn_readfirstlane(__hip_atomic_load(tmo, __ATOMIC_RELAXED, __HIP_MEMORY_SCOPE_AGENT)) == seq ||
      __builtin_amdgcn_s_memrealtime() - t0 > DEADLINE) {
    if ((threadIdx.x & 63) == 0) __hip_atomic_store(tmo, seq, __ATOMIC_RELAXED, __HIP_MEMORY_SCOPE_AGENT);
    return true;
  }
  return false;
}

__device__ __noinline__ int wait_flags(const unsigned char* wsv, unsigned fo, int sel, int k0v, int needv, unsigned seqv) {
  const unsigned long long wa = (unsigned long long)wsv;
  const unsigned whi = (unsigned)__builtin_amdgcn_readfirstlane((unsigned)(wa >> 32)), wlo = (unsigned)__builtin_amdgcn_readfirstlane((unsigned)wa);
  const unsigned char* ws = (const unsigned char*)(((unsigned long long)whi << 32) | (unsigned long long)wlo);
  const int k0 = __builtin_amdgcn_readfirstlane(k0v), need = __builtin_amdgcn_readfirstlane(needv);
  const unsigned seq = __builtin_amdgcn_readfirstlane(seqv);
  const unsigned s9 = seq << 9;
  const __amdgpu_buffer_rsrc_t rf = __builtin_amdgcn_make_buffer_rsrc((void*)(ws + OFF_FLAG + (seq & 1u) * FLAG_B), 0, FLAG_B, RSRC_W3);
  const bool past = k0 + 2 * (int)(threadIdx.x & 31) >= KSTEPS;
  const unsigned long long t0 = __builtin_amdgcn_s_memrealtime();
  for (;;) {
    asm volatile("" ::: "memory");
    const u32x4 f = __builtin_amdgcn_raw_buffer_load_b128(rf, fo, (unsigned)k0 * 8u, AUX_SYS);
    const bool h0 = (int)(f[0] - s9) >= 0 && (int)(f[2] - s9) >= 0, h1 = (int)(f[1] - s9) >= 0 && (int)(f[3] - s9) >= 0;
    const unsigned long long m = __ballot(past || ((sel == 2 || h0) && (sel == 1 || h1)));
    const unsigned miss = ~((unsigned)m & (unsigned)(m >> 32));
    int av = k0 + 2 * (miss ? __builtin_ctz(miss) : 32);
    av = av > KSTEPS ? KSTEPS : av;
    if (av >= need) return av;
    if (timed_out((unsigned*)(ws + OFF_TMO), seq, t0)) return KSTEPS;
    __builtin_amdgcn_s_sleep(4);
  }
}

__device__ __forceinline__ unsigned wave_max(unsigned m) {
#pragma unroll
  for (int o = 32; o >= 1; o >>= 1) { const unsigned t = (unsigned)__shfl_xor((int)m, o); m = t > m ? t : m; }
  return m;
}
__device__ __forceinline__ unsigned absbits(float f) { return __float_as_uint(f) & 0x7fffffffu; }

__device__ __forceinline__ unsigned take_seq(unsigned char* ws, int xcd) {
  return __hip_atomic_fetch_add((unsigned*)(ws + OFF_TICKET) + xcd * 32, 1u, __ATOMIC_RELAXED, __HIP_MEMORY_SCOPE_AGENT) / (NWG / 8) + 1u;
}

struct Src { __amdgpu_buffer_rsrc_t rs; unsigned lo, ks_b, st_a, st_b, so; };

__device__ __forceinline__ Src make_src(const Params& P, int tl8) {
  const int tid = threadIdx.x, wid = tid >> 6, lane = tid & 63, r8 = lane >> 3, c8 = lane & 7;
  const bool isw = tl8 >= 4;
  const int tl = tl8 & 3;
  Src q;
  q.rs = __builtin_amdgcn_make_buffer_rsrc((void*)(isw ? P.w + tl * 256 : P.x + (size_t)tl * 256 * KDIM), 0,
                                           isw ? (KDIM * NLOC - tl * 256) * 4 : 256 * KDIM * 4, RSRC_W3);
  q.lo = isw ? (unsigned)(wid * 8 * NLOC + lane * 4) * 4u : (unsigned)((wid * 8 + r8) * KDIM + (c8 ^ r8) * 8) * 4u;
  q.ks_b = isw ? 64u * NLOC * 4u : 64u * 4u;
  q.st_a = isw ? 2u * NLOC * 4u : 64u * KDIM * 4u;
  q.st_b = isw ? NLOC * 4u : 16u;
  q.so = isw ? (unsigned)((tid * 16 + (tid & 15)) * NLOC + ((tid * 5) & 63) * 4) * 4u
             : (unsigned)((tid >> 1) * KDIM + ((tid * 37 + 11) & 2047) * 4) * 4u;
  return q;
}
#define PLOAD(q, v, ks) do { const unsigned _k = (unsigned)(ks) * q.ks_b, _lo = (ks) < KSTEPS ? q.lo : OOB; \
    _Pragma("unroll") for (int i = 0; i < 8; ++i) \
      v[i] = __builtin_amdgcn_raw_buffer_load_b128(q.rs, _lo, _k + (i >> 1) * q.st_a + (i & 1) * q.st_b, 0); } while (0)

__device__ __forceinline__ int tile_exp(const u32x4 sv, LAS unsigned* red) {
  unsigned m = sv[0] & 0x7fffffffu;
#pragma unroll
  for (int j = 1; j < 4; ++j) { const unsigned a = sv[j] & 0x7fffffffu; m = a > m ? a : m; }
  m = wave_max(m);
  if ((threadIdx.x & 63) == 0) red[threadIdx.x >> 6] = m;
  WAIT_L(0);
  BAR;
#pragma unroll
  for (int q = 0; q < 8; ++q) { const unsigned a = red[q]; m = a > m ? a : m; }
  m = __builtin_amdgcn_readfirstlane(m);
  const int eb = (int)(m >> 23);
  return m == 0 ? 131 : eb < 5 ? 5 : eb;
}

__device__ __forceinline__ void convert(const u32x4 (&v)[8], u32x4 (&h)[4], float s, bool isw, LAS unsigned char* lb) {
  const int tid = threadIdx.x, wid = tid >> 6, lane = tid & 63, r8 = lane >> 3, c8 = lane & 7;
  if (isw) {
    const int tw = lane * 512 + ((wid ^ c8) * 16);
    const int tr = tid * 16 ^ ((r8 ^ ((wid * 2 + (r8 >> 2)) & 7)) * 16);
#pragma unroll
    for (int cc = 0; cc < 4; ++cc) {
      f16x8 t;
#pragma unroll
      for (int i = 0; i < 8; ++i) t[i] = (_Float16)(__uint_as_float(v[i][cc]) * s);
      *(LAS f16x8*)(lb + tw + cc * 128) = t;
    }
    WAIT_L(0);
    BAR;
#pragma unroll
    for (int it = 0; it < 4; ++it) h[it] = *(const LAS u32x4*)(lb + tr + it * 8192);
  } else {
#pragma unroll
    for (int it = 0; it < 4; ++it) {
      f16x8 t;
#pragma unroll
      for (int i = 0; i < 4; ++i) {
        t[i] = (_Float16)(__uint_as_float(v[2 * it][i]) * s);
        t[4 + i] = (_Float16)(__uint_as_float(v[2 * it + 1][i]) * s);
      }
      h[it] = __builtin_bit_cast(u32x4, t);
    }
  }
}

__device__ __forceinline__ unsigned prep_role(const Params& P, unsigned char* ws, int tl8, int pi, LAS unsigned char* lds) {
  const int tid = threadIdx.x;
  LAS unsigned* red = (LAS unsigned*)(lds + LDS_MAIN);
  const bool isw = tl8 >= 4;
  const int tl = tl8 & 3;
  const Src q = make_src(P, tl8);
  const u32x4 sv = __builtin_amdgcn_raw_buffer_load_b128(q.rs, q.so, 0, 0);
  u32x4 v0[8], v1[8];
  PLOAD(q, v0, pi);
  if (tid == 0) red[8] = take_seq(ws, tl8);
  const int eb = tile_exp(sv, red);
  PLOAD(q, v1, pi + NPREP);
  const unsigned seq = __builtin_amdgcn_readfirstlane(red[8]);
  const float s = __uint_as_float((unsigned)(258 - eb) << 23);
  const int slot = isw ? SLOT_W + tl : tl < 3 ? (P.rank >> 1) * 3 + tl : SLOT_OWN_AWAY;
  const unsigned par = seq & 1u;
  const __amdgpu_buffer_rsrc_t rd = __builtin_amdgcn_make_buffer_rsrc(ws + CTRL_B + (size_t)par * PAR_B + (size_t)slot * SLOT_B, 0, SLOT_B, RSRC_W3);
  unsigned* flag = (unsigned*)(ws + OFF_FLAG + par * FLAG_B) + slot * 256 + tid;
  const unsigned fval = (seq << 9) + (unsigned)(256 + eb - 131);
#define PSTEP(v, b) do { \
    u32x4 h[4]; \
    convert(v, h, s, isw, lds + (b) * KSTEP_B); \
    _Pragma("unroll") for (int it = 0; it < 4; ++it) \
      __builtin_amdgcn_raw_buffer_store_b128(h[it], rd, (unsigned)tid * 16u, (unsigned)ks * KSTEP_B + it * 8192, AUX_SYS); \
    PLOAD(q, v, ks + 2 * NPREP); \
    WAIT_V(8); \
    BAR; \
    if (tid < 2) st_sys(flag + ks * 2, fval); \
    ks += NPREP; } while (0)
  for (int ks = pi;;) {
    PSTEP(v0, 0);
    if (ks >= KSTEPS) break;
    PSTEP(v1, 1);
    if (ks >= KSTEPS) break;
  }
#undef PSTEP
  return seq;
}

template <int N, int TOT> __device__ __forceinline__ int merge_order(int c) {
  constexpr int PER = 2 * N + 1, CE = (TOT - 1) / (N + 1) * PER + 2 * ((TOT - 1) % (N + 1)) + 1;
  const int g = c / PER, r = c - g * PER;
  return c >= CE ? (c - TOT) * 2 + 1 : (((r & 1) ? N : N + 1) * g + (r >> 1)) * 2 + (r & 1);
}
#define AG_PL 5
#define AG_T2N 4
constexpr int PL = AG_PL;
constexpr int RING = 4;
constexpr int NDIR = 16;

__device__ __forceinline__ int tile_row0(int rank, int xs) {
  const int g = rank & 1;
  return xs < SLOT_AWAY ? (2 * (xs / 3) + g) * 1024 + (xs % 3) * 256 : (2 * (xs - SLOT_AWAY) + (1 - g)) * 1024 + 768;
}

__device__ __forceinline__ unsigned link_role(const Params& P, unsigned char* ws, int link, int wgi, LAS unsigned char* lds) {
  const int tid = threadIdx.x, wid = __builtin_amdgcn_readfirstlane(tid >> 6), lane = tid & 63;
  const int rank = P.rank, me = rank >> 1, peer = (rank + 1 + link) & (NR - 1);
  const bool same = ((peer ^ rank) & 1) == 0;
  const unsigned sA = same ? me * 3 : SLOT_W + (peer >> 1), dA = same ? me * 3 : SLOT_WR + me;
  const unsigned sB = same ? me * 3 + 1 : SLOT_OWN_AWAY, dB = same ? me * 3 + 1 : SLOT_AWAY + me;
  const unsigned sC = same ? me * 3 + 2 : SLOT_OWN_AWAY;
  unsigned char* pw = P.ws[peer];
  LAS unsigned* red = (LAS unsigned*)(lds + LDS_MAIN);
  const __amdgpu_buffer_rsrc_t rp = __builtin_amdgcn_make_buffer_rsrc(pw, 0, CTRL_USED, RSRC_W3);
  const unsigned one = lane == 0 ? 0u : OOB;
  unsigned seq;
  {
    const int tl8 = same ? wgi & 1 : (wgi & 1) ? 3 : 4 + (peer >> 1), ks = wgi >> 1;
    const Src q = make_src(P, tl8);
    const u32x4 sv = __builtin_amdgcn_raw_buffer_load_b128(q.rs, q.so, 0, 0);
    u32x4 v[8], h[4];
    PLOAD(q, v, ks);
    if (tid == 0) { red[8] = take_seq(ws, wgi); red[9] = 0; red[10] = 0; red[11] = 0; }
    const int eb = tile_exp(sv, red);
    seq = __builtin_amdgcn_readfirstlane(red[8]);
    convert(v, h, __uint_as_float((unsigned)(258 - eb) << 23), tl8 >= 4, lds + RING * HALF_B);
    const unsigned ds = (wgi & 1) ? dB : dA;
    const __amdgpu_buffer_rsrc_t rd = __builtin_amdgcn_make_buffer_rsrc(pw + CTRL_B + (size_t)(seq & 1u) * PAR_B + (size_t)ds * SLOT_B, 0, SLOT_B, RSRC_W3);
    TS(3);
#pragma unroll
    for (int it = 0; it < 4; ++it) {
      __builtin_amdgcn_raw_buffer_store_b128(h[it], rd, (unsigned)tid * 16u, (unsigned)ks * KSTEP_B + it * 8192, AUX_SYS);
      if (wid != 1) DRAIN();
    }
    DRAIN();
    unsigned n = 0;
    if (lane == 0) n = __hip_atomic_fetch_add(red + 9, 1u, __ATOMIC_RELAXED, __HIP_MEMORY_SCOPE_WORKGROUP);
    if (__builtin_amdgcn_readfirstlane(n) == 7)
      __builtin_amdgcn_raw_buffer_store_b32((seq << 9) + (unsigned)(256 + eb - 131), rp, lane < 2 ? lane * 4u : OOB,
                                            OFF_FLAG + (seq & 1u) * FLAG_B + ds * 1024u + (unsigned)ks * 8u, AUX_SYS);
  }
  if (wid > 1) return seq;
  const unsigned par = seq & 1u, s9 = seq << 9;
  unsigned* tmo = (unsigned*)(ws + OFF_TMO);
  volatile LAS unsigned* cnt = red;
  if (wid == 1) {
    const __amdgpu_buffer_rsrc_t rs = __builtin_amdgcn_make_buffer_rsrc(ws + CTRL_B, 0, 2 * PAR_B + OUTS_B, RSRC_W3);
    const __amdgpu_buffer_rsrc_t rc = __builtin_amdgcn_make_buffer_rsrc(ws, 0, CTRL_USED, RSRC_W3);
    const unsigned voff = (unsigned)lane * 16u, fb = OFF_FLAG + par * FLAG_B;
    const int Q = same ? 768 : 512 + NUNIT * 4;
    const unsigned qbase = (seq - 1u) * (unsigned)(Q - NDIR) - (NDIR + 16u);
    unsigned fA = 0, fB = 0, fC = 0;
    int rdy = 0;
    int p = NDIR + wgi, pn = NDIR + 8 + wgi;
    bool late = false;
    for (int n = 0;; ++n) {
      const bool live = (unsigned)p < (unsigned)Q;
      const unsigned tk = (unsigned)__builtin_amdgcn_raw_ptr_buffer_atomic_add_i32(1, rc, live ? one : OOB, OFF_QUEUE + link * 128, 16);
      u32x4 rec = {0u, 2u, 0u, 0u};
      unsigned so = OOB;
      if (live && (same || p < 512)) {
        int sel, pc;
        if (p >= 512) { sel = 2; pc = merge_order<AG_T2N, 128>(p - 512); }
        else if (same) { sel = (p >> 1) & 1; pc = (p >> 2) * 2 + (p & 1); }
        else { const int o = merge_order<7, KSTEPS - NDIR / 4>((p - NDIR) >> 1); sel = o & 1; pc = (NDIR / 4 + (o >> 1)) * 2 + (p & 1); }
        if (rdy <= (pc >> 1)) {
          DRAIN();
          cnt[10] = n;
          const unsigned long long t0 = __builtin_amdgcn_s_memrealtime();
          for (;;) {
            asm volatile("" ::: "memory");
            const u32x4 a = __builtin_amdgcn_raw_buffer_load_b128(rc, voff, fb + sA * 1024u, AUX_SYS);
            const u32x4 b = __builtin_amdgcn_raw_buffer_load_b128(rc, voff, fb + sB * 1024u, AUX_SYS);
            const u32x4 c = __builtin_amdgcn_raw_buffer_load_b128(rc, voff, fb + sC * 1024u, AUX_SYS);
            unsigned neg = 0;
#pragma unroll
            for (int j = 0; j < 4; ++j) neg |= (a[j] - s9) | (b[j] - s9) | (c[j] - s9);
            const unsigned long long miss = __ballot((int)neg < 0);
            rdy = miss ? 2 * __builtin_ctzll(miss) : KSTEPS;
            fA = __builtin_amdgcn_readfirstlane(a[0]); fB = __builtin_amdgcn_readfirstlane(b[0]); fC = __builtin_amdgcn_readfirstlane(c[0]);
            if (rdy > (pc >> 1)) break;
            if (timed_out(tmo, seq, t0)) { rdy = KSTEPS; fA = fB = fC = s9; break; }
            __builtin_amdgcn_s_sleep(4);
          }
          rdy = __builtin_amdgcn_readfirstlane(rdy);
        }
        const unsigned dn = (sel == 2 ? sC : sel ? dB : dA) * 256u + (unsigned)pc;
        so = par * PAR_B + ((sel == 2 ? sC : sel ? sB : sA) * 256u + (unsigned)pc) * HALF_B;
        rec = (u32x4){dn * HALF_B, 0u, fb + dn * 4u, sel == 2 ? fC : sel ? fB : fA};
      } else if (live) {
        const unsigned eo = OFF_ULIST + (unsigned)(peer >> 1) * 256u + (unsigned)((p - 512) >> 2) * 4u;
        unsigned e = __builtin_amdgcn_readfirstlane(__builtin_amdgcn_raw_buffer_load_b32(rc, 0, eo, AUX_SYS));
        if ((e >> 9) != (seq & 0x7fffffu)) {
          DRAIN();
          cnt[10] = n;
          const unsigned long long t0 = __builtin_amdgcn_s_memrealtime();
          for (;;) {
            asm volatile("" ::: "memory");
            e = __builtin_amdgcn_readfirstlane(__builtin_amdgcn_raw_buffer_load_b32(rc, 0, eo, AUX_SYS));
            if ((e >> 9) == (seq & 0x7fffffu) || timed_out(tmo, seq, t0)) break;
            __builtin_amdgcn_s_sleep(4);
          }
        }
        const unsigned u = e & 63u, up = u * 4u + ((unsigned)p & 3u);
        so = 2 * PAR_B + ((unsigned)(peer >> 1) * 256u + up) * HALF_B;
        rec = (u32x4){(unsigned)(tile_row0(rank, (int)(u >> 2)) + (int)((u >> 1) & 1u) * 128 + (p & 3) * 32) * (NLOC * 4u) + (unsigned)me * 1024u + (u & 1u) * 512u,
                      1u, OFF_DONE + ((unsigned)me * 256u + up) * 4u, s9};
      }
      if (!late && n - (int)__builtin_amdgcn_readfirstlane(cnt[11]) >= RING) {
        const unsigned long long t0 = __builtin_amdgcn_s_memrealtime();
        while (n - (int)__builtin_amdgcn_readfirstlane(cnt[11]) >= RING) {
          if (__builtin_amdgcn_s_memrealtime() - t0 > DEADLINE) { late = timed_out(tmo, seq, t0); break; }
          __builtin_amdgcn_s_sleep(2);
        }
      }
      so = __builtin_amdgcn_readfirstlane(so);
      const int sl = n & (RING - 1);
      if (lane == 0) *(LAS u32x4*)(red + 16 + sl * 4) = rec;
#pragma unroll
      for (int i = 0; i < 4; ++i) {
        LAS void* d = (LAS void*)(lds + sl * HALF_B + i * 4096);
        __builtin_amdgcn_raw_ptr_buffer_load_lds(rs, d, 16, voff, so + i * 4096, 0, AUX_SYS);
        __builtin_amdgcn_raw_ptr_buffer_load_lds(rs, d, 16, voff, so + i * 4096, 1024, AUX_SYS);
        __builtin_amdgcn_raw_ptr_buffer_load_lds(rs, d, 16, voff, so + i * 4096, 2048, AUX_SYS);
        __builtin_amdgcn_raw_ptr_buffer_load_lds(rs, d, 16, voff, so + i * 4096, 3072, AUX_SYS);
      }
      WAIT_V(17);
      WAIT_L(0);
      cnt[10] = live ? n : n + 1;
      if (!live) break;
      p = pn;
      pn = (int)((unsigned)__builtin_amdgcn_readfirstlane(tk) - qbase);
    }
    return seq;
  }
  const __amdgpu_buffer_rsrc_t rd = __builtin_amdgcn_make_buffer_rsrc(pw + CTRL_B + (size_t)par * PAR_B, 0, PAR_B, RSRC_W3);
  const __amdgpu_buffer_rsrc_t ro = __builtin_amdgcn_make_buffer_rsrc(P.out[peer], 0, 8192 * NLOC * 4, RSRC_W3);
  const unsigned vin = (unsigned)lane * 16u, vout = (unsigned)(lane >> 5) * (NLOC * 4u) + (unsigned)(lane & 31) * 16u;
  const unsigned spare = OFF_FLAG + par * FLAG_B + NSLOT * 1024u;
  unsigned fo = spare, fv = 0;
#define PFLAG() __builtin_amdgcn_raw_buffer_store_b32(fv, rp, one, fo, AUX_SYS)
#define PSTORE(r, vo, step) do { _Pragma("unroll") for (int i = 0; i < 16; ++i) { \
      __builtin_amdgcn_raw_buffer_store_b128(buf[i], r, vo, dst + i * (step), AUX_SYS); \
      WAIT_V(PL + (i >= PL && i < 2 * PL ? 1 : 0)); \
      if (i == PL - 1) PFLAG(); } } while (0)
  for (int n = 0;; ++n) {
    if ((int)__builtin_amdgcn_readfirstlane(cnt[10]) <= n) {
      DRAIN();
      PFLAG();
      fo = spare;
      const unsigned long long t0 = __builtin_amdgcn_s_memrealtime();
      bool dead = false;
      while ((int)__builtin_amdgcn_readfirstlane(cnt[10]) <= n) {
        if (__builtin_amdgcn_s_memrealtime() - t0 > 2 * DEADLINE) { dead = true; break; }
        __builtin_amdgcn_s_sleep(1);
      }
      if (dead) break;
    }
    const int sl = n & (RING - 1);
    const u32x4 rec = *(const LAS u32x4*)(red + 16 + sl * 4);
    u32x4 buf[16];
#pragma unroll
    for (int i = 0; i < 16; ++i) buf[i] = *(const LAS u32x4*)(lds + sl * HALF_B + i * 1024 + lane * 16);
    WAIT_L(0);
    cnt[11] = n + 1;
    const unsigned dst = __builtin_amdgcn_readfirstlane(rec[0]), kind = __builtin_amdgcn_readfirstlane(rec[1]);
    if (kind == 2) break;
    if (kind == 0) PSTORE(rd, vin, 1024u); else PSTORE(ro, vout, 2u * NLOC * 4u);
    fo = __builtin_amdgcn_readfirstlane(rec[2]);
    fv = __builtin_amdgcn_readfirstlane(rec[3]);
  }
  DRAIN();
  PFLAG();
#undef PFLAG
#undef PSTORE
  return seq;
}

struct OutTile { __amdgpu_buffer_rsrc_t ro; unsigned pitch, qa, qb; int e; bool bad; };

__device__ __forceinline__ OutTile out_tile(const Params& P, unsigned char* ws, unsigned seq, int xs, int half_a, int wsl, int half_b, int mode) {
  const int rank = P.rank;
  OutTile o;
  const int owner = mode ? 2 * (wsl - SLOT_WR) + (1 - (rank & 1)) : rank;
  const int ctile = mode ? rank >> 1 : wsl - SLOT_W;
  unsigned char* base = mode == 1 ? ws + OFF_OUTS + (size_t)((wsl - SLOT_WR) * 16 + xs) * 4 * UNIT_B
                                  : (unsigned char*)(P.out[owner] + (size_t)tile_row0(rank, xs) * NLOC + ctile * 256);
  o.ro = __builtin_amdgcn_make_buffer_rsrc(base, 0, mode == 1 ? 4 * UNIT_B : 255 * NLOC * 4 + 1024, RSRC_W3);
  o.pitch = mode == 1 ? 512u : NLOC * 4u;
  o.qa = mode == 1 ? 2 * UNIT_B : 128u * NLOC * 4u;
  o.qb = mode == 1 ? UNIT_B : 512u;
  const unsigned* fl = (const unsigned*)(ws + OFF_FLAG + (seq & 1u) * FLAG_B);
  o.e = (int)((ld_sys(fl + xs * 256 + half_a) & 511u) + (ld_sys(fl + wsl * 256 + half_b) & 511u)) - 512;
  o.bad = __hip_atomic_load((unsigned*)(ws + OFF_TMO), __ATOMIC_RELAXED, __HIP_MEMORY_SCOPE_AGENT) == seq;
  return o;
}

template <int AUX, bool PACED>
__device__ __forceinline__ void store_quarter(const OutTile& o, f32x4 (&acc)[4][2], int ai, int bj, int wr, int wc, int lane) {
  const unsigned base = (unsigned)(wr * 64 + (lane & 15)) * o.pitch + (unsigned)(wc * 32 + (lane >> 4) * 4) * 4u;
  const unsigned qo = (unsigned)ai * o.qa + (unsigned)bj * o.qb;
#pragma unroll
  for (int m = 0; m < 4; ++m)
#pragma unroll
    for (int n = 0; n < 2; ++n) {
#pragma unroll
      for (int j = 0; j < 4; ++j) {
        const float t = __builtin_amdgcn_ldexpf(acc[m][n][j], o.e);
        acc[m][n][j] = o.bad ? __builtin_nanf("") : (t < 0.f ? 0.f : t);
      }
      asm volatile("" : "+v"(acc[m][n]));
    }
  SCHED;
#pragma unroll
  for (int m = 0; m < 4; ++m)
#pragma unroll
    for (int n = 0; n < 2; ++n) {
      __builtin_amdgcn_raw_buffer_store_b128(__builtin_bit_cast(u32x4, acc[m][n]), o.ro, base + n * 64, qo + m * 16 * o.pitch, AUX);
      if (PACED) DRAIN();
    }
  SCHED;
  asm volatile("s_nop 1");
  SCHED;
}

__device__ __forceinline__ unsigned* unit_slots(unsigned char* ws, unsigned seq, int c, unsigned n) {
  const unsigned pos = __hip_atomic_fetch_add((unsigned*)(ws + OFF_UHEAD) + c * 32, n, __ATOMIC_RELAXED, __HIP_MEMORY_SCOPE_AGENT) - (seq - 1u) * NUNIT;
  return (unsigned*)(ws + OFF_ULIST) + c * 64 + (pos < 60u ? pos : 60u);
}

#define DMA2(ldsoff, rs, so) do { \
    __builtin_amdgcn_raw_ptr_buffer_load_lds(rs, (LAS void*)(lds + (ldsoff) + ldsw), 16, voff, so, 0, AUX_SYS); \
    __builtin_amdgcn_raw_ptr_buffer_load_lds(rs, (LAS void*)(lds + (ldsoff) + ldsw), 16, voff, so, 1024, AUX_SYS); } while (0)
#define MMA16(ac, At, Bt) do { __builtin_amdgcn_s_setprio(1); \
    _Pragma("unroll") for (int m = 0; m < 4; ++m) _Pragma("unroll") for (int n = 0; n < 2; ++n) _Pragma("unroll") for (int k = 0; k < 2; ++k) \
      ac[m][n] = __builtin_amdgcn_mfma_f32_16x16x32_f16(Bt[n][k], At[m][k], ac[m][n], 0, 0, 0); \
    __builtin_amdgcn_s_setprio(0); } while (0)

__device__ __forceinline__ void gemm_full(const Params& P, unsigned char* ws, unsigned seq, int xs, int wsl, LAS unsigned char* lds) {
  const int tid = threadIdx.x, wid = __builtin_amdgcn_readfirstlane(tid >> 6), lane = tid & 63;
  const int wr = wid >> 2, wc = wid & 3, fr = lane & 15, fq = lane >> 4;
  const unsigned par = seq & 1u;
  const __amdgpu_buffer_rsrc_t ra = __builtin_amdgcn_make_buffer_rsrc(ws + CTRL_B + (size_t)par * PAR_B + (size_t)xs * SLOT_B, 0, SLOT_B, RSRC_W3);
  const __amdgpu_buffer_rsrc_t rb = __builtin_amdgcn_make_buffer_rsrc(ws + CTRL_B + (size_t)par * PAR_B + (size_t)wsl * SLOT_B, 0, SLOT_B, RSRC_W3);
  const unsigned fo = (unsigned)(lane < 32 ? xs : wsl) * 1024u + (unsigned)(lane & 31) * 16u;
  const unsigned voff = (unsigned)wid * 2048u + (unsigned)lane * 16u;
  const unsigned ldsw = (unsigned)wid * 2048u;
  const int sw0 = (fq ^ (fr & 7)) * 16;
  const int aoff0 = (wr * 64 + fr) * 128 + sw0, aoff1 = aoff0 ^ 64;
  const int boff0 = 4 * HALF_B + (wc * 32 + fr) * 128 + sw0, boff1 = boff0 ^ 64;
#define SA(b, h) (((b) * 2 + (h)) * HALF_B)
#define SB(b, h) ((4 + (b) * 2 + (h)) * HALF_B)
#define STAGE(bufoff, rs, kt, h) DMA2(bufoff, rs, (unsigned)(kt) * KSTEP_B + (h) * HALF_B)
#define LDA(dst, b, h) do { _Pragma("unroll") for (int m = 0; m < 4; ++m) { \
    dst[m][0] = *(const LAS f16x8*)(lds + SA(b, h) + aoff0 + m * 2048); dst[m][1] = *(const LAS f16x8*)(lds + SA(b, h) + aoff1 + m * 2048); } } while (0)
#define LDB(dst, b, h) do { _Pragma("unroll") for (int n = 0; n < 2; ++n) { \
    dst[n][0] = *(const LAS f16x8*)(lds + SA(b, h) + boff0 + n * 2048); dst[n][1] = *(const LAS f16x8*)(lds + SA(b, h) + boff1 + n * 2048); } } while (0)
#define MMA(ai, bj, At, Bt) MMA16(acc[ai][bj], At, Bt)
  f32x4 acc[2][2][4][2];
#pragma unroll
  for (int a = 0; a < 2; ++a)
#pragma unroll
    for (int b = 0; b < 2; ++b)
#pragma unroll
      for (int m = 0; m < 4; ++m)
#pragma unroll
        for (int n = 0; n < 2; ++n) acc[a][b][m][n] = (f32x4){0.f, 0.f, 0.f, 0.f};
  f16x8 At[4][2], B0[2][2], B1[2][2];
  int avail = 0;
  if (wid == 0) avail = wait_flags(ws, fo, 0, 0, 2, seq);
  BAR;
  STAGE(SB(0, 0), rb, 0, 0); STAGE(SA(0, 0), ra, 0, 0); STAGE(SB(0, 1), rb, 0, 1); STAGE(SA(0, 1), ra, 0, 1);
  if (wr == 1) BAR;
  WAIT_V(4); BAR;
  STAGE(SB(1, 0), rb, 1, 0); STAGE(SA(1, 0), ra, 1, 0); STAGE(SB(1, 1), rb, 1, 1);
  WAIT_V(6); BAR;
  for (int t = 0; t < KSTEPS; t += 2) {
    const int t2 = (t + 2) & (KSTEPS - 1), t3 = (t + 3) & (KSTEPS - 1);
    if (wid == 0 && t + 4 > avail && t + 4 <= KSTEPS) avail = wait_flags(ws, fo, 0, avail, t + 4, seq);
    LDB(B0, 0, 0); SCHED; LDA(At, 0, 0); STAGE(SA(1, 1), ra, t + 1, 1);
    WAIT_L(8); BAR; WAIT_L(0); MMA(0, 0, At, B0); BAR; SCHED;
    LDB(B1, 0, 1); STAGE(SB(0, 0), rb, t2, 0);
    BAR; WAIT_L(0); MMA(0, 1, At, B1); BAR;
    LDA(At, 0, 1); STAGE(SA(0, 0), ra, t2, 0);
    BAR; WAIT_L(0); MMA(1, 0, At, B0); BAR; SCHED;
    STAGE(SB(0, 1), rb, t2, 1);
    WAIT_V(6); BAR; MMA(1, 1, At, B1); BAR;
    LDB(B0, 1, 0); SCHED; LDA(At, 1, 0); STAGE(SA(0, 1), ra, t2, 1);
    WAIT_L(8); BAR; WAIT_L(0); MMA(0, 0, At, B0); BAR; SCHED;
    LDB(B1, 1, 1); STAGE(SB(1, 0), rb, t3, 0);
    BAR; WAIT_L(0); MMA(0, 1, At, B1); BAR;
    LDA(At, 1, 1); STAGE(SA(1, 0), ra, t3, 0);
    BAR; WAIT_L(0); MMA(1, 0, At, B0); BAR; SCHED;
    STAGE(SB(1, 1), rb, t3, 1);
    WAIT_V(6); BAR; MMA(1, 1, At, B1); BAR;
  }
  WAIT_V(0);
  if (wr == 0) BAR;
  const bool remote = wsl >= SLOT_WR;
  int tz = threadIdx.x; asm volatile("" : "+v"(tz));
  unsigned* ul = nullptr;
  if (remote && tz == 0) ul = unit_slots(ws, seq, wsl - SLOT_WR, 4);
  const OutTile o = out_tile(P, ws, seq, xs, 0, wsl, 0, remote);
#pragma unroll
  for (int q = 0; q < 4; ++q) {
    if (remote) {
      store_quarter<AUX_SYS, false>(o, acc[q >> 1][q & 1], q >> 1, q & 1, wr, wc, tz & 63);
      if (q) {
        WAIT_V(8);
        BAR;
        if (tz == 0) st_sys(ul + q - 1, (seq << 9) + (unsigned)(xs * 4 + q - 1));
      }
    } else {
      store_quarter<0, false>(o, acc[q >> 1][q & 1], q >> 1, q & 1, wr, wc, tz & 63);
    }
  }
  if (remote) {
    DRAIN();
    BAR;
    if (tz == 0) st_sys(ul + 3, (seq << 9) + (unsigned)(xs * 4 + 3));
  }
#undef SA
#undef SB
#undef STAGE
#undef LDA
#undef LDB
#undef MMA
}

constexpr int NBQ = 4, QD = NBQ - 1;
__device__ __forceinline__ void gemm_quarter(const Params& P, unsigned char* ws, unsigned seq, int xs, int ah, int wsl, int bh, int mode, LAS unsigned char* lds) {
  const int tid = threadIdx.x, wid = __builtin_amdgcn_readfirstlane(tid >> 6), lane = tid & 63;
  const int wr = wid >> 2, wc = wid & 3, fr = lane & 15, fq = lane >> 4;
  const unsigned par = seq & 1u;
  const __amdgpu_buffer_rsrc_t ra = __builtin_amdgcn_make_buffer_rsrc(ws + CTRL_B + (size_t)par * PAR_B + (size_t)xs * SLOT_B + ah * HALF_B, 0, SLOT_B - ah * HALF_B, RSRC_W3);
  const __amdgpu_buffer_rsrc_t rb = __builtin_amdgcn_make_buffer_rsrc(ws + CTRL_B + (size_t)par * PAR_B + (size_t)wsl * SLOT_B + bh * HALF_B, 0, SLOT_B - bh * HALF_B, RSRC_W3);
  const unsigned fo = (unsigned)(lane < 32 ? xs : wsl) * 1024u + (unsigned)(lane & 31) * 16u;
  const int sel = 1 + (lane < 32 ? ah : bh);
  const unsigned voff = (unsigned)wid * 2048u + (unsigned)lane * 16u;
  const unsigned ldsw = (unsigned)wid * 2048u;
  const int sw0 = (fq ^ (fr & 7)) * 16;
  const int aoff0 = (wr * 64 + fr) * 128 + sw0, aoff1 = aoff0 ^ 64;
  const int boff0 = HALF_B + (wc * 32 + fr) * 128 + sw0, boff1 = boff0 ^ 64;
#define STAGEQ(b, kt) do { const unsigned _so = (unsigned)((kt) & (KSTEPS - 1)) * KSTEP_B; \
    DMA2((b) * KSTEP_B, ra, _so); DMA2((b) * KSTEP_B + HALF_B, rb, _so); } while (0)
  f32x4 acc[4][2];
#pragma unroll
  for (int m = 0; m < 4; ++m)
#pragma unroll
    for (int n = 0; n < 2; ++n) acc[m][n] = (f32x4){0.f, 0.f, 0.f, 0.f};
  f16x8 At[4][2], Bt[2][2];
  int avail = 0;
  if (wid == 0) avail = wait_flags(ws, fo, sel, 0, QD + 1, seq);
  BAR;
#pragma unroll
  for (int b = 0; b < QD; ++b) STAGEQ(b, b);
  WAIT_V(4 * (QD - 1)); BAR;
  if (wr == 1) BAR;
  for (int t0 = 0; t0 < KSTEPS; t0 += NBQ) {
#pragma unroll
    for (int b = 0; b < NBQ; ++b) {
      const int t = t0 + b;
#pragma unroll
      for (int n = 0; n < 2; ++n) {
        Bt[n][0] = *(const LAS f16x8*)(lds + b * KSTEP_B + boff0 + n * 2048); Bt[n][1] = *(const LAS f16x8*)(lds + b * KSTEP_B + boff1 + n * 2048);
      }
#pragma unroll
      for (int m = 0; m < 4; ++m) {
        At[m][0] = *(const LAS f16x8*)(lds + b * KSTEP_B + aoff0 + m * 2048); At[m][1] = *(const LAS f16x8*)(lds + b * KSTEP_B + aoff1 + m * 2048);
      }
      STAGEQ((b + QD) % NBQ, t + QD);
      WAIT_V(4 * (QD - 1)); WAIT_L(0); BAR;
      MMA16(acc, At, Bt);
      if (wid == 0 && t + QD + 2 > avail && avail < KSTEPS) avail = wait_flags(ws, fo, sel, avail, t + QD + 2 < KSTEPS ? t + QD + 2 : KSTEPS, seq);
      BAR; SCHED;
    }
  }
  WAIT_V(0);
  if (wr == 0) BAR;
  int tz = threadIdx.x; asm volatile("" : "+v"(tz));
  unsigned* ul = nullptr;
  if (mode == 1 && tz == 0) ul = unit_slots(ws, seq, wsl - SLOT_WR, 1);
  const OutTile o = out_tile(P, ws, seq, xs, ah, wsl, bh, mode);
  if (mode == 0) store_quarter<0, false>(o, acc, ah, bh, wr, wc, tz & 63);
  else if (mode == 1) store_quarter<AUX_SYS, false>(o, acc, ah, bh, wr, wc, tz & 63);
  else store_quarter<AUX_SYS, true>(o, acc, ah, bh, wr, wc, tz & 63);
  if (mode) {
    DRAIN();
    BAR;
    const unsigned u = (unsigned)(xs * 4 + ah * 2 + bh);
    if (mode == 1 && tz == 0) st_sys(ul, (seq << 9) + u);
    if (mode == 2 && tz < 4)
      st_sys((unsigned*)(P.ws[2 * (wsl - SLOT_WR) + (1 - (P.rank & 1))] + OFF_DONE) + (P.rank >> 1) * 256 + u * 4 + tz, seq << 9);
  }
#undef STAGEQ
}

__global__ __launch_bounds__(512, 2) void ag_gemm_kernel(Params P) {
  extern __shared__ __attribute__((aligned(16))) unsigned char shm[];
  LAS unsigned char* lds = (LAS unsigned char*)shm;
  const int tid = threadIdx.x;
  const int b = blockIdx.x, xcd = b & 7, bi = b >> 3;
  const int rank = P.rank, me = rank >> 1;
  unsigned char* ws = P.ws[rank];
  TS(0);
  if (bi < NPUSH) {
    const unsigned seq = link_role(P, ws, bi, xcd, lds);
    if (tid >= 64) return;
    TS(9);
    if (b == 0) {
      const __amdgpu_buffer_rsrc_t rdn = __builtin_amdgcn_make_buffer_rsrc(ws + OFF_DONE, 0, 4096, RSRC_W3);
      const unsigned long long t0 = __builtin_amdgcn_s_memrealtime();
      for (;;) {
        asm volatile("" ::: "memory");
        unsigned neg = 0;
#pragma unroll
        for (int c = 0; c < 4; ++c) {
          const u32x4 f = __builtin_amdgcn_raw_buffer_load_b128(rdn, tid * 16, c * 1024, AUX_SYS);
#pragma unroll
          for (int j = 0; j < 4; ++j) neg |= f[j] - (seq << 9);
        }
        if (__all((int)neg >= 0)) break;
        if (timed_out((unsigned*)(ws + OFF_TMO), seq, t0)) {
          if (tid == 0) P.out[rank][(size_t)rank * 1024 * NLOC] = __builtin_nanf("");
          break;
        }
        __builtin_amdgcn_s_sleep(1);
      }
      __builtin_amdgcn_fence(__ATOMIC_ACQUIRE, "");
      TS(12);
    }
    return;
  }
  const int cp = xcd & 3, rh = xcd >> 2;
  if (bi < NPUSH + NFULL) {
    LAS unsigned* red = (LAS unsigned*)(lds + LDS_MAIN);
    if (tid == 0) red[8] = take_seq(ws, xcd);
    WAIT_L(0);
    BAR;
    const unsigned seq = __builtin_amdgcn_readfirstlane(red[8]);
    const int j = bi - NPUSH, ri = rh * 5 + (j >> 1);
    const int xs = ri < 2 ? SLOT_AWAY + ri : ri < 8 ? ((me + 1 + ((ri - 2) >> 1)) & 3) * 3 + (ri & 1) : SLOT_AWAY + ri - 6;
    gemm_full(P, ws, seq, xs, ((j & 1) ? SLOT_WR : SLOT_W) + cp, lds);
  } else {
    const int j = bi - NPUSH - NFULL;
    const unsigned seq = prep_role(P, ws, xcd, j, lds);
    TS(2);
    if (j >= NQUART) return;
    const int hr = rh * 3 + (j >> 2);
    const bool remote = (j & 2) != 0;
    for (int it = 0; it < 2; ++it) {
      const int xs = it ? ((me + 1 + (hr >> 1)) & 3) * 3 + 2 : me * 3 + (hr >> 1);
      BAR;
      gemm_quarter(P, ws, seq, xs, hr & 1, (remote ? SLOT_WR : SLOT_W) + cp, j & 1, remote ? (it && (hr & 1) ? 2 : 1) : 0, lds);
    }
  }
}

}

extern "C" void kernel_launch(int rank, int nranks, ncclComm_t comm,
                              hipStream_t stream,
                              void* const* d_in, const int* in_sizes, int n_in,
                              void* d_out, int out_size,
                              const int* in_dtypes, int out_dtype,
                              void* const* d_out_all,
                              void* const* d_ws_all, size_t ws_bytes) {
  (void)comm; (void)in_dtypes; (void)out_dtype; (void)d_out;
  if (nranks != NR || (unsigned)rank >= (unsigned)NR || n_in < 2 || ws_bytes < WS_NEED ||
      in_sizes[0] != 1024 * KDIM || in_sizes[1] != KDIM * NLOC || out_size != 8192 * NLOC) {
    fprintf(stderr, "kernel_launch: unexpected problem shape\n");
    return;
  }
  HIP_CHECK(hipFuncSetAttribute((const void*)ag_gemm_kernel, hipFuncAttributeMaxDynamicSharedMemorySize, LDS_B));
  Params P;
  P.x = (const float*)d_in[0];
  P.w = (const float*)d_in[1];
  for (int p = 0; p < NR; ++p) { P.out[p] = (float*)d_out_all[p]; P.ws[p] = (unsigned char*)d_ws_all[p]; }
  P.rank = rank;
  hipLaunchKernelGGL(ag_gemm_kernel, dim3(NWG), dim3(512), LDS_B, stream, P);
  HIP_CHECK(hipGetLastError());
}
```

```cpp
#include <hip/hip_runtime.h>
#include <rccl/rccl.h>
#include <cstdio>

#define HIP_CHECK(expr) do { hipError_t _e = (expr); if (_e != hipSuccess) { \
    fprintf(stderr, "HIP error %s at %s:%d\n", hipGetErrorString(_e), __FILE__, __LINE__); \
    return; } } while (0)

#define LAS __attribute__((address_space(3)))
typedef _Float16 f16x8 __attribute__((ext_vector_type(8)));
typedef float f32x4 __attribute__((ext_vector_type(4)));
typedef unsigned u32x4 __attribute__((ext_vector_type(4)));

namespace {

constexpr int NR = 8;
constexpr int KDIM = 8192, NLOC = 1024;
constexpr int KSTEPS = KDIM / 64, NPAIR = KSTEPS / 2;
constexpr unsigned KK_B = 8192, PIECE_B = 16384, HALF_B = 16384, KSTEP_B = 32768, PAIR_B = 65536;
constexpr unsigned SLOT_B = NPAIR * PAIR_B;
constexpr int SLOT_V = 8, SLOT_G3 = 11, SLOT_A = 12, SLOT_V3 = 16, SLOT_T3 = 17, SLOT_W = 18, SLOT_WR = 22, NSLOT = 26;
constexpr unsigned PAR_B = NSLOT * SLOT_B;
constexpr size_t CTRL_B = 1u << 20;
constexpr unsigned UREC_B = 68 * 1024, UHDR_B = 2112;
constexpr unsigned OUTS_B = 4 * 64 * UREC_B;
constexpr unsigned OFF_OUTS = 2 * PAR_B, OFF_RCV = OFF_OUTS + OUTS_B, DATA_B = OFF_RCV + OUTS_B;
constexpr size_t WS_NEED = CTRL_B + DATA_B;
constexpr unsigned OFF_TICKET = 0;
constexpr unsigned OFF_TMO = 1024;
constexpr unsigned OFF_QUEUE = 2048;
constexpr unsigned OFF_UHEAD = 3072;
constexpr unsigned OFF_ULIST = 4096;
constexpr unsigned OFF_DIR = 6144;
constexpr unsigned OFF_DONE = 8192;
constexpr unsigned OFF_FLAG = 16384;
constexpr unsigned FLAG_B = (NSLOT + 1) * 1024;
constexpr unsigned CTRL_USED = OFF_FLAG + 2 * FLAG_B;
constexpr unsigned long long DEADLINE = 50000000ull;
constexpr int NWG = 256, NPUSH = 7, NFULL = 11, NPREP = 32 - NPUSH - NFULL;
constexpr int NUNIT = 56;
constexpr int QSAME = 704, QIN = 576, QCROSS = QIN + NUNIT * 4;
constexpr int NDIR = 32;
constexpr unsigned CBUF_B = 65536 + UHDR_B + 320;
constexpr int LDS_MAIN = 133 * 1024;
constexpr int LDS_B = 160 * 1024;
constexpr int AUX_SYS = 17;
#define AG_OAUX 0
constexpr unsigned RSRC_W3 = 0x00020000u;
constexpr unsigned OOB = 0x80000000u;
static_assert(2 * CBUF_B <= (unsigned)LDS_MAIN && DATA_B < OOB, "layout");

struct Params {
  const float* x;
  const float* w;
  float* out[NR];
  unsigned char* ws[NR];
  int rank;
  alignas(4) unsigned short ord[QSAME + QIN];
};

__device__ __forceinline__ unsigned ld_sys(const unsigned* p) { return __hip_atomic_load(p, __ATOMIC_RELAXED, __HIP_MEMORY_SCOPE_SYSTEM); }
__device__ __forceinline__ void st_sys(unsigned* p, unsigned v) { __hip_atomic_store(p, v, __ATOMIC_RELAXED, __HIP_MEMORY_SCOPE_SYSTEM); }
#define DRAIN() asm volatile("s_waitcnt vmcnt(0)" ::: "memory")
#define WAIT_V(n) asm volatile("s_waitcnt vmcnt(%0)" :: "n"(n) : "memory")
#define WAIT_L(n) asm volatile("s_waitcnt lgkmcnt(%0)" :: "n"(n) : "memory")
#define BAR __builtin_amdgcn_s_barrier()
#define SCHED __builtin_amdgcn_sched_barrier(0)
#define RFL(x) ((unsigned)__builtin_amdgcn_readfirstlane(x))
#define STORE_GAP() do { SCHED; asm volatile("s_nop 1"); SCHED; } while (0)
#define TSV(i, v)
#define TS(i)
#define TSW(i)

__device__ __forceinline__ bool timed_out(unsigned* tmo, unsigned seq, unsigned long long t0) {
  if (RFL(__hip_atomic_load(tmo, __ATOMIC_RELAXED, __HIP_MEMORY_SCOPE_AGENT)) == seq ||
      __builtin_amdgcn_s_memrealtime() - t0 > DEADLINE) {
    if ((threadIdx.x & 63) == 0) __hip_atomic_store(tmo, seq, __ATOMIC_RELAXED, __HIP_MEMORY_SCOPE_AGENT);
    return true;
  }
  return false;
}

__device__ __noinline__ int wait_flags(const unsigned char* wsv, unsigned fo, int sel, int p0v, int needv, unsigned seqv) {
  const unsigned long long wa = (unsigned long long)wsv;
  const unsigned whi = RFL((unsigned)(wa >> 32)), wlo = RFL((unsigned)wa);
  const unsigned char* ws = (const unsigned char*)(((unsigned long long)whi << 32) | (unsigned long long)wlo);
  const int p0 = __builtin_amdgcn_readfirstlane(p0v), need = __builtin_amdgcn_readfirstlane(needv);
  const unsigned seq = RFL(seqv);
  const unsigned s9 = seq << 9;
  const __amdgpu_buffer_rsrc_t rf = __builtin_amdgcn_make_buffer_rsrc((void*)(ws + OFF_FLAG + (seq & 1u) * FLAG_B), 0, FLAG_B, RSRC_W3);
  const bool past = p0 + (int)(threadIdx.x & 31) >= NPAIR;
  const unsigned long long t0 = __builtin_amdgcn_s_memrealtime();
  for (;;) {
    asm volatile("" ::: "memory");
    const u32x4 f = __builtin_amdgcn_raw_buffer_load_b128(rf, fo, (unsigned)p0 * 16u, AUX_SYS);
    int old = 0;
#pragma unroll
    for (int j = 0; j < 4; ++j) old |= (int)((f[j] - s9) >> 31) << j;
    const unsigned long long m = __ballot(past || (old & sel) == 0);
    const unsigned miss = ~((unsigned)m & (unsigned)(m >> 32));
    int av = p0 + (miss ? __builtin_ctz(miss) : 32);
    av = av > NPAIR ? NPAIR : av;
    if (av >= need) return av;
    if (timed_out((unsigned*)(ws + OFF_TMO), seq, t0)) return NPAIR;
    __builtin_amdgcn_s_sleep(4);
  }
}

__device__ __forceinline__ unsigned wave_max(unsigned m) {
#pragma unroll
  for (int o = 32; o >= 1; o >>= 1) { const unsigned t = (unsigned)__shfl_xor((int)m, o); m = t > m ? t : m; }
  return m;
}

__device__ __forceinline__ unsigned take_seq(unsigned char* ws, int xcd) {
  return __hip_atomic_fetch_add((unsigned*)(ws + OFF_TICKET) + xcd * 32, 1u, __ATOMIC_RELAXED, __HIP_MEMORY_SCOPE_AGENT) / (NWG / 8) + 1u;
}

struct Src { __amdgpu_buffer_rsrc_t rs; unsigned lo, ks_b, st_a, st_b, so; };

__device__ __forceinline__ Src make_src(const Params& P, int tl8) {
  const int tid = threadIdx.x, wid = tid >> 6, lane = tid & 63, r8 = lane >> 3, c8 = lane & 7;
  const bool isw = tl8 >= 4;
  const int tl = tl8 & 3;
  Src q;
  q.rs = __builtin_amdgcn_make_buffer_rsrc((void*)(isw ? P.w + tl * 256 : P.x + (size_t)tl * 256 * KDIM), 0,
                                           isw ? (KDIM * NLOC - tl * 256) * 4 : 256 * KDIM * 4, RSRC_W3);
  q.lo = isw ? (unsigned)(wid * 8 * NLOC + lane * 4) * 4u : (unsigned)((wid * 8 + r8) * KDIM + (c8 ^ r8) * 8) * 4u;
  q.ks_b = isw ? 64u * NLOC * 4u : 64u * 4u;
  q.st_a = isw ? 2u * NLOC * 4u : 64u * KDIM * 4u;
  q.st_b = isw ? NLOC * 4u : 16u;
  q.so = isw ? (unsigned)((tid * 16 + (tid & 15)) * NLOC + ((tid * 5) & 63) * 4) * 4u
             : (unsigned)((tid >> 1) * KDIM + ((tid * 37 + 11) & 2047) * 4) * 4u;
  return q;
}
#define PLOAD(q, v, ks) do { const unsigned _k = (unsigned)(ks) * q.ks_b, _lo = (ks) < KSTEPS ? q.lo : OOB; \
    _Pragma("unroll") for (int i = 0; i < 8; ++i) \
      v[i] = __builtin_amdgcn_raw_buffer_load_b128(q.rs, _lo, _k + (i >> 1) * q.st_a + (i & 1) * q.st_b, 0); } while (0)

__device__ __forceinline__ int tile_exp(const u32x4 sv, LAS unsigned* red) {
  unsigned m = sv[0] & 0x7fffffffu;
#pragma unroll
  for (int j = 1; j < 4; ++j) { const unsigned a = sv[j] & 0x7fffffffu; m = a > m ? a : m; }
  m = wave_max(m);
  if ((threadIdx.x & 63) == 0) red[threadIdx.x >> 6] = m;
  WAIT_L(0);
  BAR;
#pragma unroll
  for (int q = 0; q < 8; ++q) { const unsigned a = red[q]; m = a > m ? a : m; }
  m = RFL(m);
  const int eb = (int)(m >> 23);
  return m == 0 ? 131 : eb < 5 ? 5 : eb;
}

__device__ __forceinline__ void convert(const u32x4 (&v)[8], u32x4 (&h)[4], float s, bool isw, LAS unsigned char* lb) {
  const int tid = threadIdx.x, wid = tid >> 6, lane = tid & 63, r8 = lane >> 3, c8 = lane & 7;
  if (isw) {
    const int tw = lane * 512 + ((wid ^ c8) * 16);
    const int tr = tid * 16 ^ ((r8 ^ ((wid * 2 + (r8 >> 2)) & 7)) * 16);
#pragma unroll
    for (int cc = 0; cc < 4; ++cc) {
      f16x8 t;
#pragma unroll
      for (int i = 0; i < 8; ++i) t[i] = (_Float16)(__uint_as_float(v[i][cc]) * s);
      *(LAS f16x8*)(lb + tw + cc * 128) = t;
    }
    WAIT_L(0);
    BAR;
#pragma unroll
    for (int it = 0; it < 4; ++it) h[it] = *(const LAS u32x4*)(lb + tr + it * 8192);
  } else {
#pragma unroll
    for (int it = 0; it < 4; ++it) {
      f16x8 t;
#pragma unroll
      for (int i = 0; i < 4; ++i) {
        t[i] = (_Float16)(__uint_as_float(v[2 * it][i]) * s);
        t[4 + i] = (_Float16)(__uint_as_float(v[2 * it + 1][i]) * s);
      }
      h[it] = __builtin_bit_cast(u32x4, t);
    }
  }
}

__device__ __forceinline__ unsigned prep_role(const Params& P, unsigned char* ws, int tl8, int pi, LAS unsigned char* lds) {
  const int tid = threadIdx.x, me = P.rank >> 1;
  LAS unsigned* red = (LAS unsigned*)(lds + LDS_MAIN);
  const bool isw = tl8 >= 4;
  const Src q = make_src(P, tl8);
  const u32x4 sv = __builtin_amdgcn_raw_buffer_load_b128(q.rs, q.so, 0, 0);
  u32x4 v0[8], v1[8];
  PLOAD(q, v0, 2 * pi);
  PLOAD(q, v1, 2 * pi + 1);
  if (tid == 0) red[8] = take_seq(ws, tl8);
  const int eb = tile_exp(sv, red);
  const unsigned seq = RFL(red[8]);
  const float s = __uint_as_float((unsigned)(258 - eb) << 23);
  const unsigned p0 = isw ? (SLOT_W + tl8 - 4) * 256u : tl8 < 2 ? (me * 2 + tl8) * 256u : tl8 == 2 ? SLOT_V * 256u + me : SLOT_T3 * 256u;
  const unsigned gs = tl8 == 2 ? 256u : 1u;
  const unsigned par = seq & 1u;
  const __amdgpu_buffer_rsrc_t rd = __builtin_amdgcn_make_buffer_rsrc(ws + CTRL_B + (size_t)par * PAR_B, 0, PAR_B, RSRC_W3);
  unsigned* flag = (unsigned*)(ws + OFF_FLAG + par * FLAG_B) + p0 + (tid & 3) * gs;
  const unsigned fval = (seq << 9) + (unsigned)(256 + eb - 131);
  for (int pr = pi; pr < NPAIR; pr += NPREP) {
    u32x4 h0[4], h1[4];
    convert(v0, h0, s, isw, lds);
    convert(v1, h1, s, isw, lds + KSTEP_B);
#pragma unroll
    for (int it = 0; it < 4; ++it) {
      __builtin_amdgcn_raw_buffer_store_b128(h0[it], rd, (unsigned)tid * 16u, (p0 + pr * 4u + it * gs) * PIECE_B, AUX_SYS);
      __builtin_amdgcn_raw_buffer_store_b128(h1[it], rd, (unsigned)tid * 16u, (p0 + pr * 4u + it * gs) * PIECE_B + KK_B, AUX_SYS);
    }
    PLOAD(q, v0, 2 * (pr + NPREP));
    PLOAD(q, v1, 2 * (pr + NPREP) + 1);
    WAIT_V(16);
    BAR;
    if (tid < 4) st_sys(flag + pr * 4, fval);
  }
  return seq;
}

#define AG_PL 5
constexpr int PL = AG_PL;
constexpr int RING = 4;
constexpr unsigned RSLOT_B = 17 * 1024, LDS_TR = RING * RSLOT_B;
constexpr int NXW = NPUSH * 8 * 6;

__device__ __forceinline__ int group_row0(int g, int xs, int rg) {
  return xs < SLOT_V ? (2 * (xs >> 1) + g) * 1024 + (xs & 1) * 256 + rg * 64
       : xs < SLOT_G3 ? (2 * rg + g) * 1024 + 512 + (xs - SLOT_V) * 64
       : xs < SLOT_V3 ? (2 * (xs - SLOT_A) + 1 - g) * 1024 + 768 + rg * 64
                      : (2 * rg + 1 - g) * 1024 + 704;
}
__device__ __forceinline__ unsigned pack_off(unsigned cn, int w) {
  unsigned o = UHDR_B;
#pragma unroll
  for (int i = 0; i < 8; ++i) {
    const unsigned c = (unsigned)__builtin_amdgcn_readlane((int)cn, i);
    if (i < w) o += ((c > 2048u ? 2048u : c) * 4u + 63u) & ~63u;
  }
  return o;
}

__device__ __forceinline__ unsigned link_role(const Params& P, unsigned char* ws, int link, int wgi, LAS unsigned char* lds) {
  const int tid = threadIdx.x, wid = __builtin_amdgcn_readfirstlane(tid >> 6), lane = tid & 63;
  const int rank = P.rank, me = rank >> 1, peer = (rank + 1 + link) & (NR - 1);
  const bool same = ((peer ^ rank) & 1) == 0;
  const unsigned sA = same ? me * 2 : SLOT_W + (peer >> 1), dA = same ? me * 2 : SLOT_WR + me;
  const unsigned sB = same ? me * 2 + 1 : SLOT_T3, dB = same ? me * 2 + 1 : SLOT_A + me;
  const unsigned sC = same ? SLOT_V : SLOT_G3, dC = same ? SLOT_V : SLOT_V3;
  unsigned char* pw = P.ws[peer];
  LAS unsigned* red = (LAS unsigned*)(lds + LDS_MAIN);
  const __amdgpu_buffer_rsrc_t rp = __builtin_amdgcn_make_buffer_rsrc(pw, 0, CTRL_USED, RSRC_W3);
  const __amdgpu_buffer_rsrc_t rd = __builtin_amdgcn_make_buffer_rsrc(pw + CTRL_B, 0, DATA_B, RSRC_W3);
  const unsigned one = lane == 0 ? 0u : OOB;
  unsigned seq;
  {
    const int tl8 = same ? wgi & 1 : (wgi & 1) ? 3 : 4 + (peer >> 1), pr = wgi >> 1;
    const Src q = make_src(P, tl8);
    const u32x4 sv = __builtin_amdgcn_raw_buffer_load_b128(q.rs, q.so, 0, 0);
    u32x4 v0[8], v1[8], h[2][4];
    PLOAD(q, v0, 2 * pr);
    PLOAD(q, v1, 2 * pr + 1);
    if (tid == 0) { red[8] = take_seq(ws, wgi); red[9] = 0; red[10] = 0; red[11] = 0; }
    const int eb = tile_exp(sv, red);
    seq = RFL(red[8]);
    const float s = __uint_as_float((unsigned)(258 - eb) << 23);
    const unsigned pc = ((wgi & 1) ? dB : dA) * 256u + (unsigned)pr * 4u;
#pragma unroll
    for (int kk = 0; kk < 2; ++kk) {
      convert(kk ? v1 : v0, h[kk], s, tl8 >= 4, lds + LDS_TR + kk * KSTEP_B);
      if (!kk) TS(3);
#pragma unroll
      for (int it = 0; it < 4; ++it) {
        __builtin_amdgcn_raw_buffer_store_b128(h[kk][it], rd, (unsigned)tid * 16u, (seq & 1u) * PAR_B + (pc + it) * PIECE_B + kk * KK_B, AUX_SYS);
        if (wid != 1) DRAIN();
      }
    }
    DRAIN();
    unsigned n = 0;
    if (lane == 0) n = __hip_atomic_fetch_add(red + 9, 1u, __ATOMIC_RELAXED, __HIP_MEMORY_SCOPE_WORKGROUP);
    if (RFL(n) == 7)
      __builtin_amdgcn_raw_buffer_store_b32((seq << 9) + (unsigned)(256 + eb - 131), rp, lane < 4 ? lane * 4u : OOB,
                                            OFF_FLAG + (seq & 1u) * FLAG_B + pc * 4u, AUX_SYS);
  }
  const unsigned par = seq & 1u, s9 = seq << 9;
  unsigned* tmo = (unsigned*)(ws + OFF_TMO);
  volatile LAS unsigned* cnt = red;
  const __amdgpu_buffer_rsrc_t rs = __builtin_amdgcn_make_buffer_rsrc(ws + CTRL_B, 0, DATA_B, RSRC_W3);
  const __amdgpu_buffer_rsrc_t rc = __builtin_amdgcn_make_buffer_rsrc(ws, 0, CTRL_USED, RSRC_W3);
  const unsigned voff = (unsigned)lane * 16u;
  if (wid > 1) {
    const __amdgpu_buffer_rsrc_t ro = __builtin_amdgcn_make_buffer_rsrc(P.out[rank], 0, 8192 * NLOC * 4, RSRC_W3);
    LAS unsigned char* lb = lds + LDS_TR + (wid - 2) * 8192;
    {
      const unsigned long long t0 = __builtin_amdgcn_s_memrealtime();
      while (RFL(cnt[9]) < 8u && __builtin_amdgcn_s_memrealtime() - t0 < DEADLINE) __builtin_amdgcn_s_sleep(8);
    }
    for (int T = (link * 8 + wgi) * 6 + wid - 2; T < NUNIT * 32; T += NXW) {
      const unsigned ci = (unsigned)((T >> 3) & 3) * 64u + (unsigned)(T >> 5);
      const int w = T & 7;
      unsigned u;
      bool bad = false;
      const unsigned long long t0 = __builtin_amdgcn_s_memrealtime();
      for (;;) {
        asm volatile("" ::: "memory");
        const u32x4 f = __builtin_amdgcn_raw_buffer_load_b128(rc, 0, OFF_DONE + ci * 16u, AUX_SYS);
        u = RFL(f[0]);
        if ((int)RFL((f[0] - s9) | (f[1] - s9) | (f[2] - s9) | (f[3] - s9)) >= 0) break;
        if (timed_out(tmo, seq, t0)) { bad = true; break; }
        __builtin_amdgcn_s_sleep(16);
      }
      bad = RFL(bad) != 0;
      u = RFL(u);
      const unsigned rec = OFF_RCV + ci * UREC_B;
      const unsigned mk = __builtin_amdgcn_raw_buffer_load_b32(rs, lane * 32u + w * 4u, rec, AUX_SYS);
      const unsigned cn = __builtin_amdgcn_raw_buffer_load_b32(rs, 2048u + (lane & 7) * 4u, rec, AUX_SYS);
      const unsigned off = pack_off(cn, w), nb = (unsigned)__builtin_amdgcn_readlane((int)cn, w) * 4u;
#pragma unroll
      for (int i = 0; i < 2; ++i) {
        LAS void* d = (LAS void*)(lb + i * 4096);
        const unsigned so = i * 4096u < nb ? rec + off + i * 4096u : OOB, so2 = i * 4096u + 2048u < nb ? so : OOB;
        __builtin_amdgcn_raw_ptr_buffer_load_lds(rs, d, 16, voff, so, 0, AUX_SYS);
        __builtin_amdgcn_raw_ptr_buffer_load_lds(rs, d, 16, voff, so, 1024, AUX_SYS);
        __builtin_amdgcn_raw_ptr_buffer_load_lds(rs, d, 16, voff, so2, 2048, AUX_SYS);
        __builtin_amdgcn_raw_ptr_buffer_load_lds(rs, d, 16, voff, so2, 3072, AUX_SYS);
      }
      DRAIN();
      f32x4 v[8];
      unsigned run = (unsigned)(unsigned long long)lb;
#pragma unroll
      for (int i = 0; i < 32; ++i) {
        const bool k = (int)(mk << i) < 0;
        const unsigned long long bl = __builtin_amdgcn_ballot_w64(k);
        const unsigned pos = __builtin_amdgcn_mbcnt_hi((unsigned)(bl >> 32), __builtin_amdgcn_mbcnt_lo((unsigned)bl, 0u)) * 4u + run;
        const float t = *(const LAS float*)(unsigned long long)pos;
        v[i >> 2][i & 3] = k ? t : 0.f;
        run += (unsigned)__builtin_popcountll(bl) * 4u;
      }
      if (bad) {
#pragma unroll
        for (int i = 0; i < 8; ++i) v[i] = (f32x4){__builtin_nanf(""), __builtin_nanf(""), __builtin_nanf(""), __builtin_nanf("")};
      }
      const int ti = (int)(u >> 2) & 15;
      const unsigned vo = (unsigned)(lane & 15) * (NLOC * 4u) + (unsigned)(lane >> 4) * 16u;
      const unsigned so = (unsigned)group_row0(1 - (rank & 1), ti < SLOT_G3 ? ti : ti + 1, (int)(u & 2u) + (w >> 2)) * (NLOC * 4u)
                        + (ci >> 6) * 1024u + (u & 1u) * 512u + (unsigned)(w & 3) * 128u;
#pragma unroll
      for (int i = 0; i < 8; ++i) asm volatile("" : "+v"(v[i]));
      SCHED;
#pragma unroll
      for (int i = 0; i < 8; ++i)
        __builtin_amdgcn_raw_buffer_store_b128(__builtin_bit_cast(u32x4, v[i]), ro, vo + (i & 1) * 64u, so + (i >> 1) * 16u * (NLOC * 4u), AG_OAUX);
      STORE_GAP();
    }
    return seq;
  }
  if (wid == 1) {
    const unsigned fb = OFF_FLAG + par * FLAG_B;
    const int Q = same ? QSAME : QCROSS, QI = same ? QSAME : QIN;
    const unsigned* ord = (const unsigned*)P.ord + (same ? 0 : QSAME / 2);
    const unsigned qbase = (seq - 1u) * (unsigned)(Q - NDIR) - (NDIR + 16u);
    unsigned fA = 0, fB = 0, fC = 0;
    int rdy = 0;
    int p = NDIR + wgi, pn = NDIR + 8 + wgi;
    bool late = false;
    for (int n = 0;; ++n) {
      const bool live = (unsigned)p < (unsigned)Q;
      const unsigned tk = (unsigned)__builtin_amdgcn_raw_ptr_buffer_atomic_add_i32(1, rc, live ? one : OOB, OFF_QUEUE + link * 128, 16);
      u32x4 rec = {0u, 255u, 0u, 0u};
      unsigned so = OOB, kb = 16;
      if (live && p < QI) {
        const unsigned o = ord[p >> 1] >> ((p & 1) * 16), sel = (o >> 8) & 7u, pc = o & 255u;
        if (rdy <= (int)(pc >> 2)) {
          DRAIN();
          cnt[10] = n;
          const unsigned long long t0 = __builtin_amdgcn_s_memrealtime();
          for (;;) {
            asm volatile("" ::: "memory");
            const unsigned a = __builtin_amdgcn_raw_buffer_load_b32(rc, voff, fb + sA * 1024u, AUX_SYS);
            const unsigned b = __builtin_amdgcn_raw_buffer_load_b32(rc, voff, fb + sB * 1024u, AUX_SYS);
            const unsigned c = __builtin_amdgcn_raw_buffer_load_b32(rc, voff, fb + sC * 1024u + me * 4u, AUX_SYS);
            const unsigned long long miss = __ballot((int)((a - s9) | (b - s9) | (c - s9)) < 0);
            rdy = miss ? __builtin_ctzll(miss) : NPAIR;
            fA = RFL(a); fB = RFL(b); fC = RFL(c);
            if (rdy > (int)(pc >> 2)) break;
            if (timed_out(tmo, seq, t0)) { rdy = NPAIR; fA = fB = fC = s9; break; }
            __builtin_amdgcn_s_sleep(4);
          }
          rdy = __builtin_amdgcn_readfirstlane(rdy);
        }
        const unsigned dn = (sel >= 2 ? dC + sel - 2 : sel ? dB : dA) * 256u + pc;
        so = par * PAR_B + ((sel >= 2 ? sC + sel - 2 : sel ? sB : sA) * 256u + pc) * PIECE_B;
        rec = (u32x4){par * PAR_B + dn * PIECE_B, 16u, fb + dn * 4u, sel >= 2 ? fC : sel ? fB : fA};
      } else if (live) {
        const unsigned i = (unsigned)(p - QIN) >> 2, j = (unsigned)p & 3u;
        const unsigned eo = OFF_ULIST + (unsigned)(peer >> 1) * 256u + i * 4u;
        unsigned e = RFL(__builtin_amdgcn_raw_buffer_load_b32(rc, 0, eo, AUX_SYS));
        if ((e >> 16) != (seq & 0xffffu)) {
          DRAIN();
          cnt[10] = n;
          const unsigned long long t0 = __builtin_amdgcn_s_memrealtime();
          for (;;) {
            asm volatile("" ::: "memory");
            e = RFL(__builtin_amdgcn_raw_buffer_load_b32(rc, 0, eo, AUX_SYS));
            if ((e >> 16) == (seq & 0xffffu) || timed_out(tmo, seq, t0)) break;
            __builtin_amdgcn_s_sleep(4);
          }
          e = RFL(e);
        }
        unsigned nk = (e >> 8) & 127u;
        nk = nk > 68u ? 68u : nk;
        const unsigned qk = (nk + 3u) >> 2, k0 = j * qk;
        kb = k0 >= nk ? 0u : nk - k0 < qk ? nk - k0 : qk;
        so = OFF_OUTS + ((unsigned)(peer >> 1) * 64u + i) * UREC_B + k0 * 1024u;
        rec = (u32x4){OFF_RCV + ((unsigned)me * 64u + i) * UREC_B + k0 * 1024u, kb, OFF_DONE + (((unsigned)me * 64u + i) * 4u + j) * 4u, s9 + (e & 63u)};
      }
      if (!late && n - (int)RFL(cnt[11]) >= RING) {
        const unsigned long long t0 = __builtin_amdgcn_s_memrealtime();
        while (n - (int)RFL(cnt[11]) >= RING) {
          if (__builtin_amdgcn_s_memrealtime() - t0 > DEADLINE) { late = timed_out(tmo, seq, t0); break; }
          __builtin_amdgcn_s_sleep(2);
        }
      }
      so = RFL(so);
      kb = RFL(kb);
      const int sl = n & (RING - 1);
      if (lane == 0) *(LAS u32x4*)(red + 16 + sl * 4) = rec;
#pragma unroll
      for (int i = 0; i < 5; ++i) {
        LAS void* d = (LAS void*)(lds + sl * RSLOT_B + i * 4096);
        const unsigned sg = i * 4u < kb ? so + i * 4096u : OOB;
        __builtin_amdgcn_raw_ptr_buffer_load_lds(rs, d, 16, voff, sg, 0, AUX_SYS);
        if (i < 4) {
          __builtin_amdgcn_raw_ptr_buffer_load_lds(rs, d, 16, voff, sg, 1024, AUX_SYS);
          __builtin_amdgcn_raw_ptr_buffer_load_lds(rs, d, 16, voff, sg, 2048, AUX_SYS);
          __builtin_amdgcn_raw_ptr_buffer_load_lds(rs, d, 16, voff, sg, 3072, AUX_SYS);
        }
      }
      WAIT_V(18);
      WAIT_L(0);
      cnt[10] = live ? n : n + 1;
      if (!live) break;
      p = pn;
      pn = (int)(RFL(tk) - qbase);
    }
    return seq;
  }
  const unsigned spare = OFF_FLAG + par * FLAG_B + NSLOT * 1024u;
  unsigned fo = spare, fv = 0;
#define PFLAG() __builtin_amdgcn_raw_buffer_store_b32(fv, rp, one, fo, AUX_SYS)
  for (int n = 0;; ++n) {
    if ((int)RFL(cnt[10]) <= n) {
      DRAIN();
      PFLAG();
      fo = spare;
      const unsigned long long t0 = __builtin_amdgcn_s_memrealtime();
      bool dead = false;
      while ((int)RFL(cnt[10]) <= n) {
        if (__builtin_amdgcn_s_memrealtime() - t0 > 2 * DEADLINE) { dead = true; break; }
        __builtin_amdgcn_s_sleep(1);
      }
      if (dead) break;
    }
    const int sl = n & (RING - 1);
    const u32x4 rec = *(const LAS u32x4*)(red + 16 + sl * 4);
    u32x4 buf[17];
#pragma unroll
    for (int i = 0; i < 17; ++i) buf[i] = *(const LAS u32x4*)(lds + sl * RSLOT_B + i * 1024 + lane * 16);
    WAIT_L(0);
    cnt[11] = n + 1;
    const unsigned dst = RFL(rec[0]), kb = RFL(rec[1]);
    if (kb == 255u) break;
#pragma unroll
    for (int i = 0; i < 17; ++i) {
      if (i < (int)kb) {
        __builtin_amdgcn_raw_buffer_store_b128(buf[i], rd, voff, dst + i * 1024u, AUX_SYS);
        WAIT_V(PL + (i >= PL && i < 2 * PL ? 1 : 0));
        if (i == PL - 1) PFLAG();
      }
    }
    if (kb < PL) { DRAIN(); PFLAG(); }
    fo = RFL(rec[2]);
    fv = RFL(rec[3]);
  }
  DRAIN();
  PFLAG();
#undef PFLAG
  return seq;
}

__device__ __forceinline__ int flag_exp(unsigned char* ws, unsigned seq, int slot, int rg) {
  return (int)(ld_sys((const unsigned*)(ws + OFF_FLAG + (seq & 1u) * FLAG_B) + slot * 256 + rg) & 511u) - 256;
}
__device__ __forceinline__ bool call_bad(unsigned char* ws, unsigned seq) {
  return RFL(__hip_atomic_load((unsigned*)(ws + OFF_TMO), __ATOMIC_RELAXED, __HIP_MEMORY_SCOPE_AGENT)) == seq;
}

__device__ __forceinline__ void store_block(const __amdgpu_buffer_rsrc_t ro, f32x4 (&acc)[4][2], int e, bool bad, unsigned so, int lane) {
  const unsigned vo = (unsigned)(lane & 15) * (NLOC * 4u) + (unsigned)(lane >> 4) * 16u;
#pragma unroll
  for (int m = 0; m < 4; ++m)
#pragma unroll
    for (int n = 0; n < 2; ++n) {
#pragma unroll
      for (int j = 0; j < 4; ++j) {
        const float t = __builtin_amdgcn_ldexpf(acc[m][n][j], e);
        acc[m][n][j] = bad ? __builtin_nanf("") : (t < 0.f ? 0.f : t);
      }
      asm volatile("" : "+v"(acc[m][n]));
    }
  SCHED;
#pragma unroll
  for (int m = 0; m < 4; ++m)
#pragma unroll
    for (int n = 0; n < 2; ++n)
      __builtin_amdgcn_raw_buffer_store_b128(__builtin_bit_cast(u32x4, acc[m][n]), ro, vo + n * 64, so + m * 16 * (NLOC * 4u), AG_OAUX);
  STORE_GAP();
}

__device__ __forceinline__ void pack_block(f32x4 (&acc)[4][2], int e, bool bad, LAS unsigned char* cb, int wid, int lane) {
  if (bad) {
#pragma unroll
    for (int i = 0; i < 8; ++i) acc[i >> 1][i & 1] = (f32x4){__builtin_nanf(""), __builtin_nanf(""), __builtin_nanf(""), __builtin_nanf("")};
  }
  const unsigned b0 = (unsigned)(unsigned long long)(cb + wid * 8192), junk = (unsigned)(unsigned long long)(cb + 65536 + UHDR_B + lane * 4);
  unsigned run = b0, mk = 0;
#pragma unroll
  for (int i = 0; i < 32; ++i) {
    const float t = __builtin_amdgcn_ldexpf(acc[i >> 3][(i >> 2) & 1][i & 3], e);
    const bool k = !(t <= 0.f);
    const unsigned long long bl = __builtin_amdgcn_ballot_w64(k);
    const unsigned pos = __builtin_amdgcn_mbcnt_hi((unsigned)(bl >> 32), __builtin_amdgcn_mbcnt_lo((unsigned)bl, 0u)) * 4u + run;
    *(LAS float*)(unsigned long long)(k ? pos : junk) = t;
    mk = mk + mk + (k ? 1u : 0u);
    run += (unsigned)__builtin_popcountll(bl) * 4u;
  }
  *(LAS unsigned*)(cb + 65536 + lane * 32 + wid * 4) = mk;
  if (lane == 0) *(LAS unsigned*)(cb + 65536 + 2048 + wid * 4) = (run - b0) >> 2;
}
__device__ __forceinline__ unsigned send_block(unsigned char* rec, LAS unsigned char* cb, int wid, int lane, unsigned seq, int unit) {
  const unsigned cn = *(const LAS unsigned*)(cb + 65536 + 2048 + (lane & 7) * 4);
  const unsigned off = pack_off(cn, wid), end = pack_off(cn, wid + 1), all = pack_off(cn, 8);
  const __amdgpu_buffer_rsrc_t rh = __builtin_amdgcn_make_buffer_rsrc(rec, 0, UHDR_B, RSRC_W3);
  const __amdgpu_buffer_rsrc_t rv = __builtin_amdgcn_make_buffer_rsrc(rec + off, 0, end - off, RSRC_W3);
  const unsigned vo = (unsigned)lane * 16u;
  const u32x4 hd = *(const LAS u32x4*)(cb + 65536 + (wid < 2 ? wid * 1024 + vo : 2048 + (vo & 63u)));
  __builtin_amdgcn_raw_buffer_store_b128(hd, rh, vo, (unsigned)wid * 1024u, AUX_SYS);
#pragma unroll
  for (int g = 0; g < 2; ++g) {
    u32x4 v[4];
#pragma unroll
    for (int i = 0; i < 4; ++i) v[i] = *(const LAS u32x4*)(cb + wid * 8192 + g * 4096 + i * 1024 + vo);
#pragma unroll
    for (int i = 0; i < 4; ++i) __builtin_amdgcn_raw_buffer_store_b128(v[i], rv, vo + i * 1024u, g * 4096u, AUX_SYS);
    STORE_GAP();
  }
  return (seq << 16) | ((all + 1023u) >> 10 << 8) | (unsigned)unit;
}

__device__ __forceinline__ unsigned unit_slot(unsigned char* ws, unsigned seq, int c) {
  const unsigned pos = __hip_atomic_fetch_add((unsigned*)(ws + OFF_UHEAD) + c * 32, 1u, __ATOMIC_RELAXED, __HIP_MEMORY_SCOPE_AGENT) - (seq - 1u) * NUNIT;
  return (unsigned)c * 64u + (pos < 63u ? pos : 63u);
}

#define AG_LAUX AUX_SYS
#define DMA2(ldsoff, rs, so) do { \
    __builtin_amdgcn_raw_ptr_buffer_load_lds(rs, (LAS void*)(lds + (ldsoff) + ldsw), 16, voff, so, 0, AG_LAUX); \
    __builtin_amdgcn_raw_ptr_buffer_load_lds(rs, (LAS void*)(lds + (ldsoff) + ldsw), 16, voff, so, 1024, AG_LAUX); } while (0)
#define MMA16(ac, At, Bt) do { __builtin_amdgcn_s_setprio(1); \
    _Pragma("unroll") for (int m = 0; m < 4; ++m) _Pragma("unroll") for (int n = 0; n < 2; ++n) _Pragma("unroll") for (int k = 0; k < 2; ++k) \
      ac[m][n] = __builtin_amdgcn_mfma_f32_16x16x32_f16(Bt[n][k], At[m][k], ac[m][n], 0, 0, 0); \
    __builtin_amdgcn_s_setprio(0); } while (0)
#define KOFF(kt) ((unsigned)((kt) >> 1) * PAIR_B + (unsigned)((kt) & 1) * KK_B)

__device__ __forceinline__ int seen_flags(const u32x4 f, int sel, int p0, unsigned s9) {
  int old = 0;
#pragma unroll
  for (int j = 0; j < 4; ++j) old |= (int)((f[j] - s9) >> 31) << j;
  const unsigned long long m = __ballot(p0 + (int)(threadIdx.x & 31) >= NPAIR || (old & sel) == 0);
  const unsigned miss = ~((unsigned)m & (unsigned)(m >> 32));
  return p0 + (miss ? __builtin_ctz(miss) : 32);
}

__device__ __forceinline__ void gemm_full(const Params& P, unsigned char* ws, unsigned seq, int xs, int wsl, LAS unsigned char* lds) {
  const int tid = threadIdx.x, wid = __builtin_amdgcn_readfirstlane(tid >> 6), lane = tid & 63;
  const int wr = wid >> 2, wc = wid & 3, fr = lane & 15, fq = lane >> 4;
  const unsigned par = seq & 1u;
  const __amdgpu_buffer_rsrc_t ra = __builtin_amdgcn_make_buffer_rsrc(ws + CTRL_B + (size_t)par * PAR_B + (size_t)xs * SLOT_B, 0, SLOT_B, RSRC_W3);
  const __amdgpu_buffer_rsrc_t rb = __builtin_amdgcn_make_buffer_rsrc(ws + CTRL_B + (size_t)par * PAR_B + (size_t)wsl * SLOT_B, 0, SLOT_B, RSRC_W3);
  const __amdgpu_buffer_rsrc_t rf = __builtin_amdgcn_make_buffer_rsrc(ws + OFF_FLAG + par * FLAG_B, 0, FLAG_B, RSRC_W3);
  const unsigned fo = (unsigned)(lane < 32 ? xs : wsl) * 1024u + (unsigned)(lane & 31) * 16u, fa = wid == 0 ? fo : OOB;
  const unsigned voff = (unsigned)wr * PIECE_B + (unsigned)wc * 2048u + (unsigned)lane * 16u;
  const unsigned ldsw = (unsigned)wid * 2048u;
  const int sw0 = (fq ^ (fr & 7)) * 16;
  const int aoff0 = (wr * 64 + fr) * 128 + sw0, aoff1 = aoff0 ^ 64;
  const int boff0 = 4 * HALF_B + (wc * 32 + fr) * 128 + sw0, boff1 = boff0 ^ 64;
#define SA(b, h) (((b) * 2 + (h)) * HALF_B)
#define SB(b, h) ((4 + (b) * 2 + (h)) * HALF_B)
#define STAGE(bufoff, rs, po, kk, h) DMA2(bufoff, rs, (po) + (kk) * KK_B + (h) * 2 * PIECE_B)
#define LDA(dst, b, h) do { _Pragma("unroll") for (int m = 0; m < 4; ++m) { \
    dst[m][0] = *(const LAS f16x8*)(lds + SA(b, h) + aoff0 + m * 2048); dst[m][1] = *(const LAS f16x8*)(lds + SA(b, h) + aoff1 + m * 2048); } } while (0)
#define LDB(dst, b, h) do { _Pragma("unroll") for (int n = 0; n < 2; ++n) { \
    dst[n][0] = *(const LAS f16x8*)(lds + SA(b, h) + boff0 + n * 2048); dst[n][1] = *(const LAS f16x8*)(lds + SA(b, h) + boff1 + n * 2048); } } while (0)
#define MMA(ai, bj, At, Bt) MMA16(acc[ai][bj], At, Bt)
  f32x4 acc[2][2][4][2];
#pragma unroll
  for (int a = 0; a < 2; ++a)
#pragma unroll
    for (int b = 0; b < 2; ++b)
#pragma unroll
      for (int m = 0; m < 4; ++m)
#pragma unroll
        for (int n = 0; n < 2; ++n) acc[a][b][m][n] = (f32x4){0.f, 0.f, 0.f, 0.f};
  f16x8 At[4][2], B0[2][2], B1[2][2];
  int avail = 0;
  if (wid == 0) avail = wait_flags(ws, fo, 15, 0, 1, seq);
  BAR;
  STAGE(SB(0, 0), rb, 0u, 0, 0); STAGE(SA(0, 0), ra, 0u, 0, 0); STAGE(SB(0, 1), rb, 0u, 0, 1); STAGE(SA(0, 1), ra, 0u, 0, 1);
  if (wr == 1) BAR;
  WAIT_V(4); BAR;
  u32x4 ff = __builtin_amdgcn_raw_buffer_load_b128(rf, fa, 16, AUX_SYS);
  STAGE(SB(1, 0), rb, 0u, 1, 0); STAGE(SA(1, 0), ra, 0u, 1, 0); STAGE(SB(1, 1), rb, 0u, 1, 1);
  WAIT_V(6); BAR;
  for (int t = 0; t < NPAIR; ++t) {
    const unsigned po = (unsigned)t * PAIR_B, pn = po + PAIR_B;
    if (wid == 0 && t + 2 > avail && t + 2 <= NPAIR) {
      avail = seen_flags(ff, 15, t + 1, seq << 9);
      if (t + 2 > avail) avail = wait_flags(ws, fo, 15, t + 1, t + 2, seq);
    }
    LDB(B0, 0, 0); SCHED; LDA(At, 0, 0); STAGE(SA(1, 1), ra, po, 1, 1);
    WAIT_L(8); BAR; WAIT_L(0); MMA(0, 0, At, B0); BAR; SCHED;
    LDB(B1, 0, 1); STAGE(SB(0, 0), rb, pn, 0, 0);
    BAR; WAIT_L(0); MMA(0, 1, At, B1); BAR;
    LDA(At, 0, 1); STAGE(SA(0, 0), ra, pn, 0, 0);
    BAR; WAIT_L(0); MMA(1, 0, At, B0); BAR; SCHED;
    STAGE(SB(0, 1), rb, pn, 0, 1);
    WAIT_V(6); BAR; MMA(1, 1, At, B1); BAR;
    LDB(B0, 1, 0); SCHED; LDA(At, 1, 0); STAGE(SA(0, 1), ra, pn, 0, 1);
    ff = __builtin_amdgcn_raw_buffer_load_b128(rf, fa, (unsigned)(t + 2) * 16u, AUX_SYS);
    WAIT_L(8); BAR; WAIT_L(0); MMA(0, 0, At, B0); BAR; SCHED;
    LDB(B1, 1, 1); STAGE(SB(1, 0), rb, pn, 1, 0);
    BAR; WAIT_L(0); MMA(0, 1, At, B1); BAR;
    LDA(At, 1, 1); STAGE(SA(1, 0), ra, pn, 1, 0);
    BAR; WAIT_L(0); MMA(1, 0, At, B0); BAR; SCHED;
    STAGE(SB(1, 1), rb, pn, 1, 1);
    WAIT_V(6); BAR; MMA(1, 1, At, B1); BAR;
  }
  const bool remote = wsl >= SLOT_WR;
  int tz = threadIdx.x; asm volatile("" : "+v"(tz));
  LAS unsigned* red = (LAS unsigned*)(lds + LDS_MAIN);
  const int eb = flag_exp(ws, seq, wsl, 0);
  const int e0 = flag_exp(ws, seq, xs, wr) + eb, e1 = flag_exp(ws, seq, xs, 2 + wr) + eb;
  const bool bad = call_bad(ws, seq);
  WAIT_V(0);
  WAIT_L(0);
  if (wr == 0) BAR;
  if (remote) {
    BAR;
    unsigned* ul = (unsigned*)(ws + OFF_ULIST);
    const int u0 = (xs < SLOT_G3 ? xs : xs - 1) * 4;
    unsigned ent = 0, pos = 0;
#pragma unroll
    for (int q = 0; q < 4; ++q) {
      unsigned slot = 0;
      if (tz == 0) slot = unit_slot(ws, seq, wsl - SLOT_WR);
      pack_block(acc[q >> 1][q & 1], (q >> 1) ? e1 : e0, bad, lds + (q & 1) * CBUF_B, wid, tz & 63);
      DRAIN();
      if (tz == 0) red[9 + (q & 1)] = slot;
      WAIT_L(0);
      BAR;
      if (q && tz == 0) st_sys(ul + pos, ent);
      pos = RFL(red[9 + (q & 1)]);
      ent = send_block(ws + CTRL_B + OFF_OUTS + (size_t)pos * UREC_B, lds + (q & 1) * CBUF_B, wid, tz & 63, seq, u0 + q);
    }
    DRAIN();
    BAR;
    if (tz == 0) st_sys(ul + pos, ent);
  } else {
    const __amdgpu_buffer_rsrc_t ro = __builtin_amdgcn_make_buffer_rsrc(P.out[P.rank], 0, 8192 * NLOC * 4, RSRC_W3);
#pragma unroll
    for (int q = 0; q < 4; ++q)
      store_block(ro, acc[q >> 1][q & 1], (q >> 1) ? e1 : e0, bad,
                  (unsigned)group_row0(P.rank & 1, xs, (q >> 1) * 2 + wr) * (NLOC * 4u) + (unsigned)(wsl - SLOT_W) * 1024u + (q & 1) * 512u + wc * 128u, tz & 63);
  }
#undef SA
#undef SB
#undef STAGE
#undef LDA
#undef LDB
#undef MMA
}

#define AG_DW 2
constexpr int NBQ = 5;
#define AG_FW 4
#define WAIT_L0() __builtin_amdgcn_s_waitcnt(0xC07F)
__device__ __forceinline__ void gemm_quarter(const Params& P, unsigned char* ws, unsigned seq, int xs, int ah, int wsl, int bh, LAS unsigned char* lds) {
  const int tid = threadIdx.x, w8 = __builtin_amdgcn_readfirstlane(tid >> 6), lane = tid & 63;
  const unsigned par = seq & 1u;
  f32x4 acc[4][2];
  LAS unsigned char* xb = lds + 3 * KSTEP_B + (w8 & 3) * 8192 + lane * 16;
#define NEXTB(o) o = o == (NBQ - 1) * KSTEP_B ? 0u : o + KSTEP_B
  if (w8 >= 4) {
    const __amdgpu_buffer_rsrc_t ra = __builtin_amdgcn_make_buffer_rsrc(ws + CTRL_B + (size_t)par * PAR_B + (size_t)xs * SLOT_B + ah * 2 * PIECE_B, 0, SLOT_B - ah * 2 * PIECE_B, RSRC_W3);
    const __amdgpu_buffer_rsrc_t rb = __builtin_amdgcn_make_buffer_rsrc(ws + CTRL_B + (size_t)par * PAR_B + (size_t)wsl * SLOT_B + bh * 2 * PIECE_B, 0, SLOT_B - bh * 2 * PIECE_B, RSRC_W3);
    const __amdgpu_buffer_rsrc_t rf = __builtin_amdgcn_make_buffer_rsrc(ws + OFF_FLAG + par * FLAG_B, 0, FLAG_B, RSRC_W3);
    const unsigned fo = (unsigned)(lane < 32 ? xs : wsl) * 1024u + (unsigned)(lane & 31) * 16u, fa = w8 == AG_FW ? fo : OOB;
    const int sel = 3 << (2 * (lane < 32 ? ah : bh));
    const unsigned voff = (unsigned)((w8 >> 1) & 1) * PIECE_B + (unsigned)(w8 & 1) * 4096u + (unsigned)lane * 16u, ldsw = (unsigned)(w8 & 3) * 4096u;
#define DMA4(lo, rs, so) do { LAS void* _d = (LAS void*)(lds + (lo) + ldsw); \
    __builtin_amdgcn_raw_ptr_buffer_load_lds(rs, _d, 16, voff, so, 0, AG_LAUX); __builtin_amdgcn_raw_ptr_buffer_load_lds(rs, _d, 16, voff, so, 1024, AG_LAUX); \
    __builtin_amdgcn_raw_ptr_buffer_load_lds(rs, _d, 16, voff, so, 2048, AG_LAUX); __builtin_amdgcn_raw_ptr_buffer_load_lds(rs, _d, 16, voff, so, 3072, AG_LAUX); } while (0)
#define STAGEQ(bo, kt) do { const unsigned _so = KOFF(kt); DMA4(bo, ra, _so); DMA4((bo) + HALF_B, rb, _so); } while (0)
    int avail = 0;
    if (w8 == 4) avail = wait_flags(ws, fo, sel, 0, NBQ / 2, seq);
    BAR;
    u32x4 f[NBQ - 1];
#pragma unroll
    for (int b = 0; b < NBQ - 1; ++b) {
      asm volatile("" ::: "memory");
      f[b] = __builtin_amdgcn_raw_buffer_load_b128(rf, fa, 0, AUX_SYS);
      STAGEQ(b * KSTEP_B, b);
    }
    unsigned bo = (NBQ - 1) * KSTEP_B;
    for (int k0 = 0; k0 < KSTEPS; k0 += NBQ - 1) {
#pragma unroll
      for (int b = 0; b < NBQ - 1; ++b) {
        const int k = k0 + b, need = (k + NBQ - 1) / 2 + 1;
        WAIT_V(9 * (NBQ - 2));
        if (w8 == 4 && need > avail && need <= NPAIR) {
          avail = seen_flags(f[b], sel, k0 ? k >> 1 : 0, seq << 9);
          if (need > avail) avail = wait_flags(ws, fo, sel, need - 1, need, seq);
        }
        BAR;
        f[b] = __builtin_amdgcn_raw_buffer_load_b128(rf, fa, (unsigned)(need - 1) * 16u, AUX_SYS);
        STAGEQ(bo, k + NBQ - 1);
        NEXTB(bo);
      }
    }
    WAIT_V(0);
    BAR;
    BAR;
#pragma unroll
    for (int i = 0; i < 8; ++i) acc[i >> 1][i & 1] = *(const LAS f32x4*)(xb + i * 1024);
  } else {
    const int fr = lane & 15, fq = lane >> 4;
    const int sw0 = (fq ^ (fr & 7)) * 16;
    const int aoff0 = ((w8 >> 1) * 64 + fr) * 128 + sw0, aoff1 = aoff0 ^ 64;
    const int boff0 = HALF_B + ((w8 & 1) * 64 + fr) * 128 + sw0, boff1 = boff0 ^ 64;
    f32x4 ac[4][4];
#pragma unroll
    for (int m = 0; m < 4; ++m)
#pragma unroll
      for (int n = 0; n < 4; ++n) ac[m][n] = (f32x4){0.f, 0.f, 0.f, 0.f};
    f16x8 A0[4][2], B0[4][2], A1[4][2], B1[4][2];
#define RD(A, B) do { _Pragma("unroll") for (int m = 0; m < 4; ++m) { \
      B[m][0] = *(const LAS f16x8*)(lds + bo + boff0 + m * 2048); A[m][0] = *(const LAS f16x8*)(lds + bo + aoff0 + m * 2048); } \
      _Pragma("unroll") for (int m = 0; m < 4; ++m) { \
      B[m][1] = *(const LAS f16x8*)(lds + bo + boff1 + m * 2048); A[m][1] = *(const LAS f16x8*)(lds + bo + aoff1 + m * 2048); } \
      NEXTB(bo); } while (0)
#define MM(A, B) do { _Pragma("unroll") for (int k = 0; k < 2; ++k) _Pragma("unroll") for (int m = 0; m < 4; ++m) _Pragma("unroll") for (int n = 0; n < 4; ++n) \
      ac[m][n] = __builtin_amdgcn_mfma_f32_16x16x32_f16(B[n][k], A[m][k], ac[m][n], 0, 0, 0); } while (0)
#define STEP(An, Bn, A, B) do { BAR; SCHED; RD(An, Bn); MM(A, B); \
      _Pragma("unroll") for (int i = 0; i < 16; ++i) { __builtin_amdgcn_sched_group_barrier(0x100, 1, 0); __builtin_amdgcn_sched_group_barrier(0x008, 1, 0); } \
      __builtin_amdgcn_sched_group_barrier(0x008, 16, 0); SCHED; WAIT_L0(); } while (0)
    unsigned bo = 0;
    BAR;
    BAR; SCHED;
    RD(A0, B0);
    WAIT_L0();
    for (int k = 1; k < KSTEPS - 1; k += 2) {
      STEP(A1, B1, A0, B0);
      STEP(A0, B0, A1, B1);
    }
    STEP(A1, B1, A0, B0);
    MM(A1, B1);
    BAR;
#pragma unroll
    for (int i = 0; i < 8; ++i) { *(LAS f32x4*)(xb + i * 1024) = ac[i >> 1][2 + (i & 1)]; acc[i >> 1][i & 1] = ac[i >> 1][i & 1]; }
    WAIT_L(0);
    BAR;
#undef RD
#undef MM
#undef STEP
  }
  const int wr = (w8 >> 1) & 1, wc = (w8 & 1) * 2 + (w8 >> 2), wid = wr * 4 + wc;
  const bool remote = wsl >= SLOT_WR, direct = remote && xs > SLOT_V && xs < SLOT_G3;
  int tz = threadIdx.x; asm volatile("" : "+v"(tz));
  LAS unsigned* red = (LAS unsigned*)(lds + LDS_MAIN);
  unsigned slot = 0;
  if (remote && !direct && tz == 0) slot = unit_slot(ws, seq, wsl - SLOT_WR);
  const int e = flag_exp(ws, seq, xs, ah * 2 + wr) + flag_exp(ws, seq, wsl, bh * 2);
  const bool bad = call_bad(ws, seq);
  if (direct) {
    constexpr int PITCH = 528;
#pragma unroll
    for (int m = 0; m < 4; ++m)
#pragma unroll
      for (int n = 0; n < 2; ++n) {
        f32x4 v;
#pragma unroll
        for (int j = 0; j < 4; ++j) {
          const float t = __builtin_amdgcn_ldexpf(acc[m][n][j], e);
          v[j] = bad ? __builtin_nanf("") : (t < 0.f ? 0.f : t);
        }
        *(LAS f32x4*)(lds + (wr * 64 + m * 16 + (tz & 15)) * PITCH + (wc * 32 + n * 16 + ((tz >> 4) & 3) * 4) * 4) = v;
      }
    WAIT_L(0);
    BAR;
    const int owner = 2 * (wsl - SLOT_WR) + 1 - (P.rank & 1);
    const __amdgpu_buffer_rsrc_t ro = __builtin_amdgcn_make_buffer_rsrc(P.out[owner], 0, 8192 * NLOC * 4, RSRC_W3);
    const unsigned so = ((unsigned)group_row0(P.rank & 1, xs, ah * 2 + wr) + (unsigned)wc * 16u) * (NLOC * 4u) + (unsigned)(P.rank >> 1) * 1024u + bh * 512u;
    const unsigned vo = (unsigned)((tz >> 5) & 1) * (NLOC * 4u) + (unsigned)(tz & 31) * 16u;
    u32x4 v[8];
#pragma unroll
    for (int i = 0; i < 8; ++i) v[i] = *(const LAS u32x4*)(lds + (wid * 16 + i * 2 + ((tz >> 5) & 1)) * PITCH + (tz & 31) * 16);
    WAIT_L(0);
#pragma unroll
    for (int i = 0; i < 8; ++i) {
      __builtin_amdgcn_raw_buffer_store_b128(v[i], ro, vo, so + i * 2u * (NLOC * 4u), AUX_SYS);
      WAIT_V(AG_DW);
    }
    DRAIN();
    BAR;
    if (tz == 0) st_sys((unsigned*)(P.ws[owner] + OFF_DIR) + (P.rank >> 1) * 8 + (xs & 1) * 4 + ah * 2 + bh, seq << 9);
  } else if (remote) {
    if (tz == 0) red[9] = slot;
    pack_block(acc, e, bad, lds, wid, tz & 63);
    WAIT_L(0);
    BAR;
    const unsigned pos = RFL(red[9]);
    const unsigned ent = send_block(ws + CTRL_B + OFF_OUTS + (size_t)pos * UREC_B, lds, wid, tz & 63, seq, xs * 4 + ah * 2 + bh);
    DRAIN();
    BAR;
    if (tz == 0) st_sys((unsigned*)(ws + OFF_ULIST) + pos, ent);
  } else {
    const __amdgpu_buffer_rsrc_t ro = __builtin_amdgcn_make_buffer_rsrc(P.out[P.rank], 0, 8192 * NLOC * 4, RSRC_W3);
    store_block(ro, acc, e, bad, (unsigned)group_row0(P.rank & 1, xs, ah * 2 + wr) * (NLOC * 4u) + (unsigned)(wsl - SLOT_W) * 1024u + bh * 512u + wc * 128u, tz & 63);
  }
#undef STAGEQ
#undef DMA4
#undef NEXTB
}

__global__ __launch_bounds__(512, 2) void ag_gemm_kernel(Params P) {
  extern __shared__ __attribute__((aligned(16))) unsigned char shm[];
  LAS unsigned char* lds = (LAS unsigned char*)shm;
  const int tid = threadIdx.x;
  const int b = blockIdx.x, xcd = b & 7, bi = b >> 3;
  const int rank = P.rank, me = rank >> 1;
  unsigned char* ws = P.ws[rank];
  TS(0);
  if (bi < NPUSH) {
    const unsigned seq = link_role(P, ws, bi, xcd, lds);
    if (tid >= 64) return;
    TS(9);
    if (b == 0) {
      const __amdgpu_buffer_rsrc_t rdn = __builtin_amdgcn_make_buffer_rsrc(ws + OFF_DIR, 0, 128, RSRC_W3);
      const unsigned long long t0 = __builtin_amdgcn_s_memrealtime();
      for (;;) {
        asm volatile("" ::: "memory");
        const unsigned f = __builtin_amdgcn_raw_buffer_load_b32(rdn, (tid & 31) * 4, 0, AUX_SYS);
        if (__all((int)(f - (seq << 9)) >= 0)) break;
        if (timed_out((unsigned*)(ws + OFF_TMO), seq, t0)) {
          if (tid == 0) P.out[rank][(size_t)rank * 1024 * NLOC] = __builtin_nanf("");
          break;
        }
        __builtin_amdgcn_s_sleep(1);
      }
      __builtin_amdgcn_fence(__ATOMIC_ACQUIRE, "");
      TS(12);
    }
    return;
  }
  const int cp = xcd & 3, rh = xcd >> 2;
  if (bi < NPUSH + NFULL) {
    LAS unsigned* red = (LAS unsigned*)(lds + LDS_MAIN);
    if (tid == 0) red[8] = take_seq(ws, xcd);
    WAIT_L(0);
    BAR;
    const unsigned seq = RFL(red[8]);
    const int q = rh * NFULL + bi - NPUSH, ri = q >> 1;
    const int xs = ri < 5 ? SLOT_A + ri : ((me + 1 + ((ri - 5) >> 1)) & 3) * 2 + ((ri - 5) & 1);
    gemm_full(P, ws, seq, xs, ((q & 1) ? SLOT_WR : SLOT_W) + cp, lds);
    return;
  }
  const int j = bi - NPUSH - NFULL;
  const unsigned seq = prep_role(P, ws, xcd, NPREP - 1 - j, lds);
  TS(2);
  for (int it = j < 8 ? 0 : 1; it < ((j & 14) == 6 ? 1 : 2); ++it) {
    const bool far = it ? j < 4 || j >= 12 : j >= 4;
    BAR;
    gemm_quarter(P, ws, seq, it ? SLOT_V + ((j & 4) ? 2 : (j >> 1) & 1) : me * 2 + rh, it ? rh : (j >> 1) & 1, (far ? SLOT_WR : SLOT_W) + cp, j & 1, lds);
  }
}

#define AG_S0 136.8
#define AG_S1 142.8
#define AG_S2 146.3
#define AG_VR 0.65
#define AG_XW 6
void merge_streams(unsigned short* out, int n, const int* cnt, const int* size, const double* t0, const double* dt, const int* first, const int* step) {
  int done[5] = {0, 0, 0, 0, 0};
  for (;;) {
    int best = -1;
    for (int s = 0; s < n; ++s)
      if (done[s] < cnt[s] && (best < 0 || t0[s] + done[s] * dt[s] < t0[best] + done[best] * dt[best] - 1e-9)) best = s;
    if (best < 0) return;
    for (int k = 0; k < size[best]; ++k) *out++ = (unsigned short)(first[best] + done[best] * step[best] + k);
    ++done[best];
  }
}

}

extern "C" void kernel_launch(int rank, int nranks, ncclComm_t comm,
                              hipStream_t stream,
                              void* const* d_in, const int* in_sizes, int n_in,
                              void* d_out, int out_size,
                              const int* in_dtypes, int out_dtype,
                              void* const* d_out_all,
                              void* const* d_ws_all, size_t ws_bytes) {
  (void)comm; (void)in_dtypes; (void)out_dtype; (void)d_out;
  if (nranks != NR || (unsigned)rank >= (unsigned)NR || n_in < 2 || ws_bytes < WS_NEED ||
      in_sizes[0] != 1024 * KDIM || in_sizes[1] != KDIM * NLOC || out_size != 8192 * NLOC) {
    fprintf(stderr, "kernel_launch: unexpected problem shape\n");
    return;
  }
  HIP_CHECK(hipFuncSetAttribute((const void*)ag_gemm_kernel, hipFuncAttributeMaxDynamicSharedMemorySize, LDS_B));
  Params P;
  P.x = (const float*)d_in[0];
  P.w = (const float*)d_in[1];
  for (int p = 0; p < NR; ++p) { P.out[p] = (float*)d_out_all[p]; P.ws[p] = (unsigned char*)d_ws_all[p]; }
  P.rank = rank;
  const int me = rank >> 1;
  {
    const int cnt[5] = {NPAIR - 4, NPAIR - 4, NPAIR, NPAIR, NPAIR}, size[5] = {4, 4, 1, 1, 1}, step[5] = {4, 4, 4, 4, 4};
    const int first[5] = {0 << 8 | 16, 1 << 8 | 16, 2 << 8 | me, 3 << 8 | me, 4 << 8 | me};
    const double t0[5] = {4 * 2.35, 4 * 2.35, AG_S0, AG_S1, AG_S2}, dt[5] = {2.35, 2.35, AG_VR, AG_VR, AG_VR};
    merge_streams(P.ord + NDIR, 5, cnt, size, t0, dt, first, step);
  }
  for (int p = 0; p < NDIR; ++p) P.ord[p] = 0;
  for (int p = 0; p < NDIR; ++p) P.ord[QSAME + p] = 0;
  {
    const int cnt[3] = {NPAIR - 4, NPAIR - 4, NPAIR}, size[3] = {4, 4, 1}, first[3] = {0 << 8 | 16, 1 << 8 | 16, 2 << 8 | me}, step[3] = {4, 4, 4};
    const double t0[3] = {4., 4., 0.}, dt[3] = {(NPAIR - 5. - AG_XW) / (NPAIR - 5.), 1., 1.};
    merge_streams(P.ord + QSAME + NDIR, 3, cnt, size, t0, dt, first, step);
  }
  hipLaunchKernelGGL(ag_gemm_kernel, dim3(NWG), dim3(512), LDS_B, stream, P);
  HIP_CHECK(hipGetLastError());
}
```

```cpp
#include <hip/hip_runtime.h>
#include <rccl/rccl.h>
#include <cstdio>

#define HIP_CHECK(expr) do { hipError_t _e = (expr); if (_e != hipSuccess) { \
    fprintf(stderr, "HIP error %s at %s:%d\n", hipGetErrorString(_e), __FILE__, __LINE__); \
    return; } } while (0)

#define LAS __attribute__((address_space(3)))
typedef _Float16 f16x8 __attribute__((ext_vector_type(8)));
typedef float f32x4 __attribute__((ext_vector_type(4)));
typedef unsigned u32x4 __attribute__((ext_vector_type(4)));
typedef unsigned u32x2 __attribute__((ext_vector_type(2)));

namespace {

constexpr int NR = 8;
constexpr int KDIM = 8192, NLOC = 1024;
constexpr int KSTEPS = KDIM / 64, NPAIR = KSTEPS / 2;
constexpr unsigned KK_B = 8192, PIECE_B = 16384, HALF_B = 16384, KSTEP_B = 32768, PAIR_B = 65536;
constexpr unsigned SLOT_B = NPAIR * PAIR_B;
constexpr int SLOT_V = 8, SLOT_G3 = 11, SLOT_A = 12, SLOT_V3 = 16, SLOT_T3 = 17, SLOT_W = 18, SLOT_WR = 22, NSLOT = 26;
constexpr unsigned PAR_B = NSLOT * SLOT_B;
constexpr size_t CTRL_B = 1u << 20;
constexpr unsigned UREC_B = 68 * 1024, UHDR_B = 2112;
constexpr unsigned OUTS_B = 4 * 64 * UREC_B;
constexpr unsigned SH_HI = 8192, SH_EX = 14336, SH_MAX = 256, SH_LONG = 511;
constexpr unsigned OFF_OUTS = 2 * PAR_B, OFF_RCV = OFF_OUTS + OUTS_B, OFF_SH = OFF_RCV + OUTS_B;
constexpr unsigned OFF_INFO = OFF_SH + 2048 * PIECE_B;
constexpr unsigned VREC_B = 256 + 8192, OFF_VREC = OFF_INFO + 2048 * 16 + 1024;
constexpr unsigned OFF_PART = OFF_VREC + 384 * VREC_B, DATA_B = OFF_PART + 64 * 65536;
constexpr size_t WS_NEED = CTRL_B + DATA_B;
constexpr unsigned OFF_TICKET = 0;
constexpr unsigned OFF_TMO = 1024;
constexpr unsigned OFF_QUEUE = 2048;
constexpr unsigned OFF_UHEAD = 3072;
constexpr unsigned OFF_ULIST = 4096;
constexpr unsigned OFF_PFL = 6144;
constexpr unsigned OFF_VFL = 12288;
constexpr unsigned OFF_DONE = 8192;
constexpr unsigned OFF_FLAG = 16384;
constexpr unsigned FLAG_B = (NSLOT + 1) * 1024;
constexpr unsigned OFF_SFLAG = OFF_FLAG + 2 * FLAG_B;
constexpr unsigned CTRL_USED = OFF_SFLAG + 2 * FLAG_B;
constexpr unsigned long long DEADLINE = 50000000ull;
#define AG_NP 8
constexpr int NWG = 256, NPUSH = 7, NFULL = 13, NPREP = AG_NP;
constexpr int NUNIT = 48;
constexpr int QSAME = 704, QIN = 576, QCROSS = QIN + NUNIT * 2;
constexpr int NDIR = 32;
constexpr int LDS_MAIN = 137 * 1024;
constexpr int LDS_B = 160 * 1024;
constexpr int AUX_SYS = 17;
#define AG_OAUX 17
constexpr unsigned RSRC_W3 = 0x00020000u;
constexpr unsigned OOB = 0x80000000u;
static_assert(DATA_B < OOB, "layout");

struct Params {
  const float* x;
  const float* w;
  float* out[NR];
  unsigned char* ws[NR];
  int rank;
  alignas(4) unsigned short ord[QSAME + QIN];
};

__device__ __forceinline__ unsigned ld_sys(const unsigned* p) { return __hip_atomic_load(p, __ATOMIC_RELAXED, __HIP_MEMORY_SCOPE_SYSTEM); }
__device__ __forceinline__ void st_sys(unsigned* p, unsigned v) { __hip_atomic_store(p, v, __ATOMIC_RELAXED, __HIP_MEMORY_SCOPE_SYSTEM); }
#define DRAIN() asm volatile("s_waitcnt vmcnt(0)" ::: "memory")
#define WAIT_V(n) asm volatile("s_waitcnt vmcnt(%0)" :: "n"(n) : "memory")
#define WAIT_L(n) asm volatile("s_waitcnt lgkmcnt(%0)" :: "n"(n) : "memory")
#define BAR __builtin_amdgcn_s_barrier()
#define SCHED __builtin_amdgcn_sched_barrier(0)
#define RFL(x) ((unsigned)__builtin_amdgcn_readfirstlane(x))
#define STORE_GAP() do { SCHED; asm volatile("s_nop 1"); SCHED; } while (0)
#define TSV(i, v)
#define TS(i)
#define TSW(i)
#define TR(ci, k, v)

__device__ __forceinline__ bool timed_out(unsigned* tmo, unsigned seq, unsigned long long t0) {
  if (RFL(__hip_atomic_load(tmo, __ATOMIC_RELAXED, __HIP_MEMORY_SCOPE_AGENT)) == seq ||
      __builtin_amdgcn_s_memrealtime() - t0 > DEADLINE) {
    if ((threadIdx.x & 63) == 0) __hip_atomic_store(tmo, seq, __ATOMIC_RELAXED, __HIP_MEMORY_SCOPE_AGENT);
    return true;
  }
  return false;
}

__device__ __noinline__ int wait_flags(const unsigned char* wsv, unsigned fo, int sel, int p0v, int needv, unsigned seqv) {
  const unsigned long long wa = (unsigned long long)wsv;
  const unsigned whi = RFL((unsigned)(wa >> 32)), wlo = RFL((unsigned)wa);
  const unsigned char* ws = (const unsigned char*)(((unsigned long long)whi << 32) | (unsigned long long)wlo);
  const int p0 = __builtin_amdgcn_readfirstlane(p0v), need = __builtin_amdgcn_readfirstlane(needv);
  const unsigned seq = RFL(seqv);
  const unsigned s9 = seq << 9;
  const __amdgpu_buffer_rsrc_t rf = __builtin_amdgcn_make_buffer_rsrc((void*)(ws + OFF_FLAG + (seq & 1u) * FLAG_B), 0, FLAG_B, RSRC_W3);
  const bool past = p0 + (int)(threadIdx.x & 31) >= NPAIR;
  const unsigned long long t0 = __builtin_amdgcn_s_memrealtime();
  for (;;) {
    asm volatile("" ::: "memory");
    const u32x4 f = __builtin_amdgcn_raw_buffer_load_b128(rf, fo, (unsigned)p0 * 16u, AUX_SYS);
    int old = 0;
#pragma unroll
    for (int j = 0; j < 4; ++j) old |= (int)((f[j] - s9) >> 31) << j;
    const unsigned long long m = __ballot(past || (old & sel) == 0);
    const unsigned miss = ~((unsigned)m & (unsigned)(m >> 32));
    int av = p0 + (miss ? __builtin_ctz(miss) : 32);
    av = av > NPAIR ? NPAIR : av;
    if (av >= need) return av;
    if (timed_out((unsigned*)(ws + OFF_TMO), seq, t0)) return NPAIR;
    __builtin_amdgcn_s_sleep(4);
  }
}

__device__ __forceinline__ unsigned wave_max(unsigned m) {
#pragma unroll
  for (int o = 32; o >= 1; o >>= 1) { const unsigned t = (unsigned)__shfl_xor((int)m, o); m = t > m ? t : m; }
  return m;
}

__device__ __forceinline__ unsigned take_seq(unsigned char* ws, int xcd) {
  return __hip_atomic_fetch_add((unsigned*)(ws + OFF_TICKET) + xcd * 32, 1u, __ATOMIC_RELAXED, __HIP_MEMORY_SCOPE_AGENT) / (NWG / 8) + 1u;
}

struct Src { __amdgpu_buffer_rsrc_t rs; unsigned lo, ks_b, st_a, st_b, so; };

__device__ __forceinline__ Src make_src(const Params& P, int tl8) {
  const int tid = threadIdx.x, wid = tid >> 6, lane = tid & 63, r8 = lane >> 3, c8 = lane & 7;
  const bool isw = tl8 >= 4;
  const int tl = tl8 & 3;
  Src q;
  q.rs = __builtin_amdgcn_make_buffer_rsrc((void*)(isw ? P.w + tl * 256 : P.x + (size_t)tl * 256 * KDIM), 0,
                                           isw ? (KDIM * NLOC - tl * 256) * 4 : 256 * KDIM * 4, RSRC_W3);
  q.lo = isw ? (unsigned)(wid * 8 * NLOC + lane * 4) * 4u : (unsigned)((wid * 32 + r8) * KDIM + (c8 ^ r8) * 8) * 4u;
  q.ks_b = isw ? 64u * NLOC * 4u : 64u * 4u;
  q.st_a = isw ? 2u * NLOC * 4u : 8u * KDIM * 4u;
  q.st_b = isw ? NLOC * 4u : 16u;
  q.so = isw ? (unsigned)((tid * 16 + (tid & 15)) * NLOC + ((tid * 5) & 63) * 4) * 4u
             : (unsigned)((tid >> 1) * KDIM + ((tid * 37 + 11) & 2047) * 4) * 4u;
  return q;
}
#define PLOAD(q, v, ks) do { const unsigned _k = (unsigned)(ks) * q.ks_b, _lo = (ks) < KSTEPS ? q.lo : OOB; \
    _Pragma("unroll") for (int i = 0; i < 8; ++i) \
      v[i] = __builtin_amdgcn_raw_buffer_load_b128(q.rs, _lo, _k + (i >> 1) * q.st_a + (i & 1) * q.st_b, 0); } while (0)

__device__ __forceinline__ int tile_exp(const u32x4 sv, LAS unsigned* red) {
  unsigned m = 0, l = 0;
#pragma unroll
  for (int j = 0; j < 4; ++j) { const unsigned a = sv[j] & 0x7fffffffu; m = a > m ? a : m; l += a >> 15; }
  m = wave_max(m);
#pragma unroll
  for (int o = 32; o >= 1; o >>= 1) l += (unsigned)__shfl_xor((int)l, o);
  if ((threadIdx.x & 63) == 0) { red[threadIdx.x >> 6] = m; red[32 + (threadIdx.x >> 6)] = l; }
  WAIT_L(0);
  BAR;
  l = 0;
#pragma unroll
  for (int q = 0; q < 8; ++q) { const unsigned a = red[q]; m = a > m ? a : m; l += red[32 + q]; }
  m = RFL(m);
  l = RFL(l);
  const int ea = (int)(m >> 23), eg = (int)(((l >> 11) + 588u + 255u) >> 8) + 3;
  const int eb = eg > ea ? eg : ea;
  return m == 0 ? 131 : eb < 5 ? 5 : eb > 254 ? 254 : eb;
}

__device__ __forceinline__ void convert(const u32x4 (&v)[8], u32x4 (&h)[4], float s, bool isw, LAS unsigned char* lb) {
  const int tid = threadIdx.x, wid = tid >> 6, lane = tid & 63, r8 = lane >> 3, c8 = lane & 7;
  if (isw) {
    const int tw = lane * 512 + ((wid ^ c8) * 16);
#pragma unroll
    for (int cc = 0; cc < 4; ++cc) {
      f16x8 t;
#pragma unroll
      for (int i = 0; i < 8; ++i) t[i] = (_Float16)(__uint_as_float(v[i][cc]) * s);
      *(LAS f16x8*)(lb + tw + cc * 128) = t;
    }
    WAIT_L(0);
    BAR;
#pragma unroll
    for (int it = 0; it < 4; ++it) h[it] = *(const LAS u32x4*)(lb + wid * 4096 + it * 1024 + (lane * 16 ^ ((r8 ^ (it * 2 + (r8 >> 2))) * 16)));
  } else {
#pragma unroll
    for (int it = 0; it < 4; ++it) {
      f16x8 t;
#pragma unroll
      for (int i = 0; i < 4; ++i) {
        t[i] = (_Float16)(__uint_as_float(v[2 * it][i]) * s);
        t[4 + i] = (_Float16)(__uint_as_float(v[2 * it + 1][i]) * s);
      }
      h[it] = __builtin_bit_cast(u32x4, t);
    }
  }
}

#define AG_PP 170
#define AG_PO 4
__device__ __forceinline__ unsigned prep_role(const Params& P, unsigned char* ws, int tl8, int pi, LAS unsigned char* lds) {
  const int tid = threadIdx.x, me = P.rank >> 1;
  LAS unsigned* red = (LAS unsigned*)(lds + LDS_MAIN);
  const bool isw = tl8 >= 4;
  const Src q = make_src(P, tl8);
  const u32x4 sv = __builtin_amdgcn_raw_buffer_load_b128(q.rs, q.so, 0, 0);
  u32x4 v0[8], v1[8];
  PLOAD(q, v0, 2 * pi);
  PLOAD(q, v1, 2 * pi + 1);
  if (tid == 0) red[8] = take_seq(ws, tl8);
  if (tid >= 40 && tid < 48) red[tid] = 0;
  const int eb = tile_exp(sv, red);
  const unsigned seq = RFL(red[8]);
  const float s = __uint_as_float((unsigned)(258 - eb) << 23);
  const unsigned p0 = isw ? (SLOT_W + tl8 - 4) * 256u : tl8 < 2 ? (me * 2 + tl8) * 256u : tl8 == 2 ? SLOT_V * 256u + me : SLOT_T3 * 256u;
  const unsigned gs = tl8 == 2 ? 256u : 1u;
  const unsigned par = seq & 1u;
  const __amdgpu_buffer_rsrc_t rd = __builtin_amdgcn_make_buffer_rsrc(ws + CTRL_B + (size_t)par * PAR_B, 0, PAR_B, RSRC_W3);
  unsigned* flag = (unsigned*)(ws + OFF_FLAG + par * FLAG_B) + p0 + (tid & 3) * gs;
  const unsigned fval = (seq << 9) + (unsigned)(256 + eb - 131);
  const __amdgpu_buffer_rsrc_t rh = __builtin_amdgcn_make_buffer_rsrc(ws + CTRL_B + OFF_SH, 0, DATA_B - OFF_SH, RSRC_W3);
  const int wid = __builtin_amdgcn_readfirstlane(tid >> 6), grp = wid >> 1;
  const unsigned vo = (unsigned)(wid & 1) * 4096u + (unsigned)(tid & 63) * 16u;
  const unsigned long long t0 = __builtin_amdgcn_s_memrealtime();
  int odd = 0;
  for (int pr = pi; pr < NPAIR; pr += NPREP, odd ^= 1) {
    u32x4 h0[4], h1[4], lo[4], hi[3];
    convert(v0, h0, s, isw, lds);
    convert(v1, h1, s, isw, lds + KSTEP_B);
    LAS unsigned* cn = red + 40 + odd * 4 + grp;
    LAS unsigned char* ex = lds + 2 * KSTEP_B + odd * 2048 + grp * 512;
    const unsigned dl = (p0 + pr * 4u + grp * gs) * PIECE_B, sp = (unsigned)tl8 * 256u + (unsigned)pr * 4u + grp;
    unsigned e[4], ne = 0;
#pragma unroll
    for (int it = 0; it < 4; ++it) {
      __builtin_amdgcn_raw_buffer_store_b128(h0[it], rd, vo, dl + it * 1024u, AUX_SYS);
      __builtin_amdgcn_raw_buffer_store_b128(h1[it], rd, vo, dl + it * 1024u + KK_B, AUX_SYS);
      unsigned H[4];
#pragma unroll
      for (int d = 0; d < 4; ++d) {
        const unsigned a = d < 2 ? h0[it][2 * d] : h1[it][2 * d - 4], b = d < 2 ? h0[it][2 * d + 1] : h1[it][2 * d - 3];
        lo[it][d] = __builtin_amdgcn_perm(b, a, 0x06040200u);
        H[d] = __builtin_amdgcn_perm(b, a, 0x07050301u);
      }
      const unsigned g = (H[3] & ~0x20202020u) | ((H[3] >> 2) & 0x20202020u);
#pragma unroll
      for (int d = 0; d < 3; ++d) hi[(it * 3 + d) >> 2][(it * 3 + d) & 3] = (H[d] & ~0x60606060u) | ((g << (5 - 2 * d)) & 0x60606060u);
      e[it] = (((H[0] & 0x60606060u) >> 5) | ((H[1] & 0x60606060u) >> 3) | ((H[2] & 0x60606060u) >> 1) | ((H[3] & 0x60606060u) << 1)) ^ 0x55555555u;
      ne += (unsigned)__builtin_popcount((e[it] | (e[it] >> 1)) & 0x55555555u);
    }
    if (ne) {
      unsigned at = __hip_atomic_fetch_add(cn, ne, __ATOMIC_RELAXED, __HIP_MEMORY_SCOPE_WORKGROUP);
#pragma unroll
      for (int it = 0; it < 4; ++it)
        for (unsigned x = e[it]; x; ++at) {
          const unsigned f = (unsigned)__builtin_ctz(x) >> 1, x2 = (x >> (2 * f)) & 3u;
          x &= ~(3u << (2 * f));
          if (at < SH_MAX) *(LAS unsigned short*)(ex + at * 2) = (unsigned short)(x2 << 13 | (f & 2u) << 11 | (vo + it * 1024u) >> 1 | (f & 1u) << 2 | f >> 2);
        }
    }
    WAIT_L(0);
    BAR;
    ne = RFL(*cn);
    const u32x4 ev = *(const LAS u32x4*)(ex + (tid & 31) * 16);
    if (tid < 4) red[40 + (odd ^ 1) * 4 + tid] = 0;
    if (ne <= SH_MAX) {
#pragma unroll
      for (int it = 0; it < 4; ++it) __builtin_amdgcn_raw_buffer_store_b128(lo[it], rh, vo, sp * PIECE_B + it * 1024u, AUX_SYS);
#pragma unroll
      for (int j = 0; j < 3; ++j) __builtin_amdgcn_raw_buffer_store_b128(hi[j], rh, vo - (unsigned)(wid & 1) * 1024u, sp * PIECE_B + SH_HI + j * 1024u, AUX_SYS);
      __builtin_amdgcn_raw_buffer_store_b128(ev, rh, (tid & 127) < 32 ? vo : OOB, sp * PIECE_B + SH_EX, AUX_SYS);
    } else {
      ne = SH_LONG;
#pragma unroll
      for (int it = 0; it < 4; ++it) {
        __builtin_amdgcn_raw_buffer_store_b128(h0[it], rh, vo, sp * PIECE_B + it * 1024u, AUX_SYS);
        __builtin_amdgcn_raw_buffer_store_b128(h1[it], rh, vo, sp * PIECE_B + it * 1024u + KK_B, AUX_SYS);
      }
    }
    __builtin_amdgcn_raw_buffer_store_b32(ne, rh, (tid & 127) == 0 ? 0u : OOB, OFF_INFO - OFF_SH + sp * 16u, AUX_SYS);
    DRAIN();
    BAR;
    if (tid < 4) st_sys(flag + pr * 4, fval);
    while (pr + NPREP < NPAIR && (int)(__builtin_amdgcn_s_memrealtime() - t0) < (pr + NPREP - 11 + AG_PO) * AG_PP) __builtin_amdgcn_s_sleep(32);
    PLOAD(q, v0, 2 * (pr + NPREP));
    PLOAD(q, v1, 2 * (pr + NPREP) + 1);
  }
  return seq;
}

#define AG_PL 5
#define AG_DS 8
#define AG_ST 0
#define AG_SW 0
constexpr int PL = AG_PL;
#define AG_PU 7
constexpr int PU = AG_PU;
#define AG_PN 999
#define AG_P2 6
constexpr int RING = 4;
constexpr unsigned RSLOT_B = 18 * 1024, LDS_TR = RING * RSLOT_B;
constexpr int NXW = 4 * 8 * 6;

__device__ __forceinline__ int group_row0(int g, int xs, int rg) {
  return xs < SLOT_V ? (2 * (xs >> 1) + g) * 1024 + (xs & 1) * 256 + rg * 64
       : xs < SLOT_G3 ? (2 * rg + g) * 1024 + 512 + (xs - SLOT_V) * 64
       : xs < SLOT_V3 ? (2 * (xs - SLOT_A) + 1 - g) * 1024 + 768 + rg * 64
                      : (2 * rg + 1 - g) * 1024 + 704;
}

__device__ __forceinline__ unsigned link_role(const Params& P, unsigned char* ws, int link, int wgi, LAS unsigned char* lds) {
  const int tid = threadIdx.x, wid = __builtin_amdgcn_readfirstlane(tid >> 6), lane = tid & 63;
  const int rank = P.rank, me = rank >> 1, peer = (rank + 1 + link) & (NR - 1);
  const bool same = ((peer ^ rank) & 1) == 0;
  const unsigned sA = same ? me * 2 : SLOT_W + (peer >> 1), dA = same ? me * 2 : SLOT_WR + me;
  const unsigned sB = same ? me * 2 + 1 : SLOT_T3, dB = same ? me * 2 + 1 : SLOT_A + me;
  const unsigned sC = same ? SLOT_V : SLOT_G3, dC = same ? SLOT_V : SLOT_V3;
  unsigned char* pw = P.ws[peer];
  LAS unsigned* red = (LAS unsigned*)(lds + LDS_MAIN);
  const __amdgpu_buffer_rsrc_t rp = __builtin_amdgcn_make_buffer_rsrc(pw, 0, CTRL_USED, RSRC_W3);
  const __amdgpu_buffer_rsrc_t rd = __builtin_amdgcn_make_buffer_rsrc(pw + CTRL_B, 0, DATA_B, RSRC_W3);
  const unsigned one = lane == 0 ? 0u : OOB;
  unsigned seq;
  {
    const int tl8 = same ? wgi & 1 : (wgi & 1) ? 3 : 4 + (peer >> 1), pr = wgi >> 1;
    const Src q = make_src(P, tl8);
    const u32x4 sv = __builtin_amdgcn_raw_buffer_load_b128(q.rs, q.so, 0, 0);
    u32x4 v0[8], v1[8], h[2][4];
    PLOAD(q, v0, 2 * pr);
    PLOAD(q, v1, 2 * pr + 1);
    if (tid == 0) { red[8] = take_seq(ws, wgi); red[9] = 0; red[10] = 0; red[11] = 0; }
    const int eb = tile_exp(sv, red);
    seq = RFL(red[8]);
    const float s = __uint_as_float((unsigned)(258 - eb) << 23);
    const unsigned pc = ((wgi & 1) ? dB : dA) * 256u + (unsigned)pr * 4u;
#pragma unroll
    for (int kk = 0; kk < 2; ++kk) {
      convert(kk ? v1 : v0, h[kk], s, tl8 >= 4, lds + LDS_TR + kk * KSTEP_B);
      if (!kk) {
        TS(3);
        const unsigned long long t0 = __builtin_amdgcn_s_memrealtime();
        while (__builtin_amdgcn_s_memrealtime() - t0 < (unsigned long long)(pr * AG_ST)) __builtin_amdgcn_s_sleep(2);
      }
#pragma unroll
      for (int it = 0; it < 4; ++it) {
        __builtin_amdgcn_raw_buffer_store_b128(h[kk][it], rd, (unsigned)(wid & 1) * 4096u + (unsigned)lane * 16u, (seq & 1u) * PAR_B + (pc + (wid >> 1)) * PIECE_B + kk * KK_B + it * 1024u, AUX_SYS);
        WAIT_V(AG_SW);
      }
    }
    DRAIN();
    unsigned n = 0;
    if (lane == 0) n = __hip_atomic_fetch_add(red + 9, 1u, __ATOMIC_RELAXED, __HIP_MEMORY_SCOPE_WORKGROUP);
    if (RFL(n) == 7)
      __builtin_amdgcn_raw_buffer_store_b32((seq << 9) + (unsigned)(256 + eb - 131), rp, lane < 4 ? lane * 4u : OOB,
                                            OFF_FLAG + (seq & 1u) * FLAG_B + pc * 4u, AUX_SYS);
  }
  const unsigned par = seq & 1u, s9 = seq << 9;
  unsigned* tmo = (unsigned*)(ws + OFF_TMO);
  volatile LAS unsigned* cnt = red;
  const __amdgpu_buffer_rsrc_t rs = __builtin_amdgcn_make_buffer_rsrc(ws + CTRL_B, 0, DATA_B, RSRC_W3);
  const __amdgpu_buffer_rsrc_t rc = __builtin_amdgcn_make_buffer_rsrc(ws, 0, CTRL_USED, RSRC_W3);
  const unsigned voff = (unsigned)lane * 16u;
  if (wid > 1) {
    const __amdgpu_buffer_rsrc_t ro = __builtin_amdgcn_make_buffer_rsrc(P.out[rank], 0, 8192 * NLOC * 4, RSRC_W3);
    LAS unsigned char* lb = lds + LDS_TR + (wid - 2) * 8192;
    {
      const unsigned long long t0 = __builtin_amdgcn_s_memrealtime();
      while (RFL(cnt[9]) < 8u && __builtin_amdgcn_s_memrealtime() - t0 < DEADLINE) __builtin_amdgcn_s_sleep(8);
    }
    const unsigned* ord = (const unsigned*)P.ord + (same ? 0 : QSAME / 2);
    const unsigned pm = (unsigned)peer >> 1;
    for (int p = NDIR + (wid - 2) * 8 + wgi; p < (same ? QSAME : QIN); p += 48) {
      const unsigned o = ord[p >> 1] >> ((p & 1) * 16), sel = (o >> 8) & 7u;
      if (o & 0x800u) continue;
      const unsigned dn = sel >= 2 ? (dC + sel - 2) * 256u + (o & 252u) + pm
                                   : (same ? pm * 2 + sel : (sel ? SLOT_A : SLOT_WR) + pm) * 256u + (o & 255u);
      unsigned f;
      const unsigned long long t0 = __builtin_amdgcn_s_memrealtime();
      for (;;) {
        asm volatile("" ::: "memory");
        f = RFL(__builtin_amdgcn_raw_buffer_load_b32(rc, 0, OFF_SFLAG + par * FLAG_B + dn * 4u, AUX_SYS));
        if ((int)(f - (seq << 18)) >= 0) break;
        if (timed_out(tmo, seq, t0)) { f = SH_LONG << 9; break; }
        __builtin_amdgcn_s_sleep(AG_DS);
      }
      f = RFL(f);
      const unsigned ne = (f >> 9) & 511u, base = par * PAR_B + dn * PIECE_B;
      if (ne <= SH_MAX) {
        u32x4 lo[8], hw[6];
#pragma unroll
        for (int i = 0; i < 8; ++i) lo[i] = __builtin_amdgcn_raw_buffer_load_b128(rs, voff, base + i * 1024u, AUX_SYS);
#pragma unroll
        for (int i = 0; i < 6; ++i) hw[i] = __builtin_amdgcn_raw_buffer_load_b128(rs, voff, base + SH_HI + i * 1024u, AUX_SYS);
        const u32x2 ex = __builtin_amdgcn_raw_buffer_load_b64(rs, lane * 8u, base + SH_EX, AUX_SYS);
        DRAIN();
#pragma unroll
        for (int kk = 0; kk < 2; ++kk) {
#pragma unroll
          for (int i = 0; i < 8; ++i) {
            const unsigned h0 = hw[(i * 3) >> 2][(i * 3) & 3], h1 = hw[(i * 3 + 1) >> 2][(i * 3 + 1) & 3], h2 = hw[(i * 3 + 2) >> 2][(i * 3 + 2) & 3];
            unsigned a, b;
            if (kk == 0) {
              a = (h0 & 0x9f9f9f9fu) | 0x20202020u;
              b = (h1 & 0x9f9f9f9fu) | 0x20202020u;
            } else {
              a = (h2 & 0x9f9f9f9fu) | 0x20202020u;
              b = ((h0 >> 5) & 0x03030303u) | ((h1 >> 3) & 0x0c0c0c0cu) | ((h2 >> 1) & 0x10101010u) | ((h2 << 1) & 0x80808080u) | 0x20202020u;
            }
            const u32x4 v = {__builtin_amdgcn_perm(a, lo[i][2 * kk], 0x05010400u), __builtin_amdgcn_perm(a, lo[i][2 * kk], 0x07030602u),
                             __builtin_amdgcn_perm(b, lo[i][2 * kk + 1], 0x05010400u), __builtin_amdgcn_perm(b, lo[i][2 * kk + 1], 0x07030602u)};
            *(LAS u32x4*)(lb + i * 1024 + voff) = v;
          }
          asm volatile("" ::: "memory");
#pragma unroll
          for (int j = 0; j < 4; ++j) {
            const unsigned x = ex[j >> 1] >> ((j & 1) * 16);
            const bool on = (unsigned)lane * 4u + j < ne && ((x >> 12) & 1u) == (unsigned)kk;
            __hip_atomic_fetch_xor((LAS unsigned*)(lb + ((x & 0xffeu) << 1)), on ? (x & 0x6000u) << ((x & 1u) * 16u) : 0u, __ATOMIC_RELAXED, __HIP_MEMORY_SCOPE_WORKGROUP);
          }
          asm volatile("" ::: "memory");
          u32x4 v[8];
#pragma unroll
          for (int i = 0; i < 8; ++i) v[i] = *(const LAS u32x4*)(lb + i * 1024 + voff);
          WAIT_L(0);
          SCHED;
#pragma unroll
          for (int i = 0; i < 8; ++i) __builtin_amdgcn_raw_buffer_store_b128(v[i], rs, voff, base + kk * KK_B + i * 1024u, AUX_SYS);
          STORE_GAP();
        }
        DRAIN();
      }
      __builtin_amdgcn_raw_buffer_store_b32(s9 + (f & 511u), rc, one, OFF_FLAG + par * FLAG_B + dn * 4u, AUX_SYS);
    }
    TSW(20 + wid);
    const int T0 = ((link >> 1) * 8 + wgi) * 6 + wid - 2;
    for (int T = T0 < 128 || !same ? T0 : 384; T < (same ? 384 : NUNIT * 32); T += same ? 128 : NXW) {
      const unsigned ci = (unsigned)((T >> 3) & 3) * 64u + (unsigned)(T >> 5);
      const unsigned vi = ((unsigned)((T >> 3) & 3) * 12u + (unsigned)(T >> 5)) * 8u + (unsigned)(T & 7);
      const int w = T & 7;
      unsigned u;
      bool bad = false;
      const unsigned long long t0 = __builtin_amdgcn_s_memrealtime();
      for (;;) {
        asm volatile("" ::: "memory");
        u32x2 f = __builtin_amdgcn_raw_buffer_load_b64(rc, 0, same ? OFF_VFL + vi * 4u : OFF_DONE + ci * 16u, AUX_SYS);
        if (same) f[1] = f[0];
        u = RFL(f[0]);
        if ((int)RFL((f[0] - s9) | (f[1] - s9)) >= 0) break;
        if (timed_out(tmo, seq, t0)) { bad = true; break; }
        if (same) __builtin_amdgcn_s_sleep(4); else __builtin_amdgcn_s_sleep(16);
      }
      bad = RFL(bad) != 0;
      u = RFL(u);
      if (!same && w == 0) TR(ci, 10, NOW());
      const unsigned rec = same ? OFF_VREC + vi * VREC_B : OFF_RCV + ci * UREC_B;
      const unsigned mk = __builtin_amdgcn_raw_buffer_load_b32(rs, lane * 4u, same ? rec : rec + w * 256u, AUX_SYS);
      unsigned off = 256u, nb = (u & 511u) * 64u;
      if (same) {
        u = SLOT_V * 4u + (unsigned)(T >> 5);
      } else {
        const u32x2 ob = __builtin_amdgcn_raw_buffer_load_b64(rs, 0, rec + 2048u + w * 8u, AUX_SYS);
        off = UHDR_B + (RFL(ob[0]) & 0xffffu);
        nb = RFL(ob[1]);
      }
#pragma unroll
      for (int i = 0; i < 2; ++i) {
        LAS void* d = (LAS void*)(lb + i * 4096);
        const unsigned so = i * 4096u < nb ? rec + off + i * 4096u : OOB, so2 = i * 4096u + 2048u < nb ? so : OOB;
        __builtin_amdgcn_raw_ptr_buffer_load_lds(rs, d, 16, voff, so, 0, AUX_SYS);
        __builtin_amdgcn_raw_ptr_buffer_load_lds(rs, d, 16, voff, so, 1024, AUX_SYS);
        __builtin_amdgcn_raw_ptr_buffer_load_lds(rs, d, 16, voff, so2, 2048, AUX_SYS);
        __builtin_amdgcn_raw_ptr_buffer_load_lds(rs, d, 16, voff, so2, 3072, AUX_SYS);
      }
      DRAIN();
      f32x4 v[8];
      unsigned run = (unsigned)(unsigned long long)lb;
#pragma unroll
      for (int i = 0; i < 32; ++i) {
        const bool k = (int)(mk << i) < 0;
        const unsigned long long bl = __builtin_amdgcn_ballot_w64(k);
        const unsigned pos = __builtin_amdgcn_mbcnt_hi((unsigned)(bl >> 32), __builtin_amdgcn_mbcnt_lo((unsigned)bl, 0u)) * 4u + run;
        const float t = *(const LAS float*)(unsigned long long)pos;
        v[i >> 2][i & 3] = k ? t : 0.f;
        run += (unsigned)__builtin_popcountll(bl) * 4u;
      }
      if (bad) {
#pragma unroll
        for (int i = 0; i < 8; ++i) v[i] = (f32x4){__builtin_nanf(""), __builtin_nanf(""), __builtin_nanf(""), __builtin_nanf("")};
      }
      const int ti = (int)(u >> 2) & 15;
      const unsigned vo = (unsigned)(lane & 15) * (NLOC * 4u) + (unsigned)(lane >> 4) * 16u;
      const unsigned so = (unsigned)group_row0(1 - (rank & 1), ti < SLOT_G3 ? ti : ti + 1, (int)(u & 2u) + (w >> 2)) * (NLOC * 4u)
                        + (ci >> 6) * 1024u + (u & 1u) * 512u + (unsigned)(w & 3) * 128u;
#pragma unroll
      for (int i = 0; i < 8; ++i) asm volatile("" : "+v"(v[i]));
      SCHED;
#pragma unroll
      for (int i = 0; i < 8; ++i)
        __builtin_amdgcn_raw_buffer_store_b128(__builtin_bit_cast(u32x4, v[i]), ro, vo + (i & 1) * 64u, so + (i >> 1) * 16u * (NLOC * 4u), AG_OAUX);
      STORE_GAP();
      if (!same && w == 7) TR(ci, 11, NOW());
    }
    return seq;
  }
  if (wid == 1) {
    const unsigned fb = OFF_FLAG + par * FLAG_B;
    const int Q = same ? QSAME : QCROSS, QI = same ? QSAME : QIN;
    const unsigned* ord = (const unsigned*)P.ord + (same ? 0 : QSAME / 2);
    const unsigned qbase = (seq - 1u) * (unsigned)(Q - NDIR) - (NDIR + 16u);
    unsigned fA = 0, fB = 0, fC = 0;
    int rdy = 0;
    int p = NDIR + wgi, pn = NDIR + 8 + wgi;
    bool late = false;
    unsigned e = 0;
    unsigned k1 = 0;
    int held = 0;
    for (int n = 0;; ++n) {
      const bool live = (unsigned)p < (unsigned)Q;
      const unsigned tk = (unsigned)__builtin_amdgcn_raw_ptr_buffer_atomic_add_i32(1, rc, live && k1 == 0u ? one : OOB, OFF_QUEUE + link * 128, 16);
      const unsigned en = __builtin_amdgcn_raw_buffer_load_b32(rc, 0, (unsigned)(pn - QI) < (unsigned)(Q - QI) ? OFF_ULIST + (unsigned)(peer >> 1) * 256u + (unsigned)(pn - QIN) / 2u * 4u : OOB, AUX_SYS);
      u32x4 rec = {0u, 255u, 0u, 0u};
      unsigned so = OOB, kb = 16, so4 = OOB, so5 = OOB;
      bool part = false;
      if (live && p < QI) {
        const unsigned o = ord[p >> 1] >> ((p & 1) * 16), sel = (o >> 8) & 7u, pc = o & 255u;
        if (rdy <= (int)(pc >> 2)) {
          DRAIN();
          cnt[10] = n;
          const unsigned long long t0 = __builtin_amdgcn_s_memrealtime();
          for (;;) {
            asm volatile("" ::: "memory");
            const unsigned a = __builtin_amdgcn_raw_buffer_load_b32(rc, voff, fb + sA * 1024u, AUX_SYS);
            const unsigned b = __builtin_amdgcn_raw_buffer_load_b32(rc, voff, fb + sB * 1024u, AUX_SYS);
            const unsigned c = __builtin_amdgcn_raw_buffer_load_b32(rc, voff, fb + sC * 1024u + me * 4u, AUX_SYS);
            const unsigned long long miss = __ballot((int)((a - s9) | (b - s9) | (c - s9)) < 0);
            rdy = miss ? __builtin_ctzll(miss) : NPAIR;
            fA = RFL(a); fB = RFL(b); fC = RFL(c);
            if (rdy > (int)(pc >> 2)) break;
            if (timed_out(tmo, seq, t0)) { rdy = NPAIR; fA = fB = fC = s9; break; }
            __builtin_amdgcn_s_sleep(4);
          }
          rdy = __builtin_amdgcn_readfirstlane(rdy);
        }
        const unsigned dn = (sel >= 2 ? dC + sel - 2 : sel ? dB : dA) * 256u + pc;
        const unsigned fv = sel >= 2 ? fC : sel ? fB : fA;
        if (o & 0x800u) {
          so = par * PAR_B + ((sel >= 2 ? sC + sel - 2 : sel ? sB : sA) * 256u + pc) * PIECE_B;
          rec = (u32x4){par * PAR_B + dn * PIECE_B, 16u, fb + dn * 4u, fv};
        } else {
          const unsigned sp = sel >= 2 ? 512u + (pc & 252u) + (same ? sel - 2 : 3u) : (same ? sel : sel ? 3u : 4u + (peer >> 1)) * 256u + pc;
          so = OFF_SH + sp * PIECE_B;
          so4 = OFF_INFO + sp * 16u;
          rec = (u32x4){par * PAR_B + dn * PIECE_B, 64u, fb + (OFF_SFLAG - OFF_FLAG) + dn * 4u, (seq << 18) + (fv & 511u)};
        }
      } else if (live) {
        const unsigned i = (unsigned)(p - QIN) >> 1, j = (unsigned)p & 1u;
        const unsigned eo = OFF_ULIST + (unsigned)(peer >> 1) * 256u + i * 4u;
        if ((e >> 16) != (seq & 0xffffu)) {
          DRAIN();
          cnt[10] = n;
          const unsigned long long t0 = __builtin_amdgcn_s_memrealtime();
          for (;;) {
            asm volatile("" ::: "memory");
            e = RFL(__builtin_amdgcn_raw_buffer_load_b32(rc, 0, eo, AUX_SYS));
            if ((e >> 16) == (seq & 0xffffu) || timed_out(tmo, seq, t0)) break;
            __builtin_amdgcn_s_sleep(4);
          }
          e = RFL(e);
        }
        unsigned nk = (e >> 8) & 127u;
        nk = nk > 68u ? 68u : nk;
        const unsigned qk = (nk + 1u) >> 1, k0 = j * qk + k1;
        kb = (j ? nk : qk) - k0;
        part = kb > 18u && k1 == 0u;
        kb = kb > 18u ? 18u : kb;
        so = OFF_OUTS + ((unsigned)(peer >> 1) * 64u + i) * UREC_B + k0 * 1024u;
        so4 = kb > 16u ? so + 16384u : OOB;
        so5 = kb > 17u ? so4 : OOB;
        rec = (u32x4){OFF_RCV + ((unsigned)me * 64u + i) * UREC_B + k0 * 1024u, kb,
                      part ? fb + NSLOT * 1024u : OFF_DONE + (((unsigned)me * 64u + i) * 4u + j) * 4u, s9 + (e & 63u)};
        TR((unsigned)(peer >> 1) * 64u + i, 1 + j, NOW());
      }
      if (!late && n - (int)RFL(cnt[11]) >= RING) {
        const unsigned long long t0 = __builtin_amdgcn_s_memrealtime();
        while (n - (int)RFL(cnt[11]) >= RING) {
          if (__builtin_amdgcn_s_memrealtime() - t0 > DEADLINE) { late = timed_out(tmo, seq, t0); break; }
          __builtin_amdgcn_s_sleep(2);
        }
      }
      so = RFL(so);
      so4 = RFL(so4);
      so5 = RFL(so5);
      kb = RFL(kb);
      const int sl = n & (RING - 1);
      if (lane == 0) *(LAS u32x4*)(red + 16 + sl * 4) = rec;
#pragma unroll
      for (int i = 0; i < 5; ++i) {
        LAS void* d = (LAS void*)(lds + sl * RSLOT_B + i * 4096);
        const unsigned sg = i == 4 ? so4 : i * 4u < kb ? so + i * 4096u : OOB;
        __builtin_amdgcn_raw_ptr_buffer_load_lds(rs, d, 16, voff, sg, 0, AUX_SYS);
        if (i < 4) {
          __builtin_amdgcn_raw_ptr_buffer_load_lds(rs, d, 16, voff, sg, 1024, AUX_SYS);
          __builtin_amdgcn_raw_ptr_buffer_load_lds(rs, d, 16, voff, sg, 2048, AUX_SYS);
          __builtin_amdgcn_raw_ptr_buffer_load_lds(rs, d, 16, voff, sg, 3072, AUX_SYS);
        } else {
          __builtin_amdgcn_raw_ptr_buffer_load_lds(rs, d, 16, voff, so5, 1024, AUX_SYS);
        }
      }
      WAIT_V(20);
      WAIT_L(0);
      cnt[10] = live ? n : n + 1;
      if (!live) break;
      const int t = (int)(RFL(tk) - qbase);
      const unsigned er = RFL(en);
      if (RFL(part) != 0u) {
        held = t;
        k1 = 18;
      } else {
        p = pn;
        e = er;
        pn = k1 ? held : t;
        k1 = 0;
      }
    }
    return seq;
  }
  const unsigned spare = OFF_FLAG + par * FLAG_B + NSLOT * 1024u;
  unsigned fo = spare, fv = 0;
#define PFLAG() __builtin_amdgcn_raw_buffer_store_b32(fv, rp, one, fo, AUX_SYS)
  for (int n = 0;; ++n) {
    if ((int)RFL(cnt[10]) <= n) {
      DRAIN();
      PFLAG();
      fo = spare;
      const unsigned long long t0 = __builtin_amdgcn_s_memrealtime();
      bool dead = false;
      while ((int)RFL(cnt[10]) <= n) {
        if (__builtin_amdgcn_s_memrealtime() - t0 > 2 * DEADLINE) { dead = true; break; }
        __builtin_amdgcn_s_sleep(1);
      }
      if (dead) break;
    }
    const int sl = n & (RING - 1);
    const u32x4 rec = *(const LAS u32x4*)(red + 16 + sl * 4);
    u32x4 buf[18];
#pragma unroll
    for (int i = 0; i < 18; ++i) buf[i] = *(const LAS u32x4*)(lds + sl * RSLOT_B + i * 1024 + lane * 16);
    WAIT_L(0);
    cnt[11] = n + 1;
    const unsigned dst = RFL(rec[0]);
    unsigned kb = RFL(rec[1]), ne = 0, v14 = OOB;
    if (kb == 255u) break;
    if (kb == 64u) {
      ne = RFL(buf[16][0]) & 511u;
      kb = ne > SH_MAX ? 16u : 14u;
      if (ne <= SH_MAX && (unsigned)(lane >> 2) * 32u < ne) v14 = voff;
    }
#define PIECE(W) do { _Pragma("unroll") for (int i = 0; i < 18; ++i) { \
      if (i < (int)kb) { \
        __builtin_amdgcn_raw_buffer_store_b128(buf[i], rd, voff, dst + i * 1024u, AUX_SYS); \
        WAIT_V(W + (i >= W && i < 2 * W ? 2 : 0)); \
        if (i == W - 1) { \
          PFLAG(); \
          __builtin_amdgcn_raw_buffer_store_b128(buf[14], rd, v14, dst + SH_EX, AUX_SYS); \
        } \
      } \
    } \
    if (kb < W) { DRAIN(); PFLAG(); } } while (0)
    if (dst >= OFF_OUTS) PIECE(PU); else if (n >= AG_PN) PIECE(AG_P2); else PIECE(PL);
#undef PIECE
    fo = RFL(rec[2]);
    fv = RFL(rec[3]) + (ne << 9);
  }
  DRAIN();
  PFLAG();
#undef PFLAG
  return seq;
}

__device__ __forceinline__ int flag_exp(unsigned char* ws, unsigned seq, int slot, int rg) {
  return (int)(ld_sys((const unsigned*)(ws + OFF_FLAG + (seq & 1u) * FLAG_B) + slot * 256 + rg) & 511u) - 256;
}
__device__ __forceinline__ bool call_bad(unsigned char* ws, unsigned seq) {
  return RFL(__hip_atomic_load((unsigned*)(ws + OFF_TMO), __ATOMIC_RELAXED, __HIP_MEMORY_SCOPE_AGENT)) == seq;
}

__device__ __forceinline__ void store_block(const __amdgpu_buffer_rsrc_t ro, f32x4 (&acc)[4][2], int e, bool bad, unsigned so, int lane) {
  const unsigned vo = (unsigned)(lane & 15) * (NLOC * 4u) + (unsigned)(lane >> 4) * 16u;
#pragma unroll
  for (int m = 0; m < 4; ++m)
#pragma unroll
    for (int n = 0; n < 2; ++n) {
#pragma unroll
      for (int j = 0; j < 4; ++j) {
        const float t = __builtin_amdgcn_ldexpf(acc[m][n][j], e);
        acc[m][n][j] = bad ? __builtin_nanf("") : (t < 0.f ? 0.f : t);
      }
      asm volatile("" : "+v"(acc[m][n]));
    }
  SCHED;
#pragma unroll
  for (int m = 0; m < 4; ++m)
#pragma unroll
    for (int n = 0; n < 2; ++n)
      __builtin_amdgcn_raw_buffer_store_b128(__builtin_bit_cast(u32x4, acc[m][n]), ro, vo + n * 64, so + m * 16 * (NLOC * 4u), AG_OAUX);
  STORE_GAP();
}

__device__ __forceinline__ unsigned unit_ticket(unsigned char* ws, int c, bool take) {
  const __amdgpu_buffer_rsrc_t rc = __builtin_amdgcn_make_buffer_rsrc(ws + OFF_UHEAD, 0, 512, RSRC_W3);
  return (unsigned)__builtin_amdgcn_raw_ptr_buffer_atomic_add_i32(1, rc, take && threadIdx.x == 0 ? 0u : OOB, c * 128, 16);
}
__device__ __forceinline__ unsigned lds_add(LAS unsigned* p, unsigned v) {
  unsigned r = 0;
  if ((threadIdx.x & 63) == 0) r = __hip_atomic_fetch_add(p, v, __ATOMIC_RELAXED, __HIP_MEMORY_SCOPE_WORKGROUP);
  return RFL(r);
}
__device__ __forceinline__ unsigned gather_block(f32x4 (&acc)[4][2], int e, bool bad, LAS unsigned char* cb, unsigned& mk) {
  if (bad) {
    asm volatile("");
#pragma unroll
    for (int i = 0; i < 8; ++i) acc[i >> 1][i & 1] = (f32x4){__builtin_nanf(""), __builtin_nanf(""), __builtin_nanf(""), __builtin_nanf("")};
  }
  const unsigned b0 = RFL((unsigned)(unsigned long long)cb);
  unsigned run = b0;
  mk = 0;
#pragma unroll
  for (int i = 0; i < 32; i += 2) {
    float t0, t1;
    unsigned p0, p1, c0;
    asm volatile(
        "v_ldexp_f32 %0, %7, %9\n\t"
        "v_ldexp_f32 %1, %8, %9\n\t"
        "v_cmp_nge_f32 vcc, 0, %0\n\t"
        "v_cmp_nge_f32 s[92:93], 0, %1\n\t"
        "s_bcnt1_i32_b64 %4, vcc\n\t"
        "v_mbcnt_lo_u32_b32 %2, vcc_lo, 0\n\t"
        "v_mbcnt_lo_u32_b32 %3, s92, 0\n\t"
        "v_mbcnt_hi_u32_b32 %2, vcc_hi, %2\n\t"
        "v_mbcnt_hi_u32_b32 %3, s93, %3\n\t"
        "v_addc_co_u32 %5, s[94:95], %5, %5, vcc\n\t"
        "v_lshl_add_u32 %2, %2, 2, %6\n\t"
        "s_lshl2_add_u32 %6, %4, %6\n\t"
        "v_addc_co_u32 %5, s[94:95], %5, %5, s[92:93]\n\t"
        "v_lshl_add_u32 %3, %3, 2, %6\n\t"
        "s_bcnt1_i32_b64 %4, s[92:93]\n\t"
        "s_mov_b64 exec, vcc\n\t"
        "ds_write_b32 %2, %0\n\t"
        "s_mov_b64 exec, s[92:93]\n\t"
        "ds_write_b32 %3, %1\n\t"
        "s_mov_b64 exec, -1\n\t"
        "s_lshl2_add_u32 %6, %4, %6"
        : "=&v"(t0), "=&v"(t1), "=&v"(p0), "=&v"(p1), "=&s"(c0), "+v"(mk), "+s"(run)
        : "v"(acc[i >> 3][(i >> 2) & 1][i & 3]), "v"(acc[i >> 3][(i >> 2) & 1][(i & 3) + 1]), "v"(e)
        : "vcc", "scc", "s92", "s93", "s94", "s95", "memory");
  }
  WAIT_L(0);
  return run - b0;
}
__device__ __forceinline__ void pack_block(unsigned char* ws, unsigned seq, int c, unsigned& tk, bool more, f32x4 (&acc)[4][2], int e, bool bad, LAS unsigned char* cb, LAS unsigned* u, int wid, int lane) {
  unsigned mk;
  const unsigned nb = gather_block(acc, e, bad, cb, mk), n64 = (nb + 63u) & ~63u;
  const unsigned off = lds_add(u + 1, n64);
  const unsigned vo = (unsigned)lane * 16u;
  u32x4 v[8];
#pragma unroll
  for (int i = 0; i < 8; ++i) v[i] = *(const LAS u32x4*)(cb + i * 1024 + vo);
  unsigned pos = tk - (seq - 1u) * NUNIT;
  if (threadIdx.x == 0) *(volatile LAS unsigned*)u = (unsigned)c * 64u + (pos < 63u ? pos : 63u) + 1u;
  while ((pos = RFL(*(volatile LAS unsigned*)u)) == 0u) __builtin_amdgcn_s_sleep(1);
  unsigned char* rec = ws + CTRL_B + OFF_OUTS + (size_t)(pos - 1u) * UREC_B;
  const __amdgpu_buffer_rsrc_t rh = __builtin_amdgcn_make_buffer_rsrc(rec, 0, UHDR_B, RSRC_W3);
  const __amdgpu_buffer_rsrc_t rv = __builtin_amdgcn_make_buffer_rsrc(rec + UHDR_B + off, 0, n64, RSRC_W3);
  tk = unit_ticket(ws, c, more);
  __builtin_amdgcn_raw_buffer_store_b32(mk, rh, (unsigned)lane * 4u, (unsigned)wid * 256u, AUX_SYS);
  __builtin_amdgcn_raw_buffer_store_b64((u32x2){off, nb}, rh, lane == 0 ? 0u : OOB, 2048u + (unsigned)wid * 8u, AUX_SYS);
#pragma unroll
  for (int i = 0; i < 8; ++i) __builtin_amdgcn_raw_buffer_store_b128(v[i], rv, vo + (i & 3) * 1024u, (i >> 2) * 4096u, AUX_SYS);
  STORE_GAP();
}
__device__ __forceinline__ void post_unit(unsigned char* ws, unsigned seq, LAS unsigned* u, int unit) {
  if (lds_add(u + 2, 1u) == 7u) {
    const unsigned pos = *(volatile LAS unsigned*)u - 1u, all = UHDR_B + *(volatile LAS unsigned*)(u + 1);
    if ((threadIdx.x & 63) == 0) st_sys((unsigned*)(ws + OFF_ULIST) + pos, (seq << 16) | ((all + 1023u) >> 10 << 8) | (unsigned)unit);
    TR(pos, 0, NOW()); TR(pos, 9, (unsigned long long)unit);
  }
}

#define AG_LAUX AUX_SYS
#define DMA2(ldsoff, rs, so) do { \
    __builtin_amdgcn_raw_ptr_buffer_load_lds(rs, (LAS void*)(lds + (ldsoff) + ldsw), 16, voff, so, 0, AG_LAUX); \
    __builtin_amdgcn_raw_ptr_buffer_load_lds(rs, (LAS void*)(lds + (ldsoff) + ldsw), 16, voff, so, 1024, AG_LAUX); } while (0)
#define MMA16(ac, At, Bt) do { __builtin_amdgcn_s_setprio(1); \
    _Pragma("unroll") for (int m = 0; m < 4; ++m) _Pragma("unroll") for (int n = 0; n < 2; ++n) _Pragma("unroll") for (int k = 0; k < 2; ++k) \
      ac[m][n] = __builtin_amdgcn_mfma_f32_16x16x32_f16(Bt[n][k], At[m][k], ac[m][n], 0, 0, 0); \
    __builtin_amdgcn_s_setprio(0); } while (0)
#define KOFF(kt) ((unsigned)((kt) >> 1) * PAIR_B + (unsigned)((kt) & 1) * KK_B)

__device__ __forceinline__ int seen_flags(const u32x4 f, int sel, int p0, unsigned s9) {
  int old = 0;
#pragma unroll
  for (int j = 0; j < 4; ++j) old |= (int)((f[j] - s9) >> 31) << j;
  const unsigned long long m = __ballot(p0 + (int)(threadIdx.x & 31) >= NPAIR || (old & sel) == 0);
  const unsigned miss = ~((unsigned)m & (unsigned)(m >> 32));
  return p0 + (miss ? __builtin_ctz(miss) : 32);
}

__device__ __forceinline__ void gemm_full(const Params& P, unsigned char* ws, unsigned seq, int xs, int wsl, LAS unsigned char* lds) {
  const int tid = threadIdx.x, wid = __builtin_amdgcn_readfirstlane(tid >> 6), lane = tid & 63;
  const int wr = wid >> 2, wc = wid & 3, fr = lane & 15, fq = lane >> 4;
  const unsigned par = seq & 1u;
  const __amdgpu_buffer_rsrc_t ra = __builtin_amdgcn_make_buffer_rsrc(ws + CTRL_B + (size_t)par * PAR_B + (size_t)xs * SLOT_B, 0, SLOT_B, RSRC_W3);
  const __amdgpu_buffer_rsrc_t rb = __builtin_amdgcn_make_buffer_rsrc(ws + CTRL_B + (size_t)par * PAR_B + (size_t)wsl * SLOT_B, 0, SLOT_B, RSRC_W3);
  const __amdgpu_buffer_rsrc_t rf = __builtin_amdgcn_make_buffer_rsrc(ws + OFF_FLAG + par * FLAG_B, 0, FLAG_B, RSRC_W3);
  const unsigned fo = (unsigned)(lane < 32 ? xs : wsl) * 1024u + (unsigned)(lane & 31) * 16u, fa = wid == 0 ? fo : OOB;
  const unsigned voff = (unsigned)wr * PIECE_B + (unsigned)wc * 2048u + (unsigned)lane * 16u;
  const unsigned ldsw = (unsigned)wid * 2048u;
  const int sw0 = (fq ^ (fr & 7)) * 16;
  const int aoff0 = (wr * 64 + fr) * 128 + sw0, aoff1 = aoff0 ^ 64;
  const int boff0 = 4 * HALF_B + (wc * 32 + fr) * 128 + sw0, boff1 = boff0 ^ 64;
#define SA(b, h) (((b) * 2 + (h)) * HALF_B)
#define SB(b, h) ((4 + (b) * 2 + (h)) * HALF_B)
#define STAGE(bufoff, rs, po, kk, h) DMA2(bufoff, rs, (po) + (kk) * KK_B + (h) * 2 * PIECE_B)
#define LDA(dst, b, h) do { _Pragma("unroll") for (int m = 0; m < 4; ++m) { \
    dst[m][0] = *(const LAS f16x8*)(lds + SA(b, h) + aoff0 + m * 2048); dst[m][1] = *(const LAS f16x8*)(lds + SA(b, h) + aoff1 + m * 2048); } } while (0)
#define LDB(dst, b, h) do { _Pragma("unroll") for (int n = 0; n < 2; ++n) { \
    dst[n][0] = *(const LAS f16x8*)(lds + SA(b, h) + boff0 + n * 2048); dst[n][1] = *(const LAS f16x8*)(lds + SA(b, h) + boff1 + n * 2048); } } while (0)
#define MMA(ai, bj, At, Bt) MMA16(acc[ai][bj], At, Bt)
  f32x4 acc[2][2][4][2];
#pragma unroll
  for (int a = 0; a < 2; ++a)
#pragma unroll
    for (int b = 0; b < 2; ++b)
#pragma unroll
      for (int m = 0; m < 4; ++m)
#pragma unroll
        for (int n = 0; n < 2; ++n) acc[a][b][m][n] = (f32x4){0.f, 0.f, 0.f, 0.f};
  f16x8 At[4][2], B0[2][2], B1[2][2];
  int avail = 0;
  if (wid == 0) avail = wait_flags(ws, fo, 15, 0, 1, seq);
  BAR;
  u32x4 ff = __builtin_amdgcn_raw_buffer_load_b128(rf, fa, 16, AUX_SYS);
  SCHED;
  STAGE(SB(0, 0), rb, 0u, 0, 0); STAGE(SA(0, 0), ra, 0u, 0, 0); STAGE(SB(0, 1), rb, 0u, 0, 1); STAGE(SA(0, 1), ra, 0u, 0, 1);
  if (wr == 1) BAR;
  WAIT_V(4); BAR;
  STAGE(SB(1, 0), rb, 0u, 1, 0); STAGE(SA(1, 0), ra, 0u, 1, 0); STAGE(SB(1, 1), rb, 0u, 1, 1);
  WAIT_V(6); BAR;
  for (int t = 0; t < NPAIR; ++t) {
    const unsigned po = (unsigned)t * PAIR_B, pn = po + PAIR_B;
    if (wid == 0 && t + 2 > avail && t + 2 <= NPAIR) {
      avail = seen_flags(ff, 15, t + 1, seq << 9);
      if (t + 2 > avail) avail = wait_flags(ws, fo, 15, t + 1, t + 2, seq);
    }
    LDB(B0, 0, 0); SCHED; LDA(At, 0, 0); STAGE(SA(1, 1), ra, po, 1, 1);
    WAIT_V(10); WAIT_L(8); BAR; WAIT_L(0); MMA(0, 0, At, B0); BAR; SCHED;
    LDB(B1, 0, 1); STAGE(SB(0, 0), rb, pn, 0, 0);
    WAIT_V(10); BAR; WAIT_L(0); MMA(0, 1, At, B1); BAR;
    LDA(At, 0, 1);
    ff = __builtin_amdgcn_raw_buffer_load_b128(rf, fa, (unsigned)(t + 2) * 16u, AUX_SYS);
    STAGE(SA(0, 0), ra, pn, 0, 0);
    BAR; WAIT_L(0); MMA(1, 0, At, B0); BAR; SCHED;
    STAGE(SB(0, 1), rb, pn, 0, 1);
    WAIT_V(11); BAR; MMA(1, 1, At, B1); BAR;
    LDB(B0, 1, 0); SCHED; LDA(At, 1, 0); STAGE(SA(0, 1), ra, pn, 0, 1);
    WAIT_V(11); WAIT_L(8); BAR; WAIT_L(0); MMA(0, 0, At, B0); BAR; SCHED;
    LDB(B1, 1, 1); STAGE(SB(1, 0), rb, pn, 1, 0);
    WAIT_V(11); BAR; WAIT_L(0); MMA(0, 1, At, B1); BAR;
    LDA(At, 1, 1); STAGE(SA(1, 0), ra, pn, 1, 0);
    BAR; WAIT_L(0); MMA(1, 0, At, B0); BAR; SCHED;
    STAGE(SB(1, 1), rb, pn, 1, 1);
    WAIT_V(10); BAR; MMA(1, 1, At, B1); BAR;
  }
  asm volatile("" :: "v"(ff));
  const bool remote = wsl >= SLOT_WR;
  int tz = threadIdx.x; asm volatile("" : "+v"(tz));
  LAS unsigned* red = (LAS unsigned*)(lds + LDS_MAIN);
  unsigned tk = unit_ticket(ws, wsl - SLOT_WR, remote);
  const int eb = flag_exp(ws, seq, wsl, 0);
  const int e0 = flag_exp(ws, seq, xs, wr) + eb, e1 = flag_exp(ws, seq, xs, 2 + wr) + eb;
  const bool bad = call_bad(ws, seq);
  WAIT_V(0);
  WAIT_L(0);
  if (wr == 0) BAR;
  if (remote) {
    BAR;
    const int u0 = (xs < SLOT_G3 ? xs : xs - 1) * 4;
#pragma unroll
    for (int q = 0; q < 4; ++q) {
      pack_block(ws, seq, wsl - SLOT_WR, tk, q < 3, acc[q >> 1][q & 1], (q >> 1) ? e1 : e0, bad, lds + wid * 8192, red + 16 + q * 4, wid, tz & 63);
      if (q) { WAIT_V(11); post_unit(ws, seq, red + 12 + q * 4, u0 + q - 1); }
    }
    DRAIN();
    post_unit(ws, seq, red + 28, u0 + 3);
  } else {
    const __amdgpu_buffer_rsrc_t ro = __builtin_amdgcn_make_buffer_rsrc(P.out[P.rank], 0, 8192 * NLOC * 4, RSRC_W3);
#pragma unroll
    for (int q = 0; q < 4; ++q)
      store_block(ro, acc[q >> 1][q & 1], (q >> 1) ? e1 : e0, bad,
                  (unsigned)(xs == SLOT_T3 ? P.rank * 1024 + 768 + ((q >> 1) * 2 + wr) * 64 : group_row0(P.rank & 1, xs, (q >> 1) * 2 + wr)) * (NLOC * 4u)
                  + (unsigned)(wsl - SLOT_W) * 1024u + (q & 1) * 512u + wc * 128u, tz & 63);
  }
#undef SA
#undef SB
#undef STAGE
#undef LDA
#undef LDB
#undef MMA
}

#define AG_DW 2
constexpr int NBQ = 4;
constexpr unsigned LDS_F = NBQ * KSTEP_B;
#define WAIT_L0() __builtin_amdgcn_s_waitcnt(0xC07F)
#define CTZ(m) __builtin_ctzll(m)
__device__ __forceinline__ void gemm_quarter(const Params& P, unsigned char* ws, unsigned seq, int xs, int ah, int wsl, int bh, unsigned long long mask, int role, LAS unsigned char* lds) {
  const int tid = threadIdx.x, w8 = __builtin_amdgcn_readfirstlane(tid >> 6), lane = tid & 63;
  const unsigned par = seq & 1u;
  const int nk = 2 * __builtin_popcountll(mask);
  const unsigned job = (unsigned)(((xs - SLOT_V - 1) & 1) * 32 + (wsl - SLOT_W) * 4 + ah * 2 + bh);
  const __amdgpu_buffer_rsrc_t rq = __builtin_amdgcn_make_buffer_rsrc(ws + CTRL_B + OFF_PART + job * 65536u + (w8 & 3) * 16384u, 0, 16384, RSRC_W3);
  f32x4 acc[4][2];
  LAS unsigned char* xb = lds + 3 * KSTEP_B + (w8 & 3) * 8192 + lane * 16;
#define NEXTB(o) o = o == (NBQ - 1) * KSTEP_B ? 0u : o + KSTEP_B
  if (w8 >= 4) {
    const __amdgpu_buffer_rsrc_t ra = __builtin_amdgcn_make_buffer_rsrc(ws + CTRL_B + (size_t)par * PAR_B + (size_t)xs * SLOT_B + ah * 2 * PIECE_B, 0, SLOT_B - ah * 2 * PIECE_B, RSRC_W3);
    const __amdgpu_buffer_rsrc_t rb = __builtin_amdgcn_make_buffer_rsrc(ws + CTRL_B + (size_t)par * PAR_B + (size_t)wsl * SLOT_B + bh * 2 * PIECE_B, 0, SLOT_B - bh * 2 * PIECE_B, RSRC_W3);
    const unsigned fo = (unsigned)(lane < 32 ? xs : wsl) * 1024u + (unsigned)(lane & 31) * 16u;
    const int g = (w8 >> 1) & 1, sel = 1 << (2 * (lane < 32 ? ah : bh) + g);
    const volatile LAS unsigned* lf = (const volatile LAS unsigned*)(lds + LDS_F + lane * 16);
    const volatile LAS unsigned* lp = (const volatile LAS unsigned*)(lds + LDS_F + 2048 + job * 4);
    const unsigned voff = (unsigned)((w8 >> 1) & 1) * PIECE_B + (unsigned)(w8 & 1) * 4096u + (unsigned)lane * 16u, ldsw = (unsigned)(w8 & 3) * 4096u;
#define DMA4(lo, rs, vo, so) do { LAS void* _d = (LAS void*)(lds + (lo)); \
    __builtin_amdgcn_raw_ptr_buffer_load_lds(rs, _d, 16, vo, so, 0, AG_LAUX); __builtin_amdgcn_raw_ptr_buffer_load_lds(rs, _d, 16, vo, so, 1024, AG_LAUX); \
    __builtin_amdgcn_raw_ptr_buffer_load_lds(rs, _d, 16, vo, so, 2048, AG_LAUX); __builtin_amdgcn_raw_ptr_buffer_load_lds(rs, _d, 16, vo, so, 3072, AG_LAUX); } while (0)
#define FILL(top) for (; ki < (top) && CTZ(rem) < avail; ++ki) { \
      const unsigned _so = (unsigned)CTZ(rem) * PAIR_B + (unsigned)(ki & 1) * KK_B; \
      DMA4(bo + ldsw, ra, voff, _so); DMA4(bo + HALF_B + ldsw, rb, voff, _so); \
      if (ki & 1) rem &= rem - 1; \
      NEXTB(bo); }
#define PARTS() do { DMA4(bo + 2 * ldsw, rq, lane * 16u, 0u); DMA4(bo + 2 * ldsw + 4096u, rq, lane * 16u, 4096u); \
      _Pragma("unroll") for (int i = 0; i < 8; ++i) pt[i] = __builtin_amdgcn_raw_buffer_load_b128(rq, lane * 16u, 8192u + i * 1024u, AUX_SYS); \
      owed = false; } while (0)
    *(LAS u32x2*)(lds + LDS_F + (tid - 256) * 8) = (u32x2){(seq - 1u) << 9, (seq - 1u) << 9};
    *(LAS unsigned*)(lds + LDS_F + 2048 + (tid - 256) * 4) = seq - 1u;
    WAIT_L(0);
    BAR;
    int avail = 0, ki = 0;
    unsigned long long rem = mask;
    unsigned bo = 0;
    u32x4 pt[8];
#pragma unroll
    for (int i = 0; i < 8; ++i) pt[i] = (u32x4){0u, 0u, 0u, 0u};
    bool owed = role == 1;
    for (int k = 0; k < nk; ++k) {
      if (ki == k) {
        if (CTZ(rem) >= avail) avail = __builtin_amdgcn_readfirstlane(wait_flags(ws, fo, sel, CTZ(rem), CTZ(rem) + 1, seq));
        FILL(k + NBQ - 1 < nk ? k + NBQ - 1 : nk);
      }
      if (owed || role != 1) {
        if (ki - k == 3) WAIT_V(16); else if (ki - k == 2) WAIT_V(8); else WAIT_V(0);
      } else {
        if (ki - k == 2) WAIT_V(24); else WAIT_V(16);
      }
      BAR;
      if (ki < nk) {
        const unsigned long long miss = rem & __ballot((int)((lf[2 * ah + g] - (seq << 9)) | (lf[256 + 2 * bh + g] - (seq << 9))) < 0);
        const int av = miss ? CTZ(miss) : NPAIR;
        avail = av > avail ? av : avail;
        FILL(k + NBQ < nk ? k + NBQ : nk);
      }
      if (owed && ki == nk && k + 3 >= nk && RFL(*lp) == seq) PARTS();
    }
    if (owed) {
      const unsigned long long t0 = __builtin_amdgcn_s_memrealtime();
      while (RFL(ld_sys((const unsigned*)(ws + OFF_PFL) + job)) != seq && !timed_out((unsigned*)(ws + OFF_TMO), seq, t0)) __builtin_amdgcn_s_sleep(2);
      PARTS();
    }
    WAIT_V(0);
    BAR;
    BAR;
#pragma unroll
    for (int i = 0; i < 8; ++i) acc[i >> 1][i & 1] = *(const LAS f32x4*)(xb + i * 1024) + __builtin_bit_cast(f32x4, pt[i]);
  } else {
    const int fr = lane & 15, fq = lane >> 4;
    const int sw0 = (fq ^ (fr & 7)) * 16;
    const int aoff0 = ((w8 >> 1) * 64 + fr) * 128 + sw0, aoff1 = aoff0 ^ 64;
    const int boff0 = HALF_B + ((w8 & 1) * 64 + fr) * 128 + sw0, boff1 = boff0 ^ 64;
    f32x4 ac[4][4];
#pragma unroll
    for (int m = 0; m < 4; ++m)
#pragma unroll
      for (int n = 0; n < 4; ++n) ac[m][n] = (f32x4){0.f, 0.f, 0.f, 0.f};
    f16x8 A0[4][2], B0[4][2], A1[4][2], B1[4][2];
    const __amdgpu_buffer_rsrc_t rf = __builtin_amdgcn_make_buffer_rsrc(w8 == 2 ? ws + OFF_PFL : ws + OFF_FLAG + par * FLAG_B + (w8 ? wsl : xs) * 1024, 0, w8 == 3 ? 0 : 1024, RSRC_W3);
#define RD(A, B) do { _Pragma("unroll") for (int m = 0; m < 4; ++m) { \
      B[m][0] = *(const LAS f16x8*)(lds + bo + boff0 + m * 2048); A[m][0] = *(const LAS f16x8*)(lds + bo + aoff0 + m * 2048); } \
      _Pragma("unroll") for (int m = 0; m < 4; ++m) { \
      B[m][1] = *(const LAS f16x8*)(lds + bo + boff1 + m * 2048); A[m][1] = *(const LAS f16x8*)(lds + bo + aoff1 + m * 2048); } \
      NEXTB(bo); } while (0)
#define MM(A, B) do { _Pragma("unroll") for (int k = 0; k < 2; ++k) _Pragma("unroll") for (int m = 0; m < 4; ++m) _Pragma("unroll") for (int n = 0; n < 4; ++n) \
      ac[m][n] = __builtin_amdgcn_mfma_f32_16x16x32_f16(B[n][k], A[m][k], ac[m][n], 0, 0, 0); } while (0)
#define STEP(An, Bn, A, B) do { BAR; SCHED; __builtin_amdgcn_raw_ptr_buffer_load_lds(rf, (LAS void*)(lds + LDS_F + w8 * 1024), 16, lane * 16, 0, 0, AUX_SYS); RD(An, Bn); MM(A, B); \
      _Pragma("unroll") for (int i = 0; i < 16; ++i) { __builtin_amdgcn_sched_group_barrier(0x100, 1, 0); __builtin_amdgcn_sched_group_barrier(0x008, 1, 0); } \
      __builtin_amdgcn_sched_group_barrier(0x008, 16, 0); SCHED; WAIT_L0(); } while (0)
    unsigned bo = 0;
    BAR;
    BAR; SCHED;
    RD(A0, B0);
    WAIT_L0();
    for (int k = 1; k < nk - 1; k += 2) {
      STEP(A1, B1, A0, B0);
      STEP(A0, B0, A1, B1);
    }
    STEP(A1, B1, A0, B0);
    MM(A1, B1);
    if (role == 2) {
#pragma unroll
      for (int i = 0; i < 16; ++i) asm volatile("" : "+v"(ac[(i >> 1) & 3][(i >> 3) * 2 + (i & 1)]));
      SCHED;
#pragma unroll
      for (int i = 0; i < 16; ++i)
        __builtin_amdgcn_raw_buffer_store_b128(__builtin_bit_cast(u32x4, ac[(i >> 1) & 3][(i >> 3) * 2 + (i & 1)]), rq, lane * 16u, i * 1024u, AUX_SYS);
      STORE_GAP();
      DRAIN();
    }
    BAR;
#pragma unroll
    for (int i = 0; i < 8; ++i) {
      *(LAS f32x4*)(xb + i * 1024) = ac[i >> 1][2 + (i & 1)];
      acc[i >> 1][i & 1] = ac[i >> 1][i & 1];
      if (role == 1) acc[i >> 1][i & 1] += *(const LAS f32x4*)(lds + bo + w8 * 8192 + i * 1024 + lane * 16);
    }
    WAIT_V(0);
    WAIT_L(0);
    BAR;
#undef RD
#undef MM
#undef STEP
  }
  if (role == 2) {
    if (tid == 0) st_sys((unsigned*)(ws + OFF_PFL) + job, seq);
    return;
  }
  const int wr = (w8 >> 1) & 1, wc = (w8 & 1) * 2 + (w8 >> 2), wid = wr * 4 + wc;
  int tz = threadIdx.x; asm volatile("" : "+v"(tz));
  const int e = flag_exp(ws, seq, xs, ah * 2 + wr) + flag_exp(ws, seq, wsl, bh * 2);
  const bool bad = call_bad(ws, seq);
  if (wsl >= SLOT_WR) {
    unsigned mk;
    LAS unsigned char* cb = lds + w8 * 8192;
    const unsigned n64 = (gather_block(acc, e, bad, cb, mk) + 63u) & ~63u;
    const unsigned vo = (unsigned)(tz & 63) * 16u, vi = (unsigned)((((P.rank >> 1) * 3 + xs - SLOT_V) * 4 + ah * 2 + bh) * 8 + wid);
    unsigned char* pw = P.ws[2 * (wsl - SLOT_WR) + 1 - (P.rank & 1)];
    const __amdgpu_buffer_rsrc_t rv = __builtin_amdgcn_make_buffer_rsrc(pw + CTRL_B + OFF_VREC + vi * VREC_B, 0, 256u + n64, RSRC_W3);
    u32x4 v[8];
#pragma unroll
    for (int i = 0; i < 8; ++i) v[i] = *(const LAS u32x4*)(cb + i * 1024 + vo);
    __builtin_amdgcn_raw_buffer_store_b32(mk, rv, vo >> 2, 0, AUX_SYS);
#pragma unroll
    for (int i = 0; i < 8; ++i)
      if (i * 1024u < n64) {
        __builtin_amdgcn_raw_buffer_store_b128(v[i], rv, vo, 256u + i * 1024u, AUX_SYS);
        WAIT_V(AG_DW);
      }
    DRAIN();
    if ((tz & 63) == 0) st_sys((unsigned*)(pw + OFF_VFL) + vi, (seq << 9) + (n64 >> 6));
  } else {
    const __amdgpu_buffer_rsrc_t ro = __builtin_amdgcn_make_buffer_rsrc(P.out[P.rank], 0, 8192 * NLOC * 4, RSRC_W3);
    store_block(ro, acc, e, bad, (unsigned)group_row0(P.rank & 1, xs, ah * 2 + wr) * (NLOC * 4u) + (unsigned)(wsl - SLOT_W) * 1024u + bh * 512u + wc * 128u, tz & 63);
  }
#undef FILL
#undef PARTS
#undef DMA4
#undef NEXTB
}

#define AG_SWO 1
#define AG_H1 21
#define AG_H2 21
#define AG_HE 60
constexpr unsigned long long helped(int a) { return 0xaaaaaaaaaaaaaaaaull & ~((1ull << a) - 1ull) & ((1ull << AG_HE) - 1ull); }

__global__ __launch_bounds__(512, 2) void ag_gemm_kernel(Params P) {
  extern __shared__ __attribute__((aligned(16))) unsigned char shm[];
  LAS unsigned char* lds = (LAS unsigned char*)shm;
  const int tid = threadIdx.x;
  const int b = blockIdx.x, xcd = b & 7, bi = b >> 3;
  const int rank = P.rank, me = rank >> 1;
  unsigned char* ws = P.ws[rank];
  TS(0);
  if (bi < NPUSH) {
    const unsigned seq = link_role(P, ws, bi, xcd, lds);
    if (tid >= 64) return;
    TS(9);
    return;
  }
  const int cp = xcd & 3, rh = xcd >> 2;
  unsigned seq;
  int j;
  bool helper = false;
  j = bi - NPUSH - NFULL;
  if (bi < 32 - NPREP) {
    LAS unsigned* red = (LAS unsigned*)(lds + LDS_MAIN);
    if (tid == 0) red[8] = take_seq(ws, xcd);
    if (tid >= 16 && tid < 32) red[tid] = 0;
    WAIT_L(0);
    BAR;
    seq = RFL(red[8]);
  } else {
    seq = prep_role(P, ws, xcd, 31 - bi, lds);
    TS(2);
  }
  if (j < 0) {
    const int f = bi - NPUSH, ri = f >> 1;
    const int xs = ri < 2 ? SLOT_A + rh * 2 + ri : ri < 6 ? ((me + 1 + rh * 2 + (ri & 1)) & 3) * 2 + ((ri >> 2) ^ (rh & ri & AG_SWO)) : SLOT_V3;
    const bool far = f < 12 ? f & 1 : rh;
    const bool home = far && xs == SLOT_A + cp;
    gemm_full(P, ws, seq, home ? SLOT_T3 : xs, (far && !home ? SLOT_WR : SLOT_W) + cp, lds);
    if ((f >> 2) != 1) return;
    j = 4 + f;
    helper = true;
  }
  for (;;) {
    WAIT_L(0);
    BAR;
    const unsigned long long hm = j < 4 ? 0ull : j < 8 ? helped(AG_H1) : helped(AG_H2);
    gemm_quarter(P, ws, seq, SLOT_V + (j >> 2), rh, ((j & 2) ? SLOT_WR : SLOT_W) + cp, j & 1, helper ? hm : ~hm, helper ? 2 : j < 4 ? 0 : 1, lds);
    if (j >= 4) return;
    j += 4;
    helper = true;
  }
}

#define AG_FR0 2.2
#define AG_FR1 2.2
#define AG_E0 142.
#define AG_E1 156.
#define AG_S0 148.
#define AG_R0 1.3
#define AG_S1 190.
#define AG_R1 0.9
#define AG_S2 190.
#define AG_R2 0.9
#define AG_NA 0
#define AG_RA 0.72
#define AG_XW 8
#define AG_L0 2
#define AG_L1 2
#define AG_LV0 4
#define AG_LV1 16
#define AG_LV2 24
#define AG_LW 0
#define AG_LA 2
void merge_streams(unsigned short* out, int n, const int* cnt, const int* size, const double* t0, const double* dt, const int* first, const int* step, const int* lng) {
  int done[6] = {0, 0, 0, 0, 0, 0};
  for (;;) {
    int best = -1;
    for (int s = 0; s < n; ++s)
      if (done[s] < cnt[s] && (best < 0 || t0[s] + done[s] * dt[s] < t0[best] + done[best] * dt[best] - 1e-9)) best = s;
    if (best < 0) return;
    for (int k = 0; k < size[best]; ++k) *out++ = (unsigned short)(first[best] + done[best] * step[best] + k + (done[best] >= cnt[best] - lng[best] ? 0x800 : 0));
    ++done[best];
  }
}

}

extern "C" void kernel_launch(int rank, int nranks, ncclComm_t comm,
                              hipStream_t stream,
                              void* const* d_in, const int* in_sizes, int n_in,
                              void* d_out, int out_size,
                              const int* in_dtypes, int out_dtype,
                              void* const* d_out_all,
                              void* const* d_ws_all, size_t ws_bytes) {
  (void)comm; (void)in_dtypes; (void)out_dtype; (void)d_out;
  if (nranks != NR || (unsigned)rank >= (unsigned)NR || n_in < 2 || ws_bytes < WS_NEED ||
      in_sizes[0] != 1024 * KDIM || in_sizes[1] != KDIM * NLOC || out_size != 8192 * NLOC) {
    fprintf(stderr, "kernel_launch: unexpected problem shape\n");
    return;
  }
  HIP_CHECK(hipFuncSetAttribute((const void*)ag_gemm_kernel, hipFuncAttributeMaxDynamicSharedMemorySize, LDS_B));
  Params P;
  P.x = (const float*)d_in[0];
  P.w = (const float*)d_in[1];
  for (int p = 0; p < NR; ++p) { P.out[p] = (float*)d_out_all[p]; P.ws[p] = (unsigned char*)d_ws_all[p]; }
  P.rank = rank;
  const int me = rank >> 1;
  {
    const int cnt[6] = {NPAIR - 4, NPAIR - 4, NPAIR, NPAIR, AG_NA, NPAIR - AG_NA}, size[6] = {4, 4, 1, 1, 1, 1}, step[6] = {4, 4, 4, 4, 4, 4};
    const int first[6] = {0 << 8 | 16, 1 << 8 | 16, 2 << 8 | me, 3 << 8 | me, 4 << 8 | me, 4 << 8 | (AG_NA * 4 + me)};
    const double t2 = AG_S2 - (NPAIR - AG_NA) * AG_R2;
    const double t0[6] = {AG_E0 - 60 * AG_FR0, AG_E1 - 60 * AG_FR1, AG_S0 - 64 * AG_R0, AG_S1 - 64 * AG_R1, t2 - AG_NA * AG_RA, t2};
    const double dt[6] = {AG_FR0, AG_FR1, AG_R0, AG_R1, AG_RA, AG_R2};
    const int lng[6] = {AG_L0, AG_L1, AG_LV0, AG_LV1, 0, AG_LV2};
    merge_streams(P.ord + NDIR, 6, cnt, size, t0, dt, first, step, lng);
  }
  for (int p = 0; p < NDIR; ++p) P.ord[p] = 0;
  for (int p = 0; p < NDIR; ++p) P.ord[QSAME + p] = 0;
  {
    const int cnt[3] = {NPAIR - 4, NPAIR - 4, NPAIR}, size[3] = {4, 4, 1}, first[3] = {0 << 8 | 16, 1 << 8 | 16, 2 << 8 | me}, step[3] = {4, 4, 4};
    const double t0[3] = {4., 4., 0.}, dt[3] = {(NPAIR - 5. - AG_XW) / (NPAIR - 5.), 1., 1.};
    const int lng[3] = {AG_LW, AG_LA, AG_LA};
    merge_streams(P.ord + QSAME + NDIR, 3, cnt, size, t0, dt, first, step, lng);
  }
  hipLaunchKernelGGL(ag_gemm_kernel, dim3(NWG), dim3(512), LDS_B, stream, P);
  HIP_CHECK(hipGetLastError());
}
```

```cpp
#include <hip/hip_runtime.h>
#include <rccl/rccl.h>
#include <cstdio>

#define HIP_CHECK(expr) do { hipError_t _e = (expr); if (_e != hipSuccess) { \
    fprintf(stderr, "HIP error %s at %s:%d\n", hipGetErrorString(_e), __FILE__, __LINE__); \
    return; } } while (0)

#define LAS __attribute__((address_space(3)))
typedef _Float16 f16x8 __attribute__((ext_vector_type(8)));
typedef float f32x4 __attribute__((ext_vector_type(4)));
typedef unsigned u32x4 __attribute__((ext_vector_type(4)));
typedef unsigned u32x2 __attribute__((ext_vector_type(2)));

namespace {

constexpr int NR = 8;
constexpr int KDIM = 8192, NLOC = 1024;
constexpr int KSTEPS = KDIM / 64, NPAIR = KSTEPS / 2;
constexpr unsigned KK_B = 8192, PIECE_B = 16384, HALF_B = 16384, KSTEP_B = 32768, PAIR_B = 65536;
constexpr unsigned SLOT_B = NPAIR * PAIR_B;
constexpr int SLOT_V = 8, SLOT_G3 = 11, SLOT_A = 12, SLOT_V3 = 16, SLOT_T3 = 17, SLOT_W = 18, SLOT_WR = 22, NSLOT = 26;
constexpr unsigned PAR_B = NSLOT * SLOT_B;
constexpr size_t CTRL_B = 1u << 20;
constexpr unsigned UREC_B = 68 * 1024, UHDR_B = 2112;
constexpr unsigned OUTS_B = 4 * 64 * UREC_B;
constexpr unsigned SH_HI = 8192, SH_EX = 14336, SH_MAX = 256, SH_LONG = 511;
constexpr unsigned OFF_OUTS = 2 * PAR_B, OFF_RCV = OFF_OUTS + OUTS_B, OFF_SH = OFF_RCV + OUTS_B;
constexpr unsigned OFF_INFO = OFF_SH + 2048 * PIECE_B;
constexpr unsigned VREC_B = 256 + 8192, OFF_VREC = OFF_INFO + 2048 * 16 + 1024;
constexpr unsigned OFF_PART = OFF_VREC + 384 * VREC_B, DATA_B = OFF_PART + 96 * 65536;
constexpr size_t WS_NEED = CTRL_B + DATA_B;
constexpr unsigned OFF_TICKET = 0;
constexpr unsigned OFF_TMO = 1024;
constexpr unsigned OFF_QUEUE = 2048;
constexpr unsigned OFF_UHEAD = 3072;
constexpr unsigned OFF_ULIST = 4096;
constexpr unsigned OFF_PFL = 5120;
constexpr unsigned OFF_VFL = 12288;
constexpr unsigned OFF_DONE = 8192;
constexpr unsigned OFF_FLAG = 16384;
constexpr unsigned FLAG_B = (NSLOT + 1) * 1024;
constexpr unsigned OFF_SFLAG = OFF_FLAG + 2 * FLAG_B;
constexpr unsigned CTRL_USED = OFF_SFLAG + 2 * FLAG_B;
constexpr unsigned long long DEADLINE = 50000000ull;
#define AG_NP 8
constexpr int NWG = 256, NPUSH = 7, NFULL = 13, NPREP = AG_NP;
constexpr int NUNIT = 48;
constexpr int QSAME = 704, QIN = 576, QCROSS = QIN + NUNIT * 2;
constexpr int NDIR = 32;
constexpr int LDS_MAIN = 137 * 1024;
constexpr int LDS_B = 160 * 1024;
constexpr int AUX_SYS = 17;
#define AG_OAUX 17
constexpr unsigned RSRC_W3 = 0x00020000u;
constexpr unsigned OOB = 0x80000000u;
static_assert(DATA_B < OOB, "layout");

struct Params {
  const float* x;
  const float* w;
  float* out[NR];
  unsigned char* ws[NR];
  int rank;
  alignas(4) unsigned short ord[QSAME + QIN];
};

__device__ __forceinline__ unsigned ld_sys(const unsigned* p) { return __hip_atomic_load(p, __ATOMIC_RELAXED, __HIP_MEMORY_SCOPE_SYSTEM); }
__device__ __forceinline__ void st_sys(unsigned* p, unsigned v) { __hip_atomic_store(p, v, __ATOMIC_RELAXED, __HIP_MEMORY_SCOPE_SYSTEM); }
#define DRAIN() asm volatile("s_waitcnt vmcnt(0)" ::: "memory")
#define WAIT_V(n) asm volatile("s_waitcnt vmcnt(%0)" :: "n"(n) : "memory")
#define WAIT_L(n) asm volatile("s_waitcnt lgkmcnt(%0)" :: "n"(n) : "memory")
#define BAR __builtin_amdgcn_s_barrier()
#define SCHED __builtin_amdgcn_sched_barrier(0)
#define RFL(x) ((unsigned)__builtin_amdgcn_readfirstlane(x))
#define STORE_GAP() do { SCHED; asm volatile("s_nop 1"); SCHED; } while (0)
#define TSV(i, v)
#define TS(i)
#define TSW(i)
#define TR(ci, k, v)

__device__ __forceinline__ bool timed_out(unsigned* tmo, unsigned seq, unsigned long long t0) {
  if (RFL(__hip_atomic_load(tmo, __ATOMIC_RELAXED, __HIP_MEMORY_SCOPE_AGENT)) == seq ||
      __builtin_amdgcn_s_memrealtime() - t0 > DEADLINE) {
    if ((threadIdx.x & 63) == 0) __hip_atomic_store(tmo, seq, __ATOMIC_RELAXED, __HIP_MEMORY_SCOPE_AGENT);
    return true;
  }
  return false;
}

__device__ __noinline__ int wait_flags(const unsigned char* wsv, unsigned fo, int sel, int p0v, int needv, unsigned seqv) {
  const unsigned long long wa = (unsigned long long)wsv;
  const unsigned whi = RFL((unsigned)(wa >> 32)), wlo = RFL((unsigned)wa);
  const unsigned char* ws = (const unsigned char*)(((unsigned long long)whi << 32) | (unsigned long long)wlo);
  const int p0 = __builtin_amdgcn_readfirstlane(p0v), need = __builtin_amdgcn_readfirstlane(needv);
  const unsigned seq = RFL(seqv);
  const unsigned s9 = seq << 9;
  const __amdgpu_buffer_rsrc_t rf = __builtin_amdgcn_make_buffer_rsrc((void*)(ws + OFF_FLAG + (seq & 1u) * FLAG_B), 0, FLAG_B, RSRC_W3);
  const bool past = p0 + (int)(threadIdx.x & 31) >= NPAIR;
  const unsigned long long t0 = __builtin_amdgcn_s_memrealtime();
  for (;;) {
    asm volatile("" ::: "memory");
    const u32x4 f = __builtin_amdgcn_raw_buffer_load_b128(rf, fo, (unsigned)p0 * 16u, AUX_SYS);
    int old = 0;
#pragma unroll
    for (int j = 0; j < 4; ++j) old |= (int)((f[j] - s9) >> 31) << j;
    const unsigned long long m = __ballot(past || (old & sel) == 0);
    const unsigned miss = ~((unsigned)m & (unsigned)(m >> 32));
    int av = p0 + (miss ? __builtin_ctz(miss) : 32);
    av = av > NPAIR ? NPAIR : av;
    if (av >= need) return av;
    if (timed_out((unsigned*)(ws + OFF_TMO), seq, t0)) return NPAIR;
    __builtin_amdgcn_s_sleep(4);
  }
}

__device__ __forceinline__ unsigned wave_max(unsigned m) {
#pragma unroll
  for (int o = 32; o >= 1; o >>= 1) { const unsigned t = (unsigned)__shfl_xor((int)m, o); m = t > m ? t : m; }
  return m;
}

__device__ __forceinline__ unsigned take_seq(unsigned char* ws, int xcd) {
  return __hip_atomic_fetch_add((unsigned*)(ws + OFF_TICKET) + xcd * 32, 1u, __ATOMIC_RELAXED, __HIP_MEMORY_SCOPE_AGENT) / (NWG / 8) + 1u;
}

struct Src { __amdgpu_buffer_rsrc_t rs; unsigned lo, ks_b, st_a, st_b, so; };

__device__ __forceinline__ Src make_src(const Params& P, int tl8) {
  const int tid = threadIdx.x, wid = tid >> 6, lane = tid & 63, r8 = lane >> 3, c8 = lane & 7;
  const bool isw = tl8 >= 4;
  const int tl = tl8 & 3;
  Src q;
  q.rs = __builtin_amdgcn_make_buffer_rsrc((void*)(isw ? P.w + tl * 256 : P.x + (size_t)tl * 256 * KDIM), 0,
                                           isw ? (KDIM * NLOC - tl * 256) * 4 : 256 * KDIM * 4, RSRC_W3);
  q.lo = isw ? (unsigned)(wid * 8 * NLOC + lane * 4) * 4u : (unsigned)((wid * 32 + r8) * KDIM + (c8 ^ r8) * 8) * 4u;
  q.ks_b = isw ? 64u * NLOC * 4u : 64u * 4u;
  q.st_a = isw ? 2u * NLOC * 4u : 8u * KDIM * 4u;
  q.st_b = isw ? NLOC * 4u : 16u;
  q.so = isw ? (unsigned)((tid * 16 + (tid & 15)) * NLOC + ((tid * 5) & 63) * 4) * 4u
             : (unsigned)((tid >> 1) * KDIM + ((tid * 37 + 11) & 2047) * 4) * 4u;
  return q;
}
#define PLOAD(q, v, ks) do { const unsigned _k = (unsigned)(ks) * q.ks_b, _lo = (ks) < KSTEPS ? q.lo : OOB; \
    _Pragma("unroll") for (int i = 0; i < 8; ++i) \
      v[i] = __builtin_amdgcn_raw_buffer_load_b128(q.rs, _lo, _k + (i >> 1) * q.st_a + (i & 1) * q.st_b, 0); } while (0)

__device__ __forceinline__ int tile_exp(const u32x4 sv, LAS unsigned* red) {
  unsigned m = 0, l = 0;
#pragma unroll
  for (int j = 0; j < 4; ++j) { const unsigned a = sv[j] & 0x7fffffffu; m = a > m ? a : m; l += a >> 15; }
  m = wave_max(m);
#pragma unroll
  for (int o = 32; o >= 1; o >>= 1) l += (unsigned)__shfl_xor((int)l, o);
  if ((threadIdx.x & 63) == 0) { red[threadIdx.x >> 6] = m; red[32 + (threadIdx.x >> 6)] = l; }
  WAIT_L(0);
  BAR;
  l = 0;
#pragma unroll
  for (int q = 0; q < 8; ++q) { const unsigned a = red[q]; m = a > m ? a : m; l += red[32 + q]; }
  m = RFL(m);
  l = RFL(l);
  const int ea = (int)(m >> 23), eg = (int)(((l >> 11) + 588u + 255u) >> 8) + 3;
  const int eb = eg > ea ? eg : ea;
  return m == 0 ? 131 : eb < 5 ? 5 : eb > 254 ? 254 : eb;
}

__device__ __forceinline__ void convert(const u32x4 (&v)[8], u32x4 (&h)[4], float s, bool isw, LAS unsigned char* lb) {
  const int tid = threadIdx.x, wid = tid >> 6, lane = tid & 63, r8 = lane >> 3, c8 = lane & 7;
  if (isw) {
    const int tw = lane * 512 + ((wid ^ c8) * 16);
#pragma unroll
    for (int cc = 0; cc < 4; ++cc) {
      f16x8 t;
#pragma unroll
      for (int i = 0; i < 8; ++i) t[i] = (_Float16)(__uint_as_float(v[i][cc]) * s);
      *(LAS f16x8*)(lb + tw + cc * 128) = t;
    }
    WAIT_L(0);
    BAR;
#pragma unroll
    for (int it = 0; it < 4; ++it) h[it] = *(const LAS u32x4*)(lb + wid * 4096 + it * 1024 + (lane * 16 ^ ((r8 ^ (it * 2 + (r8 >> 2))) * 16)));
  } else {
#pragma unroll
    for (int it = 0; it < 4; ++it) {
      f16x8 t;
#pragma unroll
      for (int i = 0; i < 4; ++i) {
        t[i] = (_Float16)(__uint_as_float(v[2 * it][i]) * s);
        t[4 + i] = (_Float16)(__uint_as_float(v[2 * it + 1][i]) * s);
      }
      h[it] = __builtin_bit_cast(u32x4, t);
    }
  }
}

#define AG_PP 170
#define AG_PO 4
__device__ __forceinline__ unsigned prep_role(const Params& P, unsigned char* ws, int tl8, int pi, LAS unsigned char* lds) {
  const int tid = threadIdx.x, me = P.rank >> 1;
  LAS unsigned* red = (LAS unsigned*)(lds + LDS_MAIN);
  const bool isw = tl8 >= 4;
  const Src q = make_src(P, tl8);
  const u32x4 sv = __builtin_amdgcn_raw_buffer_load_b128(q.rs, q.so, 0, 0);
  u32x4 v0[8], v1[8];
  PLOAD(q, v0, 2 * pi);
  PLOAD(q, v1, 2 * pi + 1);
  if (tid == 0) red[8] = take_seq(ws, tl8);
  if (tid >= 40 && tid < 48) red[tid] = 0;
  const int eb = tile_exp(sv, red);
  const unsigned seq = RFL(red[8]);
  const float s = __uint_as_float((unsigned)(258 - eb) << 23);
  const unsigned p0 = isw ? (SLOT_W + tl8 - 4) * 256u : tl8 < 2 ? (me * 2 + tl8) * 256u : tl8 == 2 ? SLOT_V * 256u + me : SLOT_T3 * 256u;
  const unsigned gs = tl8 == 2 ? 256u : 1u;
  const unsigned par = seq & 1u;
  const __amdgpu_buffer_rsrc_t rd = __builtin_amdgcn_make_buffer_rsrc(ws + CTRL_B + (size_t)par * PAR_B, 0, PAR_B, RSRC_W3);
  unsigned* flag = (unsigned*)(ws + OFF_FLAG + par * FLAG_B) + p0 + (tid & 3) * gs;
  const unsigned fval = (seq << 9) + (unsigned)(256 + eb - 131);
  const __amdgpu_buffer_rsrc_t rh = __builtin_amdgcn_make_buffer_rsrc(ws + CTRL_B + OFF_SH, 0, DATA_B - OFF_SH, RSRC_W3);
  const int wid = __builtin_amdgcn_readfirstlane(tid >> 6), grp = wid >> 1;
  const unsigned vo = (unsigned)(wid & 1) * 4096u + (unsigned)(tid & 63) * 16u;
  const unsigned long long t0 = __builtin_amdgcn_s_memrealtime();
  int odd = 0;
  for (int pr = pi; pr < NPAIR; pr += NPREP, odd ^= 1) {
    u32x4 h0[4], h1[4], lo[4], hi[3];
    convert(v0, h0, s, isw, lds);
    convert(v1, h1, s, isw, lds + KSTEP_B);
    LAS unsigned* cn = red + 40 + odd * 4 + grp;
    LAS unsigned char* ex = lds + 2 * KSTEP_B + odd * 2048 + grp * 512;
    const unsigned dl = (p0 + pr * 4u + grp * gs) * PIECE_B, sp = (unsigned)tl8 * 256u + (unsigned)pr * 4u + grp;
    unsigned e[4], ne = 0;
#pragma unroll
    for (int it = 0; it < 4; ++it) {
      __builtin_amdgcn_raw_buffer_store_b128(h0[it], rd, vo, dl + it * 1024u, AUX_SYS);
      __builtin_amdgcn_raw_buffer_store_b128(h1[it], rd, vo, dl + it * 1024u + KK_B, AUX_SYS);
      unsigned H[4];
#pragma unroll
      for (int d = 0; d < 4; ++d) {
        const unsigned a = d < 2 ? h0[it][2 * d] : h1[it][2 * d - 4], b = d < 2 ? h0[it][2 * d + 1] : h1[it][2 * d - 3];
        lo[it][d] = __builtin_amdgcn_perm(b, a, 0x06040200u);
        H[d] = __builtin_amdgcn_perm(b, a, 0x07050301u);
      }
      const unsigned g = (H[3] & ~0x20202020u) | ((H[3] >> 2) & 0x20202020u);
#pragma unroll
      for (int d = 0; d < 3; ++d) hi[(it * 3 + d) >> 2][(it * 3 + d) & 3] = (H[d] & ~0x60606060u) | ((g << (5 - 2 * d)) & 0x60606060u);
      e[it] = (((H[0] & 0x60606060u) >> 5) | ((H[1] & 0x60606060u) >> 3) | ((H[2] & 0x60606060u) >> 1) | ((H[3] & 0x60606060u) << 1)) ^ 0x55555555u;
      ne += (unsigned)__builtin_popcount((e[it] | (e[it] >> 1)) & 0x55555555u);
    }
    if (ne) {
      unsigned at = __hip_atomic_fetch_add(cn, ne, __ATOMIC_RELAXED, __HIP_MEMORY_SCOPE_WORKGROUP);
#pragma unroll
      for (int it = 0; it < 4; ++it)
        for (unsigned x = e[it]; x; ++at) {
          const unsigned f = (unsigned)__builtin_ctz(x) >> 1, x2 = (x >> (2 * f)) & 3u;
          x &= ~(3u << (2 * f));
          if (at < SH_MAX) *(LAS unsigned short*)(ex + at * 2) = (unsigned short)(x2 << 13 | (f & 2u) << 11 | (vo + it * 1024u) >> 1 | (f & 1u) << 2 | f >> 2);
        }
    }
    WAIT_L(0);
    BAR;
    ne = RFL(*cn);
    const u32x4 ev = *(const LAS u32x4*)(ex + (tid & 31) * 16);
    if (tid < 4) red[40 + (odd ^ 1) * 4 + tid] = 0;
    if (ne <= SH_MAX) {
#pragma unroll
      for (int it = 0; it < 4; ++it) __builtin_amdgcn_raw_buffer_store_b128(lo[it], rh, vo, sp * PIECE_B + it * 1024u, AUX_SYS);
#pragma unroll
      for (int j = 0; j < 3; ++j) __builtin_amdgcn_raw_buffer_store_b128(hi[j], rh, vo - (unsigned)(wid & 1) * 1024u, sp * PIECE_B + SH_HI + j * 1024u, AUX_SYS);
      __builtin_amdgcn_raw_buffer_store_b128(ev, rh, (tid & 127) < 32 ? vo : OOB, sp * PIECE_B + SH_EX, AUX_SYS);
    } else {
      ne = SH_LONG;
#pragma unroll
      for (int it = 0; it < 4; ++it) {
        __builtin_amdgcn_raw_buffer_store_b128(h0[it], rh, vo, sp * PIECE_B + it * 1024u, AUX_SYS);
        __builtin_amdgcn_raw_buffer_store_b128(h1[it], rh, vo, sp * PIECE_B + it * 1024u + KK_B, AUX_SYS);
      }
    }
    STORE_GAP();
    __builtin_amdgcn_raw_buffer_store_b32(ne, rh, (tid & 127) == 0 ? 0u : OOB, OFF_INFO - OFF_SH + sp * 16u, AUX_SYS);
    DRAIN();
    BAR;
    if (tid < 4) st_sys(flag + pr * 4, fval);
    while (pr + NPREP < NPAIR && (int)(__builtin_amdgcn_s_memrealtime() - t0) < (pr + NPREP - 11 + AG_PO) * AG_PP) __builtin_amdgcn_s_sleep(32);
    PLOAD(q, v0, 2 * (pr + NPREP));
    PLOAD(q, v1, 2 * (pr + NPREP) + 1);
  }
  return seq;
}

#define AG_PL 5
#define AG_DS 8
#define AG_ST 0
#define AG_SW 0
constexpr int PL = AG_PL;
#define AG_PU 7
constexpr int PU = AG_PU;
#define AG_PN 999
#define AG_P2 6
constexpr int RING = 4;
constexpr unsigned RSLOT_B = 18 * 1024, LDS_TR = RING * RSLOT_B;
constexpr int NUW = 6;
constexpr int NXW = 4 * 8 * NUW;

__device__ __forceinline__ int group_row0(int g, int xs, int rg) {
  return xs < SLOT_V ? (2 * (xs >> 1) + g) * 1024 + (xs & 1) * 256 + rg * 64
       : xs < SLOT_G3 ? (2 * rg + g) * 1024 + 512 + (xs - SLOT_V) * 64
       : xs < SLOT_V3 ? (2 * (xs - SLOT_A) + 1 - g) * 1024 + 768 + rg * 64
                      : (2 * rg + 1 - g) * 1024 + 704;
}

__device__ __forceinline__ void unpack_block(const Params& P, unsigned char* ws, unsigned seq, int T, bool same, LAS unsigned char* lb) {
  const int lane = threadIdx.x & 63, rank = P.rank;
  const unsigned s9 = seq << 9, voff = (unsigned)lane * 16u;
  unsigned* tmo = (unsigned*)(ws + OFF_TMO);
  const __amdgpu_buffer_rsrc_t ro = __builtin_amdgcn_make_buffer_rsrc(P.out[rank], 0, 8192 * NLOC * 4, RSRC_W3);
  const __amdgpu_buffer_rsrc_t rs = __builtin_amdgcn_make_buffer_rsrc(ws + CTRL_B, 0, DATA_B, RSRC_W3);
  const __amdgpu_buffer_rsrc_t rc = __builtin_amdgcn_make_buffer_rsrc(ws, 0, CTRL_USED, RSRC_W3);
  const unsigned ci = (unsigned)((T >> 3) & 3) * 64u + (unsigned)(T >> 5);
  const unsigned vi = ((unsigned)((T >> 3) & 3) * 12u + (unsigned)(T >> 5)) * 8u + (unsigned)(T & 7);
  const int w = T & 7;
  unsigned u;
  bool bad = false;
  const unsigned long long t0 = __builtin_amdgcn_s_memrealtime();
  for (;;) {
    asm volatile("" ::: "memory");
    u32x2 f = __builtin_amdgcn_raw_buffer_load_b64(rc, 0, same ? OFF_VFL + vi * 4u : OFF_DONE + ci * 16u, AUX_SYS);
    if (same) f[1] = f[0];
    u = RFL(f[0]);
    if ((int)RFL((f[0] - s9) | (f[1] - s9)) >= 0) break;
    if (timed_out(tmo, seq, t0)) { bad = true; break; }
    if (same) __builtin_amdgcn_s_sleep(4); else __builtin_amdgcn_s_sleep(16);
  }
  bad = RFL(bad) != 0;
  u = RFL(u);
  if (!same && w == 0) TR(ci, 10, NOW());
  const unsigned rec = same ? OFF_VREC + vi * VREC_B : OFF_RCV + ci * UREC_B;
  const unsigned mk = __builtin_amdgcn_raw_buffer_load_b32(rs, lane * 4u, same ? rec : rec + w * 256u, AUX_SYS);
  unsigned off = 256u, nb = (u & 511u) * 64u;
  if (same) {
    u = SLOT_V * 4u + (unsigned)(T >> 5);
  } else {
    const u32x2 ob = __builtin_amdgcn_raw_buffer_load_b64(rs, 0, rec + 2048u + w * 8u, AUX_SYS);
    off = UHDR_B + (RFL(ob[0]) & 0xffffu);
    nb = RFL(ob[1]);
  }
#pragma unroll
  for (int i = 0; i < 2; ++i) {
    LAS void* d = (LAS void*)(lb + i * 4096);
    const unsigned so = i * 4096u < nb ? rec + off + i * 4096u : OOB, so2 = i * 4096u + 2048u < nb ? so : OOB;
    __builtin_amdgcn_raw_ptr_buffer_load_lds(rs, d, 16, voff, so, 0, AUX_SYS);
    __builtin_amdgcn_raw_ptr_buffer_load_lds(rs, d, 16, voff, so, 1024, AUX_SYS);
    __builtin_amdgcn_raw_ptr_buffer_load_lds(rs, d, 16, voff, so2, 2048, AUX_SYS);
    __builtin_amdgcn_raw_ptr_buffer_load_lds(rs, d, 16, voff, so2, 3072, AUX_SYS);
  }
  DRAIN();
  f32x4 v[8];
  unsigned run = (unsigned)(unsigned long long)lb;
#pragma unroll
  for (int i = 0; i < 32; ++i) {
    const bool k = (int)(mk << i) < 0;
    const unsigned long long bl = __builtin_amdgcn_ballot_w64(k);
    const unsigned pos = __builtin_amdgcn_mbcnt_hi((unsigned)(bl >> 32), __builtin_amdgcn_mbcnt_lo((unsigned)bl, 0u)) * 4u + run;
    const float t = *(const LAS float*)(unsigned long long)pos;
    v[i >> 2][i & 3] = k ? t : 0.f;
    run += (unsigned)__builtin_popcountll(bl) * 4u;
  }
  if (bad) {
#pragma unroll
    for (int i = 0; i < 8; ++i) v[i] = (f32x4){__builtin_nanf(""), __builtin_nanf(""), __builtin_nanf(""), __builtin_nanf("")};
  }
  const int ti = (int)(u >> 2) & 15;
  const unsigned vo = (unsigned)(lane & 15) * (NLOC * 4u) + (unsigned)(lane >> 4) * 16u;
  const unsigned so = (unsigned)group_row0(1 - (rank & 1), ti < SLOT_G3 ? ti : ti + 1, (int)(u & 2u) + (w >> 2)) * (NLOC * 4u)
                    + (ci >> 6) * 1024u + (u & 1u) * 512u + (unsigned)(w & 3) * 128u;
#pragma unroll
  for (int i = 0; i < 8; ++i) asm volatile("" : "+v"(v[i]));
  SCHED;
#pragma unroll
  for (int i = 0; i < 8; ++i)
    __builtin_amdgcn_raw_buffer_store_b128(__builtin_bit_cast(u32x4, v[i]), ro, vo + (i & 1) * 64u, so + (i >> 1) * 16u * (NLOC * 4u), AG_OAUX);
  STORE_GAP();
  if (!same && w == 7) TR(ci, 11, NOW());
}

__device__ __forceinline__ unsigned link_role(const Params& P, unsigned char* ws, int link, int wgi, LAS unsigned char* lds) {
  const int tid = threadIdx.x, wid = __builtin_amdgcn_readfirstlane(tid >> 6), lane = tid & 63;
  const int rank = P.rank, me = rank >> 1, peer = (rank + 1 + link) & (NR - 1);
  const bool same = ((peer ^ rank) & 1) == 0;
  const unsigned sA = same ? me * 2 : SLOT_W + (peer >> 1), dA = same ? me * 2 : SLOT_WR + me;
  const unsigned sB = same ? me * 2 + 1 : SLOT_T3, dB = same ? me * 2 + 1 : SLOT_A + me;
  const unsigned sC = same ? SLOT_V : SLOT_G3, dC = same ? SLOT_V : SLOT_V3;
  unsigned char* pw = P.ws[peer];
  LAS unsigned* red = (LAS unsigned*)(lds + LDS_MAIN);
  const __amdgpu_buffer_rsrc_t rp = __builtin_amdgcn_make_buffer_rsrc(pw, 0, CTRL_USED, RSRC_W3);
  const __amdgpu_buffer_rsrc_t rd = __builtin_amdgcn_make_buffer_rsrc(pw + CTRL_B, 0, DATA_B, RSRC_W3);
  const unsigned one = lane == 0 ? 0u : OOB;
  unsigned seq;
  {
    const int tl8 = same ? wgi & 1 : (wgi & 1) ? 3 : 4 + (peer >> 1), pr = wgi >> 1;
    const Src q = make_src(P, tl8);
    const u32x4 sv = __builtin_amdgcn_raw_buffer_load_b128(q.rs, q.so, 0, 0);
    u32x4 v0[8], v1[8], h[2][4];
    PLOAD(q, v0, 2 * pr);
    PLOAD(q, v1, 2 * pr + 1);
    if (tid == 0) { red[8] = take_seq(ws, wgi); red[9] = 0; red[10] = 0; red[11] = 0; }
    const int eb = tile_exp(sv, red);
    seq = RFL(red[8]);
    const float s = __uint_as_float((unsigned)(258 - eb) << 23);
    const unsigned pc = ((wgi & 1) ? dB : dA) * 256u + (unsigned)pr * 4u;
#pragma unroll
    for (int kk = 0; kk < 2; ++kk) {
      convert(kk ? v1 : v0, h[kk], s, tl8 >= 4, lds + LDS_TR + kk * KSTEP_B);
      if (!kk) {
        TS(3);
        const unsigned long long t0 = __builtin_amdgcn_s_memrealtime();
        while (__builtin_amdgcn_s_memrealtime() - t0 < (unsigned long long)(pr * AG_ST)) __builtin_amdgcn_s_sleep(2);
      }
#pragma unroll
      for (int it = 0; it < 4; ++it) {
        __builtin_amdgcn_raw_buffer_store_b128(h[kk][it], rd, (unsigned)(wid & 1) * 4096u + (unsigned)lane * 16u, (seq & 1u) * PAR_B + (pc + (wid >> 1)) * PIECE_B + kk * KK_B + it * 1024u, AUX_SYS);
        WAIT_V(AG_SW);
      }
    }
    DRAIN();
    unsigned n = 0;
    if (lane == 0) n = __hip_atomic_fetch_add(red + 9, 1u, __ATOMIC_RELAXED, __HIP_MEMORY_SCOPE_WORKGROUP);
    if (RFL(n) == 7)
      __builtin_amdgcn_raw_buffer_store_b32((seq << 9) + (unsigned)(256 + eb - 131), rp, lane < 4 ? lane * 4u : OOB,
                                            OFF_FLAG + (seq & 1u) * FLAG_B + pc * 4u, AUX_SYS);
  }
  const unsigned par = seq & 1u, s9 = seq << 9;
  unsigned* tmo = (unsigned*)(ws + OFF_TMO);
  volatile LAS unsigned* cnt = red;
  const __amdgpu_buffer_rsrc_t rs = __builtin_amdgcn_make_buffer_rsrc(ws + CTRL_B, 0, DATA_B, RSRC_W3);
  const __amdgpu_buffer_rsrc_t rc = __builtin_amdgcn_make_buffer_rsrc(ws, 0, CTRL_USED, RSRC_W3);
  const unsigned voff = (unsigned)lane * 16u;
  if (wid > 1) {
    LAS unsigned char* lb = lds + LDS_TR + (wid - 2) * 8192;
    {
      const unsigned long long t0 = __builtin_amdgcn_s_memrealtime();
      while (RFL(cnt[9]) < 8u && __builtin_amdgcn_s_memrealtime() - t0 < DEADLINE) __builtin_amdgcn_s_sleep(8);
    }
    const unsigned* ord = (const unsigned*)P.ord + (same ? 0 : QSAME / 2);
    const unsigned pm = (unsigned)peer >> 1;
    for (int p = NDIR + (wid - 2) * 8 + wgi; p < (same ? QSAME : QIN); p += 8 * NUW) {
      const unsigned o = ord[p >> 1] >> ((p & 1) * 16), sel = (o >> 8) & 7u;
      if (o & 0x800u) continue;
      const unsigned dn = sel >= 2 ? (dC + sel - 2) * 256u + (o & 252u) + pm
                                   : (same ? pm * 2 + sel : (sel ? SLOT_A : SLOT_WR) + pm) * 256u + (o & 255u);
      unsigned f;
      const unsigned long long t0 = __builtin_amdgcn_s_memrealtime();
      for (;;) {
        asm volatile("" ::: "memory");
        f = RFL(__builtin_amdgcn_raw_buffer_load_b32(rc, 0, OFF_SFLAG + par * FLAG_B + dn * 4u, AUX_SYS));
        if ((int)(f - (seq << 18)) >= 0) break;
        if (timed_out(tmo, seq, t0)) { f = SH_LONG << 9; break; }
        __builtin_amdgcn_s_sleep(AG_DS);
      }
      f = RFL(f);
      const unsigned ne = (f >> 9) & 511u, base = par * PAR_B + dn * PIECE_B;
      if (ne <= SH_MAX) {
        u32x4 lo[8], hw[6];
#pragma unroll
        for (int i = 0; i < 8; ++i) lo[i] = __builtin_amdgcn_raw_buffer_load_b128(rs, voff, base + i * 1024u, AUX_SYS);
#pragma unroll
        for (int i = 0; i < 6; ++i) hw[i] = __builtin_amdgcn_raw_buffer_load_b128(rs, voff, base + SH_HI + i * 1024u, AUX_SYS);
        const u32x2 ex = __builtin_amdgcn_raw_buffer_load_b64(rs, lane * 8u, base + SH_EX, AUX_SYS);
        DRAIN();
#pragma unroll
        for (int kk = 0; kk < 2; ++kk) {
#pragma unroll
          for (int i = 0; i < 8; ++i) {
            const unsigned h0 = hw[(i * 3) >> 2][(i * 3) & 3], h1 = hw[(i * 3 + 1) >> 2][(i * 3 + 1) & 3], h2 = hw[(i * 3 + 2) >> 2][(i * 3 + 2) & 3];
            unsigned a, b;
            if (kk == 0) {
              a = (h0 & 0x9f9f9f9fu) | 0x20202020u;
              b = (h1 & 0x9f9f9f9fu) | 0x20202020u;
            } else {
              a = (h2 & 0x9f9f9f9fu) | 0x20202020u;
              b = ((h0 >> 5) & 0x03030303u) | ((h1 >> 3) & 0x0c0c0c0cu) | ((h2 >> 1) & 0x10101010u) | ((h2 << 1) & 0x80808080u) | 0x20202020u;
            }
            const u32x4 v = {__builtin_amdgcn_perm(a, lo[i][2 * kk], 0x05010400u), __builtin_amdgcn_perm(a, lo[i][2 * kk], 0x07030602u),
                             __builtin_amdgcn_perm(b, lo[i][2 * kk + 1], 0x05010400u), __builtin_amdgcn_perm(b, lo[i][2 * kk + 1], 0x07030602u)};
            *(LAS u32x4*)(lb + i * 1024 + voff) = v;
          }
          asm volatile("" ::: "memory");
#pragma unroll
          for (int j = 0; j < 4; ++j) {
            const unsigned x = ex[j >> 1] >> ((j & 1) * 16);
            const bool on = (unsigned)lane * 4u + j < ne && ((x >> 12) & 1u) == (unsigned)kk;
            __hip_atomic_fetch_xor((LAS unsigned*)(lb + ((x & 0xffeu) << 1)), on ? (x & 0x6000u) << ((x & 1u) * 16u) : 0u, __ATOMIC_RELAXED, __HIP_MEMORY_SCOPE_WORKGROUP);
          }
          asm volatile("" ::: "memory");
          u32x4 v[8];
#pragma unroll
          for (int i = 0; i < 8; ++i) v[i] = *(const LAS u32x4*)(lb + i * 1024 + voff);
          WAIT_L(0);
          SCHED;
#pragma unroll
          for (int i = 0; i < 8; ++i) __builtin_amdgcn_raw_buffer_store_b128(v[i], rs, voff, base + kk * KK_B + i * 1024u, AUX_SYS);
          STORE_GAP();
        }
        DRAIN();
      }
      __builtin_amdgcn_raw_buffer_store_b32(s9 + (f & 511u), rc, one, OFF_FLAG + par * FLAG_B + dn * 4u, AUX_SYS);
    }
    TSW(20 + wid);
    if (!same)
      for (int T = ((link >> 1) * 8 + wgi) * NUW + wid - 2; T < NUNIT * 32; T += NXW) unpack_block(P, ws, seq, T, false, lb);
    return seq;
  }
  if (wid == 1) {
    const unsigned fb = OFF_FLAG + par * FLAG_B;
    const int Q = same ? QSAME : QCROSS, QI = same ? QSAME : QIN;
    const unsigned* ord = (const unsigned*)P.ord + (same ? 0 : QSAME / 2);
    const unsigned qbase = (seq - 1u) * (unsigned)(Q - NDIR) - (NDIR + 16u);
    unsigned fA = 0, fB = 0, fC = 0;
    int rdy = 0;
    int p = NDIR + wgi, pn = NDIR + 8 + wgi;
    bool late = false;
    unsigned e = 0;
    unsigned k1 = 0;
    int held = 0;
    for (int n = 0;; ++n) {
      const bool live = (unsigned)p < (unsigned)Q;
      const unsigned tk = (unsigned)__builtin_amdgcn_raw_ptr_buffer_atomic_add_i32(1, rc, live && k1 == 0u ? one : OOB, OFF_QUEUE + link * 128, 16);
      const unsigned en = __builtin_amdgcn_raw_buffer_load_b32(rc, 0, (unsigned)(pn - QI) < (unsigned)(Q - QI) ? OFF_ULIST + (unsigned)(peer >> 1) * 256u + (unsigned)(pn - QIN) / 2u * 4u : OOB, AUX_SYS);
      u32x4 rec = {0u, 255u, 0u, 0u};
      unsigned so = OOB, kb = 16, so4 = OOB, so5 = OOB;
      bool part = false;
      if (live && p < QI) {
        const unsigned o = ord[p >> 1] >> ((p & 1) * 16), sel = (o >> 8) & 7u, pc = o & 255u;
        if (rdy <= (int)(pc >> 2)) {
          DRAIN();
          cnt[10] = n;
          const unsigned long long t0 = __builtin_amdgcn_s_memrealtime();
          for (;;) {
            asm volatile("" ::: "memory");
            const unsigned a = __builtin_amdgcn_raw_buffer_load_b32(rc, voff, fb + sA * 1024u, AUX_SYS);
            const unsigned b = __builtin_amdgcn_raw_buffer_load_b32(rc, voff, fb + sB * 1024u, AUX_SYS);
            const unsigned c = __builtin_amdgcn_raw_buffer_load_b32(rc, voff, fb + sC * 1024u + me * 4u, AUX_SYS);
            const unsigned long long miss = __ballot((int)((a - s9) | (b - s9) | (c - s9)) < 0);
            rdy = miss ? __builtin_ctzll(miss) : NPAIR;
            fA = RFL(a); fB = RFL(b); fC = RFL(c);
            if (rdy > (int)(pc >> 2)) break;
            if (timed_out(tmo, seq, t0)) { rdy = NPAIR; fA = fB = fC = s9; break; }
            __builtin_amdgcn_s_sleep(4);
          }
          rdy = __builtin_amdgcn_readfirstlane(rdy);
        }
        const unsigned dn = (sel >= 2 ? dC + sel - 2 : sel ? dB : dA) * 256u + pc;
        const unsigned fv = sel >= 2 ? fC : sel ? fB : fA;
        if (o & 0x800u) {
          so = par * PAR_B + ((sel >= 2 ? sC + sel - 2 : sel ? sB : sA) * 256u + pc) * PIECE_B;
          rec = (u32x4){par * PAR_B + dn * PIECE_B, 16u, fb + dn * 4u, fv};
        } else {
          const unsigned sp = sel >= 2 ? 512u + (pc & 252u) + (same ? sel - 2 : 3u) : (same ? sel : sel ? 3u : 4u + (peer >> 1)) * 256u + pc;
          so = OFF_SH + sp * PIECE_B;
          so4 = OFF_INFO + sp * 16u;
          rec = (u32x4){par * PAR_B + dn * PIECE_B, 64u, fb + (OFF_SFLAG - OFF_FLAG) + dn * 4u, (seq << 18) + (fv & 511u)};
        }
      } else if (live) {
        const unsigned i = (unsigned)(p - QIN) >> 1, j = (unsigned)p & 1u;
        const unsigned eo = OFF_ULIST + (unsigned)(peer >> 1) * 256u + i * 4u;
        if ((e >> 16) != (seq & 0xffffu)) {
          DRAIN();
          cnt[10] = n;
          const unsigned long long t0 = __builtin_amdgcn_s_memrealtime();
          for (;;) {
            asm volatile("" ::: "memory");
            e = RFL(__builtin_amdgcn_raw_buffer_load_b32(rc, 0, eo, AUX_SYS));
            if ((e >> 16) == (seq & 0xffffu) || timed_out(tmo, seq, t0)) break;
            __builtin_amdgcn_s_sleep(4);
          }
          e = RFL(e);
        }
        unsigned nk = (e >> 8) & 127u;
        nk = nk > 68u ? 68u : nk;
        const unsigned qk = (nk + 1u) >> 1, k0 = j * qk + k1;
        kb = (j ? nk : qk) - k0;
        part = kb > 18u && k1 == 0u;
        kb = kb > 18u ? 18u : kb;
        so = OFF_OUTS + ((unsigned)(peer >> 1) * 64u + i) * UREC_B + k0 * 1024u;
        so4 = kb > 16u ? so + 16384u : OOB;
        so5 = kb > 17u ? so4 : OOB;
        rec = (u32x4){OFF_RCV + ((unsigned)me * 64u + i) * UREC_B + k0 * 1024u, kb,
                      part ? fb + NSLOT * 1024u : OFF_DONE + (((unsigned)me * 64u + i) * 4u + j) * 4u, s9 + (e & 63u)};
        TR((unsigned)(peer >> 1) * 64u + i, 1 + j, NOW());
      }
      if (!late && n - (int)RFL(cnt[11]) >= RING) {
        const unsigned long long t0 = __builtin_amdgcn_s_memrealtime();
        while (n - (int)RFL(cnt[11]) >= RING) {
          if (__builtin_amdgcn_s_memrealtime() - t0 > DEADLINE) { late = timed_out(tmo, seq, t0); break; }
          __builtin_amdgcn_s_sleep(2);
        }
      }
      so = RFL(so);
      so4 = RFL(so4);
      so5 = RFL(so5);
      kb = RFL(kb);
      const int sl = n & (RING - 1);
      if (lane == 0) *(LAS u32x4*)(red + 16 + sl * 4) = rec;
#pragma unroll
      for (int i = 0; i < 5; ++i) {
        LAS void* d = (LAS void*)(lds + sl * RSLOT_B + i * 4096);
        const unsigned sg = i == 4 ? so4 : i * 4u < kb ? so + i * 4096u : OOB;
        __builtin_amdgcn_raw_ptr_buffer_load_lds(rs, d, 16, voff, sg, 0, AUX_SYS);
        if (i < 4) {
          __builtin_amdgcn_raw_ptr_buffer_load_lds(rs, d, 16, voff, sg, 1024, AUX_SYS);
          __builtin_amdgcn_raw_ptr_buffer_load_lds(rs, d, 16, voff, sg, 2048, AUX_SYS);
          __builtin_amdgcn_raw_ptr_buffer_load_lds(rs, d, 16, voff, sg, 3072, AUX_SYS);
        } else {
          __builtin_amdgcn_raw_ptr_buffer_load_lds(rs, d, 16, voff, so5, 1024, AUX_SYS);
        }
      }
      WAIT_V(20);
      WAIT_L(0);
      cnt[10] = live ? n : n + 1;
      if (!live) break;
      const int t = (int)(RFL(tk) - qbase);
      const unsigned er = RFL(en);
      if (RFL(part) != 0u) {
        held = t;
        k1 = 18;
      } else {
        p = pn;
        e = er;
        pn = k1 ? held : t;
        k1 = 0;
      }
    }
    return seq;
  }
  const unsigned spare = OFF_FLAG + par * FLAG_B + NSLOT * 1024u;
  unsigned fo = spare, fv = 0;
#define PFLAG() __builtin_amdgcn_raw_buffer_store_b32(fv, rp, one, fo, AUX_SYS)
  for (int n = 0;; ++n) {
    if ((int)RFL(cnt[10]) <= n) {
      DRAIN();
      PFLAG();
      fo = spare;
      const unsigned long long t0 = __builtin_amdgcn_s_memrealtime();
      bool dead = false;
      while ((int)RFL(cnt[10]) <= n) {
        if (__builtin_amdgcn_s_memrealtime() - t0 > 2 * DEADLINE) { dead = true; break; }
        __builtin_amdgcn_s_sleep(1);
      }
      if (dead) break;
    }
    const int sl = n & (RING - 1);
    const u32x4 rec = *(const LAS u32x4*)(red + 16 + sl * 4);
    u32x4 buf[18];
#pragma unroll
    for (int i = 0; i < 18; ++i) buf[i] = *(const LAS u32x4*)(lds + sl * RSLOT_B + i * 1024 + lane * 16);
    WAIT_L(0);
    cnt[11] = n + 1;
    const unsigned dst = RFL(rec[0]);
    unsigned kb = RFL(rec[1]), ne = 0, v14 = OOB;
    if (kb == 255u) break;
    if (kb == 64u) {
      ne = RFL(buf[16][0]) & 511u;
      kb = ne > SH_MAX ? 16u : 14u;
      if (ne <= SH_MAX && (unsigned)(lane >> 2) * 32u < ne) v14 = voff;
    }
#define PIECE(W) do { _Pragma("unroll") for (int i = 0; i < 18; ++i) { \
      if (i < (int)kb) { \
        __builtin_amdgcn_raw_buffer_store_b128(buf[i], rd, voff, dst + i * 1024u, AUX_SYS); \
        WAIT_V(W + (i >= W && i < 2 * W ? 2 : 0)); \
        if (i == W - 1) { \
          PFLAG(); \
          __builtin_amdgcn_raw_buffer_store_b128(buf[14], rd, v14, dst + SH_EX, AUX_SYS); \
        } \
      } \
    } \
    if (kb < W) { DRAIN(); PFLAG(); } } while (0)
    if (dst >= OFF_OUTS) PIECE(PU); else if (n >= AG_PN) PIECE(AG_P2); else PIECE(PL);
#undef PIECE
    fo = RFL(rec[2]);
    fv = RFL(rec[3]) + (ne << 9);
  }
  DRAIN();
  PFLAG();
#undef PFLAG
  return seq;
}

__device__ __forceinline__ int flag_exp(unsigned char* ws, unsigned seq, int slot, int rg) {
  return (int)(ld_sys((const unsigned*)(ws + OFF_FLAG + (seq & 1u) * FLAG_B) + slot * 256 + rg) & 511u) - 256;
}
__device__ __forceinline__ bool call_bad(unsigned char* ws, unsigned seq) {
  return RFL(__hip_atomic_load((unsigned*)(ws + OFF_TMO), __ATOMIC_RELAXED, __HIP_MEMORY_SCOPE_AGENT)) == seq;
}

__device__ __forceinline__ void store_block(const __amdgpu_buffer_rsrc_t ro, f32x4 (&acc)[4][2], int e, bool bad, unsigned so, int lane) {
  const unsigned vo = (unsigned)(lane & 15) * (NLOC * 4u) + (unsigned)(lane >> 4) * 16u;
#pragma unroll
  for (int m = 0; m < 4; ++m)
#pragma unroll
    for (int n = 0; n < 2; ++n) {
#pragma unroll
      for (int j = 0; j < 4; ++j) {
        const float t = __builtin_amdgcn_ldexpf(acc[m][n][j], e);
        acc[m][n][j] = bad ? __builtin_nanf("") : (t < 0.f ? 0.f : t);
      }
      asm volatile("" : "+v"(acc[m][n]));
    }
  SCHED;
#pragma unroll
  for (int m = 0; m < 4; ++m)
#pragma unroll
    for (int n = 0; n < 2; ++n)
      __builtin_amdgcn_raw_buffer_store_b128(__builtin_bit_cast(u32x4, acc[m][n]), ro, vo + n * 64, so + m * 16 * (NLOC * 4u), AG_OAUX);
  STORE_GAP();
}

__device__ __forceinline__ unsigned unit_ticket(unsigned char* ws, int c, bool take) {
  const __amdgpu_buffer_rsrc_t rc = __builtin_amdgcn_make_buffer_rsrc(ws + OFF_UHEAD, 0, 512, RSRC_W3);
  return (unsigned)__builtin_amdgcn_raw_ptr_buffer_atomic_add_i32(1, rc, take && threadIdx.x == 0 ? 0u : OOB, c * 128, 16);
}
__device__ __forceinline__ unsigned lds_add(LAS unsigned* p, unsigned v) {
  unsigned r = 0;
  if ((threadIdx.x & 63) == 0) r = __hip_atomic_fetch_add(p, v, __ATOMIC_RELAXED, __HIP_MEMORY_SCOPE_WORKGROUP);
  return RFL(r);
}
__device__ __forceinline__ unsigned gather_block(f32x4 (&acc)[4][2], int e, bool bad, LAS unsigned char* cb, unsigned& mk) {
  if (bad) {
    asm volatile("");
#pragma unroll
    for (int i = 0; i < 8; ++i) acc[i >> 1][i & 1] = (f32x4){__builtin_nanf(""), __builtin_nanf(""), __builtin_nanf(""), __builtin_nanf("")};
  }
  const unsigned b0 = RFL((unsigned)(unsigned long long)cb);
  unsigned run = b0;
  mk = 0;
#pragma unroll
  for (int i = 0; i < 32; i += 2) {
    float t0, t1;
    unsigned p0, p1, c0;
    asm volatile(
        "v_ldexp_f32 %0, %7, %9\n\t"
        "v_ldexp_f32 %1, %8, %9\n\t"
        "v_cmp_nge_f32 vcc, 0, %0\n\t"
        "v_cmp_nge_f32 s[92:93], 0, %1\n\t"
        "s_bcnt1_i32_b64 %4, vcc\n\t"
        "v_mbcnt_lo_u32_b32 %2, vcc_lo, 0\n\t"
        "v_mbcnt_lo_u32_b32 %3, s92, 0\n\t"
        "v_mbcnt_hi_u32_b32 %2, vcc_hi, %2\n\t"
        "v_mbcnt_hi_u32_b32 %3, s93, %3\n\t"
        "v_addc_co_u32 %5, s[94:95], %5, %5, vcc\n\t"
        "v_lshl_add_u32 %2, %2, 2, %6\n\t"
        "s_lshl2_add_u32 %6, %4, %6\n\t"
        "v_addc_co_u32 %5, s[94:95], %5, %5, s[92:93]\n\t"
        "v_lshl_add_u32 %3, %3, 2, %6\n\t"
        "s_bcnt1_i32_b64 %4, s[92:93]\n\t"
        "s_mov_b64 exec, vcc\n\t"
        "ds_write_b32 %2, %0\n\t"
        "s_mov_b64 exec, s[92:93]\n\t"
        "ds_write_b32 %3, %1\n\t"
        "s_mov_b64 exec, -1\n\t"
        "s_lshl2_add_u32 %6, %4, %6"
        : "=&v"(t0), "=&v"(t1), "=&v"(p0), "=&v"(p1), "=&s"(c0), "+v"(mk), "+s"(run)
        : "v"(acc[i >> 3][(i >> 2) & 1][i & 3]), "v"(acc[i >> 3][(i >> 2) & 1][(i & 3) + 1]), "v"(e)
        : "vcc", "scc", "s92", "s93", "s94", "s95", "memory");
  }
  WAIT_L(0);
  return run - b0;
}
__device__ __forceinline__ void pack_block(unsigned char* ws, unsigned seq, int c, unsigned& tk, bool more, f32x4 (&acc)[4][2], int e, bool bad, LAS unsigned char* cb, LAS unsigned* u, int wid, int lane) {
  unsigned mk;
  const unsigned nb = gather_block(acc, e, bad, cb, mk), n64 = (nb + 63u) & ~63u;
  const unsigned off = lds_add(u + 1, n64);
  const unsigned vo = (unsigned)lane * 16u;
  u32x4 v[8];
#pragma unroll
  for (int i = 0; i < 8; ++i) v[i] = *(const LAS u32x4*)(cb + i * 1024 + vo);
  unsigned pos = tk - (seq - 1u) * NUNIT;
  if (threadIdx.x == 0) *(volatile LAS unsigned*)u = (unsigned)c * 64u + (pos < 63u ? pos : 63u) + 1u;
  while ((pos = RFL(*(volatile LAS unsigned*)u)) == 0u) __builtin_amdgcn_s_sleep(1);
  unsigned char* rec = ws + CTRL_B + OFF_OUTS + (size_t)(pos - 1u) * UREC_B;
  const __amdgpu_buffer_rsrc_t rh = __builtin_amdgcn_make_buffer_rsrc(rec, 0, UHDR_B, RSRC_W3);
  const __amdgpu_buffer_rsrc_t rv = __builtin_amdgcn_make_buffer_rsrc(rec + UHDR_B + off, 0, n64, RSRC_W3);
  tk = unit_ticket(ws, c, more);
  __builtin_amdgcn_raw_buffer_store_b32(mk, rh, (unsigned)lane * 4u, (unsigned)wid * 256u, AUX_SYS);
  __builtin_amdgcn_raw_buffer_store_b64((u32x2){off, nb}, rh, lane == 0 ? 0u : OOB, 2048u + (unsigned)wid * 8u, AUX_SYS);
#pragma unroll
  for (int i = 0; i < 8; ++i) __builtin_amdgcn_raw_buffer_store_b128(v[i], rv, vo + (i & 3) * 1024u, (i >> 2) * 4096u, AUX_SYS);
  STORE_GAP();
}
__device__ __forceinline__ void post_unit(unsigned char* ws, unsigned seq, LAS unsigned* u, int unit) {
  if (lds_add(u + 2, 1u) == 7u) {
    const unsigned pos = *(volatile LAS unsigned*)u - 1u, all = UHDR_B + *(volatile LAS unsigned*)(u + 1);
    if ((threadIdx.x & 63) == 0) st_sys((unsigned*)(ws + OFF_ULIST) + pos, (seq << 16) | ((all + 1023u) >> 10 << 8) | (unsigned)unit);
    TR(pos, 0, NOW()); TR(pos, 9, (unsigned long long)unit);
  }
}

#define AG_LAUX AUX_SYS
#define DMA2(ldsoff, rs, so) do { \
    __builtin_amdgcn_raw_ptr_buffer_load_lds(rs, (LAS void*)(lds + (ldsoff) + ldsw), 16, voff, so, 0, AG_LAUX); \
    __builtin_amdgcn_raw_ptr_buffer_load_lds(rs, (LAS void*)(lds + (ldsoff) + ldsw), 16, voff, so, 1024, AG_LAUX); } while (0)
#define MMA16(ac, At, Bt) do { __builtin_amdgcn_s_setprio(1); \
    _Pragma("unroll") for (int m = 0; m < 4; ++m) _Pragma("unroll") for (int n = 0; n < 2; ++n) _Pragma("unroll") for (int k = 0; k < 2; ++k) \
      ac[m][n] = __builtin_amdgcn_mfma_f32_16x16x32_f16(Bt[n][k], At[m][k], ac[m][n], 0, 0, 0); \
    __builtin_amdgcn_s_setprio(0); } while (0)

__device__ __forceinline__ int seen_flags(const u32x4 f, int sel, int p0, unsigned s9) {
  int old = 0;
#pragma unroll
  for (int j = 0; j < 4; ++j) old |= (int)((f[j] - s9) >> 31) << j;
  const unsigned long long m = __ballot(p0 + (int)(threadIdx.x & 31) >= NPAIR || (old & sel) == 0);
  const unsigned miss = ~((unsigned)m & (unsigned)(m >> 32));
  return p0 + (miss ? __builtin_ctz(miss) : 32);
}

__device__ __forceinline__ void gemm_full(const Params& P, unsigned char* ws, unsigned seq, int xs, int wsl, LAS unsigned char* lds) {
  const int tid = threadIdx.x, wid = __builtin_amdgcn_readfirstlane(tid >> 6), lane = tid & 63;
  const int wr = wid >> 2, wc = wid & 3, fr = lane & 15, fq = lane >> 4;
  const unsigned par = seq & 1u;
  const __amdgpu_buffer_rsrc_t ra = __builtin_amdgcn_make_buffer_rsrc(ws + CTRL_B + (size_t)par * PAR_B + (size_t)xs * SLOT_B, 0, SLOT_B, RSRC_W3);
  const __amdgpu_buffer_rsrc_t rb = __builtin_amdgcn_make_buffer_rsrc(ws + CTRL_B + (size_t)par * PAR_B + (size_t)wsl * SLOT_B, 0, SLOT_B, RSRC_W3);
  const __amdgpu_buffer_rsrc_t rf = __builtin_amdgcn_make_buffer_rsrc(ws + OFF_FLAG + RFL(par * FLAG_B), 0, FLAG_B, RSRC_W3);
  const unsigned fo = (unsigned)(lane < 32 ? xs : wsl) * 1024u + (unsigned)(lane & 31) * 16u, fa = wid == 0 ? fo : OOB;
  const unsigned voff = (unsigned)wr * PIECE_B + (unsigned)wc * 2048u + (unsigned)lane * 16u;
  const unsigned ldsw = (unsigned)wid * 2048u;
  const int sw0 = (fq ^ (fr & 7)) * 16;
  const int aoff0 = (wr * 64 + fr) * 128 + sw0, aoff1 = aoff0 ^ 64;
  const int boff0 = 4 * HALF_B + (wc * 32 + fr) * 128 + sw0, boff1 = boff0 ^ 64;
#define SA(b, h) (((b) * 2 + (h)) * HALF_B)
#define SB(b, h) ((4 + (b) * 2 + (h)) * HALF_B)
#define STAGE(bufoff, rs, po, kk, h) DMA2(bufoff, rs, (po) + (kk) * KK_B + (h) * 2 * PIECE_B)
#define LDA(dst, b, h) do { _Pragma("unroll") for (int m = 0; m < 4; ++m) { \
    dst[m][0] = *(const LAS f16x8*)(lds + SA(b, h) + aoff0 + m * 2048); dst[m][1] = *(const LAS f16x8*)(lds + SA(b, h) + aoff1 + m * 2048); } } while (0)
#define LDB(dst, b, h) do { _Pragma("unroll") for (int n = 0; n < 2; ++n) { \
    dst[n][0] = *(const LAS f16x8*)(lds + SA(b, h) + boff0 + n * 2048); dst[n][1] = *(const LAS f16x8*)(lds + SA(b, h) + boff1 + n * 2048); } } while (0)
#define MMA(ai, bj, At, Bt) MMA16(acc[ai][bj], At, Bt)
  f32x4 acc[2][2][4][2];
#pragma unroll
  for (int a = 0; a < 2; ++a)
#pragma unroll
    for (int b = 0; b < 2; ++b)
#pragma unroll
      for (int m = 0; m < 4; ++m)
#pragma unroll
        for (int n = 0; n < 2; ++n) acc[a][b][m][n] = (f32x4){0.f, 0.f, 0.f, 0.f};
  f16x8 At[4][2], B0[2][2], B1[2][2];
  int avail = 0;
  if (wid == 0) avail = wait_flags(ws, fo, 15, 0, 1, seq);
  BAR;
  u32x4 ff = __builtin_amdgcn_raw_buffer_load_b128(rf, fa, 16, AUX_SYS);
  SCHED;
  STAGE(SB(0, 0), rb, 0u, 0, 0); STAGE(SA(0, 0), ra, 0u, 0, 0); STAGE(SB(0, 1), rb, 0u, 0, 1); STAGE(SA(0, 1), ra, 0u, 0, 1);
  if (wr == 1) BAR;
  WAIT_V(4); BAR;
  STAGE(SB(1, 0), rb, 0u, 1, 0); STAGE(SA(1, 0), ra, 0u, 1, 0); STAGE(SB(1, 1), rb, 0u, 1, 1);
  WAIT_V(6); BAR;
  for (int t = 0; t < NPAIR; ++t) {
    const unsigned po = (unsigned)t * PAIR_B, pn = po + PAIR_B;
    if (wid == 0 && t + 2 > avail && t + 2 <= NPAIR) {
      avail = seen_flags(ff, 15, t + 1, seq << 9);
      if (t + 2 > avail) avail = wait_flags(ws, fo, 15, t + 1, t + 2, seq);
    }
    LDB(B0, 0, 0); SCHED; LDA(At, 0, 0); STAGE(SA(1, 1), ra, po, 1, 1);
    WAIT_V(10); WAIT_L(8); BAR; WAIT_L(0); MMA(0, 0, At, B0); BAR; SCHED;
    LDB(B1, 0, 1); STAGE(SB(0, 0), rb, pn, 0, 0);
    WAIT_V(10); BAR; WAIT_L(0); MMA(0, 1, At, B1); BAR;
    LDA(At, 0, 1);
    ff = __builtin_amdgcn_raw_buffer_load_b128(rf, fa, (unsigned)(t + 2) * 16u, AUX_SYS);
    STAGE(SA(0, 0), ra, pn, 0, 0);
    BAR; WAIT_L(0); MMA(1, 0, At, B0); BAR; SCHED;
    STAGE(SB(0, 1), rb, pn, 0, 1);
    WAIT_V(11); BAR; MMA(1, 1, At, B1); BAR;
    LDB(B0, 1, 0); SCHED; LDA(At, 1, 0); STAGE(SA(0, 1), ra, pn, 0, 1);
    WAIT_V(11); WAIT_L(8); BAR; WAIT_L(0); MMA(0, 0, At, B0); BAR; SCHED;
    LDB(B1, 1, 1); STAGE(SB(1, 0), rb, pn, 1, 0);
    WAIT_V(11); BAR; WAIT_L(0); MMA(0, 1, At, B1); BAR;
    LDA(At, 1, 1); STAGE(SA(1, 0), ra, pn, 1, 0);
    BAR; WAIT_L(0); MMA(1, 0, At, B0); BAR; SCHED;
    STAGE(SB(1, 1), rb, pn, 1, 1);
    WAIT_V(10); BAR; MMA(1, 1, At, B1); BAR;
  }
  asm volatile("" :: "v"(ff));
  const bool remote = wsl >= SLOT_WR;
  int tz = threadIdx.x; asm volatile("" : "+v"(tz));
  LAS unsigned* red = (LAS unsigned*)(lds + LDS_MAIN);
  unsigned tk = unit_ticket(ws, wsl - SLOT_WR, remote);
  const int eb = flag_exp(ws, seq, wsl, 0);
  const int e0 = flag_exp(ws, seq, xs, wr) + eb, e1 = flag_exp(ws, seq, xs, 2 + wr) + eb;
  const bool bad = call_bad(ws, seq);
  WAIT_V(0);
  WAIT_L(0);
  if (wr == 0) BAR;
  if (remote) {
    BAR;
    const int u0 = (xs < SLOT_G3 ? xs : xs - 1) * 4;
#pragma unroll
    for (int q = 0; q < 4; ++q) {
      pack_block(ws, seq, wsl - SLOT_WR, tk, q < 3, acc[q >> 1][q & 1], (q >> 1) ? e1 : e0, bad, lds + wid * 8192, red + 16 + q * 4, wid, tz & 63);
      if (q) { WAIT_V(11); post_unit(ws, seq, red + 12 + q * 4, u0 + q - 1); }
    }
    DRAIN();
    post_unit(ws, seq, red + 28, u0 + 3);
  } else {
    const __amdgpu_buffer_rsrc_t ro = __builtin_amdgcn_make_buffer_rsrc(P.out[P.rank], 0, 8192 * NLOC * 4, RSRC_W3);
#pragma unroll
    for (int q = 0; q < 4; ++q)
      store_block(ro, acc[q >> 1][q & 1], (q >> 1) ? e1 : e0, bad,
                  (unsigned)(xs == SLOT_T3 ? P.rank * 1024 + 768 + ((q >> 1) * 2 + wr) * 64 : group_row0(P.rank & 1, xs, (q >> 1) * 2 + wr)) * (NLOC * 4u)
                  + (unsigned)(wsl - SLOT_W) * 1024u + (q & 1) * 512u + wc * 128u, tz & 63);
  }
#undef SA
#undef SB
#undef STAGE
#undef LDA
#undef LDB
#undef MMA
}

__device__ __forceinline__ void gemm_half(const Params& P, unsigned char* ws, unsigned seq, int xs, int ah, int wsl, LAS unsigned char* lds) {
  const int tid = threadIdx.x, wid = __builtin_amdgcn_readfirstlane(tid >> 6), lane = tid & 63;
  const int wr = wid >> 2, wc = wid & 3, fr = lane & 15, fq = lane >> 4;
  const unsigned par = seq & 1u;
  const __amdgpu_buffer_rsrc_t ra = __builtin_amdgcn_make_buffer_rsrc(ws + CTRL_B + (size_t)par * PAR_B + (size_t)xs * SLOT_B + ah * 2 * PIECE_B, 0, SLOT_B - ah * 2 * PIECE_B, RSRC_W3);
  const __amdgpu_buffer_rsrc_t rb = __builtin_amdgcn_make_buffer_rsrc(ws + CTRL_B + (size_t)par * PAR_B + (size_t)wsl * SLOT_B, 0, SLOT_B, RSRC_W3);
  const __amdgpu_buffer_rsrc_t rf = __builtin_amdgcn_make_buffer_rsrc(ws + OFF_FLAG + RFL(par * FLAG_B), 0, FLAG_B, RSRC_W3);
  const unsigned fo = (unsigned)(lane < 32 ? xs : wsl) * 1024u + (unsigned)(lane & 31) * 16u, fa = wid == 0 ? fo : OOB;
  const int sel = lane < 32 ? 3 << (2 * ah) : 15;
  const unsigned voff = (unsigned)wr * PIECE_B + (unsigned)wc * 2048u + (unsigned)lane * 16u;
  const unsigned ldsw = (unsigned)wid * 2048u;
  const int sw0 = (fq ^ (fr & 7)) * 16;
  int aoff0 = (wr * 64 + fr) * 128 + sw0, aoff1 = aoff0 ^ 64;
  const int boff0 = 4 * HALF_B + (wc * 32 + fr) * 128 + sw0, boff1 = boff0 ^ 64;
#define SB(b, h) ((4 + (b) * 2 + (h)) * HALF_B)
#define STAGE(bufoff, rs, po, kk, h) DMA2(bufoff, rs, (po) + (kk) * KK_B + (h) * 2 * PIECE_B)
#define STA(kk, i) __builtin_amdgcn_raw_ptr_buffer_load_lds(ra, (LAS void*)(lds + an + (kk) * 2 * HALF_B + ldsw), 16, voff, pn + (kk) * KK_B, (i) * 1024, AG_LAUX)
#define LDA(dst, b) do { _Pragma("unroll") for (int m = 0; m < 4; ++m) { \
    dst[m][0] = *(const LAS f16x8*)(lds + (b) * 2 * HALF_B + aoff0 + m * 2048); dst[m][1] = *(const LAS f16x8*)(lds + (b) * 2 * HALF_B + aoff1 + m * 2048); } } while (0)
#define LDB(dst, b, h) do { _Pragma("unroll") for (int n = 0; n < 2; ++n) { \
    dst[n][0] = *(const LAS f16x8*)(lds + ((b) * 2 + (h)) * HALF_B + boff0 + n * 2048); dst[n][1] = *(const LAS f16x8*)(lds + ((b) * 2 + (h)) * HALF_B + boff1 + n * 2048); } } while (0)
  f32x4 acc[2][4][2];
#pragma unroll
  for (int b = 0; b < 2; ++b)
#pragma unroll
    for (int m = 0; m < 4; ++m)
#pragma unroll
      for (int n = 0; n < 2; ++n) acc[b][m][n] = (f32x4){0.f, 0.f, 0.f, 0.f};
  f16x8 At[4][2], Bt[2][2];
  int avail = 0;
  if (wid == 0) avail = wait_flags(ws, fo, sel, 0, 1, seq);
  BAR;
  u32x4 ff = __builtin_amdgcn_raw_buffer_load_b128(rf, fa, 16, AUX_SYS);
  SCHED;
  unsigned an = 0, pn = 0;
  STAGE(SB(0, 0), rb, 0u, 0, 0); STA(0, 0); STA(0, 1); STAGE(SB(0, 1), rb, 0u, 0, 1);
  if (wr == 1) BAR;
  WAIT_V(2); BAR;
  STAGE(SB(1, 0), rb, 0u, 1, 0); STA(1, 0); STA(1, 1);
  WAIT_V(4); BAR;
  for (int t = 0; t < NPAIR; ++t) {
    const unsigned po = pn;
    pn += PAIR_B;
    an ^= HALF_B;
    if (wid == 0 && t + 2 > avail && t + 2 <= NPAIR) {
      avail = seen_flags(ff, sel, t + 1, seq << 9);
      if (t + 2 > avail) avail = wait_flags(ws, fo, sel, t + 1, t + 2, seq);
    }
    LDB(Bt, 0, 0); SCHED; LDA(At, 0); STAGE(SB(1, 1), rb, po, 1, 1);
    WAIT_V(5); WAIT_L(8); BAR; WAIT_L(0); MMA16(acc[0], At, Bt); BAR; SCHED;
    LDB(Bt, 0, 1); STA(0, 0); STA(0, 1); STAGE(SB(0, 0), rb, pn, 0, 0);
    WAIT_V(6); BAR; WAIT_L(0); MMA16(acc[1], At, Bt); BAR; SCHED;
    LDB(Bt, 1, 0); SCHED; LDA(At, 1);
    asm volatile("" :: "v"(ff));
    ff = __builtin_amdgcn_raw_buffer_load_b128(rf, fa, (unsigned)(t + 2) * 16u, AUX_SYS);
    STA(1, 0); STAGE(SB(0, 1), rb, pn, 0, 1);
    WAIT_V(8); WAIT_L(8); BAR; WAIT_L(0); MMA16(acc[0], At, Bt); BAR; SCHED;
    LDB(Bt, 1, 1); STA(1, 1); STAGE(SB(1, 0), rb, pn, 1, 0);
    WAIT_V(7); BAR; WAIT_L(0); MMA16(acc[1], At, Bt); BAR; SCHED;
    aoff0 ^= HALF_B;
    aoff1 ^= HALF_B;
  }
  asm volatile("" :: "v"(ff));
  int tz = threadIdx.x; asm volatile("" : "+v"(tz));
  LAS unsigned* red = (LAS unsigned*)(lds + LDS_MAIN);
  unsigned tk = unit_ticket(ws, wsl - SLOT_WR, true);
  const int e = flag_exp(ws, seq, xs, ah * 2 + wr) + flag_exp(ws, seq, wsl, 0);
  const bool bad = call_bad(ws, seq);
  WAIT_V(0);
  WAIT_L(0);
  if (wr == 0) BAR;
  BAR;
  const int u0 = (xs < SLOT_G3 ? xs : xs - 1) * 4 + ah * 2;
  pack_block(ws, seq, wsl - SLOT_WR, tk, true, acc[0], e, bad, lds + wid * 8192, red + 16, wid, tz & 63);
  pack_block(ws, seq, wsl - SLOT_WR, tk, false, acc[1], e, bad, lds + wid * 8192, red + 20, wid, tz & 63);
  WAIT_V(11);
  post_unit(ws, seq, red + 16, u0);
  DRAIN();
  post_unit(ws, seq, red + 20, u0 + 1);
#undef SB
#undef STAGE
#undef STA
#undef LDA
#undef LDB
}

#define AG_DW 2
constexpr int NBQ = 4;
#define AG_RT 6
#define AG_RO 1
constexpr unsigned LDS_F = NBQ * KSTEP_B;
#define WAIT_L0() __builtin_amdgcn_s_waitcnt(0xC07F)
__device__ __forceinline__ void gemm_quarter(const Params& P, unsigned char* ws, unsigned seq, int xs, int ah, int wsl, int bh, unsigned long long mask, int role, LAS unsigned char* lds) {
  const int tid = threadIdx.x, w8 = __builtin_amdgcn_readfirstlane(tid >> 6), lane = tid & 63;
  seq = RFL(seq);
  xs = (int)RFL(xs); wsl = (int)RFL(wsl); bh = (int)RFL(bh); role = (int)RFL(role);
  const unsigned par = seq & 1u;
  const int nk = (int)RFL(2 * __builtin_popcountll(mask));
  const int cq = (wsl >= SLOT_WR) * 2 + bh;
  const unsigned job = (unsigned)((blockIdx.x & 7) * 12 + (xs - SLOT_V) * 4 + cq);
  const __amdgpu_buffer_rsrc_t rq = __builtin_amdgcn_make_buffer_rsrc(ws + CTRL_B + OFF_PART + job * 65536u + (w8 & 3) * 16384u, 0, 16384, RSRC_W3);
  f32x4 acc[4][2];
  LAS unsigned char* xb = lds + 3 * KSTEP_B + (w8 & 3) * 8192 + lane * 16;
#define NEXTB(o) o = o == (NBQ - 1) * KSTEP_B ? 0u : o + KSTEP_B
  if (w8 >= 4) {
    const __amdgpu_buffer_rsrc_t ra = __builtin_amdgcn_make_buffer_rsrc(ws + CTRL_B + (size_t)par * PAR_B + (size_t)xs * SLOT_B + ah * 2 * PIECE_B, 0, SLOT_B - ah * 2 * PIECE_B, RSRC_W3);
    const __amdgpu_buffer_rsrc_t rb = __builtin_amdgcn_make_buffer_rsrc(ws + CTRL_B + (size_t)par * PAR_B + (size_t)wsl * SLOT_B + bh * 2 * PIECE_B, 0, SLOT_B - bh * 2 * PIECE_B, RSRC_W3);
    const unsigned fo = (unsigned)(lane < 32 ? xs : wsl) * 1024u + (unsigned)(lane & 31) * 16u;
    const int g = (w8 >> 1) & 1, sel = 1 << (2 * (lane < 32 ? ah : bh) + g);
    const volatile LAS unsigned* lf = (const volatile LAS unsigned*)(lds + LDS_F + lane * 16);
    const volatile LAS unsigned* lp = (const volatile LAS unsigned*)(lds + LDS_F + 2048 + job * 4);
    const unsigned voff = (unsigned)((w8 >> 1) & 1) * PIECE_B + (unsigned)(w8 & 1) * 4096u + (unsigned)lane * 16u, ldsw = (unsigned)(w8 & 3) * 4096u;
    const int rk = __builtin_popcountll(mask & ((1ull << lane) - 1ull));
    const int at = !((mask >> lane) & 1ull) ? 63 : rk < ((nk / 2 - AG_RT) & ~3) ? (rk & ~3) | ((rk - AG_RO * (cq + xs)) & 3) : rk;
    const int tab = __builtin_amdgcn_ds_permute(at * 4, lane);
#define PAIR(i) __builtin_amdgcn_readlane(tab, i)
#define DMA4(lo, rs, vo, so) do { LAS void* _d = (LAS void*)(lds + (lo)); \
    __builtin_amdgcn_raw_ptr_buffer_load_lds(rs, _d, 16, vo, so, 0, AG_LAUX); __builtin_amdgcn_raw_ptr_buffer_load_lds(rs, _d, 16, vo, so, 1024, AG_LAUX); \
    __builtin_amdgcn_raw_ptr_buffer_load_lds(rs, _d, 16, vo, so, 2048, AG_LAUX); __builtin_amdgcn_raw_ptr_buffer_load_lds(rs, _d, 16, vo, so, 3072, AG_LAUX); } while (0)
#define FILL(top) for (; ki < (top) && ((have >> pr) & 1ull); ++ki) { \
      const unsigned _so = (unsigned)pr * PAIR_B + (unsigned)(ki & 1) * KK_B; \
      DMA4(bo + ldsw, ra, voff, _so); DMA4(bo + HALF_B + ldsw, rb, voff, _so); \
      pr = PAIR(((ki + 1) >> 1) & 63); \
      NEXTB(bo); }
#define PARTS() do { DMA4(bo + 2 * ldsw, rq, lane * 16u, 0u); DMA4(bo + 2 * ldsw + 4096u, rq, lane * 16u, 4096u); \
      _Pragma("unroll") for (int i = 0; i < 8; ++i) pt[i] = __builtin_amdgcn_raw_buffer_load_b128(rq, lane * 16u, 8192u + i * 1024u, AUX_SYS); \
      owed = false; } while (0)
    *(LAS u32x2*)(lds + LDS_F + (tid - 256) * 8) = (u32x2){(seq - 1u) << 9, (seq - 1u) << 9};
    *(LAS unsigned*)(lds + LDS_F + 2048 + (tid - 256) * 4) = seq - 1u;
    WAIT_L(0);
    BAR;
    unsigned long long have = 0;
    int ki = 0, pr = PAIR(0);
    unsigned bo = 0;
    u32x4 pt[8];
#pragma unroll
    for (int i = 0; i < 8; ++i) pt[i] = (u32x4){0u, 0u, 0u, 0u};
    bool owed = role == 1;
    for (int k = 0; k < nk; ++k) {
      if (ki == k) {
        if (!((have >> pr) & 1ull)) have |= (~0ull >> (NPAIR - __builtin_amdgcn_readfirstlane(wait_flags(ws, fo, sel, pr, pr + 1, seq)))) & (~0ull << pr);
        FILL(k + NBQ - 1 < nk ? k + NBQ - 1 : nk);
      }
      if (owed || role != 1) {
        if (ki - k == 3) WAIT_V(16); else if (ki - k == 2) WAIT_V(8); else WAIT_V(0);
      } else {
        if (ki - k == 2) WAIT_V(24); else WAIT_V(16);
      }
      BAR;
      FILL(k + NBQ < nk ? k + NBQ : nk);
      if (ki < nk && !((have >> pr) & (have >> PAIR(((ki >> 1) + 1) & 63)) & 1ull)) {
        have |= __ballot((int)((lf[2 * ah + g] - (seq << 9)) | (lf[256 + 2 * bh + g] - (seq << 9))) >= 0);
        FILL(k + NBQ < nk ? k + NBQ : nk);
      }
      if (owed && ki == nk && k + 3 >= nk && RFL(*lp) == seq) PARTS();
    }
    if (owed) {
      const unsigned long long t0 = __builtin_amdgcn_s_memrealtime();
      while (RFL(ld_sys((const unsigned*)(ws + OFF_PFL) + job)) != seq && !timed_out((unsigned*)(ws + OFF_TMO), seq, t0)) __builtin_amdgcn_s_sleep(2);
      PARTS();
    }
    WAIT_V(0);
    BAR;
    BAR;
#pragma unroll
    for (int i = 0; i < 8; ++i) acc[i >> 1][i & 1] = *(const LAS f32x4*)(xb + i * 1024) + __builtin_bit_cast(f32x4, pt[i]);
  } else {
    const int fr = lane & 15, fq = lane >> 4;
    const int sw0 = (fq ^ (fr & 7)) * 16;
    const int aoff0 = ((w8 >> 1) * 64 + fr) * 128 + sw0, aoff1 = aoff0 ^ 64;
    const int boff0 = HALF_B + ((w8 & 1) * 64 + fr) * 128 + sw0, boff1 = boff0 ^ 64;
    f32x4 ac[4][4];
#pragma unroll
    for (int m = 0; m < 4; ++m)
#pragma unroll
      for (int n = 0; n < 4; ++n) ac[m][n] = (f32x4){0.f, 0.f, 0.f, 0.f};
    f16x8 A0[4][2], B0[4][2], A1[4][2], B1[4][2];
    const __amdgpu_buffer_rsrc_t rf = __builtin_amdgcn_make_buffer_rsrc(w8 == 2 ? ws + OFF_PFL : ws + OFF_FLAG + par * FLAG_B + (w8 ? wsl : xs) * 1024, 0, w8 == 3 ? 0 : 1024, RSRC_W3);
#define RD(A, B) do { _Pragma("unroll") for (int m = 0; m < 4; ++m) { \
      B[m][0] = *(const LAS f16x8*)(lds + bo + boff0 + m * 2048); A[m][0] = *(const LAS f16x8*)(lds + bo + aoff0 + m * 2048); } \
      _Pragma("unroll") for (int m = 0; m < 4; ++m) { \
      B[m][1] = *(const LAS f16x8*)(lds + bo + boff1 + m * 2048); A[m][1] = *(const LAS f16x8*)(lds + bo + aoff1 + m * 2048); } \
      NEXTB(bo); } while (0)
#define MM(A, B) do { _Pragma("unroll") for (int k = 0; k < 2; ++k) _Pragma("unroll") for (int m = 0; m < 4; ++m) _Pragma("unroll") for (int n = 0; n < 4; ++n) \
      ac[m][n] = __builtin_amdgcn_mfma_f32_16x16x32_f16(B[n][k], A[m][k], ac[m][n], 0, 0, 0); } while (0)
#define STEP(An, Bn, A, B, F) do { BAR; SCHED; if (F) __builtin_amdgcn_raw_ptr_buffer_load_lds(rf, (LAS void*)(lds + LDS_F + w8 * 1024), 16, lane * 16, 0, 0, AUX_SYS); RD(An, Bn); MM(A, B); \
      _Pragma("unroll") for (int i = 0; i < 16; ++i) { __builtin_amdgcn_sched_group_barrier(0x100, 1, 0); __builtin_amdgcn_sched_group_barrier(0x008, 1, 0); } \
      __builtin_amdgcn_sched_group_barrier(0x008, 16, 0); SCHED; WAIT_L0(); } while (0)
    unsigned bo = 0;
    BAR;
    BAR; SCHED;
    RD(A0, B0);
    WAIT_L0();
    for (int k = 1; k < nk - 1; k += 2) {
      STEP(A1, B1, A0, B0, 1);
      STEP(A0, B0, A1, B1, 0);
    }
    STEP(A1, B1, A0, B0, 0);
    MM(A1, B1);
    if (role == 2) {
#pragma unroll
      for (int i = 0; i < 16; ++i) asm volatile("" : "+v"(ac[(i >> 1) & 3][(i >> 3) * 2 + (i & 1)]));
      SCHED;
#pragma unroll
      for (int i = 0; i < 16; ++i)
        __builtin_amdgcn_raw_buffer_store_b128(__builtin_bit_cast(u32x4, ac[(i >> 1) & 3][(i >> 3) * 2 + (i & 1)]), rq, lane * 16u, i * 1024u, AUX_SYS);
      STORE_GAP();
      DRAIN();
    }
    BAR;
#pragma unroll
    for (int i = 0; i < 8; ++i) {
      *(LAS f32x4*)(xb + i * 1024) = ac[i >> 1][2 + (i & 1)];
      acc[i >> 1][i & 1] = ac[i >> 1][i & 1];
      if (role == 1) acc[i >> 1][i & 1] += *(const LAS f32x4*)(lds + bo + w8 * 8192 + i * 1024 + lane * 16);
    }
    WAIT_V(0);
    WAIT_L(0);
    BAR;
#undef RD
#undef MM
#undef STEP
  }
  if (role == 2) {
    if (tid == 0) st_sys((unsigned*)(ws + OFF_PFL) + job, seq);
    return;
  }
  const int wr = (w8 >> 1) & 1, wc = (w8 & 1) * 2 + (w8 >> 2), wid = wr * 4 + wc;
  int tz = threadIdx.x; asm volatile("" : "+v"(tz));
  const int e = flag_exp(ws, seq, xs, ah * 2 + wr) + flag_exp(ws, seq, wsl, bh * 2);
  const bool bad = call_bad(ws, seq);
  if (wsl >= SLOT_WR) {
    unsigned mk;
    LAS unsigned char* cb = lds + w8 * 8192;
    const unsigned n64 = (gather_block(acc, e, bad, cb, mk) + 63u) & ~63u;
    const unsigned vo = (unsigned)(tz & 63) * 16u, vi = (unsigned)((((P.rank >> 1) * 3 + xs - SLOT_V) * 4 + ah * 2 + bh) * 8 + wid);
    unsigned char* pw = P.ws[2 * (wsl - SLOT_WR) + 1 - (P.rank & 1)];
    const __amdgpu_buffer_rsrc_t rv = __builtin_amdgcn_make_buffer_rsrc(pw + CTRL_B + OFF_VREC + vi * VREC_B, 0, 256u + n64, RSRC_W3);
    u32x4 v[8];
#pragma unroll
    for (int i = 0; i < 8; ++i) v[i] = *(const LAS u32x4*)(cb + i * 1024 + vo);
    __builtin_amdgcn_raw_buffer_store_b32(mk, rv, vo >> 2, 0, AUX_SYS);
#pragma unroll
    for (int i = 0; i < 8; ++i)
      if (i * 1024u < n64) {
        __builtin_amdgcn_raw_buffer_store_b128(v[i], rv, vo, 256u + i * 1024u, AUX_SYS);
        WAIT_V(AG_DW);
      }
    DRAIN();
    if ((tz & 63) == 0) st_sys((unsigned*)(pw + OFF_VFL) + vi, (seq << 9) + (n64 >> 6));
  } else {
    const __amdgpu_buffer_rsrc_t ro = __builtin_amdgcn_make_buffer_rsrc(P.out[P.rank], 0, 8192 * NLOC * 4, RSRC_W3);
    store_block(ro, acc, e, bad, (unsigned)group_row0(P.rank & 1, xs, ah * 2 + wr) * (NLOC * 4u) + (unsigned)(wsl - SLOT_W) * 1024u + bh * 512u + wc * 128u, tz & 63);
  }
#undef FILL
#undef PARTS
#undef PAIR
#undef DMA4
#undef NEXTB
}

#define AG_H0 0
#define AG_H1 27
#define AG_H2 49
#define AG_HE0 60
#define AG_HE1 60
#define AG_HE2 60
constexpr unsigned long long helped(int a, int e) { return 0xaaaaaaaaaaaaaaaaull & ~((1ull << a) - 1ull) & (e < 64 ? (1ull << e) - 1ull : ~0ull); }
#define AG_HF 1
#define AG_SWO 1
__global__ __launch_bounds__(512, 2) void ag_gemm_kernel(Params P) {
  extern __shared__ __attribute__((aligned(16))) unsigned char shm[];
  LAS unsigned char* lds = (LAS unsigned char*)shm;
  const int tid = threadIdx.x;
  const int b = blockIdx.x, xcd = b & 7, bi = b >> 3;
  const int rank = P.rank, me = rank >> 1;
  unsigned char* ws = P.ws[rank];
  TS(0);
  if (bi < NPUSH) {
    const unsigned seq = link_role(P, ws, bi, xcd, lds);
    if (tid >= 64) return;
    TS(9);
    return;
  }
  const int cp = xcd & 3, rh = xcd >> 2;
  unsigned seq;
  const int j = bi - NPUSH - NFULL;
  if (bi < 32 - NPREP) {
    LAS unsigned* red = (LAS unsigned*)(lds + LDS_MAIN);
    if (tid == 0) red[8] = take_seq(ws, xcd);
    if (tid >= 16 && tid < 32) red[tid] = 0;
    WAIT_L(0);
    BAR;
    seq = RFL(red[8]);
  } else {
    seq = prep_role(P, ws, xcd, 31 - bi, lds);
    TS(2);
  }
#define ROWS(ri) ((ri) < 2 ? SLOT_A + rh * 2 + (ri) : (ri) < 6 ? ((me + 1 + rh * 2 + ((ri) & 1)) & 3) * 2 + (((ri) >> 2) ^ (rh & (ri) & AG_SWO)) : SLOT_V3)
  const int w8 = __builtin_amdgcn_readfirstlane(tid >> 6);
  const int f = bi - NPUSH;
  if (AG_HF && (j < 0 ? f >= 4 && f < 12 && (f & 1) : j < 4)) {
    const int ri = j < 0 ? f >> 1 : 2 + j;
    gemm_half(P, ws, seq, ROWS(ri), j >= 0, SLOT_WR + cp, lds);
  } else if (j < 0) {
    const int ri = f >> 1;
    const int xs = ROWS(ri);
    const bool far = f < 12 ? f & 1 : rh;
    const bool home = far && xs == SLOT_A + cp;
    gemm_full(P, ws, seq, home ? SLOT_T3 : xs, (far && !home ? SLOT_WR : SLOT_W) + cp, lds);
  }
  const int jq = j >= 0 ? j : f < 4 ? 8 + f : (f & 1) ? (f - 5) >> 1 : 4 + ((f - 4) >> 1);
  const unsigned long long hm = jq < 4 ? helped(AG_H0, AG_HE0) : jq < 8 ? helped(AG_H1, AG_HE1) : helped(AG_H2, AG_HE2);
  if (f != 12 && (j >= 0 || hm)) {
    WAIT_L(0);
    BAR;
    gemm_quarter(P, ws, seq, SLOT_V + (jq >> 2), rh, ((jq & 2) ? SLOT_WR : SLOT_W) + cp, jq & 1, j < 0 ? hm : ~hm, j < 0 ? 2 : hm ? 1 : 0, lds);
  }
  if (f >= 4 && f < 10) {
    unpack_block(P, ws, seq, ((f - 4) * 8 + xcd) * 8 + w8, true, lds + w8 * 8192);
  }
#undef ROWS
}

#define AG_FR0 2.16
#define AG_FR1 2.16
#define AG_E0 150.
#define AG_E1 150.
#define AG_S0 200.
#define AG_R0 0.7
#define AG_S1 200.
#define AG_R1 0.95
#define AG_S2 200.
#define AG_R2 1.15
#define AG_NA 0
#define AG_RA 0.72
#define AG_XW 8
#define AG_L0 2
#define AG_L1 2
#define AG_LV0 6
#define AG_LV1 6
#define AG_LV2 6
#define AG_LW 0
#define AG_LA 2
void merge_streams(unsigned short* out, int n, const int* cnt, const int* size, const double* t0, const double* dt, const int* first, const int* step, const int* lng) {
  int done[6] = {0, 0, 0, 0, 0, 0};
  for (;;) {
    int best = -1;
    for (int s = 0; s < n; ++s)
      if (done[s] < cnt[s] && (best < 0 || t0[s] + done[s] * dt[s] < t0[best] + done[best] * dt[best] - 1e-9)) best = s;
    if (best < 0) return;
    for (int k = 0; k < size[best]; ++k) *out++ = (unsigned short)(first[best] + done[best] * step[best] + k + (done[best] >= cnt[best] - lng[best] ? 0x800 : 0));
    ++done[best];
  }
}

}

extern "C" void kernel_launch(int rank, int nranks, ncclComm_t comm,
                              hipStream_t stream,
                              void* const* d_in, const int* in_sizes, int n_in,
                              void* d_out, int out_size,
                              const int* in_dtypes, int out_dtype,
                              void* const* d_out_all,
                              void* const* d_ws_all, size_t ws_bytes) {
  (void)comm; (void)in_dtypes; (void)out_dtype; (void)d_out;
  if (nranks != NR || (unsigned)rank >= (unsigned)NR || n_in < 2 || ws_bytes < WS_NEED ||
      in_sizes[0] != 1024 * KDIM || in_sizes[1] != KDIM * NLOC || out_size != 8192 * NLOC) {
    fprintf(stderr, "kernel_launch: unexpected problem shape\n");
    return;
  }
  HIP_CHECK(hipFuncSetAttribute((const void*)ag_gemm_kernel, hipFuncAttributeMaxDynamicSharedMemorySize, LDS_B));
  Params P;
  P.x = (const float*)d_in[0];
  P.w = (const float*)d_in[1];
  for (int p = 0; p < NR; ++p) { P.out[p] = (float*)d_out_all[p]; P.ws[p] = (unsigned char*)d_ws_all[p]; }
  P.rank = rank;
  const int me = rank >> 1;
  {
    const int cnt[6] = {NPAIR - 4, NPAIR - 4, NPAIR, NPAIR, AG_NA, NPAIR - AG_NA}, size[6] = {4, 4, 1, 1, 1, 1}, step[6] = {4, 4, 4, 4, 4, 4};
    const int first[6] = {0 << 8 | 16, 1 << 8 | 16, 2 << 8 | me, 3 << 8 | me, 4 << 8 | me, 4 << 8 | (AG_NA * 4 + me)};
    const double t2 = AG_S2 - (NPAIR - AG_NA) * AG_R2;
    const double t0[6] = {AG_E0 - 60 * AG_FR0, AG_E1 - 60 * AG_FR1, AG_S0 - 64 * AG_R0, AG_S1 - 64 * AG_R1, t2 - AG_NA * AG_RA, t2};
    const double dt[6] = {AG_FR0, AG_FR1, AG_R0, AG_R1, AG_RA, AG_R2};
    const int lng[6] = {AG_L0, AG_L1, AG_LV0, AG_LV1, 0, AG_LV2};
    merge_streams(P.ord + NDIR, 6, cnt, size, t0, dt, first, step, lng);
  }
  for (int p = 0; p < NDIR; ++p) P.ord[p] = 0;
  for (int p = 0; p < NDIR; ++p) P.ord[QSAME + p] = 0;
  {
    const int cnt[3] = {NPAIR - 4, NPAIR - 4, NPAIR}, size[3] = {4, 4, 1}, first[3] = {0 << 8 | 16, 1 << 8 | 16, 2 << 8 | me}, step[3] = {4, 4, 4};
    const double t0[3] = {4., 4., 0.}, dt[3] = {(NPAIR - 5. - AG_XW) / (NPAIR - 5.), 1., 1.};
    const int lng[3] = {AG_LW, AG_LA, AG_LA};
    merge_streams(P.ord + QSAME + NDIR, 3, cnt, size, t0, dt, first, step, lng);
  }
  hipLaunchKernelGGL(ag_gemm_kernel, dim3(NWG), dim3(512), LDS_B, stream, P);
  HIP_CHECK(hipGetLastError());
}
```

```cpp
#include <hip/hip_runtime.h>
#include <rccl/rccl.h>
#include <cstdio>

#define HIP_CHECK(expr) do { hipError_t _e = (expr); if (_e != hipSuccess) { \
    fprintf(stderr, "HIP error %s at %s:%d\n", hipGetErrorString(_e), __FILE__, __LINE__); \
    return; } } while (0)

#define LAS __attribute__((address_space(3)))
typedef _Float16 f16x8 __attribute__((ext_vector_type(8)));
typedef float f32x4 __attribute__((ext_vector_type(4)));
typedef unsigned u32x4 __attribute__((ext_vector_type(4)));
typedef unsigned u32x2 __attribute__((ext_vector_type(2)));

namespace {

constexpr int NR = 8;
constexpr int KDIM = 8192, NLOC = 1024;
constexpr int KSTEPS = KDIM / 64, NPAIR = KSTEPS / 2;
constexpr unsigned KK_B = 8192, PIECE_B = 16384, HALF_B = 16384, KSTEP_B = 32768, PAIR_B = 65536;
constexpr unsigned SLOT_B = NPAIR * PAIR_B;
constexpr int SLOT_V = 8, SLOT_G3 = 11, SLOT_A = 12, SLOT_V3 = 16, SLOT_T3 = 17, SLOT_W = 18, SLOT_WR = 22, SLOT_REL = 26, NSLOT = 27;
constexpr unsigned PAR_B = NSLOT * SLOT_B;
constexpr size_t CTRL_B = 1u << 20;
constexpr unsigned UREC_B = 68 * 1024, UHDR_B = 2112;
constexpr unsigned OUTS_B = 4 * 64 * UREC_B;
constexpr unsigned SH_HI = 8192, SH_EX = 14336, SH_MAX = 256, SH_LONG = 511;
constexpr unsigned OFF_OUTS = 2 * PAR_B, OFF_RCV = OFF_OUTS + OUTS_B, OFF_SH = OFF_RCV + OUTS_B;
constexpr unsigned OFF_INFO = OFF_SH + 2048 * PIECE_B;
constexpr unsigned VREC_B = 256 + 8192, OFF_VREC = OFF_INFO + 2048 * 16 + 1024;
constexpr unsigned OFF_PART = OFF_VREC + 384 * VREC_B, DATA_B = OFF_PART + 64 * 262144;
constexpr size_t WS_NEED = CTRL_B + DATA_B;
constexpr unsigned OFF_TICKET = 0;
constexpr unsigned OFF_TMO = 1024;
constexpr unsigned OFF_QUEUE = 2048;
constexpr unsigned OFF_UHEAD = 3072;
constexpr unsigned OFF_ULIST = 4096;
constexpr unsigned OFF_PFL = 5120;
constexpr unsigned OFF_VFL = 12288;
constexpr unsigned OFF_DONE = 8192;
constexpr unsigned OFF_FLAG = 16384;
constexpr unsigned FLAG_B = (NSLOT + 1) * 1024;
constexpr unsigned OFF_SFLAG = OFF_FLAG + 2 * FLAG_B;
constexpr unsigned CTRL_USED = OFF_SFLAG + 2 * FLAG_B;
constexpr unsigned long long DEADLINE = 50000000ull;
#define AG_NP 8
constexpr int NWG = 256, NPUSH = 7, NFULL = 13, NPREP = AG_NP;
constexpr int NUNIT = 48;
#define AG_ND 16
constexpr int NREL = AG_ND;
static_assert(NREL % 4 == 0, "relay");
constexpr int QSAME = 704 - NREL, QIN = 576 + NREL, QCROSS = QIN + NUNIT * 2;
constexpr int NDIR = 32;
constexpr int LDS_MAIN = 137 * 1024;
constexpr int LDS_B = 160 * 1024;
constexpr int AUX_SYS = 17;
#define AG_OAUX 17
constexpr unsigned RSRC_W3 = 0x00020000u;
constexpr unsigned OOB = 0x80000000u;
static_assert(DATA_B < OOB, "layout");

struct Params {
  const float* x;
  const float* w;
  float* out[NR];
  unsigned char* ws[NR];
  int rank;
  unsigned long long pairs[22];
  alignas(4) unsigned short ord[QSAME + QIN];
};

__device__ __forceinline__ unsigned ld_sys(const unsigned* p) { return __hip_atomic_load(p, __ATOMIC_RELAXED, __HIP_MEMORY_SCOPE_SYSTEM); }
__device__ __forceinline__ void st_sys(unsigned* p, unsigned v) { __hip_atomic_store(p, v, __ATOMIC_RELAXED, __HIP_MEMORY_SCOPE_SYSTEM); }
#define DRAIN() asm volatile("s_waitcnt vmcnt(0)" ::: "memory")
#define WAIT_V(n) asm volatile("s_waitcnt vmcnt(%0)" :: "n"(n) : "memory")
#define WAIT_L(n) asm volatile("s_waitcnt lgkmcnt(%0)" :: "n"(n) : "memory")
#define BAR __builtin_amdgcn_s_barrier()
#define SCHED __builtin_amdgcn_sched_barrier(0)
#define RFL(x) ((unsigned)__builtin_amdgcn_readfirstlane(x))
#define STORE_GAP() do { SCHED; asm volatile("s_nop 1"); SCHED; } while (0)
#define TSV(i, v)
#define TS(i)
#define TSW(i)
#define TR(ci, k, v)

__device__ __forceinline__ bool timed_out(unsigned* tmo, unsigned seq, unsigned long long t0) {
  if (RFL(__hip_atomic_load(tmo, __ATOMIC_RELAXED, __HIP_MEMORY_SCOPE_AGENT)) == seq ||
      __builtin_amdgcn_s_memrealtime() - t0 > DEADLINE) {
    if ((threadIdx.x & 63) == 0) __hip_atomic_store(tmo, seq, __ATOMIC_RELAXED, __HIP_MEMORY_SCOPE_AGENT);
    return true;
  }
  return false;
}

__device__ __noinline__ int wait_flags(const unsigned char* wsv, unsigned fo, int sel, int p0v, int needv, unsigned seqv) {
  const unsigned long long wa = (unsigned long long)wsv;
  const unsigned whi = RFL((unsigned)(wa >> 32)), wlo = RFL((unsigned)wa);
  const unsigned char* ws = (const unsigned char*)(((unsigned long long)whi << 32) | (unsigned long long)wlo);
  const int p0 = __builtin_amdgcn_readfirstlane(p0v), need = __builtin_amdgcn_readfirstlane(needv);
  const unsigned seq = RFL(seqv);
  const unsigned s9 = seq << 9;
  const __amdgpu_buffer_rsrc_t rf = __builtin_amdgcn_make_buffer_rsrc((void*)(ws + OFF_FLAG + (seq & 1u) * FLAG_B), 0, FLAG_B, RSRC_W3);
  const bool past = p0 + (int)(threadIdx.x & 31) >= NPAIR;
  const unsigned long long t0 = __builtin_amdgcn_s_memrealtime();
  for (;;) {
    asm volatile("" ::: "memory");
    const u32x4 f = __builtin_amdgcn_raw_buffer_load_b128(rf, fo, (unsigned)p0 * 16u, AUX_SYS);
    int old = 0;
#pragma unroll
    for (int j = 0; j < 4; ++j) old |= (int)((f[j] - s9) >> 31) << j;
    const unsigned long long m = __ballot(past || (old & sel) == 0);
    const unsigned miss = ~((unsigned)m & (unsigned)(m >> 32));
    int av = p0 + (miss ? __builtin_ctz(miss) : 32);
    av = av > NPAIR ? NPAIR : av;
    if (av >= need) return av;
    if (timed_out((unsigned*)(ws + OFF_TMO), seq, t0)) return NPAIR;
    __builtin_amdgcn_s_sleep(4);
  }
}

__device__ __forceinline__ unsigned wave_max(unsigned m) {
#pragma unroll
  for (int o = 32; o >= 1; o >>= 1) { const unsigned t = (unsigned)__shfl_xor((int)m, o); m = t > m ? t : m; }
  return m;
}

__device__ __forceinline__ unsigned take_seq(unsigned char* ws, int xcd) {
  return __hip_atomic_fetch_add((unsigned*)(ws + OFF_TICKET) + xcd * 32, 1u, __ATOMIC_RELAXED, __HIP_MEMORY_SCOPE_AGENT) / (NWG / 8) + 1u;
}

struct Src { __amdgpu_buffer_rsrc_t rs; unsigned lo, ks_b, st_a, st_b, so; };

__device__ __forceinline__ Src make_src(const Params& P, int tl8) {
  const int tid = threadIdx.x, wid = tid >> 6, lane = tid & 63, r8 = lane >> 3, c8 = lane & 7;
  const bool isw = tl8 >= 4;
  const int tl = tl8 & 3;
  Src q;
  q.rs = __builtin_amdgcn_make_buffer_rsrc((void*)(isw ? P.w + tl * 256 : P.x + (size_t)tl * 256 * KDIM), 0,
                                           isw ? (KDIM * NLOC - tl * 256) * 4 : 256 * KDIM * 4, RSRC_W3);
  q.lo = isw ? (unsigned)(wid * 8 * NLOC + lane * 4) * 4u : (unsigned)((wid * 32 + r8) * KDIM + (c8 ^ r8) * 8) * 4u;
  q.ks_b = isw ? 64u * NLOC * 4u : 64u * 4u;
  q.st_a = isw ? 2u * NLOC * 4u : 8u * KDIM * 4u;
  q.st_b = isw ? NLOC * 4u : 16u;
  q.so = isw ? (unsigned)((tid * 16 + (tid & 15)) * NLOC + ((tid * 5) & 63) * 4) * 4u
             : (unsigned)((tid >> 1) * KDIM + ((tid * 37 + 11) & 2047) * 4) * 4u;
  return q;
}
#define PLOAD(q, v, ks) do { const unsigned _k = (unsigned)(ks) * q.ks_b, _lo = (ks) < KSTEPS ? q.lo : OOB; \
    _Pragma("unroll") for (int i = 0; i < 8; ++i) \
      v[i] = __builtin_amdgcn_raw_buffer_load_b128(q.rs, _lo, _k + (i >> 1) * q.st_a + (i & 1) * q.st_b, 0); } while (0)

__device__ __forceinline__ int tile_exp(const u32x4 sv, LAS unsigned* red) {
  unsigned m = 0, l = 0;
#pragma unroll
  for (int j = 0; j < 4; ++j) { const unsigned a = sv[j] & 0x7fffffffu; m = a > m ? a : m; l += a >> 15; }
  m = wave_max(m);
#pragma unroll
  for (int o = 32; o >= 1; o >>= 1) l += (unsigned)__shfl_xor((int)l, o);
  if ((threadIdx.x & 63) == 0) { red[threadIdx.x >> 6] = m; red[32 + (threadIdx.x >> 6)] = l; }
  WAIT_L(0);
  BAR;
  l = 0;
#pragma unroll
  for (int q = 0; q < 8; ++q) { const unsigned a = red[q]; m = a > m ? a : m; l += red[32 + q]; }
  m = RFL(m);
  l = RFL(l);
  const int ea = (int)(m >> 23), eg = (int)(((l >> 11) + 588u + 255u) >> 8) + 3;
  const int eb = eg > ea ? eg : ea;
  return m == 0 ? 131 : eb < 5 ? 5 : eb > 254 ? 254 : eb;
}

__device__ __forceinline__ void convert(const u32x4 (&v)[8], u32x4 (&h)[4], float s, bool isw, LAS unsigned char* lb) {
  const int tid = threadIdx.x, wid = tid >> 6, lane = tid & 63, r8 = lane >> 3, c8 = lane & 7;
  if (isw) {
    const int tw = lane * 512 + ((wid ^ c8) * 16);
#pragma unroll
    for (int cc = 0; cc < 4; ++cc) {
      f16x8 t;
#pragma unroll
      for (int i = 0; i < 8; ++i) t[i] = (_Float16)(__uint_as_float(v[i][cc]) * s);
      *(LAS f16x8*)(lb + tw + cc * 128) = t;
    }
    WAIT_L(0);
    BAR;
#pragma unroll
    for (int it = 0; it < 4; ++it) h[it] = *(const LAS u32x4*)(lb + wid * 4096 + it * 1024 + (lane * 16 ^ ((r8 ^ (it * 2 + (r8 >> 2))) * 16)));
  } else {
#pragma unroll
    for (int it = 0; it < 4; ++it) {
      f16x8 t;
#pragma unroll
      for (int i = 0; i < 4; ++i) {
        t[i] = (_Float16)(__uint_as_float(v[2 * it][i]) * s);
        t[4 + i] = (_Float16)(__uint_as_float(v[2 * it + 1][i]) * s);
      }
      h[it] = __builtin_bit_cast(u32x4, t);
    }
  }
}

#define AG_PP 170
#define AG_PO 4
__device__ __forceinline__ unsigned prep_role(const Params& P, unsigned char* ws, int tl8, int pi, LAS unsigned char* lds) {
  const int tid = threadIdx.x, me = P.rank >> 1;
  LAS unsigned* red = (LAS unsigned*)(lds + LDS_MAIN);
  const bool isw = tl8 >= 4;
  const Src q = make_src(P, tl8);
  const u32x4 sv = __builtin_amdgcn_raw_buffer_load_b128(q.rs, q.so, 0, 0);
  u32x4 v0[8], v1[8];
  PLOAD(q, v0, 2 * pi);
  PLOAD(q, v1, 2 * pi + 1);
  if (tid == 0) red[8] = take_seq(ws, tl8);
  if (tid >= 40 && tid < 48) red[tid] = 0;
  const int eb = tile_exp(sv, red);
  const unsigned seq = RFL(red[8]);
  const float s = __uint_as_float((unsigned)(258 - eb) << 23);
  const unsigned p0 = isw ? (SLOT_W + tl8 - 4) * 256u : tl8 < 2 ? (me * 2 + tl8) * 256u : tl8 == 2 ? SLOT_V * 256u + me : SLOT_T3 * 256u;
  const unsigned gs = tl8 == 2 ? 256u : 1u;
  const unsigned par = seq & 1u;
  const __amdgpu_buffer_rsrc_t rd = __builtin_amdgcn_make_buffer_rsrc(ws + CTRL_B + (size_t)par * PAR_B, 0, PAR_B, RSRC_W3);
  unsigned* flag = (unsigned*)(ws + OFF_FLAG + par * FLAG_B) + p0 + (tid & 3) * gs;
  const unsigned fval = (seq << 9) + (unsigned)(256 + eb - 131);
  const __amdgpu_buffer_rsrc_t rh = __builtin_amdgcn_make_buffer_rsrc(ws + CTRL_B + OFF_SH, 0, DATA_B - OFF_SH, RSRC_W3);
  const int wid = __builtin_amdgcn_readfirstlane(tid >> 6), grp = wid >> 1;
  const unsigned vo = (unsigned)(wid & 1) * 4096u + (unsigned)(tid & 63) * 16u;
  const unsigned long long t0 = __builtin_amdgcn_s_memrealtime();
  int odd = 0;
  for (int pr = pi; pr < NPAIR; pr += NPREP, odd ^= 1) {
    u32x4 h0[4], h1[4], lo[4], hi[3];
    convert(v0, h0, s, isw, lds);
    convert(v1, h1, s, isw, lds + KSTEP_B);
    LAS unsigned* cn = red + 40 + odd * 4 + grp;
    LAS unsigned char* ex = lds + 2 * KSTEP_B + odd * 2048 + grp * 512;
    const unsigned dl = (p0 + pr * 4u + grp * gs) * PIECE_B, sp = (unsigned)tl8 * 256u + (unsigned)pr * 4u + grp;
    unsigned e[4], ne = 0;
#pragma unroll
    for (int it = 0; it < 4; ++it) {
      __builtin_amdgcn_raw_buffer_store_b128(h0[it], rd, vo, dl + it * 1024u, AUX_SYS);
      __builtin_amdgcn_raw_buffer_store_b128(h1[it], rd, vo, dl + it * 1024u + KK_B, AUX_SYS);
      unsigned H[4];
#pragma unroll
      for (int d = 0; d < 4; ++d) {
        const unsigned a = d < 2 ? h0[it][2 * d] : h1[it][2 * d - 4], b = d < 2 ? h0[it][2 * d + 1] : h1[it][2 * d - 3];
        lo[it][d] = __builtin_amdgcn_perm(b, a, 0x06040200u);
        H[d] = __builtin_amdgcn_perm(b, a, 0x07050301u);
      }
      const unsigned g = (H[3] & ~0x20202020u) | ((H[3] >> 2) & 0x20202020u);
#pragma unroll
      for (int d = 0; d < 3; ++d) hi[(it * 3 + d) >> 2][(it * 3 + d) & 3] = (H[d] & ~0x60606060u) | ((g << (5 - 2 * d)) & 0x60606060u);
      e[it] = (((H[0] & 0x60606060u) >> 5) | ((H[1] & 0x60606060u) >> 3) | ((H[2] & 0x60606060u) >> 1) | ((H[3] & 0x60606060u) << 1)) ^ 0x55555555u;
      ne += (unsigned)__builtin_popcount((e[it] | (e[it] >> 1)) & 0x55555555u);
    }
    if (ne) {
      unsigned at = __hip_atomic_fetch_add(cn, ne, __ATOMIC_RELAXED, __HIP_MEMORY_SCOPE_WORKGROUP);
#pragma unroll
      for (int it = 0; it < 4; ++it)
        for (unsigned x = e[it]; x; ++at) {
          const unsigned f = (unsigned)__builtin_ctz(x) >> 1, x2 = (x >> (2 * f)) & 3u;
          x &= ~(3u << (2 * f));
          if (at < SH_MAX) *(LAS unsigned short*)(ex + at * 2) = (unsigned short)(x2 << 13 | (f & 2u) << 11 | (vo + it * 1024u) >> 1 | (f & 1u) << 2 | f >> 2);
        }
    }
    WAIT_L(0);
    BAR;
    ne = RFL(*cn);
    const u32x4 ev = *(const LAS u32x4*)(ex + (tid & 31) * 16);
    if (tid < 4) red[40 + (odd ^ 1) * 4 + tid] = 0;
    if (ne <= SH_MAX) {
#pragma unroll
      for (int it = 0; it < 4; ++it) __builtin_amdgcn_raw_buffer_store_b128(lo[it], rh, vo, sp * PIECE_B + it * 1024u, AUX_SYS);
#pragma unroll
      for (int j = 0; j < 3; ++j) __builtin_amdgcn_raw_buffer_store_b128(hi[j], rh, vo - (unsigned)(wid & 1) * 1024u, sp * PIECE_B + SH_HI + j * 1024u, AUX_SYS);
      __builtin_amdgcn_raw_buffer_store_b128(ev, rh, (tid & 127) < 32 ? vo : OOB, sp * PIECE_B + SH_EX, AUX_SYS);
    } else {
      ne = SH_LONG;
#pragma unroll
      for (int it = 0; it < 4; ++it) {
        __builtin_amdgcn_raw_buffer_store_b128(h0[it], rh, vo, sp * PIECE_B + it * 1024u, AUX_SYS);
        __builtin_amdgcn_raw_buffer_store_b128(h1[it], rh, vo, sp * PIECE_B + it * 1024u + KK_B, AUX_SYS);
      }
    }
    STORE_GAP();
    __builtin_amdgcn_raw_buffer_store_b32(ne, rh, (tid & 127) == 0 ? 0u : OOB, OFF_INFO - OFF_SH + sp * 16u, AUX_SYS);
    DRAIN();
    BAR;
    if (tid < 4) st_sys(flag + pr * 4, fval);
    while (pr + NPREP < NPAIR && (int)(__builtin_amdgcn_s_memrealtime() - t0) < (pr + NPREP - 11 + AG_PO) * AG_PP) __builtin_amdgcn_s_sleep(32);
    PLOAD(q, v0, 2 * (pr + NPREP));
    PLOAD(q, v1, 2 * (pr + NPREP) + 1);
  }
  return seq;
}

#define AG_PL 5
#define AG_DS 8
#define AG_ST 0
#define AG_SW 0
constexpr int PL = AG_PL;
#define AG_PU 7
constexpr int PU = AG_PU;
#define AG_AT0 900
#define AG_XT0 AG_AT0
#define AG_XTR AG_ATR
#define AG_ATR 213
#define AG_P2 7
constexpr int RING = 4;
constexpr unsigned RSLOT_B = 18 * 1024, LDS_TR = RING * RSLOT_B;
constexpr int NUW = 6;
constexpr int NXW = 4 * 8 * NUW;

__device__ __forceinline__ int group_row0(int g, int xs, int rg) {
  return xs < SLOT_V ? (2 * (xs >> 1) + g) * 1024 + (xs & 1) * 256 + rg * 64
       : xs < SLOT_G3 ? (2 * rg + g) * 1024 + 512 + (xs - SLOT_V) * 64
       : xs < SLOT_V3 ? (2 * (xs - SLOT_A) + 1 - g) * 1024 + 768 + rg * 64
                      : (2 * rg + 1 - g) * 1024 + 704;
}

__device__ __forceinline__ void unpack_block(const Params& P, unsigned char* ws, unsigned seq, int T, bool same, LAS unsigned char* lb) {
  const int lane = threadIdx.x & 63, rank = P.rank;
  const unsigned s9 = seq << 9, voff = (unsigned)lane * 16u;
  unsigned* tmo = (unsigned*)(ws + OFF_TMO);
  const __amdgpu_buffer_rsrc_t ro = __builtin_amdgcn_make_buffer_rsrc(P.out[rank], 0, 8192 * NLOC * 4, RSRC_W3);
  const __amdgpu_buffer_rsrc_t rs = __builtin_amdgcn_make_buffer_rsrc(ws + CTRL_B, 0, DATA_B, RSRC_W3);
  const __amdgpu_buffer_rsrc_t rc = __builtin_amdgcn_make_buffer_rsrc(ws, 0, CTRL_USED, RSRC_W3);
  const unsigned ci = (unsigned)((T >> 3) & 3) * 64u + (unsigned)(T >> 5);
  const unsigned vi = ((unsigned)((T >> 3) & 3) * 12u + (unsigned)(T >> 5)) * 8u + (unsigned)(T & 7);
  const int w = T & 7;
  unsigned u;
  bool bad = false;
  const unsigned long long t0 = __builtin_amdgcn_s_memrealtime();
  for (;;) {
    asm volatile("" ::: "memory");
    u32x2 f = __builtin_amdgcn_raw_buffer_load_b64(rc, 0, same ? OFF_VFL + vi * 4u : OFF_DONE + ci * 16u, AUX_SYS);
    if (same) f[1] = f[0];
    u = RFL(f[0]);
    if ((int)RFL((f[0] - s9) | (f[1] - s9)) >= 0) break;
    if (timed_out(tmo, seq, t0)) { bad = true; break; }
    if (same) __builtin_amdgcn_s_sleep(4); else __builtin_amdgcn_s_sleep(16);
  }
  bad = RFL(bad) != 0;
  u = RFL(u);
  if (!same && w == 0) TR(ci, 10, NOW());
  const unsigned rec = same ? OFF_VREC + vi * VREC_B : OFF_RCV + ci * UREC_B;
  const unsigned mk = __builtin_amdgcn_raw_buffer_load_b32(rs, lane * 4u, same ? rec : rec + w * 256u, AUX_SYS);
  unsigned off = 256u, nb = (u & 511u) * 64u;
  if (same) {
    u = SLOT_V * 4u + (unsigned)(T >> 5);
  } else {
    const u32x2 ob = __builtin_amdgcn_raw_buffer_load_b64(rs, 0, rec + 2048u + w * 8u, AUX_SYS);
    off = UHDR_B + (RFL(ob[0]) & 0xffffu);
    nb = RFL(ob[1]);
  }
#pragma unroll
  for (int i = 0; i < 2; ++i) {
    LAS void* d = (LAS void*)(lb + i * 4096);
    const unsigned so = i * 4096u < nb ? rec + off + i * 4096u : OOB, so2 = i * 4096u + 2048u < nb ? so : OOB;
    __builtin_amdgcn_raw_ptr_buffer_load_lds(rs, d, 16, voff, so, 0, AUX_SYS);
    __builtin_amdgcn_raw_ptr_buffer_load_lds(rs, d, 16, voff, so, 1024, AUX_SYS);
    __builtin_amdgcn_raw_ptr_buffer_load_lds(rs, d, 16, voff, so2, 2048, AUX_SYS);
    __builtin_amdgcn_raw_ptr_buffer_load_lds(rs, d, 16, voff, so2, 3072, AUX_SYS);
  }
  DRAIN();
  f32x4 v[8];
  unsigned run = (unsigned)(unsigned long long)lb;
#pragma unroll
  for (int i = 0; i < 32; ++i) {
    const bool k = (int)(mk << i) < 0;
    const unsigned long long bl = __builtin_amdgcn_ballot_w64(k);
    const unsigned pos = __builtin_amdgcn_mbcnt_hi((unsigned)(bl >> 32), __builtin_amdgcn_mbcnt_lo((unsigned)bl, 0u)) * 4u + run;
    const float t = *(const LAS float*)(unsigned long long)pos;
    v[i >> 2][i & 3] = k ? t : 0.f;
    run += (unsigned)__builtin_popcountll(bl) * 4u;
  }
  if (bad) {
#pragma unroll
    for (int i = 0; i < 8; ++i) v[i] = (f32x4){__builtin_nanf(""), __builtin_nanf(""), __builtin_nanf(""), __builtin_nanf("")};
  }
  const int ti = (int)(u >> 2) & 15;
  const unsigned vo = (unsigned)(lane & 15) * (NLOC * 4u) + (unsigned)(lane >> 4) * 16u;
  const unsigned so = (unsigned)group_row0(1 - (rank & 1), ti < SLOT_G3 ? ti : ti + 1, (int)(u & 2u) + (w >> 2)) * (NLOC * 4u)
                    + (ci >> 6) * 1024u + (u & 1u) * 512u + (unsigned)(w & 3) * 128u;
#pragma unroll
  for (int i = 0; i < 8; ++i) asm volatile("" : "+v"(v[i]));
  SCHED;
#pragma unroll
  for (int i = 0; i < 8; ++i)
    __builtin_amdgcn_raw_buffer_store_b128(__builtin_bit_cast(u32x4, v[i]), ro, vo + (i & 1) * 64u, so + (i >> 1) * 16u * (NLOC * 4u), AG_OAUX);
  STORE_GAP();
  if (!same && w == 7) TR(ci, 11, NOW());
}

__device__ __forceinline__ unsigned link_role(const Params& P, unsigned char* ws, int link, int wgi, LAS unsigned char* lds) {
  const int tid = threadIdx.x, wid = __builtin_amdgcn_readfirstlane(tid >> 6), lane = tid & 63;
  const int rank = P.rank, me = rank >> 1, peer = (rank + 1 + link) & (NR - 1);
  const bool same = ((peer ^ rank) & 1) == 0;
  const unsigned sA = same ? me * 2 : SLOT_W + (peer >> 1), dA = same ? me * 2 : SLOT_WR + me;
  const unsigned sB = same ? me * 2 + 1 : SLOT_T3, dB = same ? me * 2 + 1 : SLOT_A + me;
  const unsigned sC = same ? SLOT_V : SLOT_G3, dC = same ? SLOT_V : SLOT_V3;
  unsigned char* pw = P.ws[peer];
  LAS unsigned* red = (LAS unsigned*)(lds + LDS_MAIN);
  const __amdgpu_buffer_rsrc_t rp = __builtin_amdgcn_make_buffer_rsrc(pw, 0, CTRL_USED, RSRC_W3);
  const __amdgpu_buffer_rsrc_t rd = __builtin_amdgcn_make_buffer_rsrc(pw + CTRL_B, 0, DATA_B, RSRC_W3);
  const unsigned one = lane == 0 ? 0u : OOB;
  unsigned seq;
  const unsigned ts0 = (unsigned)__builtin_amdgcn_s_memrealtime();
  {
    const int tl8 = same ? wgi & 1 : (wgi & 1) ? 3 : 4 + (peer >> 1), pr = wgi >> 1;
    const Src q = make_src(P, tl8);
    const u32x4 sv = __builtin_amdgcn_raw_buffer_load_b128(q.rs, q.so, 0, 0);
    u32x4 v0[8], v1[8], h[2][4];
    PLOAD(q, v0, 2 * pr);
    PLOAD(q, v1, 2 * pr + 1);
    if (tid == 0) { red[8] = take_seq(ws, wgi); red[9] = 0; red[10] = 0; red[11] = 0; }
    const int eb = tile_exp(sv, red);
    seq = RFL(red[8]);
    const float s = __uint_as_float((unsigned)(258 - eb) << 23);
    const unsigned pc = ((wgi & 1) ? dB : dA) * 256u + (unsigned)pr * 4u;
#pragma unroll
    for (int kk = 0; kk < 2; ++kk) {
      convert(kk ? v1 : v0, h[kk], s, tl8 >= 4, lds + LDS_TR + kk * KSTEP_B);
      if (!kk) {
        TS(3);
        const unsigned long long t0 = __builtin_amdgcn_s_memrealtime();
        while (__builtin_amdgcn_s_memrealtime() - t0 < (unsigned long long)(pr * AG_ST)) __builtin_amdgcn_s_sleep(2);
      }
#pragma unroll
      for (int it = 0; it < 4; ++it) {
        __builtin_amdgcn_raw_buffer_store_b128(h[kk][it], rd, (unsigned)(wid & 1) * 4096u + (unsigned)lane * 16u, (seq & 1u) * PAR_B + (pc + (wid >> 1)) * PIECE_B + kk * KK_B + it * 1024u, AUX_SYS);
        WAIT_V(AG_SW);
      }
    }
    DRAIN();
    unsigned n = 0;
    if (lane == 0) n = __hip_atomic_fetch_add(red + 9, 1u, __ATOMIC_RELAXED, __HIP_MEMORY_SCOPE_WORKGROUP);
    if (RFL(n) == 7)
      __builtin_amdgcn_raw_buffer_store_b32((seq << 9) + (unsigned)(256 + eb - 131), rp, lane < 4 ? lane * 4u : OOB,
                                            OFF_FLAG + (seq & 1u) * FLAG_B + pc * 4u, AUX_SYS);
  }
  const unsigned par = seq & 1u, s9 = seq << 9;
  unsigned* tmo = (unsigned*)(ws + OFF_TMO);
  volatile LAS unsigned* cnt = red;
  const __amdgpu_buffer_rsrc_t rs = __builtin_amdgcn_make_buffer_rsrc(ws + CTRL_B, 0, DATA_B, RSRC_W3);
  const __amdgpu_buffer_rsrc_t rc = __builtin_amdgcn_make_buffer_rsrc(ws, 0, CTRL_USED, RSRC_W3);
  const unsigned voff = (unsigned)lane * 16u;
  if (wid > 1) {
    LAS unsigned char* lb = lds + LDS_TR + (wid - 2) * 8192;
    {
      const unsigned long long t0 = __builtin_amdgcn_s_memrealtime();
      while (RFL(cnt[9]) < 8u && __builtin_amdgcn_s_memrealtime() - t0 < DEADLINE) __builtin_amdgcn_s_sleep(8);
    }
    const unsigned* ord = (const unsigned*)P.ord + (same ? 0 : QSAME / 2);
    const unsigned pm = (unsigned)peer >> 1;
    for (int p = NDIR + (wid - 2) * 8 + wgi; p < (same ? QSAME : QIN); p += 8 * NUW) {
      const unsigned o = ord[p >> 1] >> ((p & 1) * 16), sel = (o >> 8) & 7u;
      if ((o & 0x800u) || sel == 7u) continue;
      const unsigned dn = sel >= 5 || (!same && sel == 4) ? (SLOT_V + ((o >> 6) & 3u)) * 256u + ((o & 63u) + pm) * 4u + sel - (sel - 4u >= (unsigned)me ? 3u : 4u)
                        : sel >= 2 ? (dC + sel - 2) * 256u + (o & 252u) + pm
                                   : (same ? pm * 2 + sel : (sel ? SLOT_A : SLOT_WR) + pm) * 256u + (o & 255u);
      unsigned f;
      const unsigned long long t0 = __builtin_amdgcn_s_memrealtime();
      for (;;) {
        asm volatile("" ::: "memory");
        f = RFL(__builtin_amdgcn_raw_buffer_load_b32(rc, 0, OFF_SFLAG + par * FLAG_B + dn * 4u, AUX_SYS));
        if ((int)(f - (seq << 18)) >= 0) break;
        if (timed_out(tmo, seq, t0)) { f = SH_LONG << 9; break; }
        __builtin_amdgcn_s_sleep(AG_DS);
      }
      f = RFL(f);
      const unsigned ne = (f >> 9) & 511u, base = par * PAR_B + dn * PIECE_B;
      if (ne <= SH_MAX) {
        u32x4 lo[8], hw[6];
#pragma unroll
        for (int i = 0; i < 8; ++i) lo[i] = __builtin_amdgcn_raw_buffer_load_b128(rs, voff, base + i * 1024u, AUX_SYS);
#pragma unroll
        for (int i = 0; i < 6; ++i) hw[i] = __builtin_amdgcn_raw_buffer_load_b128(rs, voff, base + SH_HI + i * 1024u, AUX_SYS);
        const u32x2 ex = __builtin_amdgcn_raw_buffer_load_b64(rs, lane * 8u, base + SH_EX, AUX_SYS);
        DRAIN();
#pragma unroll
        for (int kk = 0; kk < 2; ++kk) {
#pragma unroll
          for (int i = 0; i < 8; ++i) {
            const unsigned h0 = hw[(i * 3) >> 2][(i * 3) & 3], h1 = hw[(i * 3 + 1) >> 2][(i * 3 + 1) & 3], h2 = hw[(i * 3 + 2) >> 2][(i * 3 + 2) & 3];
            unsigned a, b;
            if (kk == 0) {
              a = (h0 & 0x9f9f9f9fu) | 0x20202020u;
              b = (h1 & 0x9f9f9f9fu) | 0x20202020u;
            } else {
              a = (h2 & 0x9f9f9f9fu) | 0x20202020u;
              b = ((h0 >> 5) & 0x03030303u) | ((h1 >> 3) & 0x0c0c0c0cu) | ((h2 >> 1) & 0x10101010u) | ((h2 << 1) & 0x80808080u) | 0x20202020u;
            }
            const u32x4 v = {__builtin_amdgcn_perm(a, lo[i][2 * kk], 0x05010400u), __builtin_amdgcn_perm(a, lo[i][2 * kk], 0x07030602u),
                             __builtin_amdgcn_perm(b, lo[i][2 * kk + 1], 0x05010400u), __builtin_amdgcn_perm(b, lo[i][2 * kk + 1], 0x07030602u)};
            *(LAS u32x4*)(lb + i * 1024 + voff) = v;
          }
          asm volatile("" ::: "memory");
#pragma unroll
          for (int j = 0; j < 4; ++j) {
            const unsigned x = ex[j >> 1] >> ((j & 1) * 16);
            const bool on = (unsigned)lane * 4u + j < ne && ((x >> 12) & 1u) == (unsigned)kk;
            __hip_atomic_fetch_xor((LAS unsigned*)(lb + ((x & 0xffeu) << 1)), on ? (x & 0x6000u) << ((x & 1u) * 16u) : 0u, __ATOMIC_RELAXED, __HIP_MEMORY_SCOPE_WORKGROUP);
          }
          asm volatile("" ::: "memory");
          u32x4 v[8];
#pragma unroll
          for (int i = 0; i < 8; ++i) v[i] = *(const LAS u32x4*)(lb + i * 1024 + voff);
          WAIT_L(0);
          SCHED;
#pragma unroll
          for (int i = 0; i < 8; ++i) __builtin_amdgcn_raw_buffer_store_b128(v[i], rs, voff, base + kk * KK_B + i * 1024u, AUX_SYS);
          STORE_GAP();
        }
        DRAIN();
      }
      __builtin_amdgcn_raw_buffer_store_b32(s9 + (f & 511u), rc, one, OFF_FLAG + par * FLAG_B + dn * 4u, AUX_SYS);
    }
    TSW(20 + wid);
    if (!same)
      for (int T = ((link >> 1) * 8 + wgi) * NUW + wid - 2; T < NUNIT * 32; T += NXW) unpack_block(P, ws, seq, T, false, lb);
    return seq;
  }
  if (wid == 1) {
    const unsigned fb = OFF_FLAG + par * FLAG_B;
    const int Q = same ? QSAME : QCROSS, QI = same ? QSAME : QIN;
    const unsigned* ord = (const unsigned*)P.ord + (same ? 0 : QSAME / 2);
    const unsigned qbase = (seq - 1u) * (unsigned)(Q - NDIR) - (NDIR + 16u);
    unsigned fA = 0, fB = 0, fC = 0;
    int rdy = 0;
    int p = NDIR + wgi, pn = NDIR + 8 + wgi;
    bool late = false;
    unsigned e = 0;
    unsigned k1 = 0;
    int held = 0;
    for (int n = 0;; ++n) {
      const bool live = (unsigned)p < (unsigned)Q;
      const unsigned tk = (unsigned)__builtin_amdgcn_raw_ptr_buffer_atomic_add_i32(1, rc, live && k1 == 0u ? one : OOB, OFF_QUEUE + link * 128, 16);
      const unsigned en = __builtin_amdgcn_raw_buffer_load_b32(rc, 0, (unsigned)(pn - QI) < (unsigned)(Q - QI) ? OFF_ULIST + (unsigned)(peer >> 1) * 256u + (unsigned)(pn - QIN) / 2u * 4u : OOB, AUX_SYS);
      u32x4 rec = {0u, 255u, 0u, 0u};
      unsigned so = OOB, kb = 16, so4 = OOB, so5 = OOB;
      bool part = false;
      if (live && p < QI) {
        unsigned o = ord[p >> 1] >> ((p & 1) * 16), sel = (o >> 8) & 7u, pc = o & 255u;
        const bool hop = sel == 7u, fwd = !same && sel >= 4u && !hop;
        if (hop) pc = ((pc & 63u) + (peer >> 1)) * 4u;
        if (fwd) {
          const unsigned c = sel - (sel - 4u >= (unsigned)peer >> 1 ? 3u : 4u), sn = SLOT_REL * 256u + c * 64u + (pc >> 6) * 16u + ((pc & 63u) >> 2), dn = (SLOT_V + (pc >> 6)) * 256u + ((pc & 63u) + me) * 4u + c;
          DRAIN();
          cnt[10] = n;
          unsigned f;
          const unsigned long long t0 = __builtin_amdgcn_s_memrealtime();
          for (;;) {
            asm volatile("" ::: "memory");
            f = RFL(__builtin_amdgcn_raw_buffer_load_b32(rc, 0, fb + (OFF_SFLAG - OFF_FLAG) + sn * 4u, AUX_SYS));
            if ((int)(f - (seq << 18)) >= 0 || timed_out(tmo, seq, t0)) break;
            __builtin_amdgcn_s_sleep(4);
          }
          f = RFL(f);
          so = par * PAR_B + sn * PIECE_B;
          rec = (u32x4){par * PAR_B + dn * PIECE_B, 1024u + ((f >> 9) & 511u), fb + (OFF_SFLAG - OFF_FLAG) + dn * 4u, (seq << 18) + (f & 511u)};
        } else {
        if (rdy <= (int)(pc >> 2)) {
          DRAIN();
          cnt[10] = n;
          const unsigned long long t0 = __builtin_amdgcn_s_memrealtime();
          for (;;) {
            asm volatile("" ::: "memory");
            const unsigned a = __builtin_amdgcn_raw_buffer_load_b32(rc, voff, fb + sA * 1024u, AUX_SYS);
            const unsigned b = __builtin_amdgcn_raw_buffer_load_b32(rc, voff, fb + sB * 1024u, AUX_SYS);
            const unsigned c = __builtin_amdgcn_raw_buffer_load_b32(rc, voff, fb + sC * 1024u + me * 4u, AUX_SYS);
            const unsigned long long miss = __ballot((int)((a - s9) | (b - s9) | (c - s9)) < 0);
            rdy = miss ? __builtin_ctzll(miss) : NPAIR;
            fA = RFL(a); fB = RFL(b); fC = RFL(c);
            if (rdy > (int)(pc >> 2)) break;
            if (timed_out(tmo, seq, t0)) { rdy = NPAIR; fA = fB = fC = s9; break; }
            __builtin_amdgcn_s_sleep(4);
          }
          rdy = __builtin_amdgcn_readfirstlane(rdy);
        }
        const unsigned dn = hop ? SLOT_REL * 256u + me * 64u + ((o >> 6) & 3u) * 16u + (pc >> 4) : (sel >= 2 ? dC + sel - 2 : sel ? dB : dA) * 256u + pc;
        const unsigned fv = sel >= 2 ? fC : sel ? fB : fA;
        if (o & 0x800u) {
          so = par * PAR_B + ((sel >= 2 ? sC + sel - 2 : sel ? sB : sA) * 256u + pc) * PIECE_B;
          rec = (u32x4){par * PAR_B + dn * PIECE_B, 16u, fb + dn * 4u, fv};
        } else {
          const unsigned sp = sel >= 2 ? 512u + (pc & 252u) + (same ? sel - 2 : hop ? (o >> 6) & 3u : 3u) : (same ? sel : sel ? 3u : 4u + (peer >> 1)) * 256u + pc;
          so = OFF_SH + sp * PIECE_B;
          so4 = OFF_INFO + sp * 16u;
          rec = (u32x4){par * PAR_B + dn * PIECE_B, 64u, fb + (OFF_SFLAG - OFF_FLAG) + dn * 4u, (seq << 18) + (fv & 511u)};
        }
        }
      } else if (live) {
        const unsigned i = (unsigned)(p - QIN) >> 1, j = (unsigned)p & 1u;
        const unsigned eo = OFF_ULIST + (unsigned)(peer >> 1) * 256u + i * 4u;
        if ((e >> 16) != (seq & 0xffffu)) {
          DRAIN();
          cnt[10] = n;
          const unsigned long long t0 = __builtin_amdgcn_s_memrealtime();
          for (;;) {
            asm volatile("" ::: "memory");
            e = RFL(__builtin_amdgcn_raw_buffer_load_b32(rc, 0, eo, AUX_SYS));
            if ((e >> 16) == (seq & 0xffffu) || timed_out(tmo, seq, t0)) break;
            __builtin_amdgcn_s_sleep(4);
          }
          e = RFL(e);
        }
        unsigned nk = (e >> 8) & 127u;
        nk = nk > 68u ? 68u : nk;
        const unsigned qk = (nk + 1u) >> 1, k0 = j * qk + k1;
        kb = (j ? nk : qk) - k0;
        part = kb > 18u && k1 == 0u;
        kb = kb > 18u ? 18u : kb;
        so = OFF_OUTS + ((unsigned)(peer >> 1) * 64u + i) * UREC_B + k0 * 1024u;
        so4 = kb > 16u ? so + 16384u : OOB;
        so5 = kb > 17u ? so4 : OOB;
        rec = (u32x4){OFF_RCV + ((unsigned)me * 64u + i) * UREC_B + k0 * 1024u, kb,
                      part ? fb + NSLOT * 1024u : OFF_DONE + (((unsigned)me * 64u + i) * 4u + j) * 4u, s9 + (e & 63u)};
        TR((unsigned)(peer >> 1) * 64u + i, 1 + j, NOW());
      }
      if (!late && n - (int)RFL(cnt[11]) >= RING) {
        const unsigned long long t0 = __builtin_amdgcn_s_memrealtime();
        while (n - (int)RFL(cnt[11]) >= RING) {
          if (__builtin_amdgcn_s_memrealtime() - t0 > DEADLINE) { late = timed_out(tmo, seq, t0); break; }
          __builtin_amdgcn_s_sleep(2);
        }
      }
      so = RFL(so);
      so4 = RFL(so4);
      so5 = RFL(so5);
      kb = RFL(kb);
      const int sl = n & (RING - 1);
      if (lane == 0) *(LAS u32x4*)(red + 16 + sl * 4) = rec;
#pragma unroll
      for (int i = 0; i < 5; ++i) {
        LAS void* d = (LAS void*)(lds + sl * RSLOT_B + i * 4096);
        const unsigned sg = i == 4 ? so4 : i * 4u < kb ? so + i * 4096u : OOB;
        __builtin_amdgcn_raw_ptr_buffer_load_lds(rs, d, 16, voff, sg, 0, AUX_SYS);
        if (i < 4) {
          __builtin_amdgcn_raw_ptr_buffer_load_lds(rs, d, 16, voff, sg, 1024, AUX_SYS);
          __builtin_amdgcn_raw_ptr_buffer_load_lds(rs, d, 16, voff, sg, 2048, AUX_SYS);
          __builtin_amdgcn_raw_ptr_buffer_load_lds(rs, d, 16, voff, sg, 3072, AUX_SYS);
        } else {
          __builtin_amdgcn_raw_ptr_buffer_load_lds(rs, d, 16, voff, so5, 1024, AUX_SYS);
        }
      }
      WAIT_V(20);
      WAIT_L(0);
      cnt[10] = live ? n : n + 1;
      if (!live) break;
      const int t = (int)(RFL(tk) - qbase);
      const unsigned er = RFL(en);
      if (RFL(part) != 0u) {
        held = t;
        k1 = 18;
      } else {
        p = pn;
        e = er;
        pn = k1 ? held : t;
        k1 = 0;
      }
    }
    return seq;
  }
  const unsigned spare = OFF_FLAG + par * FLAG_B + NSLOT * 1024u;
  unsigned fo = spare, fv = 0;
#define PFLAG() __builtin_amdgcn_raw_buffer_store_b32(fv, rp, one, fo, AUX_SYS)
  for (int n = 0;; ++n) {
    if ((int)RFL(cnt[10]) <= n) {
      DRAIN();
      PFLAG();
      fo = spare;
      const unsigned long long t0 = __builtin_amdgcn_s_memrealtime();
      bool dead = false;
      while ((int)RFL(cnt[10]) <= n) {
        if (__builtin_amdgcn_s_memrealtime() - t0 > 2 * DEADLINE) { dead = true; break; }
        __builtin_amdgcn_s_sleep(1);
      }
      if (dead) break;
    }
    const int sl = n & (RING - 1);
    const u32x4 rec = *(const LAS u32x4*)(red + 16 + sl * 4);
    u32x4 buf[18];
#pragma unroll
    for (int i = 0; i < 18; ++i) buf[i] = *(const LAS u32x4*)(lds + sl * RSLOT_B + i * 1024 + lane * 16);
    WAIT_L(0);
    cnt[11] = n + 1;
    const unsigned dst = RFL(rec[0]);
    unsigned kb = RFL(rec[1]), ne = 0, v14 = OOB;
    if (kb == 255u) break;
    if (kb >= 64u) {
      ne = (kb == 64u ? RFL(buf[16][0]) : kb) & 511u;
      kb = ne > SH_MAX ? 16u : 14u;
      if (ne <= SH_MAX && (unsigned)(lane >> 2) * 32u < ne) v14 = voff;
    }
#define PIECE(W) do { _Pragma("unroll") for (int i = 0; i < 18; ++i) { \
      if (i < (int)kb) { \
        __builtin_amdgcn_raw_buffer_store_b128(buf[i], rd, voff, dst + i * 1024u, AUX_SYS); \
        WAIT_V(W + (i >= W && i < 2 * W ? 2 : 0)); \
        if (i == W - 1) { \
          PFLAG(); \
          __builtin_amdgcn_raw_buffer_store_b128(buf[14], rd, v14, dst + SH_EX, AUX_SYS); \
        } \
      } \
    } \
    if (kb < W) { DRAIN(); PFLAG(); } } while (0)
    if (dst >= OFF_OUTS) PIECE(PU); else if ((int)((unsigned)__builtin_amdgcn_s_memrealtime() - ts0) > (same ? AG_AT0 + n * AG_ATR : AG_XT0 + n * AG_XTR)) PIECE(AG_P2); else PIECE(PL);
#undef PIECE
    fo = RFL(rec[2]);
    fv = RFL(rec[3]) + (ne << 9);
  }
  DRAIN();
  PFLAG();
#undef PFLAG
  return seq;
}

__device__ __forceinline__ int flag_exp(unsigned char* ws, unsigned seq, int slot, int rg) {
  return (int)(ld_sys((const unsigned*)(ws + OFF_FLAG + (seq & 1u) * FLAG_B) + slot * 256 + rg) & 511u) - 256;
}
__device__ __forceinline__ bool call_bad(unsigned char* ws, unsigned seq) {
  return RFL(__hip_atomic_load((unsigned*)(ws + OFF_TMO), __ATOMIC_RELAXED, __HIP_MEMORY_SCOPE_AGENT)) == seq;
}

__device__ __forceinline__ void store_block(const __amdgpu_buffer_rsrc_t ro, f32x4 (&acc)[4][2], int e, bool bad, unsigned so, int lane) {
  const unsigned vo = (unsigned)(lane & 15) * (NLOC * 4u) + (unsigned)(lane >> 4) * 16u;
#pragma unroll
  for (int m = 0; m < 4; ++m)
#pragma unroll
    for (int n = 0; n < 2; ++n) {
#pragma unroll
      for (int j = 0; j < 4; ++j) {
        const float t = __builtin_amdgcn_ldexpf(acc[m][n][j], e);
        acc[m][n][j] = bad ? __builtin_nanf("") : (t < 0.f ? 0.f : t);
      }
      asm volatile("" : "+v"(acc[m][n]));
    }
  SCHED;
#pragma unroll
  for (int m = 0; m < 4; ++m)
#pragma unroll
    for (int n = 0; n < 2; ++n)
      __builtin_amdgcn_raw_buffer_store_b128(__builtin_bit_cast(u32x4, acc[m][n]), ro, vo + n * 64, so + m * 16 * (NLOC * 4u), AG_OAUX);
  STORE_GAP();
}

__device__ __forceinline__ unsigned unit_ticket(unsigned char* ws, int c, bool take) {
  const __amdgpu_buffer_rsrc_t rc = __builtin_amdgcn_make_buffer_rsrc(ws + OFF_UHEAD, 0, 512, RSRC_W3);
  return (unsigned)__builtin_amdgcn_raw_ptr_buffer_atomic_add_i32(1, rc, take && threadIdx.x == 0 ? 0u : OOB, c * 128, 16);
}
__device__ __forceinline__ unsigned lds_add(LAS unsigned* p, unsigned v) {
  unsigned r = 0;
  if ((threadIdx.x & 63) == 0) r = __hip_atomic_fetch_add(p, v, __ATOMIC_RELAXED, __HIP_MEMORY_SCOPE_WORKGROUP);
  return RFL(r);
}
__device__ __forceinline__ unsigned gather_block(f32x4 (&acc)[4][2], int e, bool bad, LAS unsigned char* cb, unsigned& mk) {
  if (bad) {
    asm volatile("");
#pragma unroll
    for (int i = 0; i < 8; ++i) acc[i >> 1][i & 1] = (f32x4){__builtin_nanf(""), __builtin_nanf(""), __builtin_nanf(""), __builtin_nanf("")};
  }
  const unsigned b0 = RFL((unsigned)(unsigned long long)cb);
  unsigned run = b0;
  mk = 0;
#pragma unroll
  for (int i = 0; i < 32; i += 2) {
    float t0, t1;
    unsigned p0, p1, c0;
    asm volatile(
        "v_ldexp_f32 %0, %7, %9\n\t"
        "v_ldexp_f32 %1, %8, %9\n\t"
        "v_cmp_nge_f32 vcc, 0, %0\n\t"
        "v_cmp_nge_f32 s[92:93], 0, %1\n\t"
        "s_bcnt1_i32_b64 %4, vcc\n\t"
        "v_mbcnt_lo_u32_b32 %2, vcc_lo, 0\n\t"
        "v_mbcnt_lo_u32_b32 %3, s92, 0\n\t"
        "v_mbcnt_hi_u32_b32 %2, vcc_hi, %2\n\t"
        "v_mbcnt_hi_u32_b32 %3, s93, %3\n\t"
        "v_addc_co_u32 %5, s[94:95], %5, %5, vcc\n\t"
        "v_lshl_add_u32 %2, %2, 2, %6\n\t"
        "s_lshl2_add_u32 %6, %4, %6\n\t"
        "v_addc_co_u32 %5, s[94:95], %5, %5, s[92:93]\n\t"
        "v_lshl_add_u32 %3, %3, 2, %6\n\t"
        "s_bcnt1_i32_b64 %4, s[92:93]\n\t"
        "s_mov_b64 exec, vcc\n\t"
        "ds_write_b32 %2, %0\n\t"
        "s_mov_b64 exec, s[92:93]\n\t"
        "ds_write_b32 %3, %1\n\t"
        "s_mov_b64 exec, -1\n\t"
        "s_lshl2_add_u32 %6, %4, %6"
        : "=&v"(t0), "=&v"(t1), "=&v"(p0), "=&v"(p1), "=&s"(c0), "+v"(mk), "+s"(run)
        : "v"(acc[i >> 3][(i >> 2) & 1][i & 3]), "v"(acc[i >> 3][(i >> 2) & 1][(i & 3) + 1]), "v"(e)
        : "vcc", "scc", "s92", "s93", "s94", "s95", "memory");
  }
  WAIT_L(0);
  return run - b0;
}
__device__ __forceinline__ void pack_block(unsigned char* ws, unsigned seq, int c, unsigned& tk, bool more, f32x4 (&acc)[4][2], int e, bool bad, LAS unsigned char* cb, LAS unsigned* u, int wid, int lane) {
  unsigned mk;
  const unsigned nb = gather_block(acc, e, bad, cb, mk), n64 = (nb + 63u) & ~63u;
  const unsigned off = lds_add(u + 1, n64);
  const unsigned vo = (unsigned)lane * 16u;
  u32x4 v[8];
#pragma unroll
  for (int i = 0; i < 8; ++i) v[i] = *(const LAS u32x4*)(cb + i * 1024 + vo);
  unsigned pos = tk - (seq - 1u) * NUNIT;
  if (threadIdx.x == 0) *(volatile LAS unsigned*)u = (unsigned)c * 64u + (pos < 63u ? pos : 63u) + 1u;
  while ((pos = RFL(*(volatile LAS unsigned*)u)) == 0u) __builtin_amdgcn_s_sleep(1);
  unsigned char* rec = ws + CTRL_B + OFF_OUTS + (size_t)(pos - 1u) * UREC_B;
  const __amdgpu_buffer_rsrc_t rh = __builtin_amdgcn_make_buffer_rsrc(rec, 0, UHDR_B, RSRC_W3);
  const __amdgpu_buffer_rsrc_t rv = __builtin_amdgcn_make_buffer_rsrc(rec + UHDR_B + off, 0, n64, RSRC_W3);
  tk = unit_ticket(ws, c, more);
  __builtin_amdgcn_raw_buffer_store_b32(mk, rh, (unsigned)lane * 4u, (unsigned)wid * 256u, AUX_SYS);
  __builtin_amdgcn_raw_buffer_store_b64((u32x2){off, nb}, rh, lane == 0 ? 0u : OOB, 2048u + (unsigned)wid * 8u, AUX_SYS);
#pragma unroll
  for (int i = 0; i < 8; ++i) __builtin_amdgcn_raw_buffer_store_b128(v[i], rv, vo + (i & 3) * 1024u, (i >> 2) * 4096u, AUX_SYS);
  STORE_GAP();
}
__device__ __forceinline__ void post_unit(unsigned char* ws, unsigned seq, LAS unsigned* u, int unit) {
  if (lds_add(u + 2, 1u) == 7u) {
    const unsigned pos = *(volatile LAS unsigned*)u - 1u, all = UHDR_B + *(volatile LAS unsigned*)(u + 1);
    if ((threadIdx.x & 63) == 0) st_sys((unsigned*)(ws + OFF_ULIST) + pos, (seq << 16) | ((all + 1023u) >> 10 << 8) | (unsigned)unit);
    TR(pos, 0, NOW()); TR(pos, 9, (unsigned long long)unit);
  }
}

#define AG_LAUX AUX_SYS
#define DMA2(ldsoff, rs, so) do { \
    __builtin_amdgcn_raw_ptr_buffer_load_lds(rs, (LAS void*)(lds + (ldsoff) + ldsw), 16, voff, so, 0, AG_LAUX); \
    __builtin_amdgcn_raw_ptr_buffer_load_lds(rs, (LAS void*)(lds + (ldsoff) + ldsw), 16, voff, so, 1024, AG_LAUX); } while (0)
#define MMA16(ac, At, Bt) do { __builtin_amdgcn_s_setprio(1); \
    _Pragma("unroll") for (int m = 0; m < 4; ++m) _Pragma("unroll") for (int n = 0; n < 2; ++n) _Pragma("unroll") for (int k = 0; k < 2; ++k) \
      ac[m][n] = __builtin_amdgcn_mfma_f32_16x16x32_f16(Bt[n][k], At[m][k], ac[m][n], 0, 0, 0); \
    __builtin_amdgcn_s_setprio(0); } while (0)

__device__ __forceinline__ int seen_flags(const u32x4 f, int sel, int p0, unsigned s9) {
  int old = 0;
#pragma unroll
  for (int j = 0; j < 4; ++j) old |= (int)((f[j] - s9) >> 31) << j;
  const unsigned long long m = __ballot(p0 + (int)(threadIdx.x & 31) >= NPAIR || (old & sel) == 0);
  const unsigned miss = ~((unsigned)m & (unsigned)(m >> 32));
  return p0 + (miss ? __builtin_ctz(miss) : 32);
}

__device__ __forceinline__ void gemm_full(const Params& P, unsigned char* ws, unsigned seq, int xs, int wsl, unsigned long long mask, int job, LAS unsigned char* lds) {
  const int tid = threadIdx.x, wid = __builtin_amdgcn_readfirstlane(tid >> 6), lane = tid & 63;
  const int wr = wid >> 2, wc = wid & 3, fr = lane & 15, fq = lane >> 4;
  const unsigned par = seq & 1u;
  const __amdgpu_buffer_rsrc_t ra = __builtin_amdgcn_make_buffer_rsrc(ws + CTRL_B + (size_t)par * PAR_B + (size_t)xs * SLOT_B, 0, SLOT_B, RSRC_W3);
  const __amdgpu_buffer_rsrc_t rb = __builtin_amdgcn_make_buffer_rsrc(ws + CTRL_B + (size_t)par * PAR_B + (size_t)wsl * SLOT_B, 0, SLOT_B, RSRC_W3);
  const __amdgpu_buffer_rsrc_t rf = __builtin_amdgcn_make_buffer_rsrc(ws + OFF_FLAG + RFL(par * FLAG_B), 0, FLAG_B, RSRC_W3);
  const unsigned fo = (unsigned)(lane < 32 ? xs : wsl) * 1024u + (unsigned)(lane & 31) * 16u, fa = wid == 0 ? fo : OOB;
  const unsigned voff = (unsigned)wr * PIECE_B + (unsigned)wc * 2048u + (unsigned)lane * 16u;
  const unsigned ldsw = (unsigned)wid * 2048u;
  const int sw0 = (fq ^ (fr & 7)) * 16;
  const int aoff0 = (wr * 64 + fr) * 128 + sw0, aoff1 = aoff0 ^ 64;
  const int boff0 = 4 * HALF_B + (wc * 32 + fr) * 128 + sw0, boff1 = boff0 ^ 64;
#define SA(b, h) (((b) * 2 + (h)) * HALF_B)
#define SB(b, h) ((4 + (b) * 2 + (h)) * HALF_B)
#define STAGE(bufoff, rs, po, kk, h) DMA2(bufoff, rs, (po) + (kk) * KK_B + (h) * 2 * PIECE_B)
#define LDA(dst, b, h) do { _Pragma("unroll") for (int m = 0; m < 4; ++m) { \
    dst[m][0] = *(const LAS f16x8*)(lds + SA(b, h) + aoff0 + m * 2048); dst[m][1] = *(const LAS f16x8*)(lds + SA(b, h) + aoff1 + m * 2048); } } while (0)
#define LDB(dst, b, h) do { _Pragma("unroll") for (int n = 0; n < 2; ++n) { \
    dst[n][0] = *(const LAS f16x8*)(lds + SA(b, h) + boff0 + n * 2048); dst[n][1] = *(const LAS f16x8*)(lds + SA(b, h) + boff1 + n * 2048); } } while (0)
#define MMA(ai, bj, At, Bt) MMA16(acc[ai][bj], At, Bt)
  f32x4 acc[2][2][4][2];
#pragma unroll
  for (int a = 0; a < 2; ++a)
#pragma unroll
    for (int b = 0; b < 2; ++b)
#pragma unroll
      for (int m = 0; m < 4; ++m)
#pragma unroll
        for (int n = 0; n < 2; ++n) acc[a][b][m][n] = (f32x4){0.f, 0.f, 0.f, 0.f};
  f16x8 At[4][2], B0[2][2], B1[2][2];
#define NEXTP(c) do { c = mask ? __builtin_ctzll(mask) : NPAIR; mask &= mask - 1ull; } while (0)
  int avail = 0, c1, c2;
  NEXTP(c1);
  NEXTP(c2);
  if (wid == 0) avail = wait_flags(ws, fo, 15, c1, c1 + 1, seq);
  BAR;
  u32x4 ff = __builtin_amdgcn_raw_buffer_load_b128(rf, fa, (unsigned)c2 * 16u, AUX_SYS);
  SCHED;
  unsigned pn = (unsigned)c1 * PAIR_B;
  STAGE(SB(0, 0), rb, pn, 0, 0); STAGE(SA(0, 0), ra, pn, 0, 0); STAGE(SB(0, 1), rb, pn, 0, 1); STAGE(SA(0, 1), ra, pn, 0, 1);
  if (wr == 1) BAR;
  WAIT_V(4); BAR;
  STAGE(SB(1, 0), rb, pn, 1, 0); STAGE(SA(1, 0), ra, pn, 1, 0); STAGE(SB(1, 1), rb, pn, 1, 1);
  WAIT_V(6); BAR;
  do {
    const unsigned po = pn;
    c1 = c2;
    NEXTP(c2);
    pn = (unsigned)c1 * PAIR_B;
    if (wid == 0 && c1 >= avail && c1 < NPAIR) {
      avail = seen_flags(ff, 15, c1, seq << 9);
      if (c1 >= avail) avail = wait_flags(ws, fo, 15, c1, c1 + 1, seq);
    }
    LDB(B0, 0, 0); SCHED; LDA(At, 0, 0); STAGE(SA(1, 1), ra, po, 1, 1);
    WAIT_V(10); WAIT_L(8); BAR; WAIT_L(0); MMA(0, 0, At, B0); BAR; SCHED;
    LDB(B1, 0, 1); STAGE(SB(0, 0), rb, pn, 0, 0);
    WAIT_V(10); BAR; WAIT_L(0); MMA(0, 1, At, B1); BAR;
    LDA(At, 0, 1);
    ff = __builtin_amdgcn_raw_buffer_load_b128(rf, fa, (unsigned)c2 * 16u, AUX_SYS);
    STAGE(SA(0, 0), ra, pn, 0, 0);
    BAR; WAIT_L(0); MMA(1, 0, At, B0); BAR; SCHED;
    STAGE(SB(0, 1), rb, pn, 0, 1);
    WAIT_V(11); BAR; MMA(1, 1, At, B1); BAR;
    LDB(B0, 1, 0); SCHED; LDA(At, 1, 0); STAGE(SA(0, 1), ra, pn, 0, 1);
    WAIT_V(11); WAIT_L(8); BAR; WAIT_L(0); MMA(0, 0, At, B0); BAR; SCHED;
    LDB(B1, 1, 1); STAGE(SB(1, 0), rb, pn, 1, 0);
    WAIT_V(11); BAR; WAIT_L(0); MMA(0, 1, At, B1); BAR;
    LDA(At, 1, 1); STAGE(SA(1, 0), ra, pn, 1, 0);
    BAR; WAIT_L(0); MMA(1, 0, At, B0); BAR; SCHED;
    STAGE(SB(1, 1), rb, pn, 1, 1);
    WAIT_V(10); BAR; MMA(1, 1, At, B1); BAR;
  } while (c1 < NPAIR);
#undef NEXTP
  asm volatile("" :: "v"(ff));
  const bool remote = wsl >= SLOT_WR;
  int tz = threadIdx.x; asm volatile("" : "+v"(tz));
  if (job >= 0) {
    const __amdgpu_buffer_rsrc_t rq = __builtin_amdgcn_make_buffer_rsrc(ws + CTRL_B + OFF_PART + job * 262144u + wid * 8192u, 0, 3 * 65536 + 8192, RSRC_W3);
#pragma unroll
    for (int i = 0; i < 32; ++i) asm volatile("" : "+v"(acc[i >> 4][(i >> 3) & 1][(i >> 1) & 3][i & 1]));
    SCHED;
#pragma unroll
    for (int i = 0; i < 32; ++i)
      __builtin_amdgcn_raw_buffer_store_b128(__builtin_bit_cast(u32x4, acc[i >> 4][(i >> 3) & 1][(i >> 1) & 3][i & 1]), rq, (unsigned)(tz & 63) * 16u + (i & 3) * 1024u, (i >> 3) * 65536u + (i & 4) * 1024u, AUX_SYS);
    STORE_GAP();
    DRAIN();
    if (wr == 0) BAR;
    BAR;
    if (tz == 0) st_sys((unsigned*)(ws + OFF_PFL) + job, seq);
    TS(29);
    return;
  }
  LAS unsigned* red = (LAS unsigned*)(lds + LDS_MAIN);
  unsigned tk = unit_ticket(ws, wsl - SLOT_WR, remote);
  const int eb = flag_exp(ws, seq, wsl, 0);
  const int e0 = flag_exp(ws, seq, xs, wr) + eb, e1 = flag_exp(ws, seq, xs, 2 + wr) + eb;
  const bool bad = call_bad(ws, seq);
  WAIT_V(0);
  WAIT_L(0);
  if (wr == 0) BAR;
  if (remote) {
    BAR;
    const int u0 = (xs < SLOT_G3 ? xs : xs - 1) * 4;
#pragma unroll
    for (int q = 0; q < 4; ++q) {
      pack_block(ws, seq, wsl - SLOT_WR, tk, q < 3, acc[q >> 1][q & 1], (q >> 1) ? e1 : e0, bad, lds + wid * 8192, red + 16 + q * 4, wid, tz & 63);
      if (q) { WAIT_V(11); post_unit(ws, seq, red + 12 + q * 4, u0 + q - 1); }
    }
    DRAIN();
    post_unit(ws, seq, red + 28, u0 + 3);
  } else {
    const __amdgpu_buffer_rsrc_t ro = __builtin_amdgcn_make_buffer_rsrc(P.out[P.rank], 0, 8192 * NLOC * 4, RSRC_W3);
#pragma unroll
    for (int q = 0; q < 4; ++q)
      store_block(ro, acc[q >> 1][q & 1], (q >> 1) ? e1 : e0, bad,
                  (unsigned)(xs == SLOT_T3 ? P.rank * 1024 + 768 + ((q >> 1) * 2 + wr) * 64 : group_row0(P.rank & 1, xs, (q >> 1) * 2 + wr)) * (NLOC * 4u)
                  + (unsigned)(wsl - SLOT_W) * 1024u + (q & 1) * 512u + wc * 128u, tz & 63);
  }
#undef SA
#undef SB
#undef STAGE
#undef LDA
#undef LDB
#undef MMA
}

__device__ __forceinline__ void gemm_half(const Params& P, unsigned char* ws, unsigned seq, int xs, int ah, int wsl, LAS unsigned char* lds) {
  const int tid = threadIdx.x, wid = __builtin_amdgcn_readfirstlane(tid >> 6), lane = tid & 63;
  const int wr = wid >> 2, wc = wid & 3, fr = lane & 15, fq = lane >> 4;
  const unsigned par = seq & 1u;
  const __amdgpu_buffer_rsrc_t ra = __builtin_amdgcn_make_buffer_rsrc(ws + CTRL_B + (size_t)par * PAR_B + (size_t)xs * SLOT_B + ah * 2 * PIECE_B, 0, SLOT_B - ah * 2 * PIECE_B, RSRC_W3);
  const __amdgpu_buffer_rsrc_t rb = __builtin_amdgcn_make_buffer_rsrc(ws + CTRL_B + (size_t)par * PAR_B + (size_t)wsl * SLOT_B, 0, SLOT_B, RSRC_W3);
  const __amdgpu_buffer_rsrc_t rf = __builtin_amdgcn_make_buffer_rsrc(ws + OFF_FLAG + RFL(par * FLAG_B), 0, FLAG_B, RSRC_W3);
  const unsigned fo = (unsigned)(lane < 32 ? xs : wsl) * 1024u + (unsigned)(lane & 31) * 16u, fa = wid == 0 ? fo : OOB;
  const int sel = lane < 32 ? 3 << (2 * ah) : 15;
  const unsigned voff = (unsigned)wr * PIECE_B + (unsigned)wc * 2048u + (unsigned)lane * 16u;
  const unsigned ldsw = (unsigned)wid * 2048u;
  const int sw0 = (fq ^ (fr & 7)) * 16;
  int aoff0 = (wr * 64 + fr) * 128 + sw0, aoff1 = aoff0 ^ 64;
  const int boff0 = 4 * HALF_B + (wc * 32 + fr) * 128 + sw0, boff1 = boff0 ^ 64;
#define SB(b, h) ((4 + (b) * 2 + (h)) * HALF_B)
#define STAGE(bufoff, rs, po, kk, h) DMA2(bufoff, rs, (po) + (kk) * KK_B + (h) * 2 * PIECE_B)
#define STA(kk, i) __builtin_amdgcn_raw_ptr_buffer_load_lds(ra, (LAS void*)(lds + an + (kk) * 2 * HALF_B + ldsw), 16, voff, pn + (kk) * KK_B, (i) * 1024, AG_LAUX)
#define LDA(dst, b) do { _Pragma("unroll") for (int m = 0; m < 4; ++m) { \
    dst[m][0] = *(const LAS f16x8*)(lds + (b) * 2 * HALF_B + aoff0 + m * 2048); dst[m][1] = *(const LAS f16x8*)(lds + (b) * 2 * HALF_B + aoff1 + m * 2048); } } while (0)
#define LDB(dst, b, h) do { _Pragma("unroll") for (int n = 0; n < 2; ++n) { \
    dst[n][0] = *(const LAS f16x8*)(lds + ((b) * 2 + (h)) * HALF_B + boff0 + n * 2048); dst[n][1] = *(const LAS f16x8*)(lds + ((b) * 2 + (h)) * HALF_B + boff1 + n * 2048); } } while (0)
  f32x4 acc[2][4][2];
#pragma unroll
  for (int b = 0; b < 2; ++b)
#pragma unroll
    for (int m = 0; m < 4; ++m)
#pragma unroll
      for (int n = 0; n < 2; ++n) acc[b][m][n] = (f32x4){0.f, 0.f, 0.f, 0.f};
  f16x8 At[4][2], Bt[2][2];
  int avail = 0;
  if (wid == 0) avail = wait_flags(ws, fo, sel, 0, 1, seq);
  BAR;
  u32x4 ff = __builtin_amdgcn_raw_buffer_load_b128(rf, fa, 16, AUX_SYS);
  SCHED;
  unsigned an = 0, pn = 0;
  STAGE(SB(0, 0), rb, 0u, 0, 0); STA(0, 0); STA(0, 1); STAGE(SB(0, 1), rb, 0u, 0, 1);
  if (wr == 1) BAR;
  WAIT_V(2); BAR;
  STAGE(SB(1, 0), rb, 0u, 1, 0); STA(1, 0); STA(1, 1);
  WAIT_V(4); BAR;
  for (int t = 0; t < NPAIR; ++t) {
    const unsigned po = pn;
    pn += PAIR_B;
    an ^= HALF_B;
    if (wid == 0 && t + 2 > avail && t + 2 <= NPAIR) {
      avail = seen_flags(ff, sel, t + 1, seq << 9);
      if (t + 2 > avail) avail = wait_flags(ws, fo, sel, t + 1, t + 2, seq);
    }
    LDB(Bt, 0, 0); SCHED; LDA(At, 0); STAGE(SB(1, 1), rb, po, 1, 1);
    WAIT_V(5); WAIT_L(8); BAR; WAIT_L(0); MMA16(acc[0], At, Bt); BAR; SCHED;
    LDB(Bt, 0, 1); STA(0, 0); STA(0, 1); STAGE(SB(0, 0), rb, pn, 0, 0);
    WAIT_V(6); BAR; WAIT_L(0); MMA16(acc[1], At, Bt); BAR; SCHED;
    LDB(Bt, 1, 0); SCHED; LDA(At, 1);
    asm volatile("" :: "v"(ff));
    ff = __builtin_amdgcn_raw_buffer_load_b128(rf, fa, (unsigned)(t + 2) * 16u, AUX_SYS);
    STA(1, 0); STAGE(SB(0, 1), rb, pn, 0, 1);
    WAIT_V(8); WAIT_L(8); BAR; WAIT_L(0); MMA16(acc[0], At, Bt); BAR; SCHED;
    LDB(Bt, 1, 1); STA(1, 1); STAGE(SB(1, 0), rb, pn, 1, 0);
    WAIT_V(7); BAR; WAIT_L(0); MMA16(acc[1], At, Bt); BAR; SCHED;
    aoff0 ^= HALF_B;
    aoff1 ^= HALF_B;
  }
  asm volatile("" :: "v"(ff));
  int tz = threadIdx.x; asm volatile("" : "+v"(tz));
  LAS unsigned* red = (LAS unsigned*)(lds + LDS_MAIN);
  unsigned tk = unit_ticket(ws, wsl - SLOT_WR, true);
  const int e = flag_exp(ws, seq, xs, ah * 2 + wr) + flag_exp(ws, seq, wsl, 0);
  const bool bad = call_bad(ws, seq);
  WAIT_V(0);
  WAIT_L(0);
  if (wr == 0) BAR;
  BAR;
  const int u0 = (xs < SLOT_G3 ? xs : xs - 1) * 4 + ah * 2;
  pack_block(ws, seq, wsl - SLOT_WR, tk, true, acc[0], e, bad, lds + wid * 8192, red + 16, wid, tz & 63);
  pack_block(ws, seq, wsl - SLOT_WR, tk, false, acc[1], e, bad, lds + wid * 8192, red + 20, wid, tz & 63);
  WAIT_V(11);
  post_unit(ws, seq, red + 16, u0);
  DRAIN();
  post_unit(ws, seq, red + 20, u0 + 1);
#undef SB
#undef STAGE
#undef STA
#undef LDA
#undef LDB
}

#define AG_DW 2
constexpr int NBQ = 4;
#define AG_RT 6
#define AG_RO 1
constexpr unsigned LDS_F = NBQ * KSTEP_B;
#define WAIT_L0() __builtin_amdgcn_s_waitcnt(0xC07F)
__device__ __forceinline__ void gemm_quarter(const Params& P, unsigned char* ws, unsigned seq, int xs, int ah, int wsl, int bh, unsigned long long mask, int job, int np, LAS unsigned char* lds) {
  const int tid = threadIdx.x, w8 = __builtin_amdgcn_readfirstlane(tid >> 6), lane = tid & 63;
  seq = RFL(seq);
  xs = (int)RFL(xs); wsl = (int)RFL(wsl); ah = (int)RFL(ah); bh = (int)RFL(bh); job = (int)RFL(job); np = (int)RFL(np);
  const unsigned par = seq & 1u;
  const int nk = (int)RFL(2 * __builtin_popcountll(mask));
  const int cq = ah * 2 + bh;
  f32x4 acc[4][2];
  LAS unsigned char* xb = lds + 3 * KSTEP_B + (w8 & 3) * 8192 + lane * 16;
#define NEXTB(o) o = o == (NBQ - 1) * KSTEP_B ? 0u : o + KSTEP_B
  if (w8 >= 4) {
    const __amdgpu_buffer_rsrc_t ra = __builtin_amdgcn_make_buffer_rsrc(ws + CTRL_B + (size_t)par * PAR_B + (size_t)xs * SLOT_B + ah * 2 * PIECE_B, 0, SLOT_B - ah * 2 * PIECE_B, RSRC_W3);
    const __amdgpu_buffer_rsrc_t rb = __builtin_amdgcn_make_buffer_rsrc(ws + CTRL_B + (size_t)par * PAR_B + (size_t)wsl * SLOT_B + bh * 2 * PIECE_B, 0, SLOT_B - bh * 2 * PIECE_B, RSRC_W3);
    const unsigned fo = (unsigned)(lane < 32 ? xs : wsl) * 1024u + (unsigned)(lane & 31) * 16u;
    const int g = (w8 >> 1) & 1, sel = 1 << (2 * (lane < 32 ? ah : bh) + g);
    const volatile LAS unsigned* lf = (const volatile LAS unsigned*)(lds + LDS_F + lane * 16);
    const volatile LAS unsigned* lp = (const volatile LAS unsigned*)(lds + LDS_F + 2048 + (job + (lane & 3)) * 4);
    const __amdgpu_buffer_rsrc_t rq = __builtin_amdgcn_make_buffer_rsrc(ws + CTRL_B + OFF_PART + job * 262144u + cq * 65536u + (w8 & 3) * 16384u, 0, 2 * 262144 + 16384, RSRC_W3);
    const unsigned voff = (unsigned)((w8 >> 1) & 1) * PIECE_B + (unsigned)(w8 & 1) * 4096u + (unsigned)lane * 16u, ldsw = (unsigned)(w8 & 3) * 4096u;
    const int rk = __builtin_popcountll(mask & ((1ull << lane) - 1ull));
    const int at = !((mask >> lane) & 1ull) ? 63 : rk < ((nk / 2 - AG_RT) & ~3) ? (rk & ~3) | ((rk - AG_RO * (cq + xs)) & 3) : rk;
    const int tab = __builtin_amdgcn_ds_permute(at * 4, lane);
#define PAIR(i) __builtin_amdgcn_readlane(tab, i)
#define DMA4(lo, rs, vo, so) do { LAS void* _d = (LAS void*)(lds + (lo)); \
    __builtin_amdgcn_raw_ptr_buffer_load_lds(rs, _d, 16, vo, so, 0, AG_LAUX); __builtin_amdgcn_raw_ptr_buffer_load_lds(rs, _d, 16, vo, so, 1024, AG_LAUX); \
    __builtin_amdgcn_raw_ptr_buffer_load_lds(rs, _d, 16, vo, so, 2048, AG_LAUX); __builtin_amdgcn_raw_ptr_buffer_load_lds(rs, _d, 16, vo, so, 3072, AG_LAUX); } while (0)
#define FILL(top) for (; ki < (top) && ((have >> pr) & 1ull); ++ki) { \
      const unsigned _so = (unsigned)pr * PAIR_B + (unsigned)(ki & 1) * KK_B; \
      DMA4(bo + ldsw, ra, voff, _so); DMA4(bo + HALF_B + ldsw, rb, voff, _so); \
      pr = PAIR(((ki + 1) >> 1) & 63); \
      NEXTB(bo); }
#define PARTS(rdy) do { _Pragma("unroll") for (int q = 0; q < 3; ++q) if (((rdy) >> q) & 1u) { \
      _Pragma("unroll") for (int i = 0; i < 16; ++i) pt[q][i] = __builtin_amdgcn_raw_buffer_load_b128(rq, lane * 16u + (i & 3) * 1024u, q * 262144u + (i >> 2) * 4096u, AUX_SYS); \
      kq[q] = ki; } \
      owed &= ~(rdy); } while (0)
    *(LAS u32x2*)(lds + LDS_F + (tid - 256) * 8) = (u32x2){(seq - 1u) << 9, (seq - 1u) << 9};
    *(LAS unsigned*)(lds + LDS_F + 2048 + (tid - 256) * 4) = seq - 1u;
    WAIT_L(0);
    BAR;
    unsigned long long have = 0;
    int ki = 0, pr = PAIR(0);
    unsigned bo = 0;
    u32x4 pt[3][16];
#pragma unroll
    for (int i = 0; i < 48; ++i) pt[i >> 4][i & 15] = (u32x4){0u, 0u, 0u, 0u};
    unsigned owed = (1u << np) - 1u;
    int kq[3] = {0, 0, 0};
    for (int k = 0; k < nk; ++k) {
      if (ki == k) {
        if (!((have >> pr) & 1ull)) have |= (~0ull >> (NPAIR - __builtin_amdgcn_readfirstlane(wait_flags(ws, fo, sel, pr, pr + 1, seq)))) & (~0ull << pr);
        FILL(k + NBQ - 1 < nk ? k + NBQ - 1 : nk);
      }
      {
        const int y = ki - k - 1 + 2 * ((kq[0] > k) + (kq[1] > k) + (kq[2] > k));
        if (y == 2) WAIT_V(16);
        else if (y < 2) { if (y == 1) WAIT_V(8); else WAIT_V(0); }
        else if (y < 5) { if (y == 3) WAIT_V(24); else WAIT_V(32); }
        else if (y < 7) { if (y == 5) WAIT_V(40); else WAIT_V(48); }
        else WAIT_V(56);
      }
      BAR;
      FILL(k + NBQ < nk ? k + NBQ : nk);
      if (ki < nk && !((have >> pr) & (have >> PAIR(((ki >> 1) + 1) & 63)) & 1ull)) {
        have |= __ballot((int)((lf[2 * ah + g] - (seq << 9)) | (lf[256 + 2 * bh + g] - (seq << 9))) >= 0);
        FILL(k + NBQ < nk ? k + NBQ : nk);
      }
      if (owed) {
        const unsigned rdy = (unsigned)__ballot(*lp == seq) & owed;
        PARTS(rdy);
      }
    }
    if (owed) {
      const unsigned long long t0 = __builtin_amdgcn_s_memrealtime();
      do {
        unsigned rdy = (unsigned)__ballot(ld_sys((const unsigned*)(ws + OFF_PFL) + job + (lane & 3)) == seq) & owed;
        if (!rdy && timed_out((unsigned*)(ws + OFF_TMO), seq, t0)) rdy = owed;
        rdy = RFL(rdy);
        PARTS(rdy);
      } while (owed);
    }
#define F4(x) __builtin_bit_cast(f32x4, x)
    f32x4 sl[8];
#pragma unroll
    for (int i = 0; i < 8; ++i) {
      sl[i] = F4(pt[0][i]) + F4(pt[1][i]) + F4(pt[2][i]);
      acc[i >> 1][i & 1] = F4(pt[0][8 + i]) + F4(pt[1][8 + i]) + F4(pt[2][8 + i]);
    }
#undef F4
    WAIT_V(0);
    BAR;
#pragma unroll
    for (int i = 0; i < 8; ++i) *(LAS f32x4*)(xb + i * 1024 - KSTEP_B) = sl[i];
    WAIT_L(0);
    BAR;
#pragma unroll
    for (int i = 0; i < 8; ++i) acc[i >> 1][i & 1] += *(const LAS f32x4*)(xb + i * 1024);
  } else {
    const int fr = lane & 15, fq = lane >> 4;
    const int sw0 = (fq ^ (fr & 7)) * 16;
    const int aoff0 = ((w8 >> 1) * 64 + fr) * 128 + sw0, aoff1 = aoff0 ^ 64;
    const int boff0 = HALF_B + ((w8 & 1) * 64 + fr) * 128 + sw0, boff1 = boff0 ^ 64;
    f32x4 ac[4][4];
#pragma unroll
    for (int m = 0; m < 4; ++m)
#pragma unroll
      for (int n = 0; n < 4; ++n) ac[m][n] = (f32x4){0.f, 0.f, 0.f, 0.f};
    f16x8 A0[4][2], B0[4][2], A1[4][2], B1[4][2];
    const __amdgpu_buffer_rsrc_t rf = __builtin_amdgcn_make_buffer_rsrc(w8 == 2 ? ws + OFF_PFL : ws + OFF_FLAG + par * FLAG_B + (w8 ? wsl : xs) * 1024, 0, w8 == 3 ? 0 : 1024, RSRC_W3);
#define RD(A, B) do { _Pragma("unroll") for (int m = 0; m < 4; ++m) { \
      B[m][0] = *(const LAS f16x8*)(lds + bo + boff0 + m * 2048); A[m][0] = *(const LAS f16x8*)(lds + bo + aoff0 + m * 2048); } \
      _Pragma("unroll") for (int m = 0; m < 4; ++m) { \
      B[m][1] = *(const LAS f16x8*)(lds + bo + boff1 + m * 2048); A[m][1] = *(const LAS f16x8*)(lds + bo + aoff1 + m * 2048); } \
      NEXTB(bo); } while (0)
#define MM(A, B) do { _Pragma("unroll") for (int k = 0; k < 2; ++k) _Pragma("unroll") for (int m = 0; m < 4; ++m) _Pragma("unroll") for (int n = 0; n < 4; ++n) \
      ac[m][n] = __builtin_amdgcn_mfma_f32_16x16x32_f16(B[n][k], A[m][k], ac[m][n], 0, 0, 0); } while (0)
#define STEP(An, Bn, A, B, F) do { BAR; SCHED; if (F) __builtin_amdgcn_raw_ptr_buffer_load_lds(rf, (LAS void*)(lds + LDS_F + w8 * 1024), 16, lane * 16, 0, 0, AUX_SYS); RD(An, Bn); MM(A, B); \
      _Pragma("unroll") for (int i = 0; i < 16; ++i) { __builtin_amdgcn_sched_group_barrier(0x100, 1, 0); __builtin_amdgcn_sched_group_barrier(0x008, 1, 0); } \
      __builtin_amdgcn_sched_group_barrier(0x008, 16, 0); SCHED; WAIT_L0(); } while (0)
    unsigned bo = 0;
    BAR;
    BAR; SCHED;
    RD(A0, B0);
    WAIT_L0();
    for (int k = 1; k < nk - 1; k += 2) {
      STEP(A1, B1, A0, B0, 1);
      STEP(A0, B0, A1, B1, 0);
    }
    STEP(A1, B1, A0, B0, 0);
    MM(A1, B1);
    BAR;
#pragma unroll
    for (int i = 0; i < 8; ++i) *(LAS f32x4*)(xb + i * 1024) = ac[i >> 1][2 + (i & 1)];
    WAIT_V(0);
    WAIT_L(0);
    BAR;
#pragma unroll
    for (int i = 0; i < 8; ++i) acc[i >> 1][i & 1] = ac[i >> 1][i & 1] + *(const LAS f32x4*)(xb + i * 1024 - KSTEP_B);
#undef RD
#undef MM
#undef STEP
  }
  const int wr = (w8 >> 1) & 1, wc = (w8 & 1) * 2 + (w8 >> 2), wid = wr * 4 + wc;
  int tz = threadIdx.x; asm volatile("" : "+v"(tz));
  const int e = flag_exp(ws, seq, xs, ah * 2 + wr) + flag_exp(ws, seq, wsl, bh * 2);
  const bool bad = call_bad(ws, seq);
  if (wsl >= SLOT_WR) {
    unsigned mk;
    LAS unsigned char* cb = lds + w8 * 8192;
    const unsigned n64 = (gather_block(acc, e, bad, cb, mk) + 63u) & ~63u;
    const unsigned vo = (unsigned)(tz & 63) * 16u, vi = (unsigned)((((P.rank >> 1) * 3 + xs - SLOT_V) * 4 + ah * 2 + bh) * 8 + wid);
    unsigned char* pw = P.ws[2 * (wsl - SLOT_WR) + 1 - (P.rank & 1)];
    const __amdgpu_buffer_rsrc_t rv = __builtin_amdgcn_make_buffer_rsrc(pw + CTRL_B + OFF_VREC + vi * VREC_B, 0, 256u + n64, RSRC_W3);
    u32x4 v[8];
#pragma unroll
    for (int i = 0; i < 8; ++i) v[i] = *(const LAS u32x4*)(cb + i * 1024 + vo);
    __builtin_amdgcn_raw_buffer_store_b32(mk, rv, vo >> 2, 0, AUX_SYS);
#pragma unroll
    for (int i = 0; i < 8; ++i)
      if (i * 1024u < n64) {
        __builtin_amdgcn_raw_buffer_store_b128(v[i], rv, vo, 256u + i * 1024u, AUX_SYS);
        WAIT_V(AG_DW);
      }
    DRAIN();
    if ((tz & 63) == 0) st_sys((unsigned*)(pw + OFF_VFL) + vi, (seq << 9) + (n64 >> 6));
  } else {
    const __amdgpu_buffer_rsrc_t ro = __builtin_amdgcn_make_buffer_rsrc(P.out[P.rank], 0, 8192 * NLOC * 4, RSRC_W3);
    store_block(ro, acc, e, bad, (unsigned)group_row0(P.rank & 1, xs, ah * 2 + wr) * (NLOC * 4u) + (unsigned)(wsl - SLOT_W) * 1024u + bh * 512u + wc * 128u, tz & 63);
  }
#undef FILL
#undef PARTS
#undef PAIR
#undef DMA4
#undef NEXTB
}

#define AG_SWO 1
__global__ __launch_bounds__(512, 2) void ag_gemm_kernel(Params P) {
  extern __shared__ __attribute__((aligned(16))) unsigned char shm[];
  LAS unsigned char* lds = (LAS unsigned char*)shm;
  const int tid = threadIdx.x;
  const int b = blockIdx.x, xcd = b & 7, bi = b >> 3;
  const int rank = P.rank, me = rank >> 1;
  unsigned char* ws = P.ws[rank];
  TS(0);
  if (bi < NPUSH) {
    const unsigned seq = link_role(P, ws, bi, xcd, lds);
    if (tid >= 64) return;
    TS(9);
    return;
  }
  const int cp = xcd & 3, rh = xcd >> 2;
  unsigned seq;
  const int f = bi - NPUSH, j = f - NFULL;
  if (bi < 32 - NPREP) {
    LAS unsigned* red = (LAS unsigned*)(lds + LDS_MAIN);
    if (tid == 0) red[8] = take_seq(ws, xcd);
    if (tid >= 16 && tid < 32) red[tid] = 0;
    WAIT_L(0);
    BAR;
    seq = RFL(red[8]);
  } else {
    seq = prep_role(P, ws, xcd, 31 - bi, lds);
    TS(2);
  }
#define ROWS(ri) ((ri) < 2 ? SLOT_A + rh * 2 + (ri) : (ri) < 6 ? ((me + 1 + rh * 2 + ((ri) & 1)) & 3) * 2 + (((ri) >> 2) ^ (rh & (ri) & AG_SWO)) : SLOT_V3)
  const int w8 = __builtin_amdgcn_readfirstlane(tid >> 6);
  const int job = rh ? j - 4 : j == 11 ? 8 : j + 5;
  const int v = j >= 4 ? (job < 12 ? job / 3 % 3 : job / 2 - 5) : j >= 0 ? 2 : f & 1;
  const bool fin = j < 0 ? f >= 4 && f < 12 : j < 4;
  if (fin && (j >= 0 || (f & 1))) {
    const int ri = j < 0 ? f >> 1 : 2 + j;
    gemm_half(P, ws, seq, ROWS(ri), j >= 0, SLOT_WR + cp, lds);
  } else {
    const int ri = f >> 1;
    const int xs = j < 0 ? ROWS(ri) : SLOT_V + v;
    const bool far = f < 12 ? f & 1 : f == 12 ? rh : job < 9;
    const bool home = far && xs == SLOT_A + cp;
    gemm_full(P, ws, seq, home ? SLOT_T3 : xs, (far && !home ? SLOT_WR : SLOT_W) + cp, j < 0 ? ~0ull : P.pairs[job], j < 0 ? -1 : cp * 16 + job, lds);
  }
  if (fin) {
    const int cq = j >= 0 ? j : (f - 4) >> 1;
    WAIT_L(0);
    BAR;
    gemm_quarter(P, ws, seq, SLOT_V + v, cq >> 1, (rh ? SLOT_WR : SLOT_W) + cp, cq & 1, P.pairs[16 + rh * 3 + v], cp * 16 + (rh ? v * 3 : v ? 10 + v * 2 : 9), rh || !v ? 3 : 2, lds);
  }
  if (f < 4 || f == 12 || j == 11) {
    WAIT_L(0);
    BAR;
    unpack_block(P, ws, seq, ((f < 4 ? f : f == 12 ? 4 : 5) * 8 + xcd) * 8 + w8, true, lds + w8 * 8192);
  }
#undef ROWS
}

#define AG_FR0 2.16
#define AG_FR1 2.16
#define AG_E0 146.
#define AG_E1 146.
#define AG_S0 200.
#define AG_R0 0.8
#define AG_S1 200.
#define AG_R1 0.8
#define AG_S2 200.
#define AG_R2 0.8
#define AG_PE0 0
#define AG_PE1 0
#define AG_PE2 0
#define AG_VA 30.
#define AG_VB 160.
#define AG_TH 40.
#define AG_DH 1.
#define AG_TF 64.
#define AG_DF 0.1
#define AG_NA 24
#define AG_RV 3
#define AG_RM 1
#define AG_XW 8
#define AG_L0 2
#define AG_L1 2
#define AG_LV 16
#define AG_LW 2
#define AG_LA 2
#define AG_LG 0
#define AG_GD 0.
#define AG_H0 24
#define AG_H1 20
#define AG_H2 20
#define AG_HE0 56
#define AG_HE1 56
#define AG_HE2 56
#define AG_HF0 40
#define AG_HF1 40
#define AG_HF2 40
#define AG_G0 24
#define AG_G1 7
#define AG_G2 7
#define AG_GE0 58
#define AG_GE1 56
#define AG_GE2 56
#define AG_GF0 40
#define AG_GF1 45
#define AG_GF2 45
struct Items {
  int n = 0;
  double t[512];
  int first[512], size[512];
  void add(double due, int f, int sz) {
    int i = n++;
    for (; i > 0 && t[i - 1] > due + 1e-9; --i) { t[i] = t[i - 1]; first[i] = first[i - 1]; size[i] = size[i - 1]; }
    t[i] = due; first[i] = f; size[i] = sz;
  }
  bool put(unsigned short* out, int room) const {
    for (int i = 0; i < n; ++i) room -= size[i];
    if (room != 0) { fprintf(stderr, "kernel_launch: piece list\n"); return false; }
    for (int i = 0; i < n; ++i)
      for (int k = 0; k < size[i]; ++k) *out++ = (unsigned short)(first[i] + k);
    return true;
  }
};

}

extern "C" void kernel_launch(int rank, int nranks, ncclComm_t comm,
                              hipStream_t stream,
                              void* const* d_in, const int* in_sizes, int n_in,
                              void* d_out, int out_size,
                              const int* in_dtypes, int out_dtype,
                              void* const* d_out_all,
                              void* const* d_ws_all, size_t ws_bytes) {
  (void)comm; (void)in_dtypes; (void)out_dtype; (void)d_out;
  if (nranks != NR || (unsigned)rank >= (unsigned)NR || n_in < 2 || ws_bytes < WS_NEED ||
      in_sizes[0] != 1024 * KDIM || in_sizes[1] != KDIM * NLOC || out_size != 8192 * NLOC) {
    fprintf(stderr, "kernel_launch: unexpected problem shape\n");
    return;
  }
  HIP_CHECK(hipFuncSetAttribute((const void*)ag_gemm_kernel, hipFuncAttributeMaxDynamicSharedMemorySize, LDS_B));
  Params P;
  P.x = (const float*)d_in[0];
  P.w = (const float*)d_in[1];
  for (int p = 0; p < NR; ++p) { P.out[p] = (float*)d_out_all[p]; P.ws[p] = (unsigned char*)d_ws_all[p]; }
  P.rank = rank;
  const int me = rank >> 1;
  auto detour = [](int v, int g) { const int i = g - AG_NA / 4; return i >= 0 && (AG_RV < 3 ? v == AG_RV && i < NREL / 4 : i * AG_RM + (v + 3 - i % 3) % 3 < NREL / 4 && (v + 3 - i % 3) % 3 < AG_RM); };
  {
    Items s;
    for (int p = 4; p < NPAIR; ++p) {
      s.add(AG_E0 - (NPAIR - p) * AG_FR0, 0 << 8 | p * 4 | (p >= NPAIR - AG_L0 ? 0x800 : 0), 4);
      s.add(AG_E1 - (NPAIR - p) * AG_FR1, 1 << 8 | p * 4 | (p >= NPAIR - AG_L1 ? 0x800 : 0), 4);
    }
    const double e[3] = {AG_S0, AG_S1, AG_S2}, r[3] = {AG_R0, AG_R1, AG_R2};
    const int pe[3] = {AG_PE0, AG_PE1, AG_PE2};
    for (int p = 0; p < NPAIR; ++p)
      for (int v = 0; v < 3; ++v)
        if (!detour(v, p >> 2)) s.add(p < pe[v] ? AG_VA + p * (AG_VB - AG_VA) / pe[v] : e[v] - (NPAIR - p) * r[v], (2 + v) << 8 | (p * 4 + me) | (p >= NPAIR - AG_LV ? 0x800 : 0), 1);
    if (!s.put(P.ord + NDIR, QSAME - NDIR)) return;
  }
  for (int p = 0; p < NDIR; ++p) P.ord[p] = 0;
  for (int p = 0; p < NDIR; ++p) P.ord[QSAME + p] = 0;
  {
    static_assert(AG_TF > AG_TH && AG_TF + (NREL / 4 - 1) * AG_DF > AG_TH + (NREL / 4 - 1) * AG_DH, "order");
    Items s;
    for (int p = 0; p < NPAIR; ++p) {
      if (p >= 4) s.add(4. + (p - 4) * (NPAIR - 5. - AG_XW) / (NPAIR - 5.), 0 << 8 | p * 4 | (p >= NPAIR - AG_LW ? 0x800 : 0), 4);
      if (p >= 4) s.add(p, 1 << 8 | p * 4 | (p >= NPAIR - AG_LA ? 0x800 : 0), 4);
      s.add(p - AG_GD, 2 << 8 | (p * 4 + me) | (p >= NPAIR - AG_LA || p < AG_LG ? 0x800 : 0), 1);
    }
    for (int g = 0, i = 0; g < NPAIR / 4; ++g)
      for (int v = 0; v < 3; ++v)
        if (detour(v, g)) {
          s.add(AG_TH + i * AG_DH, 7 << 8 | v << 6 | g * 4, 1);
          for (int k = 0; k < 3; ++k) s.add(AG_TF + i * AG_DF, (4 + k) << 8 | v << 6 | g * 4, 1);
          ++i;
        }
    if (!s.put(P.ord + QSAME + NDIR, QIN - NDIR)) return;
  }
  for (int t = 0, job = 0; t < 6; ++t) {
    const int a[6] = {AG_H0, AG_H1, AG_H2, AG_G0, AG_G1, AG_G2}, e[6] = {AG_HE0, AG_HE1, AG_HE2, AG_GE0, AG_GE1, AG_GE2}, n = t < 4 ? 3 : 2;
    const int fr[6] = {AG_HF0, AG_HF1, AG_HF2, AG_GF0, AG_GF1, AG_GF2};
    unsigned long long& last = P.pairs[16 + (t + 3) % 6];
    last = 0;
    for (int i = 0; i < n; ++i) P.pairs[job + i] = 0;
    for (int p = 0, i = 0; p < NPAIR; ++p) {
      if (p < a[t] || (p < e[t] && p > a[t] && (p - a[t]) * fr[t] / 100 == (p - a[t] - 1) * fr[t] / 100)) P.pairs[job + i++ % n] |= 1ull << p;
      else last |= 1ull << p;
    }
    job += n;
  }
  hipLaunchKernelGGL(ag_gemm_kernel, dim3(NWG), dim3(512), LDS_B, stream, P);
  HIP_CHECK(hipGetLastError());
}
```

```cpp
#include <hip/hip_runtime.h>
#include <rccl/rccl.h>
#include <cstdio>

#define HIP_CHECK(expr) do { hipError_t _e = (expr); if (_e != hipSuccess) { \
    fprintf(stderr, "HIP error %s at %s:%d\n", hipGetErrorString(_e), __FILE__, __LINE__); \
    return; } } while (0)

#define LAS __attribute__((address_space(3)))
typedef _Float16 f16x8 __attribute__((ext_vector_type(8)));
typedef float f32x4 __attribute__((ext_vector_type(4)));
typedef unsigned u32x4 __attribute__((ext_vector_type(4)));
typedef unsigned u32x2 __attribute__((ext_vector_type(2)));

namespace {

constexpr int NR = 8;
constexpr int KDIM = 8192, NLOC = 1024;
constexpr int KSTEPS = KDIM / 64, NPAIR = KSTEPS / 2;
constexpr unsigned KK_B = 8192, PIECE_B = 16384, HALF_B = 16384, KSTEP_B = 32768, PAIR_B = 65536;
constexpr unsigned SLOT_B = NPAIR * PAIR_B;
constexpr int SLOT_V = 8, SLOT_G3 = 11, SLOT_A = 12, SLOT_V3 = 16, SLOT_T3 = 17, SLOT_W = 18, SLOT_WR = 22, SLOT_REL = 26, NSLOT = 27;
constexpr unsigned PAR_B = NSLOT * SLOT_B;
constexpr size_t CTRL_B = 1u << 20;
constexpr unsigned UREC_B = 68 * 1024, UHDR_B = 2112;
constexpr unsigned OUTS_B = 4 * 64 * UREC_B;
constexpr unsigned SH_HI = 8192, SH_EX = 14336, SH_MAX = 256, SH_LONG = 511;
constexpr unsigned OFF_OUTS = 2 * PAR_B, OFF_RCV = OFF_OUTS + OUTS_B, OFF_SH = OFF_RCV + OUTS_B;
constexpr unsigned OFF_INFO = OFF_SH + 2048 * PIECE_B;
constexpr unsigned VREC_B = 512 + 8192, OFF_VREC = OFF_INFO + 2048 * 16 + 1024;
constexpr unsigned OFF_PART = OFF_VREC + 384 * VREC_B, OFF_HAND = OFF_PART + 72 * 262144, DATA_B = OFF_HAND + 64 * 131072;
constexpr size_t WS_NEED = CTRL_B + DATA_B;
constexpr unsigned OFF_TICKET = 0;
constexpr unsigned OFF_TMO = 1024;
constexpr unsigned OFF_QUEUE = 2048;
constexpr unsigned OFF_UHEAD = 3072;
constexpr unsigned OFF_ULIST = 4096;
constexpr unsigned OFF_PFL = 5120;
constexpr unsigned OFF_DONE = 8192;
constexpr unsigned OFF_FLAG = 16384;
constexpr unsigned FLAG_B = (NSLOT + 1) * 1024;
constexpr unsigned OFF_SFLAG = OFF_FLAG + 2 * FLAG_B;
constexpr unsigned CTRL_USED = OFF_SFLAG + 2 * FLAG_B;
constexpr unsigned OFF_QFL = CTRL_USED;
constexpr unsigned long long DEADLINE = 50000000ull;
#define AG_NP 8
constexpr int NWG = 256, NPUSH = 7, NFULL = 13, NPREP = AG_NP;
constexpr int NUNIT = 48;
#define AG_ND 24
constexpr int NREL = AG_ND;
static_assert(NREL % 4 == 0, "relay");
constexpr int QSAME = 704 - NREL, QIN = 576 + NREL, QCROSS = QIN + NUNIT * 2;
constexpr int NDIR = 32;
constexpr int LDS_MAIN = 137 * 1024;
constexpr int LDS_B = 160 * 1024;
constexpr int AUX_SYS = 17;
#define AG_OAUX 17
constexpr unsigned RSRC_W3 = 0x00020000u;
constexpr unsigned OOB = 0x80000000u;
static_assert(DATA_B < OOB, "layout");

struct Params {
  const float* x;
  const float* w;
  float* out[NR];
  unsigned char* ws[NR];
  int rank;
  unsigned long long pairs[24];
  alignas(4) unsigned short ord[QSAME + QIN];
};

__device__ __forceinline__ unsigned ld_sys(const unsigned* p) { return __hip_atomic_load(p, __ATOMIC_RELAXED, __HIP_MEMORY_SCOPE_SYSTEM); }
__device__ __forceinline__ void st_sys(unsigned* p, unsigned v) { __hip_atomic_store(p, v, __ATOMIC_RELAXED, __HIP_MEMORY_SCOPE_SYSTEM); }
#define DRAIN() asm volatile("s_waitcnt vmcnt(0)" ::: "memory")
#define WAIT_V(n) asm volatile("s_waitcnt vmcnt(%0)" :: "n"(n) : "memory")
#define WAIT_L(n) asm volatile("s_waitcnt lgkmcnt(%0)" :: "n"(n) : "memory")
#define BAR __builtin_amdgcn_s_barrier()
#define SCHED __builtin_amdgcn_sched_barrier(0)
#define RFL(x) ((unsigned)__builtin_amdgcn_readfirstlane(x))
#define STORE_GAP() do { SCHED; asm volatile("s_nop 1"); SCHED; } while (0)
#define TSV(i, v)
#define TS(i)
#define TSW(i)
#define TR(ci, k, v)
#define KT(i, v)
#define KCLR()
#define KDUMP()

#define AG_US 4
#define AG_UD 8
#define AG_Z1 4
#define AG_Z2 16
#define AG_Z3 4
#define AG_Z4 2
#define AG_Z5 8
__device__ __forceinline__ bool timed_out(unsigned* tmo, unsigned seq, unsigned long long t0) {
  const unsigned long long dt = __builtin_amdgcn_s_memrealtime() - t0;
  if (RFL(__hip_atomic_load(tmo, __ATOMIC_RELAXED, __HIP_MEMORY_SCOPE_AGENT)) == seq || dt > DEADLINE) {
    if ((threadIdx.x & 63) == 0) __hip_atomic_store(tmo, seq, __ATOMIC_RELAXED, __HIP_MEMORY_SCOPE_AGENT);
    return true;
  }
  return false;
}

__device__ __noinline__ int wait_flags(const unsigned char* wsv, unsigned fo, int sel, int p0v, int needv, unsigned seqv) {
  const unsigned long long wa = (unsigned long long)wsv;
  const unsigned whi = RFL((unsigned)(wa >> 32)), wlo = RFL((unsigned)wa);
  const unsigned char* ws = (const unsigned char*)(((unsigned long long)whi << 32) | (unsigned long long)wlo);
  const int p0 = __builtin_amdgcn_readfirstlane(p0v), need = __builtin_amdgcn_readfirstlane(needv);
  const unsigned seq = RFL(seqv);
  const unsigned s9 = seq << 9;
  const __amdgpu_buffer_rsrc_t rf = __builtin_amdgcn_make_buffer_rsrc((void*)(ws + OFF_FLAG + (seq & 1u) * FLAG_B), 0, FLAG_B, RSRC_W3);
  const bool past = p0 + (int)(threadIdx.x & 31) >= NPAIR;
  const unsigned long long t0 = __builtin_amdgcn_s_memrealtime();
  for (;;) {
    asm volatile("" ::: "memory");
    const u32x4 f = __builtin_amdgcn_raw_buffer_load_b128(rf, fo, (unsigned)p0 * 16u, AUX_SYS);
    int old = 0;
#pragma unroll
    for (int j = 0; j < 4; ++j) old |= (int)((f[j] - s9) >> 31) << j;
    const unsigned long long m = __ballot(past || (old & sel) == 0);
    const unsigned miss = ~((unsigned)m & (unsigned)(m >> 32));
    int av = p0 + (miss ? __builtin_ctz(miss) : 32);
    av = av > NPAIR ? NPAIR : av;
    if (av >= need) return av;
    if (timed_out((unsigned*)(ws + OFF_TMO), seq, t0)) return NPAIR;
    __builtin_amdgcn_s_sleep(AG_Z1);
  }
}

__device__ __forceinline__ unsigned wave_max(unsigned m) {
#pragma unroll
  for (int o = 32; o >= 1; o >>= 1) { const unsigned t = (unsigned)__shfl_xor((int)m, o); m = t > m ? t : m; }
  return m;
}

__device__ __forceinline__ unsigned take_seq(unsigned char* ws, int xcd) {
  return __hip_atomic_fetch_add((unsigned*)(ws + OFF_TICKET) + xcd * 32, 1u, __ATOMIC_RELAXED, __HIP_MEMORY_SCOPE_AGENT) / (NWG / 8) + 1u;
}

struct Src { __amdgpu_buffer_rsrc_t rs; unsigned lo, ks_b, st_a, st_b, so; };

__device__ __forceinline__ Src make_src(const Params& P, int tl8) {
  const int tid = threadIdx.x, wid = tid >> 6, lane = tid & 63, r8 = lane >> 3, c8 = lane & 7;
  const bool isw = tl8 >= 4;
  const int tl = tl8 & 3;
  Src q;
  q.rs = __builtin_amdgcn_make_buffer_rsrc((void*)(isw ? P.w + tl * 256 : P.x + (size_t)tl * 256 * KDIM), 0,
                                           isw ? (KDIM * NLOC - tl * 256) * 4 : 256 * KDIM * 4, RSRC_W3);
  q.lo = isw ? (unsigned)(wid * 8 * NLOC + lane * 4) * 4u : (unsigned)((wid * 32 + r8) * KDIM + (c8 ^ r8) * 8) * 4u;
  q.ks_b = isw ? 64u * NLOC * 4u : 64u * 4u;
  q.st_a = isw ? 2u * NLOC * 4u : 8u * KDIM * 4u;
  q.st_b = isw ? NLOC * 4u : 16u;
  q.so = isw ? (unsigned)((tid * 16 + (tid & 15)) * NLOC + ((tid * 5) & 63) * 4) * 4u
             : (unsigned)((tid >> 1) * KDIM + ((tid * 37 + 11) & 2047) * 4) * 4u;
  return q;
}
#define PLOAD(q, v, ks) do { const unsigned _k = (unsigned)(ks) * q.ks_b, _lo = (ks) < KSTEPS ? q.lo : OOB; \
    _Pragma("unroll") for (int i = 0; i < 8; ++i) \
      v[i] = __builtin_amdgcn_raw_buffer_load_b128(q.rs, _lo, _k + (i >> 1) * q.st_a + (i & 1) * q.st_b, 0); } while (0)

__device__ __forceinline__ int tile_exp(const u32x4 sv, LAS unsigned* red) {
  unsigned m = 0, l = 0;
#pragma unroll
  for (int j = 0; j < 4; ++j) { const unsigned a = sv[j] & 0x7fffffffu; m = a > m ? a : m; l += a >> 15; }
  m = wave_max(m);
#pragma unroll
  for (int o = 32; o >= 1; o >>= 1) l += (unsigned)__shfl_xor((int)l, o);
  if ((threadIdx.x & 63) == 0) { red[threadIdx.x >> 6] = m; red[32 + (threadIdx.x >> 6)] = l; }
  WAIT_L(0);
  BAR;
  l = 0;
#pragma unroll
  for (int q = 0; q < 8; ++q) { const unsigned a = red[q]; m = a > m ? a : m; l += red[32 + q]; }
  m = RFL(m);
  l = RFL(l);
  const int ea = (int)(m >> 23), eg = (int)(((l >> 11) + 588u + 255u) >> 8) + 3;
  const int eb = eg > ea ? eg : ea;
  return m == 0 ? 131 : eb < 5 ? 5 : eb > 254 ? 254 : eb;
}

__device__ __forceinline__ void convert(const u32x4 (&v)[8], u32x4 (&h)[4], float s, bool isw, LAS unsigned char* lb) {
  const int tid = threadIdx.x, wid = tid >> 6, lane = tid & 63, r8 = lane >> 3, c8 = lane & 7;
  if (isw) {
    const int tw = lane * 512 + ((wid ^ c8) * 16);
#pragma unroll
    for (int cc = 0; cc < 4; ++cc) {
      f16x8 t;
#pragma unroll
      for (int i = 0; i < 8; ++i) t[i] = (_Float16)(__uint_as_float(v[i][cc]) * s);
      *(LAS f16x8*)(lb + tw + cc * 128) = t;
    }
    WAIT_L(0);
    BAR;
#pragma unroll
    for (int it = 0; it < 4; ++it) h[it] = *(const LAS u32x4*)(lb + wid * 4096 + it * 1024 + (lane * 16 ^ ((r8 ^ (it * 2 + (r8 >> 2))) * 16)));
  } else {
#pragma unroll
    for (int it = 0; it < 4; ++it) {
      f16x8 t;
#pragma unroll
      for (int i = 0; i < 4; ++i) {
        t[i] = (_Float16)(__uint_as_float(v[2 * it][i]) * s);
        t[4 + i] = (_Float16)(__uint_as_float(v[2 * it + 1][i]) * s);
      }
      h[it] = __builtin_bit_cast(u32x4, t);
    }
  }
}

#define AG_PP 170
#define AG_PO 4
__device__ __forceinline__ unsigned prep_role(const Params& P, unsigned char* ws, int tl8, int pi, LAS unsigned char* lds) {
  const int tid = threadIdx.x, me = P.rank >> 1;
  LAS unsigned* red = (LAS unsigned*)(lds + LDS_MAIN);
  const bool isw = tl8 >= 4;
  const Src q = make_src(P, tl8);
  const u32x4 sv = __builtin_amdgcn_raw_buffer_load_b128(q.rs, q.so, 0, 0);
  u32x4 v0[8], v1[8];
  PLOAD(q, v0, 2 * pi);
  PLOAD(q, v1, 2 * pi + 1);
  if (tid == 0) red[8] = take_seq(ws, tl8);
  if ((tid >= 40 && tid < 48) || (tid >= 16 && tid < 32)) red[tid] = 0;
  const int eb = tile_exp(sv, red);
  const unsigned seq = RFL(red[8]);
  const float s = __uint_as_float((unsigned)(258 - eb) << 23);
  const unsigned p0 = isw ? (SLOT_W + tl8 - 4) * 256u : tl8 < 2 ? (me * 2 + tl8) * 256u : tl8 == 2 ? SLOT_V * 256u + me : SLOT_T3 * 256u;
  const unsigned gs = tl8 == 2 ? 256u : 1u;
  const unsigned par = seq & 1u;
  const __amdgpu_buffer_rsrc_t rd = __builtin_amdgcn_make_buffer_rsrc(ws + CTRL_B + (size_t)par * PAR_B, 0, PAR_B, RSRC_W3);
  unsigned* flag = (unsigned*)(ws + OFF_FLAG + par * FLAG_B) + p0 + (tid & 3) * gs;
  const unsigned fval = (seq << 9) + (unsigned)(256 + eb - 131);
  const __amdgpu_buffer_rsrc_t rh = __builtin_amdgcn_make_buffer_rsrc(ws + CTRL_B + OFF_SH, 0, DATA_B - OFF_SH, RSRC_W3);
  const int wid = __builtin_amdgcn_readfirstlane(tid >> 6), grp = wid >> 1;
  const unsigned vo = (unsigned)(wid & 1) * 4096u + (unsigned)(tid & 63) * 16u;
  const unsigned long long t0 = __builtin_amdgcn_s_memrealtime();
  int odd = 0;
  for (int pr = pi; pr < NPAIR; pr += NPREP, odd ^= 1) {
    u32x4 h0[4], h1[4], lo[4], hi[3];
    convert(v0, h0, s, isw, lds);
    convert(v1, h1, s, isw, lds + KSTEP_B);
    LAS unsigned* cn = red + 40 + odd * 4 + grp;
    LAS unsigned char* ex = lds + 2 * KSTEP_B + odd * 2048 + grp * 512;
    const unsigned dl = (p0 + pr * 4u + grp * gs) * PIECE_B, sp = (unsigned)tl8 * 256u + (unsigned)pr * 4u + grp;
    unsigned e[4], ne = 0;
#pragma unroll
    for (int it = 0; it < 4; ++it) {
      __builtin_amdgcn_raw_buffer_store_b128(h0[it], rd, vo, dl + it * 1024u, AUX_SYS);
      __builtin_amdgcn_raw_buffer_store_b128(h1[it], rd, vo, dl + it * 1024u + KK_B, AUX_SYS);
      unsigned H[4];
#pragma unroll
      for (int d = 0; d < 4; ++d) {
        const unsigned a = d < 2 ? h0[it][2 * d] : h1[it][2 * d - 4], b = d < 2 ? h0[it][2 * d + 1] : h1[it][2 * d - 3];
        lo[it][d] = __builtin_amdgcn_perm(b, a, 0x06040200u);
        H[d] = __builtin_amdgcn_perm(b, a, 0x07050301u);
      }
      const unsigned g = (H[3] & ~0x20202020u) | ((H[3] >> 2) & 0x20202020u);
#pragma unroll
      for (int d = 0; d < 3; ++d) hi[(it * 3 + d) >> 2][(it * 3 + d) & 3] = (H[d] & ~0x60606060u) | ((g << (5 - 2 * d)) & 0x60606060u);
      e[it] = (((H[0] & 0x60606060u) >> 5) | ((H[1] & 0x60606060u) >> 3) | ((H[2] & 0x60606060u) >> 1) | ((H[3] & 0x60606060u) << 1)) ^ 0x55555555u;
      ne += (unsigned)__builtin_popcount((e[it] | (e[it] >> 1)) & 0x55555555u);
    }
    if (ne) {
      unsigned at = __hip_atomic_fetch_add(cn, ne, __ATOMIC_RELAXED, __HIP_MEMORY_SCOPE_WORKGROUP);
#pragma unroll
      for (int it = 0; it < 4; ++it)
        for (unsigned x = e[it]; x; ++at) {
          const unsigned f = (unsigned)__builtin_ctz(x) >> 1, x2 = (x >> (2 * f)) & 3u;
          x &= ~(3u << (2 * f));
          if (at < SH_MAX) *(LAS unsigned short*)(ex + at * 2) = (unsigned short)(x2 << 13 | (f & 2u) << 11 | (vo + it * 1024u) >> 1 | (f & 1u) << 2 | f >> 2);
        }
    }
    WAIT_L(0);
    BAR;
    ne = RFL(*cn);
    const u32x4 ev = *(const LAS u32x4*)(ex + (tid & 31) * 16);
    if (tid < 4) red[40 + (odd ^ 1) * 4 + tid] = 0;
    if (ne <= SH_MAX) {
#pragma unroll
      for (int it = 0; it < 4; ++it) __builtin_amdgcn_raw_buffer_store_b128(lo[it], rh, vo, sp * PIECE_B + it * 1024u, AUX_SYS);
#pragma unroll
      for (int j = 0; j < 3; ++j) __builtin_amdgcn_raw_buffer_store_b128(hi[j], rh, vo - (unsigned)(wid & 1) * 1024u, sp * PIECE_B + SH_HI + j * 1024u, AUX_SYS);
      __builtin_amdgcn_raw_buffer_store_b128(ev, rh, (tid & 127) < 32 ? vo : OOB, sp * PIECE_B + SH_EX, AUX_SYS);
    } else {
      ne = SH_LONG;
#pragma unroll
      for (int it = 0; it < 4; ++it) {
        __builtin_amdgcn_raw_buffer_store_b128(h0[it], rh, vo, sp * PIECE_B + it * 1024u, AUX_SYS);
        __builtin_amdgcn_raw_buffer_store_b128(h1[it], rh, vo, sp * PIECE_B + it * 1024u + KK_B, AUX_SYS);
      }
    }
    STORE_GAP();
    __builtin_amdgcn_raw_buffer_store_b32(ne, rh, (tid & 127) == 0 ? 0u : OOB, OFF_INFO - OFF_SH + sp * 16u, AUX_SYS);
    DRAIN();
    BAR;
    if (tid < 4) st_sys(flag + pr * 4, fval);
    while (pr + NPREP < NPAIR && (int)(__builtin_amdgcn_s_memrealtime() - t0) < (pr + NPREP - 11 + AG_PO) * AG_PP) __builtin_amdgcn_s_sleep(32);
    PLOAD(q, v0, 2 * (pr + NPREP));
    PLOAD(q, v1, 2 * (pr + NPREP) + 1);
  }
  return seq;
}

#define AG_PL 5
#define AG_DS 8
#define AG_ST 0
#define AG_SW 0
constexpr int PL = AG_PL;
#define AG_PU 7
constexpr int PU = AG_PU;
#define AG_AT0 900
#define AG_XT0 AG_AT0
#define AG_XTR AG_ATR
#define AG_ATR 213
#define AG_P2 7
#define AG_PF 0
#define AG_PS 8
constexpr int RING = 4;
constexpr unsigned RSLOT_B = 18 * 1024, LDS_TR = RING * RSLOT_B;
constexpr int NUW = 6;

__device__ __forceinline__ void store_rows(const __amdgpu_buffer_rsrc_t ro, f32x4 (&v)[8], unsigned so, LAS unsigned char* lb, int lane) {
  const unsigned wa = (unsigned)(lane & 15) * 128u + (unsigned)((lane >> 4) ^ (lane & 7)) * 16u;
#pragma unroll
  for (int i = 0; i < 8; ++i) *(LAS f32x4*)(lb + (i >> 1) * 2048 + (wa ^ (i & 1) * 64u)) = v[i];
  const unsigned ra = (unsigned)(lane >> 3) * 128u + (unsigned)((lane ^ (lane >> 3)) & 7) * 16u;
#pragma unroll
  for (int i = 0; i < 8; ++i) v[i] = *(const LAS f32x4*)(lb + i * 1024 + ra);
  const unsigned vo = (unsigned)(lane >> 3) * (NLOC * 4u) + (unsigned)(lane & 7) * 16u;
#pragma unroll
  for (int i = 0; i < 8; ++i)
    __builtin_amdgcn_raw_buffer_store_b128(__builtin_bit_cast(u32x4, v[i]), ro, vo, so + i * 8u * (NLOC * 4u), AG_OAUX);
  STORE_GAP();
}

__device__ __forceinline__ int group_row0(int g, int xs, int rg) {
  return xs < SLOT_V ? (2 * (xs >> 1) + g) * 1024 + (xs & 1) * 256 + rg * 64
       : xs < SLOT_G3 ? (2 * rg + g) * 1024 + 512 + (xs - SLOT_V) * 64
       : xs < SLOT_V3 ? (2 * (xs - SLOT_A) + 1 - g) * 1024 + 768 + rg * 64
                      : (2 * rg + 1 - g) * 1024 + 704;
}

__device__ __forceinline__ void unpack_block(const Params& P, unsigned char* ws, unsigned seq, int T, bool same, LAS unsigned char* lb) {
  const int lane = threadIdx.x & 63, rank = P.rank;
  const unsigned s9 = seq << 9, voff = (unsigned)lane * 16u;
  unsigned* tmo = (unsigned*)(ws + OFF_TMO);
  const __amdgpu_buffer_rsrc_t ro = __builtin_amdgcn_make_buffer_rsrc(P.out[rank], 0, 8192 * NLOC * 4, RSRC_W3);
  const __amdgpu_buffer_rsrc_t rs = __builtin_amdgcn_make_buffer_rsrc(ws + CTRL_B, 0, DATA_B, RSRC_W3);
  const __amdgpu_buffer_rsrc_t rc = __builtin_amdgcn_make_buffer_rsrc(ws, 0, CTRL_USED, RSRC_W3);
  const unsigned ci = (unsigned)((T >> 3) & 3) * 64u + (unsigned)(T >> 5);
  const unsigned vi = ((unsigned)((T >> 3) & 3) * 12u + (unsigned)(T >> 5)) * 8u + (unsigned)(T & 7);
  const int w = T & 7;
  const unsigned rec = same ? OFF_VREC + vi * VREC_B : OFF_RCV + ci * UREC_B;
  unsigned u, mk = 0;
  bool bad = false;
  const unsigned long long t0 = __builtin_amdgcn_s_memrealtime();
  for (;;) {
    asm volatile("" ::: "memory");
    u32x2 f = same ? __builtin_amdgcn_raw_buffer_load_b64(rs, 0, rec, AUX_SYS) : __builtin_amdgcn_raw_buffer_load_b64(rc, 0, OFF_DONE + ci * 16u, AUX_SYS);
    if (same) f[0] = f[1];
    u = RFL(f[0]);
    if ((int)RFL((f[0] - s9) | (f[1] - s9)) >= 0) break;
    if (timed_out(tmo, seq, t0)) { bad = true; break; }
    if (same) __builtin_amdgcn_s_sleep(AG_US); else __builtin_amdgcn_s_sleep(AG_Z2);
  }
  bad = RFL(bad) != 0;
  u = RFL(u);
  if (!same && w == 0) TR(ci, 10, NOW());
  unsigned off = 512u, nb = (u & 255u) * 64u;
  if (same) {
    u = SLOT_V * 4u + (unsigned)(T >> 5);
  } else {
    mk = __builtin_amdgcn_raw_buffer_load_b32(rs, lane * 4u, rec + w * 256u, AUX_SYS);
    const u32x2 ob = __builtin_amdgcn_raw_buffer_load_b64(rs, 0, rec + 2048u + w * 8u, AUX_SYS);
    off = UHDR_B + (RFL(ob[0]) & 0xffffu);
    nb = RFL(ob[1]);
  }
  nb = nb < 8192u ? nb : 8192u;
  unsigned need = (1u << ((nb + 1023u) >> 10)) - 1u + (same ? 256u : 0u);
  while (need) {
    u32x2 f = {mk, s9};
    if (need & 256u) f = __builtin_amdgcn_raw_buffer_load_b64(rs, lane * 8u, rec, AUX_SYS);
#pragma unroll
    for (int i = 0; i < 8; ++i)
      if (need >> i & 1u) __builtin_amdgcn_raw_ptr_buffer_load_lds(rs, (LAS void*)(lb + i * 1024), 16, voff, rec + off + i * 1024u, 0, AUX_SYS);
    DRAIN();
    if (!same || bad) break;
    mk = f[0];
    if (__ballot((int)(f[1] - s9) < 0) == 0ull) need &= 255u;
#pragma unroll
    for (int i = 0; i < 8; ++i)
      if (need >> i & 1u) {
        const u32x4 x = *(const volatile LAS u32x4*)(lb + i * 1024 + voff);
        if (__ballot(i * 1024u + voff < nb && min(min(x[0], x[1]), min(x[2], x[3])) == 0u) == 0ull) need &= ~(1u << i);
      }
    if (need && timed_out(tmo, seq, t0)) { bad = true; break; }
    if (AG_UD && need) __builtin_amdgcn_s_sleep(AG_UD);
  }
  bad = RFL(bad) != 0;
  f32x4 v[8];
  unsigned run = (unsigned)(unsigned long long)lb;
#pragma unroll
  for (int i = 0; i < 32; ++i) {
    const bool k = (int)(mk << i) < 0;
    const unsigned long long bl = __builtin_amdgcn_ballot_w64(k);
    const unsigned pos = __builtin_amdgcn_mbcnt_hi((unsigned)(bl >> 32), __builtin_amdgcn_mbcnt_lo((unsigned)bl, 0u)) * 4u + run;
    const float t = *(const LAS float*)(unsigned long long)pos;
    v[i >> 2][i & 3] = k ? t : 0.f;
    run += (unsigned)__builtin_popcountll(bl) * 4u;
  }
  if (bad) {
#pragma unroll
    for (int i = 0; i < 8; ++i) v[i] = (f32x4){__builtin_nanf(""), __builtin_nanf(""), __builtin_nanf(""), __builtin_nanf("")};
  }
  const int ti = (int)(u >> 2) & 15;
  const unsigned so = (unsigned)group_row0(1 - (rank & 1), ti < SLOT_G3 ? ti : ti + 1, (int)(u & 2u) + (w >> 2)) * (NLOC * 4u)
                    + (ci >> 6) * 1024u + (u & 1u) * 512u + (unsigned)(w & 3) * 128u;
#pragma unroll
  for (int i = 0; i < 8; ++i) asm volatile("" : "+v"(v[i]));
  SCHED;
  store_rows(ro, v, so, lb, lane);
  if (same) {
    const __amdgpu_buffer_rsrc_t rz = __builtin_amdgcn_make_buffer_rsrc(ws + CTRL_B + rec + 512u, 0, nb, RSRC_W3);
    u32x4 z = {0u, 0u, 0u, 0u};
    asm volatile("" : "+v"(z));
#pragma unroll
    for (int i = 0; i < 8; ++i) __builtin_amdgcn_raw_buffer_store_b128(z, rz, voff, i * 1024u, AUX_SYS);
    STORE_GAP();
  }
  if (!same && w == 7) TR(ci, 11, NOW());
}

__device__ __forceinline__ unsigned part_words(int xs, int wsl, int cq) {
  return OFF_QFL + (unsigned)((((wsl - SLOT_W) & 3) * 6 + xs - SLOT_V + (wsl < SLOT_WR ? 3 : 0)) * 4 + cq) * 128u;
}

__device__ __forceinline__ unsigned link_role(const Params& P, unsigned char* ws, int link, int wgi, LAS unsigned char* lds) {
  const int tid = threadIdx.x, wid = __builtin_amdgcn_readfirstlane(tid >> 6), lane = tid & 63;
  const int rank = P.rank, me = rank >> 1, peer = (rank + 1 + link) & (NR - 1);
  const bool same = ((peer ^ rank) & 1) == 0;
  const unsigned sA = same ? me * 2 : SLOT_W + (peer >> 1), dA = same ? me * 2 : SLOT_WR + me;
  const unsigned sB = same ? me * 2 + 1 : SLOT_T3, dB = same ? me * 2 + 1 : SLOT_A + me;
  const unsigned sC = same ? SLOT_V : SLOT_G3, dC = same ? SLOT_V : SLOT_V3;
  unsigned char* pw = P.ws[peer];
  LAS unsigned* red = (LAS unsigned*)(lds + LDS_MAIN);
  const __amdgpu_buffer_rsrc_t rp = __builtin_amdgcn_make_buffer_rsrc(pw, 0, CTRL_USED, RSRC_W3);
  const __amdgpu_buffer_rsrc_t rd = __builtin_amdgcn_make_buffer_rsrc(pw + CTRL_B, 0, DATA_B, RSRC_W3);
  const unsigned one = lane == 0 ? 0u : OOB;
  unsigned seq;
  const unsigned ts0 = (unsigned)__builtin_amdgcn_s_memrealtime();
  {
    const int tl8 = same ? wgi & 1 : (wgi & 1) ? 3 : 4 + (peer >> 1), pr = wgi >> 1;
    const Src q = make_src(P, tl8);
    const u32x4 sv = __builtin_amdgcn_raw_buffer_load_b128(q.rs, q.so, 0, 0);
    u32x4 v0[8], v1[8], h[2][4];
    PLOAD(q, v0, 2 * pr);
    PLOAD(q, v1, 2 * pr + 1);
    if (tid == 0) { red[8] = take_seq(ws, wgi); red[9] = 0; red[10] = 0; red[11] = 0; }
    const int eb = tile_exp(sv, red);
    seq = RFL(red[8]);
    const float s = __uint_as_float((unsigned)(258 - eb) << 23);
    const unsigned pc = ((wgi & 1) ? dB : dA) * 256u + (unsigned)pr * 4u;
#pragma unroll
    for (int kk = 0; kk < 2; ++kk) {
      convert(kk ? v1 : v0, h[kk], s, tl8 >= 4, lds + LDS_TR + kk * KSTEP_B);
      if (!kk) {
        TS(3);
        const unsigned long long t0 = __builtin_amdgcn_s_memrealtime();
        while (__builtin_amdgcn_s_memrealtime() - t0 < (unsigned long long)(pr * AG_ST)) __builtin_amdgcn_s_sleep(2);
      }
#pragma unroll
      for (int it = 0; it < 4; ++it) {
        __builtin_amdgcn_raw_buffer_store_b128(h[kk][it], rd, (unsigned)(wid & 1) * 4096u + (unsigned)lane * 16u, (seq & 1u) * PAR_B + (pc + (wid >> 1)) * PIECE_B + kk * KK_B + it * 1024u, AUX_SYS);
        WAIT_V(AG_SW);
      }
    }
    DRAIN();
    unsigned n = 0;
    if (lane == 0) n = __hip_atomic_fetch_add(red + 9, 1u, __ATOMIC_RELAXED, __HIP_MEMORY_SCOPE_WORKGROUP);
    if (RFL(n) == 7)
      __builtin_amdgcn_raw_buffer_store_b32((seq << 9) + (unsigned)(256 + eb - 131), rp, lane < 4 ? lane * 4u : OOB,
                                            OFF_FLAG + (seq & 1u) * FLAG_B + pc * 4u, AUX_SYS);
  }
  const unsigned par = seq & 1u, s9 = seq << 9;
  unsigned* tmo = (unsigned*)(ws + OFF_TMO);
  volatile LAS unsigned* cnt = red;
  const __amdgpu_buffer_rsrc_t rs = __builtin_amdgcn_make_buffer_rsrc(ws + CTRL_B, 0, DATA_B, RSRC_W3);
  const __amdgpu_buffer_rsrc_t rc = __builtin_amdgcn_make_buffer_rsrc(ws, 0, CTRL_USED, RSRC_W3);
  const unsigned voff = (unsigned)lane * 16u;
  if (wid > 1) {
    LAS unsigned char* lb = lds + LDS_TR + (wid - 2) * 8192;
    {
      const unsigned long long t0 = __builtin_amdgcn_s_memrealtime();
      while (RFL(cnt[9]) < 8u && __builtin_amdgcn_s_memrealtime() - t0 < DEADLINE) __builtin_amdgcn_s_sleep(8);
    }
    const unsigned* ord = (const unsigned*)P.ord + (same ? 0 : QSAME / 2);
    const unsigned pm = (unsigned)peer >> 1;
    for (int p = NDIR + (wid - 2) * 8 + wgi; p < (same ? QSAME : QIN); p += 8 * NUW) {
      const unsigned o = ord[p >> 1] >> ((p & 1) * 16), sel = (o >> 8) & 7u;
      if ((o & 0x800u) || sel == 7u) continue;
      const unsigned dn = sel >= 5 || (!same && sel == 4) ? (SLOT_V + ((o >> 6) & 3u)) * 256u + ((o & 63u) + pm) * 4u + sel - (sel - 4u >= (unsigned)me ? 3u : 4u)
                        : sel >= 2 ? (dC + sel - 2) * 256u + (o & 252u) + pm
                                   : (same ? pm * 2 + sel : (sel ? SLOT_A : SLOT_WR) + pm) * 256u + (o & 255u);
      unsigned f;
      const unsigned long long t0 = __builtin_amdgcn_s_memrealtime();
      for (;;) {
        asm volatile("" ::: "memory");
        f = RFL(__builtin_amdgcn_raw_buffer_load_b32(rc, 0, OFF_SFLAG + par * FLAG_B + dn * 4u, AUX_SYS));
        if ((int)(f - (seq << 18)) >= 0) break;
        if (timed_out(tmo, seq, t0)) { f = SH_LONG << 9; break; }
        __builtin_amdgcn_s_sleep(AG_DS);
      }
      f = RFL(f);
      const unsigned ne = (f >> 9) & 511u, base = par * PAR_B + dn * PIECE_B;
      if (ne <= SH_MAX) {
        u32x4 lo[8], hw[6];
#pragma unroll
        for (int i = 0; i < 8; ++i) lo[i] = __builtin_amdgcn_raw_buffer_load_b128(rs, voff, base + i * 1024u, AUX_SYS);
#pragma unroll
        for (int i = 0; i < 6; ++i) hw[i] = __builtin_amdgcn_raw_buffer_load_b128(rs, voff, base + SH_HI + i * 1024u, AUX_SYS);
        const u32x2 ex = __builtin_amdgcn_raw_buffer_load_b64(rs, lane * 8u, base + SH_EX, AUX_SYS);
        DRAIN();
#pragma unroll
        for (int kk = 0; kk < 2; ++kk) {
#pragma unroll
          for (int i = 0; i < 8; ++i) {
            const unsigned h0 = hw[(i * 3) >> 2][(i * 3) & 3], h1 = hw[(i * 3 + 1) >> 2][(i * 3 + 1) & 3], h2 = hw[(i * 3 + 2) >> 2][(i * 3 + 2) & 3];
            unsigned a, b;
            if (kk == 0) {
              a = (h0 & 0x9f9f9f9fu) | 0x20202020u;
              b = (h1 & 0x9f9f9f9fu) | 0x20202020u;
            } else {
              a = (h2 & 0x9f9f9f9fu) | 0x20202020u;
              b = ((h0 >> 5) & 0x03030303u) | ((h1 >> 3) & 0x0c0c0c0cu) | ((h2 >> 1) & 0x10101010u) | ((h2 << 1) & 0x80808080u) | 0x20202020u;
            }
            const u32x4 v = {__builtin_amdgcn_perm(a, lo[i][2 * kk], 0x05010400u), __builtin_amdgcn_perm(a, lo[i][2 * kk], 0x07030602u),
                             __builtin_amdgcn_perm(b, lo[i][2 * kk + 1], 0x05010400u), __builtin_amdgcn_perm(b, lo[i][2 * kk + 1], 0x07030602u)};
            *(LAS u32x4*)(lb + i * 1024 + voff) = v;
          }
          asm volatile("" ::: "memory");
#pragma unroll
          for (int j = 0; j < 4; ++j) {
            const unsigned x = ex[j >> 1] >> ((j & 1) * 16);
            const bool on = (unsigned)lane * 4u + j < ne && ((x >> 12) & 1u) == (unsigned)kk;
            __hip_atomic_fetch_xor((LAS unsigned*)(lb + ((x & 0xffeu) << 1)), on ? (x & 0x6000u) << ((x & 1u) * 16u) : 0u, __ATOMIC_RELAXED, __HIP_MEMORY_SCOPE_WORKGROUP);
          }
          asm volatile("" ::: "memory");
          u32x4 v[8];
#pragma unroll
          for (int i = 0; i < 8; ++i) v[i] = *(const LAS u32x4*)(lb + i * 1024 + voff);
          WAIT_L(0);
          SCHED;
#pragma unroll
          for (int i = 0; i < 8; ++i) __builtin_amdgcn_raw_buffer_store_b128(v[i], rs, voff, base + kk * KK_B + i * 1024u, AUX_SYS);
          STORE_GAP();
        }
        DRAIN();
      }
      __builtin_amdgcn_raw_buffer_store_b32(s9 + (f & 511u), rc, one, OFF_FLAG + par * FLAG_B + dn * 4u, AUX_SYS);
    }
    TSW(20 + wid);
    if (same) {
      return seq;
    }
    const unsigned tb = (seq - 1u) * (unsigned)(NUNIT * 32 + 32 * NUW);
    unsigned T = RFL(__builtin_amdgcn_raw_ptr_buffer_atomic_add_i32(1, rc, one, OFF_QUEUE + NPUSH * 128, 16)) - tb;
    while (T < NUNIT * 32u) {
      const unsigned tn = (unsigned)__builtin_amdgcn_raw_ptr_buffer_atomic_add_i32(1, rc, one, OFF_QUEUE + NPUSH * 128, 16);
      unpack_block(P, ws, seq, (int)T, false, lb);
      T = RFL(tn) - tb;
    }
    return seq;
  }
  if (wid == 1) {
    const unsigned fb = OFF_FLAG + par * FLAG_B;
    const int Q = same ? QSAME : QCROSS, QI = same ? QSAME : QIN;
    const unsigned* ord = (const unsigned*)P.ord + (same ? 0 : QSAME / 2);
    const unsigned qbase = (seq - 1u) * (unsigned)(Q - NDIR) - (NDIR + 16u);
    unsigned fA = 0, fB = 0, fC = 0;
    int rdy = 0;
    int p = NDIR + wgi, pn = NDIR + 8 + wgi;
    bool late = false;
    unsigned e = 0;
    unsigned k1 = 0;
    int held = 0;
    for (int n = 0;; ++n) {
      const bool live = (unsigned)p < (unsigned)Q;
      const unsigned tk = (unsigned)__builtin_amdgcn_raw_ptr_buffer_atomic_add_i32(1, rc, live && k1 == 0u ? one : OOB, OFF_QUEUE + link * 128, 16);
      const unsigned en = __builtin_amdgcn_raw_buffer_load_b32(rc, 0, (unsigned)(pn - QI) < (unsigned)(Q - QI) ? OFF_ULIST + (unsigned)(peer >> 1) * 256u + (unsigned)(pn - QIN) / 2u * 4u : OOB, AUX_SYS);
      u32x4 rec = {0u, 255u, 0u, 0u};
      unsigned so = OOB, kb = 16, so4 = OOB, so5 = OOB;
      bool part = false;
      if (live && p < QI) {
        unsigned o = ord[p >> 1] >> ((p & 1) * 16), sel = (o >> 8) & 7u, pc = o & 255u;
        const bool hop = sel == 7u, fwd = !same && sel >= 4u && !hop;
        if (hop) pc = ((pc & 63u) + (peer >> 1)) * 4u;
        if (fwd) {
          const unsigned c = sel - (sel - 4u >= (unsigned)peer >> 1 ? 3u : 4u), sn = SLOT_REL * 256u + c * 64u + (pc >> 6) * 16u + ((pc & 63u) >> 2), dn = (SLOT_V + (pc >> 6)) * 256u + ((pc & 63u) + me) * 4u + c;
          DRAIN();
          cnt[10] = n;
          unsigned f;
          const unsigned long long t0 = __builtin_amdgcn_s_memrealtime();
          for (;;) {
            asm volatile("" ::: "memory");
            f = RFL(__builtin_amdgcn_raw_buffer_load_b32(rc, 0, fb + (OFF_SFLAG - OFF_FLAG) + sn * 4u, AUX_SYS));
            if ((int)(f - (seq << 18)) >= 0 || timed_out(tmo, seq, t0)) break;
            __builtin_amdgcn_s_sleep(AG_Z3);
          }
          f = RFL(f);
          so = par * PAR_B + sn * PIECE_B;
          rec = (u32x4){par * PAR_B + dn * PIECE_B, 1024u + ((f >> 9) & 511u), fb + (OFF_SFLAG - OFF_FLAG) + dn * 4u, (seq << 18) + (f & 511u)};
        } else {
        if (rdy <= (int)(pc >> 2)) {
          DRAIN();
          cnt[10] = n;
          const unsigned long long t0 = __builtin_amdgcn_s_memrealtime();
          for (;;) {
            asm volatile("" ::: "memory");
            const unsigned a = __builtin_amdgcn_raw_buffer_load_b32(rc, voff, fb + sA * 1024u, AUX_SYS);
            const unsigned b = __builtin_amdgcn_raw_buffer_load_b32(rc, voff, fb + sB * 1024u, AUX_SYS);
            const unsigned c = __builtin_amdgcn_raw_buffer_load_b32(rc, voff, fb + sC * 1024u + me * 4u, AUX_SYS);
            const unsigned long long miss = __ballot((int)((a - s9) | (b - s9) | (c - s9)) < 0);
            rdy = miss ? __builtin_ctzll(miss) : NPAIR;
            fA = RFL(a); fB = RFL(b); fC = RFL(c);
            if (rdy > (int)(pc >> 2)) break;
            if (timed_out(tmo, seq, t0)) { rdy = NPAIR; fA = fB = fC = s9; break; }
            __builtin_amdgcn_s_sleep(AG_Z3);
          }
          rdy = __builtin_amdgcn_readfirstlane(rdy);
        }
        const unsigned dn = hop ? SLOT_REL * 256u + me * 64u + ((o >> 6) & 3u) * 16u + (pc >> 4) : (sel >= 2 ? dC + sel - 2 : sel ? dB : dA) * 256u + pc;
        const unsigned fv = sel >= 2 ? fC : sel ? fB : fA;
        if (o & 0x800u) {
          so = par * PAR_B + ((sel >= 2 ? sC + sel - 2 : sel ? sB : sA) * 256u + pc) * PIECE_B;
          rec = (u32x4){par * PAR_B + dn * PIECE_B, 16u, fb + dn * 4u, fv};
        } else {
          const unsigned sp = sel >= 2 ? 512u + (pc & 252u) + (same ? sel - 2 : hop ? (o >> 6) & 3u : 3u) : (same ? sel : sel ? 3u : 4u + (peer >> 1)) * 256u + pc;
          so = OFF_SH + sp * PIECE_B;
          so4 = OFF_INFO + sp * 16u;
          rec = (u32x4){par * PAR_B + dn * PIECE_B, 64u, fb + (OFF_SFLAG - OFF_FLAG) + dn * 4u, (seq << 18) + (fv & 511u)};
        }
        }
      } else if (live) {
        const unsigned i = (unsigned)(p - QIN) >> 1, j = (unsigned)p & 1u;
        const unsigned eo = OFF_ULIST + (unsigned)(peer >> 1) * 256u + i * 4u;
        if ((e >> 16) != (seq & 0xffffu)) {
          DRAIN();
          cnt[10] = n;
          const unsigned long long t0 = __builtin_amdgcn_s_memrealtime();
          for (;;) {
            asm volatile("" ::: "memory");
            e = RFL(__builtin_amdgcn_raw_buffer_load_b32(rc, 0, eo, AUX_SYS));
            if ((e >> 16) == (seq & 0xffffu) || timed_out(tmo, seq, t0)) break;
            __builtin_amdgcn_s_sleep(AG_Z3);
          }
          e = RFL(e);
        }
        unsigned nk = (e >> 8) & 127u;
        nk = nk > 68u ? 68u : nk;
        const unsigned qk = (nk + 1u) >> 1, k0 = j * qk + k1;
        kb = (j ? nk : qk) - k0;
        part = kb > 18u && k1 == 0u;
        kb = kb > 18u ? 18u : kb;
        so = OFF_OUTS + ((unsigned)(peer >> 1) * 64u + i) * UREC_B + k0 * 1024u;
        so4 = kb > 16u ? so + 16384u : OOB;
        so5 = kb > 17u ? so4 : OOB;
        rec = (u32x4){OFF_RCV + ((unsigned)me * 64u + i) * UREC_B + k0 * 1024u, kb,
                      part ? fb + NSLOT * 1024u : OFF_DONE + (((unsigned)me * 64u + i) * 4u + j) * 4u, s9 + (e & 63u)};
        TR((unsigned)(peer >> 1) * 64u + i, 1 + j, NOW());
      }
      if (!late && n - (int)RFL(cnt[11]) >= RING) {
        const unsigned long long t0 = __builtin_amdgcn_s_memrealtime();
        while (n - (int)RFL(cnt[11]) >= RING) {
          if (__builtin_amdgcn_s_memrealtime() - t0 > DEADLINE) { late = timed_out(tmo, seq, t0); break; }
          __builtin_amdgcn_s_sleep(2);
        }
      }
      so = RFL(so);
      so4 = RFL(so4);
      so5 = RFL(so5);
      kb = RFL(kb);
      const int sl = n & (RING - 1);
      if (lane == 0) *(LAS u32x4*)(red + 16 + sl * 4) = rec;
#pragma unroll
      for (int i = 0; i < 5; ++i) {
        LAS void* d = (LAS void*)(lds + sl * RSLOT_B + i * 4096);
        const unsigned sg = i == 4 ? so4 : i * 4u < kb ? so + i * 4096u : OOB;
        __builtin_amdgcn_raw_ptr_buffer_load_lds(rs, d, 16, voff, sg, 0, AUX_SYS);
        if (i < 4) {
          __builtin_amdgcn_raw_ptr_buffer_load_lds(rs, d, 16, voff, sg, 1024, AUX_SYS);
          __builtin_amdgcn_raw_ptr_buffer_load_lds(rs, d, 16, voff, sg, 2048, AUX_SYS);
          __builtin_amdgcn_raw_ptr_buffer_load_lds(rs, d, 16, voff, sg, 3072, AUX_SYS);
        } else {
          __builtin_amdgcn_raw_ptr_buffer_load_lds(rs, d, 16, voff, so5, 1024, AUX_SYS);
        }
      }
      WAIT_V(20);
      WAIT_L(0);
      cnt[10] = live ? n : n + 1;
      if (!live) break;
      const int t = (int)(RFL(tk) - qbase);
      const unsigned er = RFL(en);
      if (RFL(part) != 0u) {
        held = t;
        k1 = 18;
      } else {
        p = pn;
        e = er;
        pn = k1 ? held : t;
        k1 = 0;
      }
    }
    return seq;
  }
  const unsigned spare = OFF_FLAG + par * FLAG_B + NSLOT * 1024u;
  unsigned fo = spare, fv = 0;
#define PFLAG() __builtin_amdgcn_raw_buffer_store_b32(fv, rp, one, fo, AUX_SYS)
  for (int n = 0;; ++n) {
    if ((int)RFL(cnt[10]) <= n) {
      DRAIN();
      PFLAG();
      fo = spare;
      const unsigned long long t0 = __builtin_amdgcn_s_memrealtime();
      bool dead = false;
      while ((int)RFL(cnt[10]) <= n) {
        if (__builtin_amdgcn_s_memrealtime() - t0 > 2 * DEADLINE) { dead = true; break; }
        __builtin_amdgcn_s_sleep(1);
      }
      if (dead) break;
    }
    const int sl = n & (RING - 1);
    const u32x4 rec = *(const LAS u32x4*)(red + 16 + sl * 4);
    u32x4 buf[18];
#pragma unroll
    for (int i = 0; i < 18; ++i) buf[i] = *(const LAS u32x4*)(lds + sl * RSLOT_B + i * 1024 + lane * 16);
    WAIT_L(0);
    cnt[11] = n + 1;
    const unsigned dst = RFL(rec[0]);
    unsigned kb = RFL(rec[1]), ne = 0, v14 = OOB;
    if (kb == 255u) break;
    if (kb >= 64u) {
      ne = (kb == 64u ? RFL(buf[16][0]) : kb) & 511u;
      kb = ne > SH_MAX ? 16u : 14u;
      if (ne <= SH_MAX && (unsigned)(lane >> 2) * 32u < ne) v14 = voff;
    }
#define PIECE(W) do { _Pragma("unroll") for (int i = 0; i < 18; ++i) { \
      if (i < (int)kb) { \
        __builtin_amdgcn_raw_buffer_store_b128(buf[i], rd, voff, dst + i * 1024u, AUX_SYS); \
        WAIT_V(W + (i >= W && i < 2 * W ? 2 : 0)); \
        if (i == W - 1) { \
          PFLAG(); \
          __builtin_amdgcn_raw_buffer_store_b128(buf[14], rd, v14, dst + SH_EX, AUX_SYS); \
        } \
      } \
    } \
    if (kb < W) { DRAIN(); PFLAG(); } } while (0)
    if (dst >= OFF_OUTS) PIECE(PU); else if ((int)((unsigned)__builtin_amdgcn_s_memrealtime() - ts0) > (same ? AG_AT0 + n * AG_ATR : AG_XT0 + n * AG_XTR)) PIECE(AG_P2); else PIECE(PL);
#undef PIECE
    fo = RFL(rec[2]);
    fv = RFL(rec[3]) + (ne << 9);
  }
  DRAIN();
  PFLAG();
#undef PFLAG
  return seq;
}

__device__ __forceinline__ int flag_exp(unsigned char* ws, unsigned seq, int slot, int rg) {
  return (int)(ld_sys((const unsigned*)(ws + OFF_FLAG + (seq & 1u) * FLAG_B) + slot * 256 + rg) & 511u) - 256;
}
__device__ __forceinline__ bool call_bad(unsigned char* ws, unsigned seq) {
  return RFL(__hip_atomic_load((unsigned*)(ws + OFF_TMO), __ATOMIC_RELAXED, __HIP_MEMORY_SCOPE_AGENT)) == seq;
}

__device__ __forceinline__ void store_block(const __amdgpu_buffer_rsrc_t ro, f32x4 (&acc)[4][2], int e, bool bad, unsigned so, LAS unsigned char* lb, int lane) {
  const float s1 = __int_as_float((127 + (e >> 1)) << 23), s2 = __int_as_float((127 + e - (e >> 1)) << 23);
  f32x4 v[8];
#pragma unroll
  for (int i = 0; i < 8; ++i) {
    const f32x4 t = acc[i >> 1][i & 1] * s1 * s2;
#pragma unroll
    for (int j = 0; j < 4; ++j) { const int b = __float_as_int(t[j]); v[i][j] = __int_as_float(b < 0 ? 0 : b); }
  }
  if (bad) {
    asm volatile("");
#pragma unroll
    for (int i = 0; i < 8; ++i) v[i] = (f32x4){__builtin_nanf(""), __builtin_nanf(""), __builtin_nanf(""), __builtin_nanf("")};
  }
  store_rows(ro, v, so, lb, lane);
}

__device__ __forceinline__ unsigned unit_ticket(unsigned char* ws, int c, bool take) {
  const __amdgpu_buffer_rsrc_t rc = __builtin_amdgcn_make_buffer_rsrc(ws + OFF_UHEAD, 0, 512, RSRC_W3);
  return (unsigned)__builtin_amdgcn_raw_ptr_buffer_atomic_add_i32(1, rc, take && threadIdx.x == 0 ? 0u : OOB, c * 128, 16);
}
__device__ __forceinline__ unsigned lds_add(LAS unsigned* p, unsigned v) {
  unsigned r = 0;
  if ((threadIdx.x & 63) == 0)
    asm volatile("ds_add_rtn_u32 %0, %1, %2\n\ts_waitcnt lgkmcnt(0)" : "=v"(r) : "v"((unsigned)(unsigned long long)p), "v"(v) : "memory");
  return RFL(r);
}
__device__ __forceinline__ unsigned gather_block(f32x4 (&acc)[4][2], int e, bool bad, LAS unsigned char* cb, unsigned& mk) {
  if (bad) {
    asm volatile("");
#pragma unroll
    for (int i = 0; i < 8; ++i) acc[i >> 1][i & 1] = (f32x4){__builtin_nanf(""), __builtin_nanf(""), __builtin_nanf(""), __builtin_nanf("")};
  }
  const unsigned b0 = RFL((unsigned)(unsigned long long)cb);
  unsigned run = b0;
  mk = 0;
#pragma unroll
  for (int i = 0; i < 32; i += 2) {
    float t0, t1;
    unsigned p0, p1, c0;
    asm volatile(
        "v_ldexp_f32 %0, %7, %9\n\t"
        "v_ldexp_f32 %1, %8, %9\n\t"
        "v_cmp_nge_f32 vcc, 0, %0\n\t"
        "v_cmp_nge_f32 s[92:93], 0, %1\n\t"
        "s_bcnt1_i32_b64 %4, vcc\n\t"
        "v_mbcnt_lo_u32_b32 %2, vcc_lo, 0\n\t"
        "v_mbcnt_lo_u32_b32 %3, s92, 0\n\t"
        "v_mbcnt_hi_u32_b32 %2, vcc_hi, %2\n\t"
        "v_mbcnt_hi_u32_b32 %3, s93, %3\n\t"
        "v_addc_co_u32 %5, s[94:95], %5, %5, vcc\n\t"
        "v_lshl_add_u32 %2, %2, 2, %6\n\t"
        "s_lshl2_add_u32 %6, %4, %6\n\t"
        "v_addc_co_u32 %5, s[94:95], %5, %5, s[92:93]\n\t"
        "v_lshl_add_u32 %3, %3, 2, %6\n\t"
        "s_bcnt1_i32_b64 %4, s[92:93]\n\t"
        "s_mov_b64 exec, vcc\n\t"
        "ds_write_b32 %2, %0\n\t"
        "s_mov_b64 exec, s[92:93]\n\t"
        "ds_write_b32 %3, %1\n\t"
        "s_mov_b64 exec, -1\n\t"
        "s_lshl2_add_u32 %6, %4, %6"
        : "=&v"(t0), "=&v"(t1), "=&v"(p0), "=&v"(p1), "=&s"(c0), "+v"(mk), "+s"(run)
        : "v"(acc[i >> 3][(i >> 2) & 1][i & 3]), "v"(acc[i >> 3][(i >> 2) & 1][(i & 3) + 1]), "v"(e)
        : "vcc", "scc", "s92", "s93", "s94", "s95", "memory");
  }
  WAIT_L(0);
  return run - b0;
}
__device__ __forceinline__ void pack_block(unsigned char* ws, unsigned seq, int c, unsigned& tk, bool more, f32x4 (&acc)[4][2], int e, bool bad, LAS unsigned char* cb, LAS unsigned* u, int wid, int lane) {
  unsigned mk;
  const unsigned nb = gather_block(acc, e, bad, cb, mk), n64 = (nb + 63u) & ~63u;
  const unsigned off = lds_add(u + 1, n64);
  const unsigned vo = (unsigned)lane * 16u;
  u32x4 v[8];
#pragma unroll
  for (int i = 0; i < 8; ++i) v[i] = *(const LAS u32x4*)(cb + i * 1024 + vo);
  unsigned pos = tk - (seq - 1u) * NUNIT;
  if (threadIdx.x == 0) *(volatile LAS unsigned*)u = (unsigned)c * 64u + (pos < 63u ? pos : 63u) + 1u;
  while ((pos = RFL(*(volatile LAS unsigned*)u)) == 0u) __builtin_amdgcn_s_sleep(1);
  unsigned char* rec = ws + CTRL_B + OFF_OUTS + (size_t)(pos - 1u) * UREC_B;
  const __amdgpu_buffer_rsrc_t rh = __builtin_amdgcn_make_buffer_rsrc(rec, 0, UHDR_B, RSRC_W3);
  const __amdgpu_buffer_rsrc_t rv = __builtin_amdgcn_make_buffer_rsrc(rec + UHDR_B + off, 0, n64, RSRC_W3);
  tk = unit_ticket(ws, c, more);
  __builtin_amdgcn_raw_buffer_store_b32(mk, rh, (unsigned)lane * 4u, (unsigned)wid * 256u, AUX_SYS);
  __builtin_amdgcn_raw_buffer_store_b64((u32x2){off, nb}, rh, lane == 0 ? 0u : OOB, 2048u + (unsigned)wid * 8u, AUX_SYS);
#pragma unroll
  for (int i = 0; i < 8; ++i) __builtin_amdgcn_raw_buffer_store_b128(v[i], rv, vo + (i & 3) * 1024u, (i >> 2) * 4096u, AUX_SYS);
  STORE_GAP();
}
__device__ __forceinline__ void post_unit(unsigned char* ws, unsigned seq, LAS unsigned* u, int unit) {
  if (lds_add(u + 2, 1u) == 7u) {
    const unsigned pos = *(volatile LAS unsigned*)u - 1u, all = UHDR_B + *(volatile LAS unsigned*)(u + 1);
    if ((threadIdx.x & 63) == 0) st_sys((unsigned*)(ws + OFF_ULIST) + pos, (seq << 16) | ((all + 1023u) >> 10 << 8) | (unsigned)unit);
    TR(pos, 0, NOW()); TR(pos, 9, (unsigned long long)unit);
  }
}

#define AG_LAUX AUX_SYS
#define DMA2(ldsoff, rs, so) do { \
    __builtin_amdgcn_raw_ptr_buffer_load_lds(rs, (LAS void*)(lds + (ldsoff) + ldsw), 16, voff, so, 0, AG_LAUX); \
    __builtin_amdgcn_raw_ptr_buffer_load_lds(rs, (LAS void*)(lds + (ldsoff) + ldsw), 16, voff, so, 1024, AG_LAUX); } while (0)
#define MMA16(ac, At, Bt) do { __builtin_amdgcn_s_setprio(1); \
    _Pragma("unroll") for (int m = 0; m < 4; ++m) _Pragma("unroll") for (int n = 0; n < 2; ++n) _Pragma("unroll") for (int k = 0; k < 2; ++k) \
      ac[m][n] = __builtin_amdgcn_mfma_f32_16x16x32_f16(Bt[n][k], At[m][k], ac[m][n], 0, 0, 0); \
    __builtin_amdgcn_s_setprio(0); } while (0)

__device__ __forceinline__ int seen_flags(const u32x4 f, int sel, int p0, unsigned s9) {
  int old = 0;
#pragma unroll
  for (int j = 0; j < 4; ++j) old |= (int)((f[j] - s9) >> 31) << j;
  const unsigned long long m = __ballot(p0 + (int)(threadIdx.x & 31) >= NPAIR || (old & sel) == 0);
  const unsigned miss = ~((unsigned)m & (unsigned)(m >> 32));
  return p0 + (miss ? __builtin_ctz(miss) : 32);
}

constexpr int NBQ = 4;
#define AG_RT 6
#define AG_RO 1
#define AG_QP 1
constexpr unsigned LDS_F = NBQ * KSTEP_B;
#define AG_HKA 0xffffffffu
#define AG_HKB 0xffffffffu
#define DMA4(lo, rs, vo, so) do { LAS void* _d = (LAS void*)(lds + (lo)); \
    __builtin_amdgcn_raw_ptr_buffer_load_lds(rs, _d, 16, vo, so, 0, AG_LAUX); __builtin_amdgcn_raw_ptr_buffer_load_lds(rs, _d, 16, vo, so, 1024, AG_LAUX); \
    __builtin_amdgcn_raw_ptr_buffer_load_lds(rs, _d, 16, vo, so, 2048, AG_LAUX); __builtin_amdgcn_raw_ptr_buffer_load_lds(rs, _d, 16, vo, so, 3072, AG_LAUX); } while (0)
struct Feed {
  __amdgpu_buffer_rsrc_t ra, rb;
  unsigned voff, sot, fa, fb, so, bo, s9;
  int ki, here, nk;
  __device__ __forceinline__ Feed(unsigned char* ws, unsigned seq, int xs, int ah, int wsl, int bh, unsigned long long mask, int k0) {
    const int w8 = __builtin_amdgcn_readfirstlane(threadIdx.x >> 6), lane = threadIdx.x & 63, g = (w8 >> 1) & 1;
    unsigned char* base = ws + CTRL_B + (size_t)(seq & 1u) * PAR_B;
    ra = __builtin_amdgcn_make_buffer_rsrc(base + (size_t)xs * SLOT_B + ah * 2 * PIECE_B, 0, SLOT_B - ah * 2 * PIECE_B, RSRC_W3);
    rb = __builtin_amdgcn_make_buffer_rsrc(base + (size_t)wsl * SLOT_B + bh * 2 * PIECE_B, 0, SLOT_B - bh * 2 * PIECE_B, RSRC_W3);
    nk = (int)RFL(2 * __builtin_popcountll(mask));
    voff = (unsigned)g * PIECE_B + (unsigned)(w8 & 1) * 4096u + (unsigned)lane * 16u;
    const int rk = __builtin_popcountll(mask & ((1ull << lane) - 1ull));
    const int at = !((mask >> lane) & 1ull) ? 63 : rk < ((nk / 2 - AG_RT) & ~3) ? (rk & ~3) | ((rk - (AG_RO == 2 ? wsl < SLOT_WR : AG_RO) * (ah * 2 + bh + xs)) & 3) : rk;
    const unsigned tab = (unsigned)__builtin_amdgcn_ds_permute(at * 4, lane);
    sot = tab * PAIR_B;
    fa = tab * 16u + (unsigned)(2 * ah + g) * 4u;
    fb = tab * 16u + (unsigned)(2 * bh + g) * 4u;
    s9 = seq << 9;
    ki = k0;
    here = 0;
    bo = (unsigned)(w8 & 3) * 4096u + (unsigned)k0 * KSTEP_B;
    so = (unsigned)__builtin_amdgcn_readlane(sot, k0 >> 1) | (unsigned)(k0 & 1) * KK_B;
  }
  __device__ __forceinline__ void next(LAS unsigned char* lds) {
    DMA4(bo, ra, voff, so & AG_HKA); DMA4(bo + HALF_B, rb, voff, so & AG_HKB);
    step();
  }
  __device__ __forceinline__ void step() {
    ++ki;
    so = (unsigned)__builtin_amdgcn_readlane(sot, (ki >> 1) & 63) | (unsigned)(ki & 1) * KK_B;
    bo = (bo + KSTEP_B) & (NBQ * KSTEP_B - 1u);
  }
  __device__ __forceinline__ void seen(unsigned x, unsigned y) {
    const unsigned long long m = __ballot((int)((x - s9) | (y - s9)) < 0);
    const int n = m ? 2 * __builtin_ctzll(m) : nk;
    here = n > here ? (n < nk ? n : nk) : here;
  }
};

__device__ __forceinline__ void gemm_full(const Params& P, unsigned char* ws, unsigned seq, int xs, int wsl, unsigned long long mask, int job, int hand, f32x4 (&keep)[2][4][2], int pre, LAS unsigned char* lds) {
  const int tid = threadIdx.x, wid = __builtin_amdgcn_readfirstlane(tid >> 6), lane = tid & 63;
  const int wr = wid >> 2, wc = wid & 3, fr = lane & 15, fq = lane >> 4;
  const unsigned par = seq & 1u;
  const __amdgpu_buffer_rsrc_t ra = __builtin_amdgcn_make_buffer_rsrc(ws + CTRL_B + (size_t)par * PAR_B + (size_t)xs * SLOT_B, 0, SLOT_B, RSRC_W3);
  const __amdgpu_buffer_rsrc_t rb = __builtin_amdgcn_make_buffer_rsrc(ws + CTRL_B + (size_t)par * PAR_B + (size_t)wsl * SLOT_B, 0, SLOT_B, RSRC_W3);
  const __amdgpu_buffer_rsrc_t rf = __builtin_amdgcn_make_buffer_rsrc(ws + OFF_FLAG + RFL(par * FLAG_B), 0, FLAG_B, RSRC_W3);
  const unsigned fo = (unsigned)(lane < 32 ? xs : wsl) * 1024u + (unsigned)(lane & 31) * 16u, fa = wid == 0 ? fo : OOB;
  const unsigned voff = (unsigned)wr * PIECE_B + (unsigned)wc * 2048u + (unsigned)lane * 16u;
  const unsigned ldsw = (unsigned)wid * 2048u;
  const int sw0 = (fq ^ (fr & 7)) * 16;
  const int aoff0 = (wr * 64 + fr) * 128 + sw0, aoff1 = aoff0 ^ 64;
  const int boff0 = 4 * HALF_B + (wc * 32 + fr) * 128 + sw0, boff1 = boff0 ^ 64;
#define SA(b, h) (((b) * 2 + (h)) * HALF_B)
#define SB(b, h) ((4 + (b) * 2 + (h)) * HALF_B)
#define STAGE(bufoff, rs, po, kk, h) DMA2(bufoff, rs, (po) + (kk) * KK_B + (h) * 2 * PIECE_B)
#define LDA(dst, b, h) do { _Pragma("unroll") for (int m = 0; m < 4; ++m) { \
    dst[m][0] = *(const LAS f16x8*)(lds + SA(b, h) + aoff0 + m * 2048); dst[m][1] = *(const LAS f16x8*)(lds + SA(b, h) + aoff1 + m * 2048); } } while (0)
#define LDB(dst, b, h) do { _Pragma("unroll") for (int n = 0; n < 2; ++n) { \
    dst[n][0] = *(const LAS f16x8*)(lds + SA(b, h) + boff0 + n * 2048); dst[n][1] = *(const LAS f16x8*)(lds + SA(b, h) + boff1 + n * 2048); } } while (0)
#define MMA(ai, bj, At, Bt) MMA16(acc[ai][bj], At, Bt)
  f32x4 acc[2][2][4][2];
#pragma unroll
  for (int a = 0; a < 2; ++a)
#pragma unroll
    for (int b = 0; b < 2; ++b)
#pragma unroll
      for (int m = 0; m < 4; ++m)
#pragma unroll
        for (int n = 0; n < 2; ++n) acc[a][b][m][n] = (f32x4){0.f, 0.f, 0.f, 0.f};
  f16x8 At[4][2], B0[2][2], B1[2][2];
#define NEXTP(c) do { c = mask ? __builtin_ctzll(mask) : NPAIR; mask &= mask - 1ull; } while (0)
  int avail = pre, c1, c2;
  int early = pre;
  NEXTP(c1);
  NEXTP(c2);
  if (!pre) {
    if (wid == 0) avail = wait_flags(ws, fo, 15, c1, c1 + 1, seq);
    BAR;
  }
  u32x4 ff;
  unsigned pn = (unsigned)c1 * PAIR_B;
  if (pre) {
    ff = *(const LAS u32x4*)(lds + LDS_F + (lane & 32) * 32 + ((c2 + (lane & 31)) & 63) * 16);
    if (wr == 1) BAR;
    WAIT_V(16); BAR;
    STAGE(SB(1, 0), rb, pn, 1, 0); STAGE(SA(1, 0), ra, pn, 1, 0); STAGE(SB(1, 1), rb, pn, 1, 1);
  } else {
    ff = __builtin_amdgcn_raw_buffer_load_b128(rf, fa, (unsigned)c2 * 16u, AUX_SYS);
    SCHED;
    STAGE(SB(0, 0), rb, pn, 0, 0); STAGE(SA(0, 0), ra, pn, 0, 0); STAGE(SB(0, 1), rb, pn, 0, 1); STAGE(SA(0, 1), ra, pn, 0, 1);
    if (wr == 1) BAR;
    WAIT_V(4); BAR;
    STAGE(SB(1, 0), rb, pn, 1, 0); STAGE(SA(1, 0), ra, pn, 1, 0); STAGE(SB(1, 1), rb, pn, 1, 1);
    WAIT_V(6);
  }
  BAR;
  do {
    const unsigned po = pn;
    c1 = c2;
    NEXTP(c2);
    pn = (unsigned)c1 * PAIR_B;
    if (wid == 0 && c1 >= avail && c1 < NPAIR) {
      avail = seen_flags(ff, 15, c1, seq << 9);
      if (c1 >= avail) avail = wait_flags(ws, fo, 15, c1, c1 + 1, seq);
    }
    LDB(B0, 0, 0); SCHED; LDA(At, 0, 0); STAGE(SA(1, 1), ra, po, 1, 1);
    if (!early) WAIT_V(10);
    WAIT_L(8); BAR; WAIT_L(0); MMA(0, 0, At, B0); BAR; SCHED;
    LDB(B1, 0, 1); STAGE(SB(0, 0), rb, pn, 0, 0);
    if (!early) WAIT_V(10);
    BAR; WAIT_L(0); MMA(0, 1, At, B1); BAR;
    LDA(At, 0, 1);
    asm volatile("" :: "v"(ff));
    ff = __builtin_amdgcn_raw_buffer_load_b128(rf, fa, (unsigned)c2 * 16u, AUX_SYS);
    STAGE(SA(0, 0), ra, pn, 0, 0);
    BAR; WAIT_L(0); MMA(1, 0, At, B0); BAR; SCHED;
    STAGE(SB(0, 1), rb, pn, 0, 1);
    WAIT_V(11);
    early = 0;
    if (job >= 0) asm volatile("" : "+s"(early));
    BAR; MMA(1, 1, At, B1); BAR;
    LDB(B0, 1, 0); SCHED; LDA(At, 1, 0); STAGE(SA(0, 1), ra, pn, 0, 1);
    WAIT_V(11); WAIT_L(8); BAR; WAIT_L(0); MMA(0, 0, At, B0); BAR; SCHED;
    LDB(B1, 1, 1); STAGE(SB(1, 0), rb, pn, 1, 0);
    WAIT_V(11); BAR; WAIT_L(0); MMA(0, 1, At, B1); BAR;
    LDA(At, 1, 1); STAGE(SA(1, 0), ra, pn, 1, 0);
    BAR; WAIT_L(0); MMA(1, 0, At, B0); BAR; SCHED;
    STAGE(SB(1, 1), rb, pn, 1, 1);
    WAIT_V(10); BAR; MMA(1, 1, At, B1); BAR;
  } while (c1 < NPAIR);
#undef NEXTP
  asm volatile("" :: "v"(ff));
  const bool remote = wsl >= SLOT_WR;
  int tz = threadIdx.x; asm volatile("" : "+v"(tz));
  if (job >= 0) {
    const __amdgpu_buffer_rsrc_t rq = __builtin_amdgcn_make_buffer_rsrc(ws + CTRL_B + OFF_PART + job * 262144u + wid * 8192u, 0, 3 * 65536 + 8192, RSRC_W3);
#pragma unroll
    for (int i = 0; i < 32; ++i) asm volatile("" : "+v"(acc[i >> 4][(i >> 3) & 1][(i >> 1) & 3][i & 1]));
    SCHED;
#pragma unroll
    for (int i = 0; i < 32; ++i)
      __builtin_amdgcn_raw_buffer_store_b128(__builtin_bit_cast(u32x4, acc[i >> 4][(i >> 3) & 1][(i >> 1) & 3][i & 1]), rq, (unsigned)(tz & 63) * 16u + (i & 3) * 1024u, (i >> 3) * 65536u + (i & 4) * 1024u, AUX_SYS);
    STORE_GAP();
    const __amdgpu_buffer_rsrc_t rg = __builtin_amdgcn_make_buffer_rsrc(ws + part_words(xs, wsl, 0) + (unsigned)((job % 3) * 8 + wid) * 4u, 0, 512, RSRC_W3);
    const unsigned one = (tz & 63) == 0 ? 0u : OOB;
#pragma unroll
    for (int i = 0; i < 4; ++i) {
      WAIT_V(24 - 7 * i);
      __builtin_amdgcn_raw_buffer_store_b32(seq, rg, one, i * 128, AUX_SYS);
    }
    if (wr == 0) BAR;
    TS(29);
    return;
  }
  if (hand >= 0) {
    const __amdgpu_buffer_rsrc_t rq = __builtin_amdgcn_make_buffer_rsrc(ws + CTRL_B + OFF_HAND + hand * 131072u + wid * 16384u, 0, 16384, RSRC_W3);
#pragma unroll
    for (int i = 0; i < 16; ++i) asm volatile("" : "+v"(acc[1][i >> 3][(i >> 1) & 3][i & 1]));
    SCHED;
#pragma unroll
    for (int i = 0; i < 16; ++i)
      __builtin_amdgcn_raw_buffer_store_b128(__builtin_bit_cast(u32x4, acc[1][i >> 3][(i >> 1) & 3][i & 1]), rq, (unsigned)(tz & 63) * 16u + (i & 3) * 1024u, (i >> 2) * 4096u, AUX_SYS);
    STORE_GAP();
#pragma unroll
    for (int i = 0; i < 16; ++i) keep[i >> 3][(i >> 1) & 3][i & 1] = acc[0][i >> 3][(i >> 1) & 3][i & 1];
    if (wr == 0) BAR;
    return;
  }
  LAS unsigned* red = (LAS unsigned*)(lds + LDS_MAIN);
  unsigned tk = unit_ticket(ws, wsl - SLOT_WR, remote);
  const int eb = flag_exp(ws, seq, wsl, 0);
  const int e0 = flag_exp(ws, seq, xs, wr) + eb, e1 = flag_exp(ws, seq, xs, 2 + wr) + eb;
  const bool bad = call_bad(ws, seq);
  WAIT_V(0);
  WAIT_L(0);
  if (wr == 0) BAR;
  BAR;
  if (remote) {
    const int u0 = (xs < SLOT_G3 ? xs : xs - 1) * 4;
#pragma unroll
    for (int q = 0; q < 4; ++q) {
      pack_block(ws, seq, wsl - SLOT_WR, tk, q < 3, acc[q >> 1][q & 1], (q >> 1) ? e1 : e0, bad, lds + wid * 8192, red + 16 + q * 4, wid, tz & 63);
      if (q) { WAIT_V(11); post_unit(ws, seq, red + 12 + q * 4, u0 + q - 1); }
    }
    DRAIN();
    post_unit(ws, seq, red + 28, u0 + 3);
  } else {
    const __amdgpu_buffer_rsrc_t ro = __builtin_amdgcn_make_buffer_rsrc(P.out[P.rank], 0, 8192 * NLOC * 4, RSRC_W3);
#pragma unroll
    for (int q = 0; q < 4; ++q)
      store_block(ro, acc[q >> 1][q & 1], (q >> 1) ? e1 : e0, bad,
                  (unsigned)(xs == SLOT_T3 ? P.rank * 1024 + 768 + ((q >> 1) * 2 + wr) * 64 : group_row0(P.rank & 1, xs, (q >> 1) * 2 + wr)) * (NLOC * 4u)
                  + (unsigned)(wsl - SLOT_W) * 1024u + (q & 1) * 512u + wc * 128u, lds + wid * 8192, tz & 63);
  }
#undef SA
#undef SB
#undef STAGE
#undef LDA
#undef LDB
#undef MMA
}

__device__ __forceinline__ int gemm_half(const Params& P, unsigned char* ws, unsigned seq, int xs, int ah, int wsl, int p0, int hand, f32x4 (&acc)[2][4][2],
                                         int nt, int nxs, int nws, unsigned long long nmask, int nq, LAS unsigned char* lds) {
  const int tid = threadIdx.x, wid = __builtin_amdgcn_readfirstlane(tid >> 6), lane = tid & 63;
  const int wr = wid >> 2, wc = wid & 3, fr = lane & 15, fq = lane >> 4;
  const unsigned par = seq & 1u;
  const __amdgpu_buffer_rsrc_t ra = __builtin_amdgcn_make_buffer_rsrc(ws + CTRL_B + (size_t)par * PAR_B + (size_t)xs * SLOT_B + ah * 2 * PIECE_B, 0, SLOT_B - ah * 2 * PIECE_B, RSRC_W3);
  const __amdgpu_buffer_rsrc_t rb = __builtin_amdgcn_make_buffer_rsrc(ws + CTRL_B + (size_t)par * PAR_B + (size_t)wsl * SLOT_B, 0, SLOT_B, RSRC_W3);
  const __amdgpu_buffer_rsrc_t rf = __builtin_amdgcn_make_buffer_rsrc(ws + OFF_FLAG + RFL(par * FLAG_B), 0, FLAG_B, RSRC_W3);
  const unsigned fo = (unsigned)(lane < 32 ? xs : wsl) * 1024u + (unsigned)(lane & 31) * 16u, fa = wid == 0 ? fo : OOB;
  const int sel = lane < 32 ? 3 << (2 * ah) : 15;
  const unsigned voff = (unsigned)wr * PIECE_B + (unsigned)wc * 2048u + (unsigned)lane * 16u;
  const unsigned ldsw = (unsigned)wid * 2048u;
  const int sw0 = (fq ^ (fr & 7)) * 16;
  int aoff0 = (wr * 64 + fr) * 128 + sw0, aoff1 = aoff0 ^ 64;
  const int boff0 = 4 * HALF_B + (wc * 32 + fr) * 128 + sw0, boff1 = boff0 ^ 64;
#define SB(b, h) ((4 + (b) * 2 + (h)) * HALF_B)
#define STAGE(bufoff, rs, po, kk, h) DMA2(bufoff, rs, (po) + (kk) * KK_B + (h) * 2 * PIECE_B)
#define STA(kk, i) __builtin_amdgcn_raw_ptr_buffer_load_lds(ra, (LAS void*)(lds + an + (kk) * 2 * HALF_B + ldsw), 16, voff, pn + (kk) * KK_B, (i) * 1024, AG_LAUX)
#define LDA(dst, b) do { _Pragma("unroll") for (int m = 0; m < 4; ++m) { \
    dst[m][0] = *(const LAS f16x8*)(lds + (b) * 2 * HALF_B + aoff0 + m * 2048); dst[m][1] = *(const LAS f16x8*)(lds + (b) * 2 * HALF_B + aoff1 + m * 2048); } } while (0)
#define LDB(dst, b, h) do { _Pragma("unroll") for (int n = 0; n < 2; ++n) { \
    dst[n][0] = *(const LAS f16x8*)(lds + ((b) * 2 + (h)) * HALF_B + boff0 + n * 2048); dst[n][1] = *(const LAS f16x8*)(lds + ((b) * 2 + (h)) * HALF_B + boff1 + n * 2048); } } while (0)
  f16x8 At[4][2], Bt[2][2];
  int avail = 0;
  if (wid == 0) avail = wait_flags(ws, fo, sel, p0, p0 + 1, seq);
  BAR;
  u32x4 ff = __builtin_amdgcn_raw_buffer_load_b128(rf, fa, (unsigned)(p0 + 1) * 16u, AUX_SYS);
  SCHED;
  unsigned an = 0, pn = (unsigned)p0 * PAIR_B;
  STAGE(SB(0, 0), rb, pn, 0, 0); STA(0, 0); STA(0, 1); STAGE(SB(0, 1), rb, pn, 0, 1);
  if (wr == 1) BAR;
  WAIT_V(2); BAR;
  STAGE(SB(1, 0), rb, pn, 1, 0); STA(1, 0); STA(1, 1);
  WAIT_V(4); BAR;
  unsigned* hfl = (unsigned*)(ws + OFF_PFL) + hand;
  unsigned hf = seq;
  if (p0 && !ah && tid == 256) st_sys(hfl, seq);
  for (int t = p0; t < NPAIR - 1; ++t) {
    if (t == NPAIR - 2 && p0 && ah) hf = ld_sys(hfl);
    const unsigned po = pn;
    pn += PAIR_B;
    an ^= HALF_B;
    if (wid == 0 && t + 2 > avail) {
      avail = seen_flags(ff, sel, t + 1, seq << 9);
      if (t + 2 > avail) avail = wait_flags(ws, fo, sel, t + 1, t + 2, seq);
    }
    LDB(Bt, 0, 0); SCHED; LDA(At, 0); STAGE(SB(1, 1), rb, po, 1, 1);
    WAIT_V(5); WAIT_L(8); BAR; WAIT_L(0); MMA16(acc[0], At, Bt); BAR; SCHED;
    LDB(Bt, 0, 1); STA(0, 0); STA(0, 1); STAGE(SB(0, 0), rb, pn, 0, 0);
    WAIT_V(6); BAR; WAIT_L(0); MMA16(acc[1], At, Bt); BAR; SCHED;
    LDB(Bt, 1, 0); SCHED; LDA(At, 1);
    asm volatile("" :: "v"(ff));
    ff = __builtin_amdgcn_raw_buffer_load_b128(rf, fa, (unsigned)(t + 2) * 16u, AUX_SYS);
    STA(1, 0); STAGE(SB(0, 1), rb, pn, 0, 1);
    WAIT_V(8); WAIT_L(8); BAR; WAIT_L(0); MMA16(acc[0], At, Bt); BAR; SCHED;
    LDB(Bt, 1, 1); STA(1, 1); STAGE(SB(1, 0), rb, pn, 1, 0);
    WAIT_V(7); BAR; WAIT_L(0); MMA16(acc[1], At, Bt); BAR; SCHED;
    aoff0 ^= HALF_B;
    aoff1 ^= HALF_B;
  }
  asm volatile("" :: "v"(ff));
  int tz = threadIdx.x; asm volatile("" : "+v"(tz));
  LAS unsigned* red = (LAS unsigned*)(lds + LDS_MAIN);
  const bool remote = wsl >= SLOT_WR, late = RFL(hf) != seq;
  unsigned char* hp = ws + CTRL_B + OFF_HAND + hand * 131072u + wid * 16384u;
  u32x4 hs[16];
#define HAND(n, i0, i1) do { const __amdgpu_buffer_rsrc_t rq = __builtin_amdgcn_make_buffer_rsrc(hp, 0, n, RSRC_W3); \
    _Pragma("unroll") for (int i = i0; i < i1; ++i) hs[i] = __builtin_amdgcn_raw_buffer_load_b128(rq, (unsigned)(tz & 63) * 16u + (i & 3) * 1024u, (i >> 2) * 4096u, AUX_SYS); } while (0)
  LDB(Bt, 0, 0); SCHED; LDA(At, 0); STAGE(SB(1, 1), rb, pn, 1, 1);
  SCHED;
  const __amdgpu_buffer_rsrc_t rn = __builtin_amdgcn_make_buffer_rsrc(wid == 2 ? ws + part_words(nxs, nws, nq) : ws + OFF_FLAG + par * FLAG_B + (wid ? nws : nxs) * 1024, 0, wid < 2 ? 1024 : wid == 2 ? 128 : 0, RSRC_W3);
  __builtin_amdgcn_raw_ptr_buffer_load_lds(rn, (LAS void*)(lds + LDS_F + wid * 1024), 16, (tz & 63) * 16, 0, 0, AUX_SYS);
  unsigned tk = unit_ticket(ws, wsl - SLOT_WR, remote);
  const unsigned ex = __builtin_amdgcn_raw_buffer_load_b32(rf, (tz & 32) ? wsl * 1024u : xs * 1024u + (ah * 2 + wr) * 4u, (unsigned)p0 * 16u, AUX_SYS);
  unsigned tm = __hip_atomic_load((unsigned*)(ws + OFF_TMO), __ATOMIC_RELAXED, __HIP_MEMORY_SCOPE_AGENT);
  const int hn = p0 && ah && !late ? 16384 : 0;
  HAND(hn, 0, 2);
  WAIT_V(11); WAIT_L(8); BAR; WAIT_L(0); MMA16(acc[0], At, Bt); BAR; SCHED;
  LDB(Bt, 0, 1);
  HAND(hn, 2, 9);
  WAIT_V(15); BAR; WAIT_L(0); MMA16(acc[1], At, Bt); BAR; SCHED;
  LDB(Bt, 1, 0); SCHED; LDA(At, 1);
  HAND(hn, 9, 16);
  WAIT_V(20); WAIT_L(8); BAR; WAIT_L(0); MMA16(acc[0], At, Bt); BAR; SCHED;
  LDB(Bt, 1, 1);
  BAR; WAIT_L(0); MMA16(acc[1], At, Bt); BAR; SCHED;
  if (late) {
    const unsigned long long t0 = __builtin_amdgcn_s_memrealtime();
    while (RFL(ld_sys(hfl)) != seq && !timed_out((unsigned*)(ws + OFF_TMO), seq, t0)) __builtin_amdgcn_s_sleep(AG_Z5);
    asm volatile("" ::: "memory");
    HAND(16384, 0, 16);
    DRAIN();
    tm = __hip_atomic_load((unsigned*)(ws + OFF_TMO), __ATOMIC_RELAXED, __HIP_MEMORY_SCOPE_AGENT);
  }
#undef HAND
  __builtin_amdgcn_s_waitcnt(0x0F70);
  SCHED;
  if (wr == 0) BAR;
  int pre = 0;
  if (nt == 1) {
    const int c = __builtin_ctzll(nmask);
    const int av = seen_flags(*(const LAS u32x4*)(lds + LDS_F + (tz & 32) * 32 + ((c + (tz & 31)) & 63) * 16), 15, c, seq << 9);
    if (av > c) {
      const __amdgpu_buffer_rsrc_t na = __builtin_amdgcn_make_buffer_rsrc(ws + CTRL_B + (size_t)par * PAR_B + (size_t)nxs * SLOT_B, 0, SLOT_B, RSRC_W3);
      const __amdgpu_buffer_rsrc_t nb = __builtin_amdgcn_make_buffer_rsrc(ws + CTRL_B + (size_t)par * PAR_B + (size_t)nws * SLOT_B, 0, SLOT_B, RSRC_W3);
      pn = (unsigned)c * PAIR_B;
      STAGE(SB(0, 0), nb, pn, 0, 0); STAGE(0, na, pn, 0, 0); STAGE(SB(0, 1), nb, pn, 0, 1); STAGE(HALF_B, na, pn, 0, 1);
      pre = av < NPAIR ? av : NPAIR;
    }
  } else if (nt == 2 && wr) {
    Feed q(ws, seq, nxs, nq >> 1, nws, nq & 1, nmask, 0);
    q.seen(*(const LAS unsigned*)(lds + LDS_F + q.fa), *(const LAS unsigned*)(lds + LDS_F + 1024 + q.fb));
    while (q.ki < q.here && q.ki < 2) q.next(lds);
    pre = q.ki;
  }
  if (nt == 2 && !remote) BAR;
  SCHED;
  const bool bad = RFL(tm) == seq;
  const int e = (int)(__builtin_amdgcn_readlane(ex, 0) & 511u) + (int)(__builtin_amdgcn_readlane(ex, 32) & 511u) - 512;
#pragma unroll
  for (int i = 0; i < 16; ++i) acc[i >> 3][(i >> 1) & 3][i & 1] += __builtin_bit_cast(f32x4, hs[i]);
  asm volatile("" : "+s"(wsl), "+s"(nt));
  LAS unsigned char* cb = lds + wc * 8192 + (nt == 1 ? 2 * HALF_B + wr * 4 * HALF_B : 4 * HALF_B + wr * 2 * HALF_B);
  if (wsl >= SLOT_WR) {
    const int u0 = (xs < SLOT_G3 ? xs : xs - 1) * 4 + ah * 2;
    pack_block(ws, seq, wsl - SLOT_WR, tk, true, acc[0], e, bad, cb, red + 16, wid, tz & 63);
    pack_block(ws, seq, wsl - SLOT_WR, tk, false, acc[1], e, bad, cb, red + 20, wid, tz & 63);
    WAIT_V(11);
    post_unit(ws, seq, red + 16, u0);
    DRAIN();
    __builtin_amdgcn_s_waitcnt(0x0F70);
    post_unit(ws, seq, red + 20, u0 + 1);
  } else {
    const __amdgpu_buffer_rsrc_t ro = __builtin_amdgcn_make_buffer_rsrc(P.out[P.rank], 0, 8192 * NLOC * 4, RSRC_W3);
#pragma unroll
    for (int q = 0; q < 2; ++q)
      store_block(ro, acc[q], e, bad,
                  (unsigned)(xs == SLOT_T3 ? P.rank * 1024 + 768 + (ah * 2 + wr) * 64 : group_row0(P.rank & 1, xs, ah * 2 + wr)) * (NLOC * 4u)
                  + (unsigned)(wsl - SLOT_W) * 1024u + q * 512u + wc * 128u, cb, tz & 63);
  }
  return (int)RFL(pre);
#undef SB
#undef STAGE
#undef STA
#undef LDA
#undef LDB
}

#define AG_P1 1
#define AG_DW 1
#define WAIT_L0() __builtin_amdgcn_s_waitcnt(0xC07F)
__device__ __forceinline__ void gemm_quarter(const Params& P, unsigned char* ws, unsigned seq, int xs, int ah, int wsl, int bh, unsigned long long mask, int job, int np, int pre, LAS unsigned char* lds) {
  const int tid = threadIdx.x, w8 = __builtin_amdgcn_readfirstlane(tid >> 6), lane = tid & 63;
  seq = RFL(seq);
  xs = (int)RFL(xs); wsl = (int)RFL(wsl); ah = (int)RFL(ah); bh = (int)RFL(bh); job = (int)RFL(job); np = (int)RFL(np);
  const unsigned par = seq & 1u;
  const int nk = (int)RFL(2 * __builtin_popcountll(mask));
  const int cq = ah * 2 + bh;
  f32x4 acc[4][2];
  LAS unsigned char* xb = lds + 3 * KSTEP_B + (w8 & 3) * 8192 + lane * 16;
#define NEXTB(o) o = o == (NBQ - 1) * KSTEP_B ? 0u : o + KSTEP_B
  if (w8 >= 4) {
    Feed q(ws, seq, xs, ah, wsl, bh, mask, pre);
    int &ki = q.ki, &here = q.here;
    const unsigned po = (unsigned)(lane & 3) * 32u + (unsigned)(w8 & 3) * 8u;
    const volatile LAS u32x2* lp = (const volatile LAS u32x2*)(lds + LDS_F + 2048 + po);
#define BOTH(z) (unsigned)__ballot(z[0] == seq && z[1] == seq)
    const __amdgpu_buffer_rsrc_t rq = __builtin_amdgcn_make_buffer_rsrc(ws + CTRL_B + OFF_PART + job * 262144u + cq * 65536u + (w8 & 3) * 16384u, 0, 2 * 262144 + 16384, RSRC_W3);
    const __amdgpu_buffer_rsrc_t rf = __builtin_amdgcn_make_buffer_rsrc(ws + OFF_FLAG + par * FLAG_B, 0, FLAG_B, RSRC_W3);
    const volatile LAS unsigned* la = (const volatile LAS unsigned*)(lds + LDS_F + q.fa);
    const volatile LAS unsigned* lb = (const volatile LAS unsigned*)(lds + LDS_F + 1024 + q.fb);
#define NEXT() q.next(lds)
#define SEEN(x, y) q.seen(x, y)
#define PARTS(rdy) do { _Pragma("unroll") for (int b = 0; b < 3; ++b) if (((rdy) >> b) & 1u) { \
      _Pragma("unroll") for (int i = 0; i < 16; ++i) pt[b][i] = __builtin_amdgcn_raw_buffer_load_b128(rq, lane * 16u + (i & 3) * 1024u, b * 262144u + (i >> 2) * 4096u, AUX_SYS); \
      if (w8 == 4) KT(100 + b, ki); } \
      owed &= ~(rdy); } while (0)
    if (w8 == 4) KCLR();
    __builtin_amdgcn_s_setprio(AG_QP);
    SEEN(*la, *lb);
    while (ki < here && ki < NBQ - 1) NEXT();
    int k = 0;
    for (; ki < nk; ++k) {
      if (ki == k + NBQ - 1 && ki < here && pre <= k && !(ki & 1)) {
#define QKT()
#define GO() do { WAIT_V(8 * (NBQ - 2)); BAR; SCHED; } while (0)
        unsigned so = q.so, bo = q.bo;
        GO();
        unsigned x = *la, y = *lb;
        DMA4(bo, q.ra, q.voff, so & AG_HKA);
        SCHED;
        for (;;) {
          QKT();
          SEEN(x, y);
          SCHED;
          DMA4(bo + HALF_B, q.rb, q.voff, so & AG_HKB);
          GO();
          DMA4(bo + KSTEP_B, q.ra, q.voff, (so + KK_B) & AG_HKA);
          SCHED;
          ++k;
          QKT();
          ki += 2;
          const unsigned s1 = so + KK_B, b1 = bo + KSTEP_B + HALF_B;
          so = (unsigned)__builtin_amdgcn_readlane(q.sot, (ki >> 1) & 63);
          bo ^= 2 * KSTEP_B;
          const bool out = ki >= here;
          SCHED;
          DMA4(b1, q.rb, q.voff, s1 & AG_HKB);
          if (out) break;
          GO();
          x = *la, y = *lb;
          DMA4(bo, q.ra, q.voff, so & AG_HKA);
          SCHED;
          ++k;
        }
        q.so = so;
        q.bo = bo;
        continue;
#undef GO
      }
      if (ki == k) {
        if (here == ki) {
          const unsigned long long t0 = __builtin_amdgcn_s_memrealtime();
          for (;;) {
            asm volatile("" ::: "memory");
            const unsigned x = __builtin_amdgcn_raw_buffer_load_b32(rf, q.fa, (unsigned)xs * 1024u, AUX_SYS), y = __builtin_amdgcn_raw_buffer_load_b32(rf, q.fb, (unsigned)wsl * 1024u, AUX_SYS);
            SEEN(x, y);
            if (here > ki) break;
            if (timed_out((unsigned*)(ws + OFF_TMO), seq, t0)) { here = nk; break; }
            __builtin_amdgcn_s_sleep(AG_Z4);
          }
          here = (int)RFL(here);
        }
        while (ki < here && ki < k + NBQ - 1) NEXT();
      }
      {
        const int y = ki - k - 1 + 2 * (pre > k);
        if (y == 2) WAIT_V(16);
        else if (y < 2) { if (y == 1) WAIT_V(8); else WAIT_V(0); }
        else if (y < 5) { if (y == 3) WAIT_V(24); else WAIT_V(32); }
        else WAIT_V(40);
      }
      BAR;
      QKT();
      const unsigned x = *la, y = *lb;
      if (ki < here) NEXT();
      SEEN(x, y);
      while (ki < here && ki < k + NBQ) NEXT();
    }
    u32x4 pt[3][16];
#pragma unroll
    for (int i = 0; i < 48; ++i) pt[i >> 4][i & 15] = (u32x4){0u, 0u, 0u, 0u};
    unsigned owed = (1u << np) - 1u;
    for (; k < nk; ++k) {
      if (owed && (AG_P1 || k > nk - NBQ + 1)) {
        const u32x2 z = *lp;
        unsigned rdy = BOTH(z) & owed;
        rdy &= 0u - rdy;
        PARTS(rdy);
      }
      const int y = nk - k - 1 + 2 * (np - __builtin_popcount(owed));
      if (y < 2) { if (y == 1) WAIT_V(8); else WAIT_V(0); }
      else if (y < 4) { if (y == 2) WAIT_V(16); else WAIT_V(24); }
      else if (y < 6) { if (y == 4) WAIT_V(32); else WAIT_V(40); }
      else if (y == 6) WAIT_V(48);
      else WAIT_V(56);
      BAR;
      QKT();
    }
    if (w8 == 4) KT(104, 0);
    if (owed) {
      const unsigned long long t0 = __builtin_amdgcn_s_memrealtime();
      do {
        asm volatile("" ::: "memory");
        const u32x2 z = __builtin_amdgcn_raw_buffer_load_b64(__builtin_amdgcn_make_buffer_rsrc(ws + part_words(xs, wsl, cq), 0, 128, RSRC_W3), po, 0, AUX_SYS);
        unsigned rdy = BOTH(z) & owed;
        if (!rdy && timed_out((unsigned*)(ws + OFF_TMO), seq, t0)) rdy = owed;
        rdy = RFL(rdy);
        PARTS(rdy);
      } while (owed);
    }
    const unsigned tm = __hip_atomic_load((unsigned*)(ws + OFF_TMO), __ATOMIC_RELAXED, __HIP_MEMORY_SCOPE_AGENT);
    __builtin_amdgcn_s_setprio(0);
#define F4(x) __builtin_bit_cast(f32x4, x)
    f32x4 sl[8];
#pragma unroll
    for (int i = 0; i < 8; ++i) {
      sl[i] = F4(pt[0][i]) + F4(pt[1][i]) + F4(pt[2][i]);
      acc[i >> 1][i & 1] = F4(pt[0][8 + i]) + F4(pt[1][8 + i]) + F4(pt[2][8 + i]);
    }
#undef F4
    WAIT_V(0);
    if (w8 == 4) KT(105, 0);
    BAR;
    if (w8 == 4) KT(106, 0);
#pragma unroll
    for (int i = 0; i < 8; ++i) *(LAS f32x4*)(xb + i * 1024 - KSTEP_B) = sl[i];
    *(LAS unsigned*)(lds + LDS_MAIN + 192 + (w8 & 3) * 4) = tm;
    WAIT_L(0);
    BAR;
#pragma unroll
    for (int i = 0; i < 8; ++i) acc[i >> 1][i & 1] += *(const LAS f32x4*)(xb + i * 1024);
    if (w8 == 4) { KT(107, 0); KDUMP(); }
  } else {
    const int fr = lane & 15, fq = lane >> 4;
    const int sw0 = (fq ^ (fr & 7)) * 16;
    const int aoff0 = ((w8 >> 1) * 64 + fr) * 128 + sw0, aoff1 = aoff0 ^ 64;
    const int boff0 = HALF_B + ((w8 & 1) * 64 + fr) * 128 + sw0, boff1 = boff0 ^ 64;
    f32x4 ac[4][4];
#pragma unroll
    for (int m = 0; m < 4; ++m)
#pragma unroll
      for (int n = 0; n < 4; ++n) ac[m][n] = (f32x4){0.f, 0.f, 0.f, 0.f};
    f16x8 A0[4][2], B0[4][2], A1[4][2], B1[4][2];
    const __amdgpu_buffer_rsrc_t rf = __builtin_amdgcn_make_buffer_rsrc(w8 == 2 ? ws + part_words(xs, wsl, cq) : ws + OFF_FLAG + par * FLAG_B + (w8 ? wsl : xs) * 1024, 0, w8 < 2 ? 1024 : w8 == 2 ? 128 : 0, RSRC_W3);
#define RD(A, B) do { _Pragma("unroll") for (int m = 0; m < 4; ++m) { \
      B[m][0] = *(const LAS f16x8*)(lds + bo + boff0 + m * 2048); A[m][0] = *(const LAS f16x8*)(lds + bo + aoff0 + m * 2048); } \
      _Pragma("unroll") for (int m = 0; m < 4; ++m) { \
      B[m][1] = *(const LAS f16x8*)(lds + bo + boff1 + m * 2048); A[m][1] = *(const LAS f16x8*)(lds + bo + aoff1 + m * 2048); } \
      NEXTB(bo); } while (0)
#define MM(A, B) do { _Pragma("unroll") for (int k = 0; k < 2; ++k) _Pragma("unroll") for (int m = 0; m < 4; ++m) _Pragma("unroll") for (int n = 0; n < 4; ++n) \
      ac[m][n] = __builtin_amdgcn_mfma_f32_16x16x32_f16(B[n][k], A[m][k], ac[m][n], 0, 0, 0); } while (0)
#define STEP(An, Bn, A, B, F) do { BAR; SCHED; RD(An, Bn); MM(A, B); \
      _Pragma("unroll") for (int i = 0; i < 16; ++i) { __builtin_amdgcn_sched_group_barrier(0x100, 1, 0); __builtin_amdgcn_sched_group_barrier(0x008, 1, 0); } \
      __builtin_amdgcn_sched_group_barrier(0x008, 16, 0); SCHED; \
      if (F) __builtin_amdgcn_raw_ptr_buffer_load_lds(rf, (LAS void*)(lds + LDS_F + w8 * 1024), 16, lane * 16, 0, 0, AUX_SYS); \
      WAIT_L0(); } while (0)
    unsigned bo = 0;
    BAR; SCHED;
    RD(A0, B0);
    WAIT_L0();
    for (int k = 1; k < nk - 1; k += 2) {
      STEP(A1, B1, A0, B0, 1);
      STEP(A0, B0, A1, B1, 0);
    }
    STEP(A1, B1, A0, B0, 0);
    MM(A1, B1);
    BAR;
#pragma unroll
    for (int i = 0; i < 8; ++i) *(LAS f32x4*)(xb + i * 1024) = ac[i >> 1][2 + (i & 1)];
    WAIT_V(0);
    WAIT_L(0);
    BAR;
#pragma unroll
    for (int i = 0; i < 8; ++i) acc[i >> 1][i & 1] = ac[i >> 1][i & 1] + *(const LAS f32x4*)(xb + i * 1024 - KSTEP_B);
#undef RD
#undef MM
#undef STEP
  }
  const int wr = (w8 >> 1) & 1, wc = (w8 & 1) * 2 + (w8 >> 2), wid = wr * 4 + wc;
  int tz = threadIdx.x; asm volatile("" : "+v"(tz));
  unsigned ea = *(const LAS unsigned*)(lds + LDS_F + __builtin_ctzll(mask) * 16 + (ah * 2 + wr) * 4), eb = *(const LAS unsigned*)(lds + LDS_F + 1024 + __builtin_ctzll(mask) * 16 + bh * 8);
  const bool bad = RFL(*(const LAS unsigned*)(lds + LDS_MAIN + 192 + (w8 & 3) * 4)) == seq;
  int e = (int)(ea & 511u) + (int)(eb & 511u) - 512;
  if ((int)RFL((ea - (seq << 9)) | (eb - (seq << 9))) < 0) e = flag_exp(ws, seq, xs, ah * 2 + wr) + flag_exp(ws, seq, wsl, bh * 2);
  if (wsl >= SLOT_WR) {
    unsigned mk;
    LAS unsigned char* cb = lds + w8 * 8192;
    const unsigned nb = gather_block(acc, e, bad, cb, mk), n64 = (nb + 63u) & ~63u;
    const unsigned vo = (unsigned)(tz & 63) * 16u, vi = (unsigned)((((P.rank >> 1) * 3 + xs - SLOT_V) * 4 + ah * 2 + bh) * 8 + wid);
    const __amdgpu_buffer_rsrc_t rv = __builtin_amdgcn_make_buffer_rsrc(P.ws[2 * (wsl - SLOT_WR) + 1 - (P.rank & 1)] + CTRL_B + OFF_VREC + vi * VREC_B, 0, 512u + n64, RSRC_W3);
    __builtin_amdgcn_raw_buffer_store_b64((u32x2){mk, (seq << 9) + (n64 >> 6)}, rv, vo >> 1, 0, AUX_SYS);
    if (nb + (vo >> 2) < n64) *(volatile LAS unsigned*)(cb + nb + (vo >> 2)) = 1u;
    u32x4 v[8];
#pragma unroll
    for (int i = 0; i < 8; ++i) v[i] = *(const volatile LAS u32x4*)(cb + i * 1024 + vo);
#pragma unroll
    for (int i = 0; i < 8; ++i)
      if (i * 1024u < n64) {
        __builtin_amdgcn_raw_buffer_store_b128(v[i], rv, vo, 512u + i * 1024u, AUX_SYS);
        WAIT_V(AG_DW);
      }
  } else {
    const __amdgpu_buffer_rsrc_t ro = __builtin_amdgcn_make_buffer_rsrc(P.out[P.rank], 0, 8192 * NLOC * 4, RSRC_W3);
    store_block(ro, acc, e, bad, (unsigned)group_row0(P.rank & 1, xs, ah * 2 + wr) * (NLOC * 4u) + (unsigned)(wsl - SLOT_W) * 1024u + bh * 512u + wc * 128u, lds + w8 * 8192, tz & 63);
  }
#undef NEXT
#undef SEEN
#undef PARTS
#undef BOTH
#undef NEXTB
}

#define AG_P0A 36
#define AG_P0S 34
#define AG_P0V 32
#define AG_G4 1
#define AG_SWO 1
__global__ __launch_bounds__(512, 2) void ag_gemm_kernel(Params P) {
  extern __shared__ __attribute__((aligned(16))) unsigned char shm[];
  LAS unsigned char* lds = (LAS unsigned char*)shm;
  const int tid = threadIdx.x;
  const int b = blockIdx.x, xcd = b & 7, bi = b >> 3;
  const int rank = P.rank, me = rank >> 1;
  unsigned char* ws = P.ws[rank];
  TS(0);
  if (bi < NPUSH) {
    const unsigned seq = link_role(P, ws, bi, xcd, lds);
    if (tid >= 64) return;
    TS(9);
    return;
  }
  const int cp = xcd & 3, rh = xcd >> 2;
  unsigned seq;
  const int f = bi - NPUSH, j = f - NFULL;
  if (bi < 32 - NPREP) {
    LAS unsigned* red = (LAS unsigned*)(lds + LDS_MAIN);
    if (tid == 0) red[8] = take_seq(ws, xcd);
    if (tid >= 16 && tid < 32) red[tid] = 0;
    WAIT_L(0);
    BAR;
    seq = RFL(red[8]);
  } else {
    seq = prep_role(P, ws, xcd, 31 - bi, lds);
    TS(2);
  }
#define ROWS(ri) ((ri) < 2 ? SLOT_A + rh * 2 + (ri) : (ri) < 6 ? ((me + 1 + rh * 2 + ((ri) & 1)) & 3) * 2 + (((ri) >> 2) ^ (rh & (ri) & AG_SWO)) : SLOT_V3)
  const int w8 = __builtin_amdgcn_readfirstlane(tid >> 6);
  const int g = j < 0 ? f : j < 4 ? 5 + 2 * j : j < 8 ? j - 4 : j == 11 && rh ? 12 : 2 * j - 12;
  const bool whole = g == (rh ? 10 : 12), split = g < 4 || (!(g & 1) && !whole);
  const int hand = xcd * 8 + (g < 4 ? g : g == 12 ? 7 : 2 + (g >> 1));
  const int p0 = !split ? 0 : g < 4 ? AG_P0A : g < 12 ? AG_P0S : AG_P0V;
  const bool far = g < 12 ? g & 1 : rh;
  const bool home = far && ROWS(g >> 1) == SLOT_A + cp;
  const int xs = home ? SLOT_T3 : ROWS(g >> 1), wsl = (far && !home ? SLOT_WR : SLOT_W) + cp;
  f32x4 hs[2][4][2];
#pragma unroll
  for (int i = 0; i < 16; ++i) hs[i >> 3][(i >> 1) & 3][i & 1] = (f32x4){0.f, 0.f, 0.f, 0.f};
  const int k = ((g - 4) >> 1) + (j >= 0 ? 4 : 0);
  const int job = split && g >= 4 && g < 12 ? (rh ? (k - (k >> 2)) * 3 : k < 6 ? k * 3 + 1 : (k - 6) * 3 + 2) : !rh && j >= 0 && j < 4 ? j * 3 + 8 : -1;
  const int nt = job >= 0 ? 1 : (g < 4 ? j < 0 || rh == AG_G4 : g & 1) ? 2 : 0;
  const int t = job >= 0 ? job / 3 : g < 4 ? (j >= 0 ? 4 : rh ? 5 : 3) : !rh ? 0 : j < 0 ? 1 : 2, cq = (g < 4 ? g : (g - 5) >> 1) & 3;
  const int nxs = SLOT_V + t % 3, nws = (t < 3 ? SLOT_WR : SLOT_W) + cp;
  const unsigned long long nmask = P.pairs[job >= 0 ? job : 18 + t];
  int pre = 0;
  if (j < 0 && (split || whole)) gemm_full(P, ws, seq, xs, wsl, split ? (1ull << p0) - 1ull : ~0ull, -1, split ? hand : -1, hs, 0, lds);
  if (!whole) pre = gemm_half(P, ws, seq, xs, j >= 0, wsl, p0, hand, hs, nt, nxs, nws, nmask, cq, lds);
  WAIT_L(0);
  BAR;
  if (job >= 0) {
    gemm_full(P, ws, seq, nxs, nws, nmask, cp * 18 + job, -1, hs, pre, lds);
  } else if (nt == 2) {
    gemm_quarter(P, ws, seq, nxs, cq >> 1, nws, cq & 1, nmask, cp * 18 + t * 3, 3, pre, lds);
  }
  const int un = j >= 4 && j < 8 ? (rh == AG_G4 ? -1 : j - 4) : rh ? (f == 10 ? 4 : f == 12 ? 5 : j == 11 ? 6 : f == 4 ? 7 : -1) : f == 12 ? 11 : j < 0 && job >= 1 && job < 8 ? job / 3 + 8 : -1;
  if (un >= 0) {
    WAIT_L(0);
    BAR;
    unpack_block(P, ws, seq, (un * 4 + cp) * 8 + w8, true, lds + w8 * 8192);
  }
#undef ROWS
}

#define AG_FR0 2.16
#define AG_FR1 2.16
#define AG_E0 146.
#define AG_E1 146.
#define AG_S0 200.
#define AG_R0 0.8
#define AG_S1 203.2
#define AG_R1 0.8
#define AG_S2 206.4
#define AG_R2 0.8
#define AG_PE0 0
#define AG_PE1 0
#define AG_PE2 0
#define AG_VA 30.
#define AG_VB 160.
#define AG_TH 40.
#define AG_DH 1.
#define AG_TF 64.
#define AG_DF 0.1
#define AG_NA 24
#define AG_RV 3
#define AG_RM 1
#define AG_XW 12
#define AG_L0 2
#define AG_L1 2
#define AG_LV 48
#define AG_LW 0
#define AG_LA 2
#define AG_LG 0
#define AG_GD 0.
#define AG_H0 14
#define AG_H1 14
#define AG_H2 14
#define AG_HE0 54
#define AG_HE1 54
#define AG_HE2 54
#define AG_HF0 50
#define AG_HF1 50
#define AG_HF2 50
#define AG_G0 28
#define AG_G1 28
#define AG_G2 28
#define AG_GE0 56
#define AG_GE1 56
#define AG_GE2 56
#define AG_GF0 50
#define AG_GF1 50
#define AG_GF2 50
#define AG_HB 0
#define AG_GB 0
#define AG_HX 100
#define AG_GX 100
#define AG_MO 3
struct Items {
  int n = 0;
  double t[512];
  int first[512], size[512];
  void add(double due, int f, int sz) {
    int i = n++;
    for (; i > 0 && t[i - 1] > due + 1e-9; --i) { t[i] = t[i - 1]; first[i] = first[i - 1]; size[i] = size[i - 1]; }
    t[i] = due; first[i] = f; size[i] = sz;
  }
  bool put(unsigned short* out, int room) const {
    for (int i = 0; i < n; ++i) room -= size[i];
    if (room != 0) { fprintf(stderr, "kernel_launch: piece list\n"); return false; }
    for (int i = 0; i < n; ++i)
      for (int k = 0; k < size[i]; ++k) *out++ = (unsigned short)(first[i] + k);
    return true;
  }
};

}

extern "C" void kernel_launch(int rank, int nranks, ncclComm_t comm,
                              hipStream_t stream,
                              void* const* d_in, const int* in_sizes, int n_in,
                              void* d_out, int out_size,
                              const int* in_dtypes, int out_dtype,
                              void* const* d_out_all,
                              void* const* d_ws_all, size_t ws_bytes) {
  (void)comm; (void)in_dtypes; (void)out_dtype; (void)d_out;
  if (nranks != NR || (unsigned)rank >= (unsigned)NR || n_in < 2 || ws_bytes < WS_NEED ||
      in_sizes[0] != 1024 * KDIM || in_sizes[1] != KDIM * NLOC || out_size != 8192 * NLOC) {
    fprintf(stderr, "kernel_launch: unexpected problem shape\n");
    return;
  }
  HIP_CHECK(hipFuncSetAttribute((const void*)ag_gemm_kernel, hipFuncAttributeMaxDynamicSharedMemorySize, LDS_B));
  Params P;
  P.x = (const float*)d_in[0];
  P.w = (const float*)d_in[1];
  for (int p = 0; p < NR; ++p) { P.out[p] = (float*)d_out_all[p]; P.ws[p] = (unsigned char*)d_ws_all[p]; }
  P.rank = rank;
  const int me = rank >> 1;
  auto detour = [](int v, int g) { const int i = g - AG_NA / 4; return i >= 0 && (AG_RV < 3 ? v == AG_RV && i < NREL / 4 : i * AG_RM + (v + 3 - i % 3) % 3 < NREL / 4 && (v + 3 - i % 3) % 3 < AG_RM); };
  {
    Items s;
    for (int p = 4; p < NPAIR; ++p) {
      s.add(AG_E0 - (NPAIR - p) * AG_FR0, 0 << 8 | p * 4 | (p >= NPAIR - AG_L0 ? 0x800 : 0), 4);
      s.add(AG_E1 - (NPAIR - p) * AG_FR1, 1 << 8 | p * 4 | (p >= NPAIR - AG_L1 ? 0x800 : 0), 4);
    }
    const double e[3] = {AG_S0, AG_S1, AG_S2}, r[3] = {AG_R0, AG_R1, AG_R2};
    const int pe[3] = {AG_PE0, AG_PE1, AG_PE2};
    for (int p = 0; p < NPAIR; ++p)
      for (int v = 0; v < 3; ++v)
        if (!detour(v, p >> 2)) s.add(p < pe[v] ? AG_VA + p * (AG_VB - AG_VA) / pe[v] : e[v] - (NPAIR - p) * r[v], (2 + v) << 8 | (p * 4 + me) | (p >= NPAIR - AG_LV ? 0x800 : 0), 1);
    if (!s.put(P.ord + NDIR, QSAME - NDIR)) return;
  }
  for (int p = 0; p < NDIR; ++p) P.ord[p] = 0;
  for (int p = 0; p < NDIR; ++p) P.ord[QSAME + p] = 0;
  {
    static_assert(AG_TF > AG_TH && AG_TF + (NREL / 4 - 1) * AG_DF > AG_TH + (NREL / 4 - 1) * AG_DH, "order");
    Items s;
    for (int p = 0; p < NPAIR; ++p) {
      if (p >= 4) s.add(4. + (p - 4) * (NPAIR - 5. - AG_XW) / (NPAIR - 5.), 0 << 8 | p * 4 | (p >= NPAIR - AG_LW ? 0x800 : 0), 4);
      if (p >= 4) s.add(p, 1 << 8 | p * 4 | (p >= NPAIR - AG_LA ? 0x800 : 0), 4);
      s.add(p - AG_GD, 2 << 8 | (p * 4 + me) | (p >= NPAIR - AG_LA || p < AG_LG ? 0x800 : 0), 1);
    }
    for (int g = 0, i = 0; g < NPAIR / 4; ++g)
      for (int v = 0; v < 3; ++v)
        if (detour(v, g)) {
          s.add(AG_TH + i * AG_DH, 7 << 8 | v << 6 | g * 4, 1);
          for (int k = 0; k < 3; ++k) s.add(AG_TF + i * AG_DF, (4 + k) << 8 | v << 6 | g * 4, 1);
          ++i;
        }
    if (!s.put(P.ord + QSAME + NDIR, QIN - NDIR)) return;
  }
  for (int i = 0; i < 24; ++i) P.pairs[i] = 0;
  int n[2][3] = {{0, 0, AG_MO}, {0, 0, AG_MO}};
  for (int p = 0; p < NPAIR; ++p) {
    const int a[6] = {AG_H0, AG_H1, AG_H2, AG_G0, AG_G1, AG_G2}, e[6] = {AG_HE0, AG_HE1, AG_HE2, AG_GE0, AG_GE1, AG_GE2};
    const int b[6] = {AG_HB, AG_HB, AG_HB, AG_GB, AG_GB, AG_GB}, f1[6] = {AG_HX, AG_HX, AG_HX, AG_GX, AG_GX, AG_GX};
    const int fr[6] = {AG_HF0, AG_HF1, AG_HF2, AG_GF0, AG_GF1, AG_GF2};
    bool on[6];
    for (int t = 0; t < 6; ++t) {
      const int lo = p < b[t] ? a[t] : b[t] > a[t] ? b[t] : a[t], f = p < b[t] ? f1[t] : fr[t];
      on[t] = p < a[t] || (p < e[t] && (p > lo ? (p - lo) * f / 100 == (p - lo - 1) * f / 100 : lo > a[t]));
    }
    int m[3];
    for (int q = 0; q < 3; ++q) m[q] = on[0] && on[3] ? (n[0][q] > n[1][q] ? n[0][q] : n[1][q]) * 64 + n[0][q] + n[1][q] : n[on[3]][q];
    const int i = m[0] <= m[1] && m[0] <= m[2] ? 0 : m[1] <= m[2] ? 1 : 2;
    n[0][i] += on[0];
    n[1][i] += on[3];
    for (int t = 0; t < 6; ++t) P.pairs[on[t] ? t * 3 + i : 18 + t] |= 1ull << p;
  }
  hipLaunchKernelGGL(ag_gemm_kernel, dim3(NWG), dim3(512), LDS_B, stream, P);
  HIP_CHECK(hipGetLastError());
}
```
